```python
import numpy as np

kernel_name = "gpu_gemm_rs_m4096_k4096_n4096_f32_none"


ABSMAX_RTOL = 0.001

COMPUTE_DTYPE = "float32"
EPILOGUE = "none"
IN_QUANT = None
OUT_QUANT = None
IN_DTYPES = {"x": "float32", "w_mat": "float32"}
WORLD_SIZE = 8

def setup_inputs():

    x = np.random.randn(4096, 4096).astype(np.float32)
    w_mat = (np.random.randn(4096, 4096) / np.sqrt(4096)).astype(np.float32)
    return {"x": x, "w_mat": w_mat}

def _epilogue(y):
    return y

def reference(x, w_mat):
    y = (x @ w_mat).astype(np.float32)
    return _epilogue(y)
```

```cpp
#include <hip/hip_runtime.h>
#include <rccl/rccl.h>
#include <cstdio>
#include <cstdint>

#define HIP_CHECK(expr) do { hipError_t _e = (expr); if (_e != hipSuccess) { \
    fprintf(stderr, "HIP error %s at %s:%d\n", hipGetErrorString(_e), __FILE__, __LINE__); \
    return; } } while (0)

namespace {

typedef _Float16 h8 __attribute__((ext_vector_type(8)));
typedef float f4 __attribute__((ext_vector_type(4)));
typedef unsigned u4 __attribute__((ext_vector_type(4)));
typedef unsigned long long u64;

constexpr int NR = 8;
constexpr int NCOL = 4096;
constexpr int KSH = 512;
constexpr int MSH = 512;

constexpr size_t ITEM = 16384;
constexpr size_t XS_BYTES = (size_t)8 * 32 * 2 * ITEM;
constexpr size_t WS_BYTES = (size_t)32 * 32 * ITEM;
constexpr size_t BUF_BYTES = XS_BYTES + WS_BYTES;
constexpr size_t CTRL = 2 * BUF_BYTES;
constexpr size_t FLAG_PAR = 8192;
constexpr size_t XFLAG = 0;
constexpr size_t WFLAG = 2048;
constexpr size_t YFLAG = 6144;
constexpr size_t EPOCH = CTRL + 2 * FLAG_PAR;
constexpr size_t TMO = EPOCH + 16384;
constexpr size_t STAMP = TMO + 4096;
constexpr size_t WS_NEED = STAMP + 262144;

#define NPUSH_WG 4
constexpr int NPW = NPUSH_WG * 4;
constexpr int GRID = 8 * (NPUSH_WG + 32);
static_assert(GRID * 4 * 4 <= 16384, "epoch area");

#define NFWD 8
#define NTB 8
constexpr int TA = 32 - NTB;
constexpr int TF = 32 - NFWD;

constexpr u64 DEADLINE_TICKS = 100000000ull;

constexpr int MAXITEMS = 256;
struct Tables {
    uint16_t list[3][MAXITEMS];
    int len[3];
    uint8_t order[4][32];
};

constexpr Tables make_tables() {
    Tables T{};
    for (int c = 0; c < 3; ++c) {
        int n = 0;
        for (int q = 0; q < 4; ++q) for (int a = 8; a < 16; ++a) T.list[c][n++] = (uint16_t)(0 << 8 | a << 2 | q);
        for (int q = 0; q < 4; ++q) for (int t = 0; t < TA; ++t)
            if (c != 2 || t < TF || q == 0) T.list[c][n++] = (uint16_t)(1 << 8 | t << 2 | q);
        for (int q = 0; q < 4; ++q) for (int t = TA; t < 32; ++t)
            if (c != 2 || t < TF || q == 0) T.list[c][n++] = (uint16_t)(1 << 8 | t << 2 | q);
        if (c == 1)
            for (int s = 0; s < 4; ++s) for (int t = TF; t < 32; ++t) T.list[c][n++] = (uint16_t)(2 << 8 | t << 2 | s);
        for (int q = 0; q < 4; ++q) for (int a = 0; a < 8; ++a) T.list[c][n++] = (uint16_t)(0 << 8 | a << 2 | q);
        T.len[c] = n;
    }
    for (int c = 0; c < 4; ++c) {
        int n = 0;
        for (int q = 0; q < 4; ++q) T.order[c][n++] = (uint8_t)(0 << 2 | q);
        if (c == 0) {
            for (int q = 0; q < 4; ++q) for (int j = 1; j < 8; ++j) T.order[c][n++] = (uint8_t)(j << 2 | q);
        } else if (c == 1) {
            for (int j = 1; j < 8; ++j) T.order[c][n++] = (uint8_t)(j << 2 | 0);
            for (int q = 1; q < 4; ++q) for (int j = 1; j < 4; ++j) T.order[c][n++] = (uint8_t)(j << 2 | q);
            for (int j = 4; j < 8; ++j) for (int q = 1; q < 4; ++q) T.order[c][n++] = (uint8_t)(j << 2 | q);
        } else {
            for (int q = 0; q < 4; ++q) for (int j = 4; j < 8; ++j) T.order[c][n++] = (uint8_t)(j << 2 | q);
            for (int q = 0; q < 4; ++q) for (int j = 1; j < 4; ++j) T.order[c][n++] = (uint8_t)(j << 2 | q);
        }
    }
    return T;
}

struct Params {
    const float* x;
    const float* w;
    float* out[NR];
    char* ws[NR];
    int rank;
    int len[3];
    uint32_t list[3][MAXITEMS / 2];
    uint32_t order[4][8];
};

#define RLX_SYS __ATOMIC_RELAXED, __HIP_MEMORY_SCOPE_SYSTEM
#define RLX_AGT __ATOMIC_RELAXED, __HIP_MEMORY_SCOPE_AGENT
constexpr int AUX_SYS = 17;

__device__ __forceinline__ int uni(int v) { return __builtin_amdgcn_readfirstlane(v); }

#define STAMP_AT(slot) do { } while (0)

__device__ __forceinline__ bool spin_pause(u64 t0, unsigned* tmo, unsigned call) {
    __builtin_amdgcn_s_sleep(2);
    u64 dt = __builtin_amdgcn_s_memrealtime() - t0;
    if (dt > 50000ull) {
        if (dt > DEADLINE_TICKS || (unsigned)uni((int)__hip_atomic_load(tmo, RLX_AGT)) == call) {
            __hip_atomic_store(tmo, call, RLX_AGT);
            return true;
        }
        __builtin_amdgcn_s_sleep(32);
    }
    return false;
}

__device__ __forceinline__ float wave_max(float v) {
#pragma unroll
    for (int o = 32; o > 0; o >>= 1) v = fmaxf(v, __shfl_xor(v, o, 64));
    return v;
}

__global__ void __launch_bounds__(256, 2) gemm_rs_kernel(const Params P) {
    __shared__ __attribute__((aligned(1024))) char lds[4 * ITEM];
    const int lane = threadIdx.x & 63;
    const int wave = uni(threadIdx.x >> 6);
    const int link = blockIdx.x & 7;
    const int slot = blockIdx.x >> 3;
    const int rank = P.rank & 7;
    const int peer = rank ^ link;
    char* const myws = P.ws[rank];
    unsigned* const tmo = (unsigned*)(myws + TMO);

    unsigned* const ep = (unsigned*)(myws + EPOCH) + (blockIdx.x * 4 + wave);
    const unsigned call = (unsigned)uni((int)__hip_atomic_load(ep, RLX_AGT)) + 1u;
    if (lane == 0) __hip_atomic_store(ep, call, RLX_AGT);
    const unsigned tag = call & 0xFFFFFFu;
    const size_t par = call & 1u;
    STAMP_AT(0);

    if (slot < NPUSH_WG) {
        const int cls = link == 0 ? 0 : (link < 4 ? 1 : 2);
        const int len = P.len[cls];
        char* const dws = P.ws[peer];
        char* const dbuf = dws + par * BUF_BYTES;
        unsigned* const dflag = (unsigned*)(dws + CTRL + par * FLAG_PAR);
        const int ppos = peer & 3;
        const int c16 = lane & 15, q4 = lane >> 4;
        unsigned* pend = nullptr;
        unsigned pendv = 0;
        bool dead = false;
        for (int i = slot * 4 + wave; i < len; i += NPW) {
            const unsigned code = (P.list[cls][i >> 1] >> ((i & 1) * 16)) & 0xFFFFu;
            const int type = code >> 8, a = (code >> 2) & 31, q = code & 3;
            u4 o[16];
            unsigned ecode;
            size_t doff;
            unsigned* fl;
            if (type == 2) {
                const int s = ((peer ^ (4 + q)) & 7) * 4 + (rank & 3);
                doff = XS_BYTES + ((size_t)a * 32 + s) * ITEM;
                const size_t fo = (WFLAG >> 2) + a * 32 + s;
                fl = dflag + fo;
                unsigned* src_flag = (unsigned*)(myws + CTRL + par * FLAG_PAR) + fo;
                unsigned v = (unsigned)uni((int)__hip_atomic_load(src_flag, RLX_SYS));
                asm volatile("s_waitcnt vmcnt(0)" ::: "memory");
                if (pend && lane == 0) __hip_atomic_store(pend, pendv, RLX_SYS);
                pend = nullptr;
                if ((v >> 8) != tag) {
                    const u64 t0 = __builtin_amdgcn_s_memrealtime();
                    while ((v >> 8) != tag) {
                        if (spin_pause(t0, tmo, call)) { dead = true; break; }
                        v = (unsigned)uni((int)__hip_atomic_load(src_flag, RLX_SYS));
                    }
                }
                if (dead) break;
                ecode = v & 255u;
                auto rs = __builtin_amdgcn_make_buffer_rsrc(myws + par * BUF_BYTES + doff, 0, (int)ITEM, 0x00020000);
#pragma unroll
                for (int f = 0; f < 16; ++f) o[f] = __builtin_amdgcn_raw_buffer_load_b128(rs, lane * 16, f * 1024, AUX_SYS);
            } else {
                const int qa = q ^ ppos;
                const int s = rank * 4 + qa;
                f4 v[32];
                if (type == 0) {
                    const int rb = a >> 1, half = a & 1;
                    const int row0 = (rb < 4 ? peer * MSH + rb * 128 : ((peer ^ rb) & 7) * MSH + ppos * 128) + half * 64;
                    doff = (((size_t)rb * 32 + s) * 2 + half) * ITEM;
                    fl = dflag + (XFLAG >> 2) + a * 32 + s;
                    auto rs = __builtin_amdgcn_make_buffer_rsrc((void*)(P.x + (size_t)row0 * KSH + qa * 128), 0,
                                                                64 * KSH * 4 - qa * 512, 0x00020000);
                    const int vo = c16 * (KSH * 4) + q4 * 16;
#pragma unroll
                    for (int mf = 0; mf < 4; ++mf)
#pragma unroll
                        for (int h = 0; h < 8; ++h)
                            v[mf * 8 + h] = __builtin_bit_cast(f4, __builtin_amdgcn_raw_buffer_load_b128(rs, vo + h * 64, mf * 16 * KSH * 4, 0));
                } else {
                    doff = XS_BYTES + ((size_t)a * 32 + s) * ITEM;
                    fl = dflag + (WFLAG >> 2) + a * 32 + s;
                    const int col0 = (peer >> 2) * 2048 + a * 64;
                    auto rs = __builtin_amdgcn_make_buffer_rsrc((void*)(P.w + (size_t)qa * 128 * NCOL + col0), 0,
                                                                128 * NCOL * 4 - col0 * 4, 0x00020000);
                    const int vo = q4 * 4 * (NCOL * 4) + c16 * 16;
#pragma unroll
                    for (int ks = 0; ks < 4; ++ks)
#pragma unroll
                        for (int j = 0; j < 8; ++j)
                            v[ks * 8 + j] = __builtin_bit_cast(f4, __builtin_amdgcn_raw_buffer_load_b128(
                                rs, vo, (ks * 32 + (j >> 2) * 16 + (j & 3)) * (NCOL * 4), 0));
                }
                asm volatile("s_waitcnt vmcnt(0)" ::: "memory");
                if (pend && lane == 0) __hip_atomic_store(pend, pendv, RLX_SYS);
                pend = nullptr;
                float m = 0.f;
#pragma unroll
                for (int k = 0; k < 32; ++k)
                    m = fmaxf(fmaxf(m, fmaxf(fabsf(v[k][0]), fabsf(v[k][1]))), fmaxf(fabsf(v[k][2]), fabsf(v[k][3])));
                m = wave_max(m);
                const int eb = (uni(__builtin_bit_cast(int, m)) >> 23) & 255;
                const int e8 = (eb - 134) >> 3;
                ecode = (unsigned)(e8 + 32);
                const int sh = -8 * e8;
#pragma unroll
                for (int k = 0; k < 32; ++k)
#pragma unroll
                    for (int c = 0; c < 4; ++c) v[k][c] = __builtin_amdgcn_ldexpf(v[k][c], sh);
                if (type == 0) {
#pragma unroll
                    for (int f = 0; f < 16; ++f) {
                        const f4 lo = v[2 * f], hi = v[2 * f + 1];
                        h8 t = {(_Float16)lo[0], (_Float16)lo[1], (_Float16)lo[2], (_Float16)lo[3],
                                (_Float16)hi[0], (_Float16)hi[1], (_Float16)hi[2], (_Float16)hi[3]};
                        o[f] = __builtin_bit_cast(u4, t);
                    }
                } else {
#pragma unroll
                    for (int f = 0; f < 16; ++f) {
                        const int b = (f >> 2) * 8, nf = f & 3;
                        h8 t = {(_Float16)v[b][nf], (_Float16)v[b + 1][nf], (_Float16)v[b + 2][nf], (_Float16)v[b + 3][nf],
                                (_Float16)v[b + 4][nf], (_Float16)v[b + 5][nf], (_Float16)v[b + 6][nf], (_Float16)v[b + 7][nf]};
                        o[f] = __builtin_bit_cast(u4, t);
                    }
                }
            }
            auto rd = __builtin_amdgcn_make_buffer_rsrc(dbuf + doff, 0, (int)ITEM, 0x00020000);
#pragma unroll
            for (int f = 0; f < 16; ++f) __builtin_amdgcn_raw_buffer_store_b128(o[f], rd, lane * 16, f * 1024, AUX_SYS);
            pend = fl;
            pendv = tag << 8 | ecode;
        }
        asm volatile("s_waitcnt vmcnt(0)" ::: "memory");
        if (pend && lane == 0) __hip_atomic_store(pend, pendv, RLX_SYS);
        STAMP_AT(1);
        if (blockIdx.x == 0 && wave == 0) {
            unsigned* yf = (unsigned*)(myws + CTRL + par * FLAG_PAR + YFLAG) + lane;
            const u64 t0 = __builtin_amdgcn_s_memrealtime();
            for (;;) {
                const unsigned v0 = __hip_atomic_load(yf, RLX_SYS), v1 = __hip_atomic_load(yf + 64, RLX_SYS);
                if (__all(v0 == tag && v1 == tag)) break;
                if (spin_pause(t0, tmo, call)) { dead = true; break; }
            }
            __builtin_amdgcn_fence(__ATOMIC_ACQUIRE, "");
            STAMP_AT(2);
        }
        if (dead && lane == 0) {
            P.out[rank][((rank >> 2) ^ 1) * 2048] = __builtin_nanf("");
        }
        return;
    }

    const int t = slot - NPUSH_WG;
    const int cls = (link < 4 ? 2 : 0) | (t >= TF ? 1 : 0);
    const int pos = rank & 3;
    const char* const abase = myws + par * BUF_BYTES + (size_t)link * (32 * 2 * ITEM) + wave * 8192;
    const char* const bbase = myws + par * BUF_BYTES + XS_BYTES + (size_t)t * (32 * ITEM) + wave * 4096;
    unsigned* const fptr = (unsigned*)(myws + CTRL + par * FLAG_PAR) +
        (lane < 32 ? (XFLAG >> 2) + (link * 2 + (wave >> 1)) * 32 + lane : (WFLAG >> 2) + t * 32 + (lane - 32));
    const unsigned ldsb = (unsigned)(uintptr_t)lds + wave * 4096;
    const int voa = lane * 16 + 4096, vob = lane * 16;

    f4 acc[2][4];
#pragma unroll
    for (int i = 0; i < 2; ++i)
#pragma unroll
        for (int j = 0; j < 4; ++j) acc[i][j] = f4{0.f, 0.f, 0.f, 0.f};
    h8 A[4][8];
    int E[4];
    int ecur = 0, emax = -1000;
    bool first = true;
    u64 ready = 0;
    unsigned fv = 0;
    bool dead = false;

    auto issue = [&](int u, h8 (&a)[8], int& e, int ring) {
        const unsigned oc = (P.order[cls][u >> 2] >> ((u & 3) * 8)) & 255u;
        const int s = ((rank ^ (oc >> 2)) & 7) * 4 + ((oc ^ pos) & 3);
        const u64 need = (1ull << s) | (1ull << (32 + s));
        if ((ready & need) != need) {
            const u64 t0 = __builtin_amdgcn_s_memrealtime();
            for (;;) {
                fv = __hip_atomic_load(fptr, RLX_SYS);
                ready = __ballot((fv >> 8) == tag);
                if ((ready & need) == need) break;
                if (spin_pause(t0, tmo, call)) { dead = true; ready = ~0ull; break; }
            }
        }
        e = (int)(__builtin_amdgcn_readlane(fv, s) & 255u) + (int)(__builtin_amdgcn_readlane(fv, 32 + s) & 255u) - 64;
        const char* sa = abase + (size_t)s * (2 * ITEM);
        const char* sb = bbase + (size_t)s * ITEM;
        const unsigned m0v = ldsb + ring * (unsigned)ITEM;
        unsigned keep;
        asm volatile(
            "s_mov_b32 %[keep], m0\n\t"
            "s_mov_b32 m0, %[m0v]\n\t"
            "s_nop 4\n\t"
            "global_load_lds_dwordx4 %[vob], %[sb] sc0 sc1\n\t"
            "global_load_lds_dwordx4 %[vob], %[sb] offset:1024 sc0 sc1\n\t"
            "global_load_lds_dwordx4 %[vob], %[sb] offset:2048 sc0 sc1\n\t"
            "global_load_lds_dwordx4 %[vob], %[sb] offset:3072 sc0 sc1\n\t"
            "global_load_dwordx4 %[a0], %[voa], %[sa] offset:-4096 sc0 sc1\n\t"
            "global_load_dwordx4 %[a4], %[voa], %[sa] sc0 sc1\n\t"
            "global_load_dwordx4 %[a1], %[voa], %[sa] offset:-3072 sc0 sc1\n\t"
            "global_load_dwordx4 %[a5], %[voa], %[sa] offset:1024 sc0 sc1\n\t"
            "global_load_dwordx4 %[a2], %[voa], %[sa] offset:-2048 sc0 sc1\n\t"
            "global_load_dwordx4 %[a6], %[voa], %[sa] offset:2048 sc0 sc1\n\t"
            "global_load_dwordx4 %[a3], %[voa], %[sa] offset:-1024 sc0 sc1\n\t"
            "global_load_dwordx4 %[a7], %[voa], %[sa] offset:3072 sc0 sc1\n\t"
            "s_mov_b32 m0, %[keep]"
            : [keep] "=&s"(keep), [a0] "=&v"(a[0]), [a1] "=&v"(a[1]), [a2] "=&v"(a[2]), [a3] "=&v"(a[3]),
              [a4] "=&v"(a[4]), [a5] "=&v"(a[5]), [a6] "=&v"(a[6]), [a7] "=&v"(a[7])
            : [m0v] "s"(m0v), [vob] "v"(vob), [sb] "s"(sb), [voa] "v"(voa), [sa] "s"(sa)
            : "memory");
    };

#define LANDED(N, a) asm volatile("s_waitcnt vmcnt(" #N ") lgkmcnt(0)\n\ts_barrier" \
        : "+v"(a[0]), "+v"(a[1]), "+v"(a[2]), "+v"(a[3]), "+v"(a[4]), "+v"(a[5]), "+v"(a[6]), "+v"(a[7]) :: "memory")

    auto compute = [&](h8 (&a)[8], int e, int ring) {
        if (e != ecur) {
            if (first) { ecur = e; emax = e; }
            emax = e > emax ? e : emax;
            const int et = e > emax - 7 ? e : emax - 7;
            const int sh = 8 * (ecur - et);
#pragma unroll
            for (int i = 0; i < 2; ++i)
#pragma unroll
                for (int j = 0; j < 4; ++j)
#pragma unroll
                    for (int c = 0; c < 4; ++c) acc[i][j][c] = __builtin_amdgcn_ldexpf(acc[i][j][c], sh);
            ecur = et;
            if (et != e) {
                const _Float16 f = (_Float16)__builtin_amdgcn_ldexpf(1.0f, 8 * (e - et));
#pragma unroll
                for (int i = 0; i < 8; ++i) a[i] *= f;
            }
        }
        if (first) { emax = e; first = false; }
        const h8* bp = (const h8*)(lds + ring * ITEM) + lane;
#pragma unroll
        for (int ks = 0; ks < 4; ++ks)
#pragma unroll
            for (int nf = 0; nf < 4; ++nf) {
                const h8 b = bp[(ks * 4 + nf) * 64];
                acc[0][nf] = __builtin_amdgcn_mfma_f32_16x16x32_f16(a[ks], b, acc[0][nf], 0, 0, 0);
                acc[1][nf] = __builtin_amdgcn_mfma_f32_16x16x32_f16(a[4 + ks], b, acc[1][nf], 0, 0, 0);
            }
    };

    issue(0, A[0], E[0], 0);
    issue(1, A[1], E[1], 1);
    issue(2, A[2], E[2], 2);
    STAMP_AT(1);
#pragma unroll 1
    for (int u = 0; u < 28; u += 4) {
        LANDED(24, A[0]); issue(u + 3, A[3], E[3], 3); compute(A[0], E[0], 0);
        LANDED(24, A[1]); issue(u + 4, A[0], E[0], 0); compute(A[1], E[1], 1);
        LANDED(24, A[2]); issue(u + 5, A[1], E[1], 1); compute(A[2], E[2], 2);
        LANDED(24, A[3]); issue(u + 6, A[2], E[2], 2); compute(A[3], E[3], 3);
    }
    LANDED(24, A[0]); issue(31, A[3], E[3], 3); compute(A[0], E[0], 0);
    STAMP_AT(9);
    LANDED(24, A[1]); compute(A[1], E[1], 1);
    LANDED(12, A[2]); compute(A[2], E[2], 2);
    LANDED(0, A[3]);  compute(A[3], E[3], 3);
    STAMP_AT(10);

    if (!dead) dead = (unsigned)uni((int)__hip_atomic_load(tmo, RLX_AGT)) == call;
    {
        const int orow = (link < 4 ? link : pos) * 128 + wave * 32;
        float* ob = P.out[link < 4 ? rank : peer] + (size_t)orow * NCOL + (rank >> 2) * 2048 + t * 64;
        auto ro = __builtin_amdgcn_make_buffer_rsrc(ob, 0, 31 * NCOL * 4 + 256, 0x00020000);
        const int vo = (lane >> 4) * 4 * (NCOL * 4) + (lane & 15) * 16;
        const int sh = 8 * ecur;
#pragma unroll
        for (int i = 0; i < 2; ++i)
#pragma unroll
            for (int j = 0; j < 4; ++j) {
                f4 r;
#pragma unroll
                for (int c = 0; c < 4; ++c) r[c] = dead ? __builtin_nanf("") : __builtin_amdgcn_ldexpf(acc[i][c][j], sh);
                __builtin_amdgcn_raw_buffer_store_b128(__builtin_bit_cast(u4, r), ro, vo, (i * 16 + j) * (NCOL * 4), AUX_SYS);
            }
    }
    if (link >= 4) {
        asm volatile("s_waitcnt vmcnt(0)\n\ts_barrier" ::: "memory");
        if (threadIdx.x == 0) {
            unsigned* yf = (unsigned*)(P.ws[peer] + CTRL + par * FLAG_PAR + YFLAG) + pos * 32 + t;
            __hip_atomic_store(yf, tag, RLX_SYS);
        }
    }
    STAMP_AT(11);
}

}

extern "C" void kernel_launch(int rank, int nranks, ncclComm_t comm,
                              hipStream_t stream,
                              void* const* d_in, const int* in_sizes, int n_in,
                              void* d_out, int out_size,
                              const int* in_dtypes, int out_dtype,
                              void* const* d_out_all,
                              void* const* d_ws_all, size_t ws_bytes) {
    (void)comm; (void)in_dtypes; (void)out_dtype; (void)d_out;
    if (nranks != NR || (unsigned)rank >= (unsigned)NR || n_in < 2 || ws_bytes < WS_NEED ||
        in_sizes[0] != 4096 * KSH || in_sizes[1] != KSH * NCOL || out_size != MSH * NCOL) {
        fprintf(stderr, "kernel_launch: unexpected problem shape\n");
        return;
    }
    constexpr Tables T = make_tables();
    Params P;
    P.x = (const float*)d_in[0];
    P.w = (const float*)d_in[1];
    for (int p = 0; p < NR; ++p) { P.out[p] = (float*)d_out_all[p]; P.ws[p] = (char*)d_ws_all[p]; }
    P.rank = rank;
    for (int c = 0; c < 3; ++c) {
        P.len[c] = T.len[c];
        for (int i = 0; i < MAXITEMS / 2; ++i) P.list[c][i] = (uint32_t)T.list[c][2 * i] | (uint32_t)T.list[c][2 * i + 1] << 16;
    }
    for (int c = 0; c < 4; ++c)
        for (int i = 0; i < 8; ++i)
            P.order[c][i] = (uint32_t)T.order[c][4 * i] | (uint32_t)T.order[c][4 * i + 1] << 8 |
                            (uint32_t)T.order[c][4 * i + 2] << 16 | (uint32_t)T.order[c][4 * i + 3] << 24;
    gemm_rs_kernel<<<GRID, 256, 0, stream>>>(P);
    HIP_CHECK(hipGetLastError());
}
```

```cpp
#include <hip/hip_runtime.h>
#include <rccl/rccl.h>
#include <cstdio>
#include <cstdint>

#define HIP_CHECK(expr) do { hipError_t _e = (expr); if (_e != hipSuccess) { \
    fprintf(stderr, "HIP error %s at %s:%d\n", hipGetErrorString(_e), __FILE__, __LINE__); \
    return; } } while (0)

namespace {

typedef _Float16 h8 __attribute__((ext_vector_type(8)));
typedef float f4 __attribute__((ext_vector_type(4)));
typedef unsigned u4 __attribute__((ext_vector_type(4)));
typedef unsigned long long u64;

constexpr int NR = 8;
constexpr int NCOL = 4096;
constexpr int KSH = 512;
constexpr int MSH = 512;

constexpr size_t ITEM = 16384;
constexpr size_t XS_BYTES = (size_t)8 * 32 * 2 * ITEM;
constexpr size_t WS_BYTES = (size_t)32 * 32 * ITEM;
constexpr size_t BUF_BYTES = XS_BYTES + WS_BYTES;
constexpr size_t CTRL = 2 * BUF_BYTES;
constexpr size_t FLAG_PAR = 8192;
constexpr size_t XFLAG = 0;
constexpr size_t WFLAG = 2048;
constexpr size_t YFLAG = 6144;
constexpr size_t EPOCH = CTRL + 2 * FLAG_PAR;
constexpr size_t TMO = EPOCH + 16384;
constexpr size_t STAMP = TMO + 4096;
constexpr size_t WS_NEED = STAMP + (1 << 20);

constexpr u64 DEADLINE_TICKS = 100000000ull;

constexpr int MAXITEMS = 256;

struct Params {
    const float* x;
    const float* w;
    char* ws[NR];
    float* out[NR];
    int rank;
    int nwg, pw, per;
    int pace, lead;
    int ybase;
    int pfstart, map1, rot, safe;
    uint32_t list[3][MAXITEMS / 2];
};

#define RLX_SYS __ATOMIC_RELAXED, __HIP_MEMORY_SCOPE_SYSTEM
#define RLX_AGT __ATOMIC_RELAXED, __HIP_MEMORY_SCOPE_AGENT
constexpr int AUX_SYS = 17;

__device__ __forceinline__ int uni(int v) { return __builtin_amdgcn_readfirstlane(v); }

#define STAMP_V(slot, val) do { } while (0)
#define STAMP_AT(slot) do { } while (0)
#define STAMP_OUT() do { } while (0)

__device__ __forceinline__ bool spin_pause(u64 t0, unsigned* tmo, unsigned call) {
    __builtin_amdgcn_s_sleep(2);
    u64 dt = __builtin_amdgcn_s_memrealtime() - t0;
    if (dt > 50000ull) {
        if (dt > DEADLINE_TICKS || (unsigned)uni((int)__hip_atomic_load(tmo, RLX_AGT)) == call) {
            __hip_atomic_store(tmo, call, RLX_AGT);
            return true;
        }
        __builtin_amdgcn_s_sleep(32);
    }
    return false;
}

__device__ __forceinline__ float wave_max(float v) {
#pragma unroll
    for (int o = 32; o > 0; o >>= 1) v = fmaxf(v, __shfl_xor(v, o, 64));
    return v;
}

__device__ __forceinline__ void wait_until(unsigned tstart, int ticks) {
    while ((int)((unsigned)__builtin_amdgcn_s_memrealtime() - tstart) < ticks) __builtin_amdgcn_s_sleep(1);
}

__global__ void __launch_bounds__(256, 2) gemm_rs_kernel(const Params P) {
    __shared__ __attribute__((aligned(1024))) char lds[4 * ITEM + 64];
    const unsigned tstart = (unsigned)__builtin_amdgcn_s_memrealtime();
    const int lane = threadIdx.x & 63;
    const int wave = uni(threadIdx.x >> 6);
    const int link = blockIdx.x & 7;
    const int slot = blockIdx.x >> 3;
    const int rank = P.rank & 7;
    const int peer = rank ^ link;
    char* const myws = P.ws[0];
    unsigned* const tmo = (unsigned*)(myws + TMO);
    const int nwg = P.nwg, pw = P.pw, per = P.per, pace = P.pace, lead = P.lead, ybase = P.ybase, pfstart = P.pfstart, map1 = P.map1, prot = P.rot, safe = P.safe;
    const bool pusher = slot < nwg;
    if (pusher && wave >= pw) return;

    unsigned* const ep = (unsigned*)(myws + EPOCH) + (blockIdx.x * 4 + wave);
    const unsigned epv = __hip_atomic_load(ep, RLX_AGT);

    if (pusher) {
        const int cls = link == 0 ? 0 : (link < 4 ? 1 : 2);
        const int npw = nwg * pw;
        const int widx = wave * nwg + slot;
        char* const dws = P.ws[link];
        const int ppos = peer & 3;
        const int c16 = lane & 15, q4 = lane >> 4;
        bool dead = false;
        f4 v[32];
        auto code_of = [&](int k) -> unsigned {
            const int e = widx * per + k;
            return k < per ? (P.list[cls][e >> 1] >> ((e & 1) * 16)) & 0xFFFFu : 0xFFFFu;
        };
        auto fetch = [&](unsigned code) {
            const int type = code >> 8, a = (code >> 2) & 31, qa = (code & 3) ^ ppos;
            const float* base;
            int bytes, s0, s1, s2, vo;
            if (type == 0) {
                const int rb = a >> 1, half = a & 1;
                const int row0 = (rb < 4 ? peer * MSH + rb * 128 : ((peer ^ rb) & 7) * MSH + ppos * 128) + half * 64;
                base = P.x + (size_t)row0 * KSH + qa * 128;
                bytes = 64 * KSH * 4 - qa * 512;
                s0 = 64; s1 = 256; s2 = 16 * KSH * 4;
                vo = c16 * (KSH * 4) + q4 * 16;
            } else {
                const int col0 = (peer >> 2) * 2048 + a * 64;
                base = P.w + (size_t)qa * 128 * NCOL + col0;
                bytes = type == 1 ? 128 * NCOL * 4 - col0 * 4 : 0;
                s0 = NCOL * 4; s1 = 16 * NCOL * 4; s2 = 32 * NCOL * 4;
                vo = q4 * 4 * (NCOL * 4) + c16 * 16;
            }
            auto rs = __builtin_amdgcn_make_buffer_rsrc((void*)base, 0, bytes, 0x00020000);
#pragma unroll
            for (int j = 0; j < 32; ++j)
                v[j] = __builtin_bit_cast(f4, __builtin_amdgcn_raw_buffer_load_b128(
                    rs, vo, (j >> 3) * s2 + ((j >> 2) & 1) * s1 + (j & 3) * s0, 0));
        };
        unsigned code = code_of(0);
        fetch(code);
        const unsigned call = (unsigned)uni((int)epv) + 1u;
        if (lane == 0) __hip_atomic_store(ep, call, RLX_AGT);
        const unsigned tag = call & 0xFFFFFFu;
        const size_t par = call & 1u;
        char* const dbuf = dws + par * BUF_BYTES;
        unsigned* const dflag = (unsigned*)(dws + CTRL + par * FLAG_PAR);
        for (int k = 0; code != 0xFFFFu; ++k) {
            const int type = code >> 8, a = (code >> 2) & 31, q = code & 3;
            const int s = type == 2 ? ((peer ^ (4 + q)) & 7) * 4 + (rank & 3) : rank * 4 + (q ^ ppos);
            const size_t doff = type == 0 ? (((size_t)(a >> 1) * 32 + s) * 2 + (a & 1)) * ITEM : XS_BYTES + ((size_t)a * 32 + s) * ITEM;
            const int fo = (int)((type == 0 ? XFLAG : WFLAG) >> 2) + a * 32 + s;
            unsigned ecode;
            auto send = [&](u4 (&o)[16]) {
                asm volatile("" : "+v"(o[0]), "+v"(o[1]), "+v"(o[2]), "+v"(o[3]), "+v"(o[4]), "+v"(o[5]), "+v"(o[6]), "+v"(o[7]),
                                  "+v"(o[8]), "+v"(o[9]), "+v"(o[10]), "+v"(o[11]), "+v"(o[12]), "+v"(o[13]), "+v"(o[14]), "+v"(o[15]));
                STAMP_AT(k * 4 + 1);
                if (link) wait_until(tstart, lead + (((widx + k * npw) * pace) >> 8));
                STAMP_AT(k * 4 + 2);
                auto rd = __builtin_amdgcn_make_buffer_rsrc(dbuf + doff, 0, (int)ITEM, 0x00020000);
#pragma unroll
                for (int f = 0; f < 16; ++f) __builtin_amdgcn_raw_buffer_store_b128(o[f], rd, lane * 16, f * 1024, AUX_SYS);
                asm volatile("s_nop 1" :: "v"(o[0]), "v"(o[1]), "v"(o[2]), "v"(o[3]), "v"(o[4]), "v"(o[5]), "v"(o[6]), "v"(o[7]),
                             "v"(o[8]), "v"(o[9]), "v"(o[10]), "v"(o[11]), "v"(o[12]), "v"(o[13]), "v"(o[14]), "v"(o[15]) : "memory");
            };
            if (type == 2) {
                unsigned* src_flag = (unsigned*)(myws + CTRL + par * FLAG_PAR) + fo;
                unsigned fv = (unsigned)uni((int)__hip_atomic_load(src_flag, RLX_SYS));
                if ((fv >> 8) != tag) {
                    const u64 t0 = __builtin_amdgcn_s_memrealtime();
                    while ((fv >> 8) != tag && !dead) {
                        dead = spin_pause(t0, tmo, call);
                        fv = (unsigned)uni((int)__hip_atomic_load(src_flag, RLX_SYS));
                    }
                }
                ecode = fv & 255u;
                auto rs = __builtin_amdgcn_make_buffer_rsrc(myws + par * BUF_BYTES + doff, 0, (int)ITEM, 0x00020000);
                u4 o[16];
#pragma unroll
                for (int f = 0; f < 16; ++f) o[f] = __builtin_amdgcn_raw_buffer_load_b128(rs, lane * 16, f * 1024, AUX_SYS);
                STAMP_AT(k * 4 + 0);
                send(o);
            } else {
                float m = 0.f;
#pragma unroll
                for (int j = 0; j < 32; ++j)
                    m = fmaxf(fmaxf(m, fmaxf(fabsf(v[j][0]), fabsf(v[j][1]))), fmaxf(fabsf(v[j][2]), fabsf(v[j][3])));
                STAMP_AT(k * 4 + 0);
                m = wave_max(m);
                const int eb = (uni(__builtin_bit_cast(int, m)) >> 23) & 255;
                const int e8 = (eb - 134) >> 3;
                ecode = (unsigned)(e8 + 32);
                const int sh = -8 * e8;
                auto cv = [&](float f) { return (_Float16)__builtin_amdgcn_ldexpf(f, sh); };
                u4 o[16];
                if (type == 0) {
#pragma unroll
                    for (int f = 0; f < 16; ++f) {
                        const f4 lo = v[2 * f], hi = v[2 * f + 1];
                        h8 t = {cv(lo[0]), cv(lo[1]), cv(lo[2]), cv(lo[3]), cv(hi[0]), cv(hi[1]), cv(hi[2]), cv(hi[3])};
                        o[f] = __builtin_bit_cast(u4, t);
                    }
                    send(o);
                } else {
#pragma unroll
                    for (int f = 0; f < 16; ++f) {
                        const int b = (f >> 2) * 8, nf = f & 3;
                        h8 t = {cv(v[b][nf]), cv(v[b + 1][nf]), cv(v[b + 2][nf]), cv(v[b + 3][nf]),
                                cv(v[b + 4][nf]), cv(v[b + 5][nf]), cv(v[b + 6][nf]), cv(v[b + 7][nf])};
                        o[f] = __builtin_bit_cast(u4, t);
                    }
                    send(o);
                }
            }
            code = code_of(k + 1);
            if (safe == 2 || (safe == 1 && (code >> 8) >= 2)) {
                asm volatile("s_waitcnt vmcnt(0)" ::: "memory");
                if (lane == 0 && !dead) __hip_atomic_store(dflag + fo, tag << 8 | ecode, RLX_SYS);
                STAMP_AT(k * 4 + 3);
            }
            asm volatile("" ::: "memory");
            fetch(code);
            if (safe == 2 || (safe == 1 && (code >> 8) >= 2)) continue;
            asm volatile("s_waitcnt vmcnt(32)" ::: "memory");
            if (lane == 0 && !dead) __hip_atomic_store(dflag + fo, tag << 8 | ecode, RLX_SYS);
            STAMP_AT(k * 4 + 3);
        }
        if (blockIdx.x == 0 && wave == 0) {
            unsigned* yf = (unsigned*)(myws + CTRL + par * FLAG_PAR + YFLAG) + lane;
            const u64 t0 = __builtin_amdgcn_s_memrealtime();
            for (;;) {
                const unsigned v0 = __hip_atomic_load(yf, RLX_SYS), v1 = __hip_atomic_load(yf + 64, RLX_SYS);
                if (__all(v0 == tag && v1 == tag)) break;
                if (spin_pause(t0, tmo, call)) { dead = true; break; }
            }
            __builtin_amdgcn_fence(__ATOMIC_ACQUIRE, "");
            STAMP_AT(61);
        }
        if (dead && lane == 0) {
            P.out[0][((rank >> 2) ^ 1) * 2048] = __builtin_nanf("");
        }
        STAMP_OUT();
        return;
    }

    const unsigned call = (unsigned)uni((int)epv) + 1u;
    if (lane == 0) __hip_atomic_store(ep, call, RLX_AGT);
    const unsigned tag = call & 0xFFFFFFu;
    const size_t par = call & 1u;
    const int g = slot - nwg;
    const bool own = link < 4;
    const int rb = (own && map1) ? (g & 3) : link;
    const int t = (own && map1) ? link * 8 + (g >> 2) : g;
    const int pos = rank & 3;
    const char* const atile = myws + par * BUF_BYTES + (size_t)rb * (32 * 2 * ITEM);
    const char* const btile = myws + par * BUF_BYTES + XS_BYTES + (size_t)t * (32 * ITEM);
    const char* const abase = atile + wave * 8192;
    const char* const bbase = btile + wave * 4096;
    unsigned* const fxp = (unsigned*)(myws + CTRL + par * FLAG_PAR + XFLAG) + rb * 64 + lane;
    unsigned* const fwp = (unsigned*)(myws + CTRL + par * FLAG_PAR + WFLAG) + t * 32 + (lane & 31);
    const unsigned ldsb = (unsigned)(uintptr_t)lds + wave * 4096;
    typedef volatile __attribute__((address_space(3))) unsigned* ldsu;
    const ldsu dec = (ldsu)(lds + 4 * ITEM);
    const int voa = lane * 16 + 4096, vob = lane * 16;
    const int rot = (g * prot) & 31;

    f4 acc[2][4];
#pragma unroll
    for (int i = 0; i < 2; ++i)
#pragma unroll
        for (int j = 0; j < 4; ++j) acc[i][j] = f4{0.f, 0.f, 0.f, 0.f};
    h8 A[4][8];
    int E[4];
    int ecur = 0, emax = -1000;
    bool first = true;
    unsigned issued = 0, rx = 0, rw = 0, pfd = 0, fx = 0, fw = 0;
    bool dead = false;

    auto decide = [&](int u) {
        if (wave != 0) return;
        unsigned avail = rx & rw & ~issued;
        if (!avail) {
            const u64 t0 = __builtin_amdgcn_s_memrealtime();
            const char* pfp = nullptr;
            int pfo = 0;
            for (;;) {
                if (pfp) {
                    unsigned d;
                    asm volatile("global_load_dword %0, %3, %5 sc0 sc1\n\t"
                                 "global_load_dword %0, %4, %5 sc0 sc1\n\t"
                                 "global_load_dword %1, %6, off sc0 sc1\n\t"
                                 "global_load_dword %2, %7, off sc0 sc1\n\t"
                                 "s_waitcnt vmcnt(0)"
                                 : "=&v"(d), "=&v"(fx), "=&v"(fw) : "v"(lane * 128), "v"(lane * 128 + pfo), "s"(pfp), "v"(fxp), "v"(fwp) : "memory");
                } else {
                    fx = __hip_atomic_load(fxp, RLX_SYS);
                    fw = __hip_atomic_load(fwp, RLX_SYS);
                }
                const u64 bx = __ballot((fx >> 8) == tag);
                rx = (unsigned)bx & (unsigned)(bx >> 32);
                rw = (unsigned)__ballot((fw >> 8) == tag);
                avail = rx & rw & ~issued;
                if (avail) break;
                const unsigned cand = (rx | rw) & ~issued & ~pfd;
                pfp = nullptr;
                if (u >= pfstart && cand) {
                    const int s = __builtin_ctz(cand);
                    pfd |= 1u << s;
                    pfp = ((rw >> s) & 1) ? btile + (size_t)s * ITEM : atile + (size_t)s * (2 * ITEM) + (g & 1) * ITEM;
                    pfo = 8192;
                } else if (spin_pause(t0, tmo, call)) { dead = true; rx = rw = ~0u; avail = ~issued; break; }
            }
        }
        const unsigned r = __builtin_amdgcn_alignbit(avail, avail, rot);
        const int s = (__builtin_ctz(uni(r)) + rot) & 31;
        issued |= 1u << s;
        const unsigned word = s | (__builtin_amdgcn_readlane(fx, s) & 255u) << 8 | (__builtin_amdgcn_readlane(fx, 32 + s) & 255u) << 16 |
                              (__builtin_amdgcn_readlane(fw, s) & 255u) << 24;
        dec[u & 1] = word;
        STAMP_AT(u);
    };

    auto issue = [&](int u, h8 (&a)[8], int& e, int ring) {
        const unsigned word = (unsigned)uni((int)dec[u & 1]);
        const int s = word & 31;
        e = (int)((word >> (8 + (wave >> 1) * 8)) & 255u) + (int)(word >> 24) - 64;
        const char* sa = abase + (size_t)s * (2 * ITEM);
        const char* sb = bbase + (size_t)s * ITEM;
        const unsigned m0v = ldsb + ring * (unsigned)ITEM;
        unsigned keep;
        asm volatile(
            "s_mov_b32 %[keep], m0\n\t"
            "s_mov_b32 m0, %[m0v]\n\t"
            "s_nop 4\n\t"
            "global_load_lds_dwordx4 %[vob], %[sb] sc0 sc1\n\t"
            "global_load_lds_dwordx4 %[vob], %[sb] offset:1024 sc0 sc1\n\t"
            "global_load_lds_dwordx4 %[vob], %[sb] offset:2048 sc0 sc1\n\t"
            "global_load_lds_dwordx4 %[vob], %[sb] offset:3072 sc0 sc1\n\t"
            "global_load_dwordx4 %[a0], %[voa], %[sa] offset:-4096 sc0 sc1\n\t"
            "global_load_dwordx4 %[a4], %[voa], %[sa] sc0 sc1\n\t"
            "global_load_dwordx4 %[a1], %[voa], %[sa] offset:-3072 sc0 sc1\n\t"
            "global_load_dwordx4 %[a5], %[voa], %[sa] offset:1024 sc0 sc1\n\t"
            "global_load_dwordx4 %[a2], %[voa], %[sa] offset:-2048 sc0 sc1\n\t"
            "global_load_dwordx4 %[a6], %[voa], %[sa] offset:2048 sc0 sc1\n\t"
            "global_load_dwordx4 %[a3], %[voa], %[sa] offset:-1024 sc0 sc1\n\t"
            "global_load_dwordx4 %[a7], %[voa], %[sa] offset:3072 sc0 sc1\n\t"
            "s_mov_b32 m0, %[keep]"
            : [keep] "=&s"(keep), [a0] "=&v"(a[0]), [a1] "=&v"(a[1]), [a2] "=&v"(a[2]), [a3] "=&v"(a[3]),
              [a4] "=&v"(a[4]), [a5] "=&v"(a[5]), [a6] "=&v"(a[6]), [a7] "=&v"(a[7])
            : [m0v] "s"(m0v), [vob] "v"(vob), [sb] "s"(sb), [voa] "v"(voa), [sa] "s"(sa)
            : "memory");
    };

#define BAR() asm volatile("s_waitcnt lgkmcnt(0)\n\ts_barrier" ::: "memory")
#define LANDED(N, a) asm volatile("s_waitcnt vmcnt(" #N ") lgkmcnt(0)\n\ts_barrier" \
        : "+v"(a[0]), "+v"(a[1]), "+v"(a[2]), "+v"(a[3]), "+v"(a[4]), "+v"(a[5]), "+v"(a[6]), "+v"(a[7]) :: "memory")

    auto compute = [&](h8 (&a)[8], int e, int ring) {
        if (e != ecur) {
            if (first) { ecur = e; emax = e; }
            emax = e > emax ? e : emax;
            const int et = e > emax - 7 ? e : emax - 7;
            const int sh = 8 * (ecur - et);
#pragma unroll
            for (int i = 0; i < 2; ++i)
#pragma unroll
                for (int j = 0; j < 4; ++j)
#pragma unroll
                    for (int c = 0; c < 4; ++c) acc[i][j][c] = __builtin_amdgcn_ldexpf(acc[i][j][c], sh);
            ecur = et;
            if (et != e) {
                const _Float16 f = (_Float16)__builtin_amdgcn_ldexpf(1.0f, 8 * (e - et));
#pragma unroll
                for (int i = 0; i < 8; ++i) a[i] *= f;
            }
        }
        if (first) { emax = e; first = false; }
        const h8* bp = (const h8*)(lds + ring * ITEM) + lane;
#pragma unroll
        for (int ks = 0; ks < 4; ++ks)
#pragma unroll
            for (int nf = 0; nf < 4; ++nf) {
                const h8 b = bp[(ks * 4 + nf) * 64];
                acc[0][nf] = __builtin_amdgcn_mfma_f32_16x16x32_f16(a[ks], b, acc[0][nf], 0, 0, 0);
                acc[1][nf] = __builtin_amdgcn_mfma_f32_16x16x32_f16(a[4 + ks], b, acc[1][nf], 0, 0, 0);
            }
    };

    decide(0); BAR(); issue(0, A[0], E[0], 0);
    decide(1); BAR(); issue(1, A[1], E[1], 1);
    decide(2); BAR(); issue(2, A[2], E[2], 2);
#pragma unroll 1
    for (int u = 0; u < 28; u += 4) {
        decide(u + 3); LANDED(24, A[0]); issue(u + 3, A[3], E[3], 3); compute(A[0], E[0], 0);
        decide(u + 4); LANDED(24, A[1]); issue(u + 4, A[0], E[0], 0); compute(A[1], E[1], 1);
        decide(u + 5); LANDED(24, A[2]); issue(u + 5, A[1], E[1], 1); compute(A[2], E[2], 2);
        decide(u + 6); LANDED(24, A[3]); issue(u + 6, A[2], E[2], 2); compute(A[3], E[3], 3);
    }
    decide(31); LANDED(24, A[0]); issue(31, A[3], E[3], 3); compute(A[0], E[0], 0);
    LANDED(24, A[1]); compute(A[1], E[1], 1);
    LANDED(12, A[2]); compute(A[2], E[2], 2);
    LANDED(0, A[3]);
    STAMP_AT(33);
    const unsigned tmv = __hip_atomic_load(tmo, RLX_AGT);
    compute(A[3], E[3], 3);

    if (!dead) dead = (unsigned)uni((int)tmv) == call;
    {
        const int orow = (own ? rb : pos) * 128 + wave * 32;
        float* ob = P.out[own ? 0 : link] + (size_t)orow * NCOL + (rank >> 2) * 2048 + t * 64;
        auto ro = __builtin_amdgcn_make_buffer_rsrc(ob, 0, 31 * NCOL * 4 + 256, 0x00020000);
        const int vo = (lane >> 4) * 4 * (NCOL * 4) + (lane & 15) * 16;
        const int sh = 8 * ecur;
        u4 r[8];
#pragma unroll
        for (int i = 0; i < 2; ++i)
#pragma unroll
            for (int j = 0; j < 4; ++j) {
                f4 y;
#pragma unroll
                for (int c = 0; c < 4; ++c) y[c] = dead ? __builtin_nanf("") : __builtin_amdgcn_ldexpf(acc[i][c][j], sh);
                r[i * 4 + j] = __builtin_bit_cast(u4, y);
            }
        STAMP_AT(34);
        if (!own) wait_until(tstart, lead + (((ybase + 2 * t) * pace) >> 8));
        STAMP_AT(35);
#pragma unroll
        for (int i = 0; i < 8; ++i)
            __builtin_amdgcn_raw_buffer_store_b128(r[i], ro, vo, ((i >> 2) * 16 + (i & 3)) * (NCOL * 4), AUX_SYS);
        asm volatile("s_nop 1" :: "v"(r[0]), "v"(r[1]), "v"(r[2]), "v"(r[3]), "v"(r[4]), "v"(r[5]), "v"(r[6]), "v"(r[7]) : "memory");
    }
    if (!own) {
        asm volatile("s_waitcnt vmcnt(0)\n\ts_barrier" ::: "memory");
        if (threadIdx.x == 0) {
            unsigned* yf = (unsigned*)(P.ws[link] + CTRL + par * FLAG_PAR + YFLAG) + pos * 32 + t;
            __hip_atomic_store(yf, tag, RLX_SYS);
        }
    }
}

struct Tune { int nwg = 32, pw = 1, pace = 8000, lead = 50, yadv = 0, pfstart = 16, map1 = 1, rot = 3, nf = 8, nb = 8, order = 0, safe = 2; };
#define TUNE_FIELDS(F) F(nwg) F(pw) F(pace) F(lead) F(yadv) F(pfstart) F(map1) F(rot) F(nf) F(nb) F(order) F(safe)
constexpr Tune g_tune{};

void build_lists(const Tune& T, uint16_t (&L)[3][MAXITEMS], int (&len)[3]) {
    const int nf = T.nf, nb = T.nb, ta = nf, tb = 32 - nb;
    for (int c = 0; c < 3; ++c) {
        int n = 0;
        auto X = [&](int a0) { for (int q = 0; q < 4; ++q) for (int a = a0; a < a0 + 8; ++a) L[c][n++] = (uint16_t)(0 << 8 | a << 2 | q); };
        auto W = [&](int t0, int t1) { for (int q = 0; q < 4; ++q) for (int t = t0; t < t1; ++t) L[c][n++] = (uint16_t)(1 << 8 | t << 2 | q); };
        if (c == 0) { X(8); W(0, ta); W(ta, tb); W(tb, 32); X(0); }
        if (c == 1) {
            X(8); W(0, ta);
            for (int s = 0; s < 4; ++s) for (int t = 0; t < ta; ++t) L[c][n++] = (uint16_t)(2 << 8 | t << 2 | s);
            W(ta, tb); W(tb, 32); X(0);
        }
        if (c == 2) {
            for (int t = 0; t < ta; ++t) L[c][n++] = (uint16_t)(1 << 8 | t << 2 | 0);
            X(8); X(0); W(tb, 32); W(ta, tb);
        }
        len[c] = n;
    }
}

}

extern "C" void kernel_launch(int rank, int nranks, ncclComm_t comm,
                              hipStream_t stream,
                              void* const* d_in, const int* in_sizes, int n_in,
                              void* d_out, int out_size,
                              const int* in_dtypes, int out_dtype,
                              void* const* d_out_all,
                              void* const* d_ws_all, size_t ws_bytes) {
    (void)comm; (void)in_dtypes; (void)out_dtype; (void)d_out;
    if (nranks != NR || (unsigned)rank >= (unsigned)NR || n_in < 2 || ws_bytes < WS_NEED ||
        in_sizes[0] != 4096 * KSH || in_sizes[1] != KSH * NCOL || out_size != MSH * NCOL) {
        fprintf(stderr, "kernel_launch: unexpected problem shape\n");
        return;
    }
    const Tune T = g_tune;
    uint16_t L[3][MAXITEMS];
    int len[3];
    build_lists(T, L, len);
    Params P;
    P.x = (const float*)d_in[0];
    P.w = (const float*)d_in[1];
    for (int j = 0; j < NR; ++j) { P.out[j] = (float*)d_out_all[rank ^ j]; P.ws[j] = (char*)d_ws_all[rank ^ j]; }
    P.rank = rank;
    P.nwg = T.nwg; P.pw = T.pw;
    const int npw = T.nwg * T.pw;
    P.per = MAXITEMS / npw;
    P.pace = T.pace; P.lead = T.lead; P.ybase = len[2] - T.yadv; P.pfstart = T.pfstart; P.map1 = T.map1; P.rot = T.rot; P.safe = T.safe;
    for (int c = 0; c < 3; ++c) {
        uint16_t R[MAXITEMS];
        for (int i = 0; i < MAXITEMS; ++i) R[i] = 0xFFFF;
        for (int i = 0; i < len[c]; ++i) {
            if (i / npw >= P.per) { fprintf(stderr, "item table overflow\n"); return; }
            R[(i % npw) * P.per + i / npw] = L[c][i];
        }
        for (int i = 0; i < MAXITEMS / 2; ++i) P.list[c][i] = (uint32_t)R[2 * i] | (uint32_t)R[2 * i + 1] << 16;
    }
    gemm_rs_kernel<<<8 * (T.nwg + 32), 256, 0, stream>>>(P);
    HIP_CHECK(hipGetLastError());
}
```

```cpp
#include <hip/hip_runtime.h>
#include <rccl/rccl.h>
#include <cstdio>
#include <cstdint>

#define HIP_CHECK(expr) do { hipError_t _e = (expr); if (_e != hipSuccess) { \
    fprintf(stderr, "HIP error %s at %s:%d\n", hipGetErrorString(_e), __FILE__, __LINE__); \
    return; } } while (0)

namespace {

typedef _Float16 h8 __attribute__((ext_vector_type(8)));
typedef float f4 __attribute__((ext_vector_type(4)));
typedef unsigned u4 __attribute__((ext_vector_type(4)));
typedef unsigned long long u64;

constexpr int NR = 8;
constexpr int NCOL = 4096;
constexpr int KSH = 512;
constexpr int MSH = 512;
constexpr int NWORK = 28, NSEND = 4;

constexpr unsigned ITEM = 16384;
constexpr unsigned XR_BYTES = 8 * 32 * 2 * ITEM;
constexpr unsigned WR_BYTES = 32 * 32 * ITEM;
constexpr unsigned BUF_BYTES = XR_BYTES + WR_BYTES;
constexpr unsigned STG = 2 * BUF_BYTES;
constexpr unsigned YST = STG + 512 * ITEM;
constexpr unsigned CTRL = YST + 256 * ITEM;
constexpr unsigned FLAG_PAR = 131072;
constexpr unsigned YFLAG = 1536;
constexpr unsigned SFLAG = CTRL + 2 * FLAG_PAR;
constexpr unsigned TFLAG = SFLAG + 2048;
constexpr unsigned EPOCH = TFLAG + 2048;
constexpr unsigned TMO = EPOCH + 8192;
constexpr unsigned STAMP = TMO + 4096;
constexpr unsigned WS_NEED = STAMP + (1 << 20);

constexpr u64 DEADLINE_TICKS = 100000000ull;

constexpr int MAXITEMS = 256;

struct Params {
    const float* x;
    const float* w;
    int rank;
    int stag0, stag;
    int fs;
    int rot, xrot, omap, fastu, slp0, slp1;
    char* ws[NR];
    float* out[NR];
    unsigned conv[256];
    uint16_t list[2][MAXITEMS];
};

__host__ __device__ inline int staged_id(int code, int peer, int link, int xrot) {
    const int a = (code >> 2) & 31, qa = (code & 3) ^ (peer & 3);
    if ((code >> 8) == 0) {
        const int rb = a >> 1;
        const int grb = rb < 4 ? peer * 4 + ((rb + link * xrot) & 3) : ((peer ^ rb) & 7) * 4 + (peer & 3);
        return (grb * 4 + qa) * 2 + (a & 1);
    }
    return 256 + ((peer >> 2) * 32 + a) * 4 + qa;
}
__host__ __device__ inline int recv_slot(int id, int src, int dst) {
    if (id < 256) {
        const int grb = id >> 3, qa = (id >> 1) & 3, o = grb >> 2, p = grb & 3;
        const int rb = o == dst ? p : o ^ dst;
        if (o != dst && (rb < 4 || p != (dst & 3))) return -1;
        return (rb * 32 + src * 4 + qa) * 2 + (id & 1);
    }
    const int ct = (id - 256) >> 2;
    if ((ct >> 5) != (dst >> 2)) return -1;
    return 512 + (ct & 31) * 32 + src * 4 + (id & 3);
}
__host__ __device__ inline unsigned flag_of_slot(int slot, int fs) {
    const int i = slot < 512 ? (slot >> 6) * 64 + (slot & 1) * 32 + ((slot >> 1) & 31) : slot;
    return (i >> 6) * fs + (i & 63) * 4;
}

#define RLX_SYS __ATOMIC_RELAXED, __HIP_MEMORY_SCOPE_SYSTEM
#define RLX_AGT __ATOMIC_RELAXED, __HIP_MEMORY_SCOPE_AGENT
constexpr int AUX_SYS = 17;

__device__ __forceinline__ int uni(int v) { return __builtin_amdgcn_readfirstlane(v); }

#define STAMP_V(slot, val) do { } while (0)
#define STAMP_AT(slot) do { } while (0)
#define STAMP_OUT() do { } while (0)

__device__ __forceinline__ bool spin_pause(u64 t0, unsigned* tmo, unsigned call) {
    __builtin_amdgcn_s_sleep(2);
    u64 dt = __builtin_amdgcn_s_memrealtime() - t0;
    if (dt > 50000ull) {
        if (dt > DEADLINE_TICKS || (unsigned)uni((int)__hip_atomic_load(tmo, RLX_AGT)) == call) {
            __hip_atomic_store(tmo, call, RLX_AGT);
            return true;
        }
        __builtin_amdgcn_s_sleep(32);
    }
    return false;
}

__device__ __forceinline__ unsigned wave_max(unsigned v) {
    v = max(v, (unsigned)__builtin_amdgcn_update_dpp(0, (int)v, 0x111, 0xf, 0xf, true));
    v = max(v, (unsigned)__builtin_amdgcn_update_dpp(0, (int)v, 0x112, 0xf, 0xf, true));
    v = max(v, (unsigned)__builtin_amdgcn_update_dpp(0, (int)v, 0x114, 0xf, 0xf, true));
    v = max(v, (unsigned)__builtin_amdgcn_update_dpp(0, (int)v, 0x118, 0xf, 0xf, true));
    v = max(v, (unsigned)__builtin_amdgcn_update_dpp(0, (int)v, 0x142, 0xa, 0xf, true));
    v = max(v, (unsigned)__builtin_amdgcn_update_dpp(0, (int)v, 0x143, 0xc, 0xf, true));
    return v;
}

__device__ __forceinline__ void wait_until(unsigned tstart, int ticks) {
    while ((int)((unsigned)__builtin_amdgcn_s_memrealtime() - tstart) < ticks) __builtin_amdgcn_s_sleep(1);
}

#define BAR() asm volatile("s_waitcnt lgkmcnt(0)\n\ts_barrier" ::: "memory")

constexpr unsigned RDY_VALID = 0x100, RDY_DEAD = 0x200;

typedef float f32v __attribute__((ext_vector_type(32)));

constexpr int C_DEAD = 16, C_MAX = 64, C_CNT = 72, C_GO = 75;

__global__ void __launch_bounds__(384, 2) gemm_rs_kernel(const Params P) {
    __shared__ __attribute__((aligned(1024))) char lds[4 * ITEM + 512];
    const u64 tstart64 = __builtin_amdgcn_s_memrealtime();
    const unsigned tstart = (unsigned)tstart64;
    const int link = blockIdx.x & 7;
    const int slot = blockIdx.x >> 3;
    const float* const px = P.x;
    const float* const pw = P.w;
    char* const myws = P.ws[0];
    char* const prws = P.ws[link];
    const unsigned cv = P.conv[blockIdx.x];
    const int rank = P.rank & 7, stag0 = P.stag0, stag = P.stag, fs = P.fs;
    asm volatile("" :: "s"(tstart64), "s"(px), "s"(pw), "s"(myws), "s"(prws), "s"(cv), "s"(rank), "s"(stag0), "s"(stag), "s"(fs));
    const int lane = threadIdx.x & 63;
    const int wave = uni(threadIdx.x >> 6);
    const int peer = rank ^ link;
    const int pos = rank & 3;
    const bool sender = slot < NSEND && link != 0;
    const int g = slot - NSEND;
    unsigned* const tmo = (unsigned*)(myws + TMO);
    typedef volatile __attribute__((address_space(3))) unsigned* ldsu;
    const ldsu ctl = (ldsu)(lds + 4 * ITEM);
    if (wave == 5 && !sender) return;

    unsigned* const ep = (unsigned*)(myws + EPOCH) + (blockIdx.x * 8 + wave);
    const int c16 = lane & 15, q4 = lane >> 4;
    unsigned call, tag, par;
    int code = 0;

    if (wave < 4) {
        auto fetch = [&](unsigned id, f4 (&d)[8]) {
            const float* base;
            int bytes, s0, s1, vo;
            if (id < 256) {
                const int qa = (id >> 1) & 3;
                base = px + (size_t)((id >> 3) * 128 + (id & 1) * 64 + wave * 16) * KSH + qa * 128;
                bytes = 16 * KSH * 4 - qa * 512;
                s0 = 64; s1 = 256;
                vo = c16 * (KSH * 4) + q4 * 16;
            } else {
                const int col0 = ((id >> 2) & 63) * 64;
                base = pw + (size_t)((id & 3) * 128 + wave * 32) * NCOL + col0;
                bytes = 32 * NCOL * 4 - col0 * 4;
                s0 = NCOL * 4; s1 = 16 * NCOL * 4;
                vo = q4 * 4 * (NCOL * 4) + c16 * 16;
            }
            auto rs = __builtin_amdgcn_make_buffer_rsrc((void*)base, 0, bytes, 0x00020000);
#pragma unroll
            for (int j = 0; j < 8; ++j)
                d[j] = __builtin_bit_cast(f4, __builtin_amdgcn_raw_buffer_load_b128(rs, vo, (j >> 2) * s1 + (j & 3) * s0, 0));
        };
        auto convert = [&](unsigned id, f4 (&v)[8], u4 (&o)[4], int c) -> unsigned {
            float m = 0.f;
#pragma unroll
            for (int j = 0; j < 8; ++j) m = fmaxf(fmaxf(fmaxf(fmaxf(m, fabsf(v[j][0])), fabsf(v[j][1])), fabsf(v[j][2])), fabsf(v[j][3]));
            STAMP_AT(64 + c * 4);
            const unsigned wm = wave_max(__builtin_bit_cast(unsigned, m));
            if (lane == 63) ctl[C_MAX + c * 4 + wave] = wm;
            BAR();
            const u4 mp = *(volatile __attribute__((address_space(3))) u4*)(ctl + C_MAX + c * 4);
            const int e8 = max(((uni((int)max(max(mp[0], mp[1]), max(mp[2], mp[3]))) >> 23) - 134) >> 3, -15);
            const float sc = __builtin_bit_cast(float, (127 - 8 * e8) << 23);
#pragma unroll
            for (int j = 0; j < 8; ++j) v[j] *= sc;
            auto h = [](float f) { return (_Float16)f; };
            if (id < 256) {
                asm volatile("" : "+v"(v[0]), "+v"(v[7]));
#pragma unroll
                for (int f = 0; f < 4; ++f) {
                    const f4 lo = v[2 * f], hi = v[2 * f + 1];
                    h8 t = {h(lo[0]), h(lo[1]), h(lo[2]), h(lo[3]), h(hi[0]), h(hi[1]), h(hi[2]), h(hi[3])};
                    o[f] = __builtin_bit_cast(u4, t);
                }
            } else {
#pragma unroll
                for (int f = 0; f < 4; ++f) {
                    h8 t = {h(v[0][f]), h(v[1][f]), h(v[2][f]), h(v[3][f]), h(v[4][f]), h(v[5][f]), h(v[6][f]), h(v[7][f])};
                    o[f] = __builtin_bit_cast(u4, t);
                }
            }
            asm volatile("" : "+v"(o[0]), "+v"(o[1]), "+v"(o[2]), "+v"(o[3]));
            STAMP_AT(64 + c * 4 + 1);
            return (unsigned)(e8 + 32);
        };
        auto last_in = [&](int c) {
            return lane == 0 && __hip_atomic_fetch_add((__attribute__((address_space(3))) unsigned*)(ctl + c), 1u, __ATOMIC_RELAXED, __HIP_MEMORY_SCOPE_WORKGROUP) == 3u;
        };
        const unsigned ida = cv & 511u, idb = (cv >> 16) & 511u;
        f4 va[8], vb[8];
        const unsigned epv = __hip_atomic_load(ep, RLX_AGT);
        if (!sender) wait_until(tstart, stag0 + (((g < 0 ? 0 : g) * stag) >> 4));
        fetch(ida, va);
        fetch(idb, vb);
        call = (unsigned)uni((int)epv) + 1u;
        tag = call & 0xFFFFFFu;
        par = call & 1u;
        const int sla = recv_slot(ida, rank, peer) & 2047, slb = recv_slot(idb, rank, peer) & 2047;
        char* const la = myws + STG + ida * ITEM + wave * 4096;
        char* const lb = myws + STG + idb * ITEM + wave * 4096;
        auto ra = __builtin_amdgcn_make_buffer_rsrc(sender ? prws + par * BUF_BYTES + sla * ITEM + wave * 4096 : la, 0, 4096, 0x00020000);
        auto rb = __builtin_amdgcn_make_buffer_rsrc(sender ? prws + par * BUF_BYTES + slb * ITEM + wave * 4096 : lb, 0, 4096, 0x00020000);
        auto staged = [&](unsigned id, unsigned fv) {
            __hip_atomic_store((unsigned*)(myws + SFLAG) + id, fv, RLX_SYS);
            const int ls = recv_slot(id, rank, rank);
            if (ls >= 0) __hip_atomic_store((unsigned*)(myws + CTRL + par * FLAG_PAR + flag_of_slot(ls, fs)), fv, RLX_SYS);
        };
        u4 oa[4], ob[4];
        const unsigned fa = tag << 8 | convert(ida, va, oa, 0);
#pragma unroll
        for (int f = 0; f < 4; ++f) __builtin_amdgcn_raw_buffer_store_b128(oa[f], ra, lane * 16 + f * 1024, 0, AUX_SYS);
        __builtin_amdgcn_sched_barrier(0);
        asm volatile("" : "+v"(vb[0]) :: "memory");
        const unsigned fb = tag << 8 | convert(idb, vb, ob, 1);
#pragma unroll
        for (int f = 0; f < 4; ++f) {
            asm volatile("s_waitcnt vmcnt(3)" ::: "memory");
            __builtin_amdgcn_raw_buffer_store_b128(ob[f], rb, lane * 16 + f * 1024, 0, AUX_SYS);
        }
        STAMP_AT(64 + 2);
        if (last_in(C_CNT)) {
            if (sender) __hip_atomic_store((unsigned*)(prws + CTRL + par * FLAG_PAR + flag_of_slot(sla, fs)), fa, RLX_SYS);
            else staged(ida, fa);
        }
        if (sender) {
            asm volatile("s_waitcnt vmcnt(2)" ::: "memory");
            (void)last_in(C_GO);
            auto r2a = __builtin_amdgcn_make_buffer_rsrc(la, 0, 4096, 0x00020000);
            auto r2b = __builtin_amdgcn_make_buffer_rsrc(lb, 0, 4096, 0x00020000);
#pragma unroll
            for (int f = 0; f < 4; ++f) __builtin_amdgcn_raw_buffer_store_b128(oa[f], r2a, lane * 16 + f * 1024, 0, AUX_SYS);
#pragma unroll
            for (int f = 0; f < 4; ++f) __builtin_amdgcn_raw_buffer_store_b128(ob[f], r2b, lane * 16 + f * 1024, 0, AUX_SYS);
            asm volatile("s_waitcnt vmcnt(8)" ::: "memory");
            if (last_in(C_CNT + 1)) __hip_atomic_store((unsigned*)(prws + CTRL + par * FLAG_PAR + flag_of_slot(slb, fs)), fb, RLX_SYS);
        }
        asm volatile("s_waitcnt vmcnt(0)" :: "v"(oa[0]), "v"(oa[1]), "v"(oa[2]), "v"(oa[3]), "v"(ob[0]), "v"(ob[1]), "v"(ob[2]), "v"(ob[3]) : "memory");
        STAMP_AT(64 + 4 + 2);
        if (last_in(C_CNT + 2)) {
            if (sender) staged(ida, fa);
            staged(idb, fb);
        }
        if (lane == 0) __hip_atomic_store(ep, call, RLX_AGT);
        if (g < 0) { STAMP_OUT(); return; }
    } else {
        const unsigned epv = __hip_atomic_load(ep, RLX_AGT);
        const int idx = 2 * NSEND + slot + NSEND * lane;
        code = P.list[link >> 2][idx < MAXITEMS ? idx : MAXITEMS - 1];
        if (wave == 4) { if (lane < 4) ctl[C_CNT + lane] = 0; }
        else ctl[lane] = 0;
        BAR(); BAR();
        call = (unsigned)uni((int)epv) + 1u;
        if (lane == 0) __hip_atomic_store(ep, call, RLX_AGT);
        tag = call & 0xFFFFFFu;
        par = call & 1u;
    }
    call = (unsigned)uni((int)call);
    tag = call & 0xFFFFFFu;
    par = call & 1u;

    if (g < 0) {
        const int sj = slot;
        if (link == 0) {
            if (sj != 0) return;
            const unsigned* yf = (const unsigned*)(myws + CTRL + par * FLAG_PAR + flag_of_slot(YFLAG + (lane & 15), fs));
            const u64 t0 = __builtin_amdgcn_s_memrealtime();
            bool dead = false;
            for (;;) {
                unsigned a, b, c, n;
                u64 m;
                asm volatile("; OPAQUE\n\t"
                             "s_movk_i32 %[n], 64\n\t"
                             "global_load_dword %[a], %[p], off sc0 sc1\n\t"
                             "s_sleep 6\n\t"
                             "global_load_dword %[b], %[p], off sc0 sc1\n\t"
                             "s_sleep 6\n\t"
                             "global_load_dword %[c], %[p], off sc0 sc1\n\t"
                             ".Lfin%=:\n\t"
#define FIN(r) "s_waitcnt vmcnt(2)\n\tv_cmp_ne_u32 %[m], %[t], %[" #r "]\n\ts_cmp_eq_u64 %[m], 0\n\ts_cbranch_scc1 .Lfin_end%=\n\t" \
               "global_load_dword %[" #r "], %[p], off sc0 sc1\n\ts_sleep 6\n\t"
                             FIN(a) FIN(b) FIN(c)
                             "s_sub_u32 %[n], %[n], 1\n\t"
                             "s_cbranch_scc0 .Lfin%=\n\t"
                             "s_waitcnt vmcnt(0)\n\t"
                             ".Lfin_end%=:"
                             : [a] "=&v"(a), [b] "=&v"(b), [c] "=&v"(c), [n] "=&s"(n), [m] "=&s"(m)
                             : [p] "v"(yf), [t] "s"(tag << 8) : "memory", "scc");
                if (m == 0) break;
                if (spin_pause(t0, tmo, call)) { dead = true; break; }
            }
            STAMP_AT(125);
            if (dead && lane < 4) P.out[0][(size_t)(lane * 128 + 127) * NCOL + ((rank >> 2) ^ 1) * 2048 + 2047] = __builtin_nanf("");
            STAMP_OUT();
            return;
        }
        const int n = __builtin_popcountll(__ballot(code != 0xFFFF));
        const int type = code >> 8;
        unsigned src, dst, sfl, dfl;
        if (type == 3) {
            const unsigned quad = (code >> 3) & 7, rg = code & 7;
            src = YST + ((link & 3) * 128 + rg * 16) * 8192 + quad * 1024;
            dst = (pos * 128 + rg * 16) * (NCOL * 4) + (rank >> 2) * 8192 + quad * 1024;
            sfl = TFLAG + ((link & 3) * 32 + quad * 4) * 4;
            dfl = lane == n - 1 ? CTRL + par * FLAG_PAR + flag_of_slot(YFLAG + pos * 4 + sj, fs) : 0;
        } else if (type == 2) {
            const unsigned sl = 512 + ((code >> 2) & 31) * 32 + ((peer ^ (4 + (code & 3))) & 7) * 4 + pos;
            src = dst = par * BUF_BYTES + sl * ITEM;
            sfl = dfl = CTRL + par * FLAG_PAR + flag_of_slot(sl, fs);
        } else {
            const int id = staged_id(code & 0x1FF, peer, link, P.xrot);
            const int sl = recv_slot(id, rank, peer) & 2047;
            src = STG + id * ITEM;
            dst = par * BUF_BYTES + sl * ITEM;
            sfl = SFLAG + id * 4;
            dfl = CTRL + par * FLAG_PAR + flag_of_slot(sl, fs);
        }
        auto rws = __builtin_amdgcn_make_buffer_rsrc(myws, 0, (int)STAMP, 0x00020000);
        if (wave == 5) {
            bool done = lane >= n;
            u64 t0 = __builtin_amdgcn_s_memrealtime();
            for (;;) {
                const u64 pend = __ballot(!done);
                if (!pend) break;
                const int first = __builtin_ctzll(pend);
                bool ok = false;
                if (!done && lane < first + 6) {
                    const u4 f = __builtin_amdgcn_raw_buffer_load_b128(rws, sfl & ~15u, 0, AUX_SYS);
                    const unsigned c = (sfl >> 2) & 3;
                    const unsigned v = c == 0 ? f[0] : c == 1 ? f[1] : c == 2 ? f[2] : f[3];
                    ok = (v >> 8) == tag;
                    if (type == 3) ok = ok && (f[1] >> 8) == tag && (f[2] >> 8) == tag && (f[3] >> 8) == tag;
                    if (ok) { ctl[lane] = (v & 255u) | RDY_VALID; done = true; }
                }
                if (__any(ok)) t0 = __builtin_amdgcn_s_memrealtime();
                else if (spin_pause(t0, tmo, call)) { if (!done) ctl[lane] = RDY_DEAD; break; }
            }
            STAMP_OUT();
            return;
        }
        auto rpw = __builtin_amdgcn_make_buffer_rsrc(prws, 0, (int)STAMP, 0x00020000);
        const float* const po = P.out[link];
        const int vo = lane * 16;
        const unsigned sstr = type == 3 ? 8192u : 1024u, dstr = type == 3 ? NCOL * 4u : 1024u;
        int k = 0;
        while (k < n) {
            unsigned r;
            while ((r = (unsigned)uni((int)ctl[k])) == 0) __builtin_amdgcn_s_sleep(1);
            if (r & RDY_DEAD) break;
            u4 R[16];
            unsigned la = (unsigned)(uintptr_t)lds + 4 * ITEM + k * 4, vt, pfv, so, ss, dso, ds, k1, rn, pfo;
            u64 pm;
#define LD(f) "buffer_load_dwordx4 %" #f ", %[vo], %[rs], %[so] offen sc0 sc1\n\ts_add_u32 %[so], %[so], %[ss]\n\t"
#define ST(f) "buffer_store_dwordx4 %" #f ", %[vo], s[88:91], %[dso] offen sc0 sc1\n\ts_add_u32 %[dso], %[dso], %[ds]\n\t"
#define WT(n) "s_waitcnt vmcnt(" #n ")\n\t"
#define SL(f) WT(23) ST(f) LD(f)
#define FLAG "s_mov_b64 exec, %[pm]\n\tbuffer_store_dword %[pfv], off, %[rf], %[pfo] sc0 sc1\n\ts_mov_b64 exec, -1\n\t"
#define NEXTFLAG "v_readlane_b32 %[pfo], %[dfl], %[k]\n\ts_and_b32 %[r], %[r], 0xff\n\ts_or_b32 %[r], %[r], %[tag8]\n\t" \
                 "v_mov_b32 %[pfv], %[r]\n\ts_cmp_lg_u32 %[pfo], 0\n\ts_cselect_b64 %[pm], 1, 0\n\ts_mov_b32 %[k], %[k1]\n\t"
#define ASTAMP
#define ASTAMP_OP
#define ASTAMP_CL
            asm volatile("; OPAQUE\n\t"
                         "v_readlane_b32 %[so], %[src], %[k]\n\t"
                         "v_readlane_b32 %[ss], %[sstr], %[k]\n\t"
                         "s_mov_b64 %[pm], 0\n\t"
                         "s_mov_b32 %[pfo], 0\n\t"
                         "s_mov_b32 s91, 0x20000\n\t"
                         "s_nop 4\n\t"
                         LD(0) LD(1) LD(2) LD(3) LD(4) LD(5) LD(6) LD(7) LD(8) LD(9) LD(10) LD(11) LD(12) LD(13) LD(14) LD(15)
                         ".Lsend_go%=:\n\t"
                         "ds_read_b32 %[vt], %[lgo]\n\t"
                         "s_waitcnt lgkmcnt(0)\n\t"
                         "v_readfirstlane_b32 %[rn], %[vt]\n\t"
                         "s_cmp_lt_u32 %[rn], 4\n\t"
                         "s_cbranch_scc1 .Lsend_go%=\n\t"
                         WT(0)
                         ".Lsend_loop%=:\n\t"
                         "ds_read_b32 %[vt], %[la] offset:4\n\t"
                         "s_add_u32 %[k1], %[k], 1\n\t"
                         "v_readlane_b32 %[dso], %[dst], %[k]\n\t"
                         "v_readlane_b32 %[ds], %[dstr], %[k]\n\t"
                         "v_readlane_b32 %[so], %[src], %[k1]\n\t"
                         "v_readlane_b32 %[ss], %[sstr], %[k1]\n\t"
                         "s_cmp_eq_u32 %[ds], 0x400\n\t"
                         "s_cselect_b64 s[88:89], %[pw], %[po]\n\t"
                         "s_cselect_b32 s90, %[szw], %[szo]\n\t"
                         "s_waitcnt lgkmcnt(0)\n\t"
                         "v_readfirstlane_b32 %[rn], %[vt]\n\t"
                         "v_add_u32 %[la], 4, %[la]\n\t"
                         ASTAMP
                         "s_bitcmp1_b32 %[rn], 8\n\t"
                         "s_cbranch_scc0 .Lsend_last%=\n\t"
                         SL(0) SL(1) SL(2) SL(3) SL(4) SL(5) SL(6) SL(7) SL(8) SL(9) SL(10)
                         WT(23) FLAG ST(11) LD(11) SL(12) SL(13) SL(14) SL(15)
                         NEXTFLAG
                         "s_mov_b32 %[r], %[rn]\n\t"
                         "s_branch .Lsend_loop%=\n\t"
                         ".Lsend_last%=:\n\t"
                         WT(23) ST(0) WT(22) ST(1) WT(21) ST(2) WT(20) ST(3) WT(19) ST(4) WT(18) ST(5) WT(17) ST(6) WT(16) ST(7)
                         WT(15) ST(8) WT(14) ST(9) WT(13) ST(10) WT(12) FLAG ST(11) WT(11) ST(12) WT(11) ST(13) WT(11) ST(14) WT(11) ST(15)
                         NEXTFLAG
                         WT(0)
                         FLAG
                         "s_nop 1"
                         : "=&v"(R[0]), "=&v"(R[1]), "=&v"(R[2]), "=&v"(R[3]), "=&v"(R[4]), "=&v"(R[5]), "=&v"(R[6]), "=&v"(R[7]),
                           "=&v"(R[8]), "=&v"(R[9]), "=&v"(R[10]), "=&v"(R[11]), "=&v"(R[12]), "=&v"(R[13]), "=&v"(R[14]), "=&v"(R[15]),
                           [la] "+v"(la), [vt] "=&v"(vt), [pfv] "=&v"(pfv), [so] "=&s"(so), [ss] "=&s"(ss), [dso] "=&s"(dso), [ds] "=&s"(ds),
                           [k1] "=&s"(k1), [rn] "=&s"(rn), [pfo] "=&s"(pfo), [pm] "=&s"(pm), [k] "+s"(k), [r] "+s"(r) ASTAMP_OP
                         : [vo] "v"(vo), [src] "v"(src), [dst] "v"(dst), [dfl] "v"(dfl), [sstr] "v"(sstr), [dstr] "v"(dstr),
                           [lgo] "v"((unsigned)(uintptr_t)lds + 4 * ITEM + C_GO * 4),
                           [rs] "s"(rws), [rf] "s"(rpw), [pw] "s"(prws), [po] "s"(po), [szw] "s"(STAMP), [szo] "s"(MSH * NCOL * 4), [tag8] "s"(tag << 8)
                         : "memory", "scc", "s88", "s89", "s90", "s91" ASTAMP_CL);
            STAMP_AT(63 + k);
        }
        STAMP_OUT();
        return;
    }

    const bool own = g >= 12;
    const int o = g - 12, omap = P.omap;
    const int rb = own ? (omap ? (o & 3) : (link & 1) * 2 + (o & 1)) : 4 + (link >> 1);
    const int grb = own ? rank * 4 + rb : ((rank ^ rb) & 7) * 4 + pos;
    const int njob = g < 4 ? 2 : 1;
    auto tile = [&](int job) {
        return own ? (omap ? link * 4 + (o >> 2) : (link >> 1) * 8 + (o >> 1)) : (job ? 3 : g >> 2) * 8 + (link & 1) * 4 + (g & 3);
    };
    if (wave == 4) {
#pragma unroll 1
        for (int job = 0; job < njob; ++job) {
            const int t = tile(job);
            const int rot = (g * P.rot) & 31, fastu = P.fastu, slp0 = P.slp0, slp1 = P.slp1;
            const int ls = lane & 31, lh = lane >> 5;
            const char* const fxp = myws + CTRL + par * FLAG_PAR + rb * fs + lane * 4;
            const char* const fwp = myws + CTRL + par * FLAG_PAR + flag_of_slot(512 + t * 32 + ls, fs);
            const bool loc = (ls >> 2) == rank;
            const unsigned offa = loc ? STG + ((grb * 4 + (ls & 3)) * 2 + lh) * ITEM : par * BUF_BYTES + ((rb * 32 + ls) * 2 + lh) * ITEM;
            const unsigned offb = loc ? STG + (256 + ((rank >> 2) * 32 + t) * 4 + (ls & 3)) * ITEM : par * BUF_BYTES + XR_BYTES + (t * 32 + ls) * ITEM;
            unsigned issued = 0, xa, wa, xb, wb;
            int u = 0;
            bool dead = false;
            u64 t0 = __builtin_amdgcn_s_memrealtime();
#define POLL(fx, fw) asm volatile("; POLL\n\tglobal_load_dword %0, %2, off sc0 sc1\n\tglobal_load_dword %1, %3, off sc0 sc1" \
                                  : "=&v"(fx), "=&v"(fw) : "v"(fxp), "v"(fwp) : "memory")
            auto tick = [&](unsigned& px_, unsigned& pw_) {
                unsigned fx, fw;
                asm volatile("; GOT\n\ts_waitcnt vmcnt(2)\n\tv_mov_b32 %0, %2\n\tv_mov_b32 %1, %3\n\t"
                             "global_load_dword %2, %4, off sc0 sc1\n\tglobal_load_dword %3, %5, off sc0 sc1"
                             : "=&v"(fx), "=&v"(fw), "+v"(px_), "+v"(pw_) : "v"(fxp), "v"(fwp) : "memory");
                const u64 bx = __ballot((fx >> 8) == tag);
                unsigned rdy = (unsigned)bx & (unsigned)(bx >> 32) & (unsigned)__ballot((fw >> 8) == tag);
                if (dead) rdy = ~0u;
                unsigned avail = rdy & ~issued;
                if (!avail) {
                    const int n = u < fastu ? slp0 : slp1;
                    for (int i = 0; i < n; ++i) __builtin_amdgcn_s_sleep(2);
                    dead = spin_pause(t0, tmo, call);
                    return;
                }
                const u4 rec = {offa, offb, (fx & 255u) + (fw & 255u) - 64u, 0u};
                do {
                    const int s = (__builtin_ctz(__builtin_amdgcn_alignbit(avail, avail, rot)) + rot) & 31;
                    issued |= 1u << s;
                    avail &= ~(1u << s);
                    if (ls == s) *(volatile __attribute__((address_space(3))) u4*)(ctl + (u & 1) * 8 + lh * 4) = rec;
                    STAMP_AT(job * 32 + u);
                    STAMP_V(64 + job * 32 + u, s);
                    BAR();
                    ++u;
                } while (__builtin_popcount(avail) > (rdy == ~0u ? 0 : 2));
                t0 = __builtin_amdgcn_s_memrealtime();
            };
            POLL(xa, wa);
            POLL(xb, wb);
            for (;;) {
                tick(xa, wa);
                if (u == 32) break;
                tick(xb, wb);
                if (u == 32) break;
            }
            ctl[C_DEAD] = dead;
            BAR(); BAR(); BAR();
            asm volatile("s_waitcnt vmcnt(0)" :: "v"(xa), "v"(wa), "v"(xb), "v"(wb) : "memory");
            if (own) break;
            BAR();
            if (lane == 0) __hip_atomic_store((unsigned*)(myws + TFLAG) + (rb & 3) * 32 + t, tag << 8, RLX_SYS);
        }
        STAMP_OUT();
        return;
    }
#pragma unroll 1
    for (int job = 0; job < njob; ++job) {
        const int t = tile(job);
        f32v acc, d1, d2, d3, d4, d5;
        u4 dc;
        int ecur, emax, cnt, e0, e1, e2, e3, ta, tb;
        unsigned keep;
#define S_(x) #x
#define S(x) S_(x)
#define VR(b) "v[" S(b) ":" S(b) "+3]"
#define ACC(i, nf) VR(64 + i * 16 + nf * 4)
#define BF(ks, nf) VR(96 + (ks & 1) * 16 + nf * 4)
#define AF(c, f) VR(128 + c * 32 + (f) * 4)
#define RDB(c, ks, nf) "ds_read_b128 " BF(ks, nf) ", %[vl] offset:" S(c * 16384 + ks * 4096 + nf * 1024) "\n\t"
#define MM(c, ks, nf, w) "s_waitcnt lgkmcnt(" #w ")\n\t" \
        "v_mfma_f32_16x16x32_f16 " ACC(0, nf) ", " AF(c, ks) ", " BF(ks, nf) ", " ACC(0, nf) "\n\t" \
        "v_mfma_f32_16x16x32_f16 " ACC(1, nf) ", " AF(c, 4 + ks) ", " BF(ks, nf) ", " ACC(1, nf) "\n\t"
#define LDB(j) "global_load_lds_dwordx4 v61, %[ws] offset:" S(j * 1024) " sc0 sc1\n\t"
#define LDA(n, f) "global_load_dwordx4 " AF(n, f) ", v60, %[ws] offset:" S((f) * 1024 - 4096) " sc0 sc1\n\t"
#define REC(n) "ds_read_b96 v[60:62], %[vc] offset:" S((n & 1) * 32) "\n\t"
#define ADR(n) "s_add_u32 m0, %[lb], " S(n * 16384) "\n\tv_add_u32 v60, v60, %[voa]\n\tv_add_u32 v61, v61, %[vob]\n\tv_readfirstlane_b32 %[e" #n "], v62\n\t"
#define PRO(n) "s_barrier\n\t" REC(n) "s_waitcnt lgkmcnt(0)\n\t" ADR(n) LDB(0) LDB(1) LDB(2) LDB(3) \
        LDA(n, 0) LDA(n, 4) LDA(n, 1) LDA(n, 5) LDA(n, 2) LDA(n, 6) LDA(n, 3) LDA(n, 7)
#define SK "s_cbranch_scc1 2\n\t"
#define STEP(c, n, lim, tailwait) "s_waitcnt vmcnt(24)\n\t" tailwait "s_barrier\n\t" REC(n) \
        RDB(c, 0, 0) RDB(c, 0, 1) RDB(c, 0, 2) RDB(c, 0, 3) RDB(c, 1, 0) RDB(c, 1, 1) RDB(c, 1, 2) RDB(c, 1, 3) \
        "s_cmp_lg_u32 %[e" #c "], %[ecur]\n\ts_cbranch_scc1 .Lslow" #c "_%=\n\t.Lfast" #c "_%=:\n\t" \
        "s_waitcnt lgkmcnt(8)\n\t" ADR(n) "s_cmp_ge_u32 %[cnt], " #lim "\n\t" \
        MM(c, 0, 0, 7) RDB(c, 2, 0) SK LDB(0) MM(c, 0, 1, 7) RDB(c, 2, 1) SK LDB(1) MM(c, 0, 2, 7) RDB(c, 2, 2) SK LDB(2) MM(c, 0, 3, 7) RDB(c, 2, 3) \
        MM(c, 1, 0, 7) RDB(c, 3, 0) SK LDB(3) MM(c, 1, 1, 7) RDB(c, 3, 1) SK LDA(n, 0) MM(c, 1, 2, 7) RDB(c, 3, 2) SK LDA(n, 4) MM(c, 1, 3, 7) RDB(c, 3, 3) \
        MM(c, 2, 0, 7) SK LDA(n, 1) MM(c, 2, 1, 6) SK LDA(n, 5) MM(c, 2, 2, 5) SK LDA(n, 2) MM(c, 2, 3, 4) \
        MM(c, 3, 0, 3) SK LDA(n, 6) MM(c, 3, 1, 2) SK LDA(n, 3) MM(c, 3, 2, 1) SK LDA(n, 7) MM(c, 3, 3, 0)
#define TW(n) "s_cbranch_scc0 1\n\ts_waitcnt vmcnt(" #n ")\n\t"
#define R4(M, b) M(b) M(b + 1) M(b + 2) M(b + 3)
#define R16(M, b) R4(M, b) R4(M, b + 4) R4(M, b + 8) R4(M, b + 12)
#define R32(M, b) R16(M, b) R16(M, b + 16)
#define ZERO(r) "v_mov_b32 v[" S(r) "], 0\n\t"
#define LDX(r) "v_ldexp_f32 v[" S(r) "], v[" S(r) "], %[tb]\n\t"
#define PKM(r) "v_pk_mul_f16 v[" S(r) "], v[" S(r) "], v63 op_sel_hi:[1,0]\n\t"
#define SLOW(c) ".Lslow" #c "_%=:\n\ts_max_i32 %[emax], %[emax], %[e" #c "]\n\ts_add_i32 %[ta], %[emax], -7\n\ts_max_i32 %[ta], %[ta], %[e" #c "]\n\t" \
        "s_sub_i32 %[tb], %[ecur], %[ta]\n\ts_lshl_b32 %[tb], %[tb], 3\n\ts_mov_b32 %[ecur], %[ta]\n\t" R32(LDX, 64) \
        "s_sub_i32 %[tb], %[e" #c "], %[ta]\n\ts_lshl_b32 %[tb], %[tb], 3\n\tv_ldexp_f32 v63, 1.0, %[tb]\n\tv_cvt_f16_f32 v63, v63\n\ts_nop 0\n\t" \
        R32(PKM, 128 + c * 32) "s_nop 1\n\ts_branch .Lfast" #c "_%=\n\t"
        asm volatile("; OPAQUE\n\t"
                     "s_mov_b32 %[keep], m0\n\t"
                     "s_mov_b32 %[cnt], 0\n\t"
                     R32(ZERO, 64)
                     PRO(0) PRO(1) PRO(2)
                     "s_mov_b32 %[ecur], %[e0]\n\t"
                     "s_mov_b32 %[emax], %[e0]\n\t"
                     ".Lgemm%=:\n\t"
                     STEP(0, 3, 8, "") STEP(1, 0, 7, "") STEP(2, 1, 7, TW(12)) STEP(3, 2, 7, TW(0))
                     "s_add_u32 %[cnt], %[cnt], 1\n\t"
                     "s_cmp_lt_u32 %[cnt], 8\n\t"
                     "s_cbranch_scc1 .Lgemm%=\n\t"
                     "s_branch .Lgemm_end%=\n\t"
                     SLOW(0) SLOW(1) SLOW(2) SLOW(3)
                     ".Lgemm_end%=:\n\t"
                     "s_mov_b32 m0, %[keep]\n\t"
                     "s_nop 5"
                     : "=&{v[64:95]}"(acc), "=&{v[96:127]}"(d1), "=&{v[128:159]}"(d2), "=&{v[160:191]}"(d3),
                       "=&{v[192:223]}"(d4), "=&{v[224:255]}"(d5), "=&{v[60:63]}"(dc),
                       [ecur] "=&s"(ecur), [emax] "=&s"(emax), [cnt] "=&s"(cnt), [e0] "=&s"(e0), [e1] "=&s"(e1), [e2] "=&s"(e2), [e3] "=&s"(e3),
                       [ta] "=&s"(ta), [tb] "=&s"(tb), [keep] "=&s"(keep)
                     : [vl] "v"((unsigned)(uintptr_t)lds + lane * 16), [vc] "v"((unsigned)(uintptr_t)lds + 4 * ITEM + (wave >> 1) * 16),
                       [voa] "v"(lane * 16 + 4096 + (wave & 1) * 8192), [vob] "v"(lane * 16 + wave * 4096),
                       [lb] "s"((unsigned)(uintptr_t)lds + wave * 4096), [ws] "s"(myws)
                     : "memory", "scc");
        STAMP_AT(job * 4 + 0);

        {
            const int rs = own ? NCOL * 4 : 8192;
            char* ob = own ? (char*)P.out[0] + (size_t)(rb * 128 + wave * 32) * (NCOL * 4) + (rank >> 2) * 8192 + t * 256
                           : myws + YST + (size_t)((rb & 3) * 128 + wave * 32) * 8192 + t * 256;
            auto ro = __builtin_amdgcn_make_buffer_rsrc(ob, 0, 31 * rs + 256, 0x00020000);
            const int vo = q4 * 4 * rs + c16 * 16;
            float sc;
            if (ecur < -15 || ecur > 15) {
                sc = __builtin_bit_cast(float, (127 + 4 * ecur) << 23);
                acc *= sc;
            } else sc = __builtin_bit_cast(float, (127 + 8 * ecur) << 23);
            if (ctl[C_DEAD]) sc = __builtin_nanf("");
            u4 r[8];
#pragma unroll
            for (int i = 0; i < 2; ++i)
#pragma unroll
                for (int j = 0; j < 4; ++j) {
                    f4 y;
#pragma unroll
                    for (int c = 0; c < 4; ++c) y[c] = acc[i * 16 + c * 4 + j] * sc;
                    r[i * 4 + j] = __builtin_bit_cast(u4, y);
                }
            asm volatile("" : "+v"(r[0]), "+v"(r[1]), "+v"(r[2]), "+v"(r[3]), "+v"(r[4]), "+v"(r[5]), "+v"(r[6]), "+v"(r[7]));
            STAMP_AT(job * 4 + 1);
#pragma unroll
            for (int i = 0; i < 8; ++i)
                __builtin_amdgcn_raw_buffer_store_b128(r[i], ro, vo, ((i >> 2) * 16 + (i & 3)) * rs, AUX_SYS);
            asm volatile("s_nop 1" :: "v"(r[0]), "v"(r[1]), "v"(r[2]), "v"(r[3]), "v"(r[4]), "v"(r[5]), "v"(r[6]), "v"(r[7]) : "memory");
        }
        if (!own) asm volatile("s_waitcnt vmcnt(0)\n\ts_barrier" ::: "memory");
    }
    STAMP_OUT();
}

struct Tune { int stag0 = 0, stag = 100, fs = 4352, rot = 3, xrot = 1, omap = 0, fastu = 20, slp0 = 6, slp1 = 2, nf = 8, xe = 0; };

void build_lists(const Tune& T, uint16_t (&L)[3][MAXITEMS], int (&len)[3]) {
    const int nf = T.nf;
    for (int c = 0; c < 3; ++c) {
        int n = 0;
        auto X = [&](int a0, int i0 = 0, int i1 = 32) { for (int i = i0; i < i1; ++i) L[c][n++] = (uint16_t)(0 << 8 | (a0 + (i & 7)) << 2 | i >> 3); };
        auto W = [&](int t0, int t1) { for (int q = 0; q < 4; ++q) for (int t = t0; t < t1; ++t) L[c][n++] = (uint16_t)(1 << 8 | t << 2 | q); };
        if (c == 0) { X(8); W(0, 8); W(8, 16); W(16, 24); W(24, 32); X(0); }
        if (c == 1) {
            X(8); W(0, 8);
            for (int s = 0; s < 4; ++s) for (int t = 0; t < nf; ++t) L[c][n++] = (uint16_t)(2 << 8 | t << 2 | s);
            W(8, 16); W(16, 24); X(0, 0, T.xe); W(24, 32); X(0, T.xe, 32);
        }
        if (c == 2) {
            for (int t = 0; t < nf; ++t) L[c][n++] = (uint16_t)(1 << 8 | t << 2 | 0);
            X(8);
            if (nf < 8) W(nf, 8);
            W(nf > 8 ? nf : 8, 16); W(16, 24); W(24, 32); X(0);
            for (int i = 0; i < 64; ++i) L[c][n++] = (uint16_t)(3 << 8 | i);
        }
        len[c] = n;
        while (n < MAXITEMS) L[c][n++] = 0xFFFF;
    }
}

}

extern "C" void kernel_launch(int rank, int nranks, ncclComm_t comm,
                              hipStream_t stream,
                              void* const* d_in, const int* in_sizes, int n_in,
                              void* d_out, int out_size,
                              const int* in_dtypes, int out_dtype,
                              void* const* d_out_all,
                              void* const* d_ws_all, size_t ws_bytes) {
    (void)comm; (void)in_dtypes; (void)out_dtype; (void)d_out;
    if (nranks != NR || (unsigned)rank >= (unsigned)NR || n_in < 2 || ws_bytes < WS_NEED ||
        in_sizes[0] != 4096 * KSH || in_sizes[1] != KSH * NCOL || out_size != MSH * NCOL) {
        fprintf(stderr, "kernel_launch: unexpected problem shape\n");
        return;
    }
    const Tune T;
    uint16_t L[3][MAXITEMS];
    int len[3];
    build_lists(T, L, len);
    Params P;
    P.x = (const float*)d_in[0];
    P.w = (const float*)d_in[1];
    for (int j = 0; j < NR; ++j) { P.out[j] = (float*)d_out_all[rank ^ j]; P.ws[j] = (char*)d_ws_all[rank ^ j]; }
    P.rank = rank;
    P.stag0 = T.stag0; P.stag = T.stag; P.fs = T.fs;
    P.rot = T.rot; P.xrot = T.xrot; P.omap = T.omap; P.fastu = T.fastu; P.slp0 = T.slp0; P.slp1 = T.slp1;
    for (int c = 0; c < 2; ++c) for (int i = 0; i < MAXITEMS; ++i) P.list[c][i] = L[c + 1][i];
    bool taken[512] = {};
    int next = 0;
    for (int i = 0; i < 256; ++i) P.conv[i] = 0;
    for (int p = 0; p < MAXITEMS; ++p)
        for (int l = 0; l < 8; ++l) {
            const int code = L[l == 0 ? 0 : 1 + (l >> 2)][p];
            if ((code >> 8) > 1) { if (p < 2 * NSEND) { fprintf(stderr, "bad list head\n"); return; } continue; }
            const int id = staged_id(code, rank ^ l, l, T.xrot);
            if (taken[id]) { if (p < 2 * NSEND) { fprintf(stderr, "bad list head\n"); return; } continue; }
            taken[id] = true;
            if (p < 2 * NSEND) P.conv[(p & 3) * 8 + l] |= (unsigned)id << (p >> 2) * 16;
            else {
                P.conv[8 * NSEND + next % (8 * NWORK)] |= (unsigned)id << (next / (8 * NWORK)) * 16;
                ++next;
            }
        }
    if (next != 16 * NWORK) { fprintf(stderr, "bad conversion table\n"); return; }
    gemm_rs_kernel<<<8 * (NWORK + NSEND), 384, 0, stream>>>(P);
    HIP_CHECK(hipGetLastError());
}
```

```cpp
#include <hip/hip_runtime.h>
#include <rccl/rccl.h>
#include <cstdio>
#include <cstdint>
#include <cstddef>

#define HIP_CHECK(expr) do { hipError_t _e = (expr); if (_e != hipSuccess) { \
    fprintf(stderr, "HIP error %s at %s:%d\n", hipGetErrorString(_e), __FILE__, __LINE__); \
    return; } } while (0)

namespace {

typedef _Float16 h8 __attribute__((ext_vector_type(8)));
typedef float f4 __attribute__((ext_vector_type(4)));
typedef unsigned u4 __attribute__((ext_vector_type(4)));
typedef unsigned long long u64;

constexpr int NR = 8;
constexpr int NCOL = 4096;
constexpr int KSH = 512;
constexpr int MSH = 512;
constexpr int NWORK = 28, NSEND = 4;

constexpr unsigned ITEM = 16384;
constexpr unsigned XR_BYTES = 8 * 32 * 2 * ITEM;
constexpr unsigned WR_BYTES = 32 * 32 * ITEM;
constexpr unsigned STG = XR_BYTES + WR_BYTES;
constexpr unsigned CTRL = STG + 512 * ITEM;
constexpr unsigned YRDY = CTRL + 110592;
constexpr unsigned RDY = CTRL + 114688;
constexpr unsigned BUF_BYTES = CTRL + 131072;
constexpr unsigned YST = 2 * BUF_BYTES;
constexpr unsigned EPOCH = YST + 256 * ITEM;
constexpr unsigned TMO = EPOCH + 8192;
constexpr unsigned STAMP = TMO + 4096;
constexpr unsigned WS_NEED = STAMP + (1 << 20);

constexpr u64 DEADLINE_TICKS = 100000000ull;

constexpr int MAXITEMS = 256;

struct Params {
    uint16_t list[2][MAXITEMS];
    unsigned conv[256];
    char* wsx[NR];
    char* ws[NR];
    const float* x;
    const float* w;
    float* out;
    int rank;
    int stag0, stag;
    int bdel, stagb;
    int fs;
    int rot, xrot, omap, fastu, slp0, slp1;
};

__host__ __device__ inline int staged_id(int code, int peer, int link, int xrot) {
    const int a = (code >> 2) & 31, qa = (code & 3) ^ (peer & 3);
    if ((code >> 8) == 0) {
        const int rb = a >> 1;
        const int grb = rb < 4 ? peer * 4 + ((rb + link * xrot) & 3) : ((peer ^ rb) & 7) * 4 + (peer & 3);
        return (grb * 4 + qa) * 2 + (a & 1);
    }
    return 256 + ((peer >> 2) * 32 + a) * 4 + qa;
}
__host__ __device__ inline int recv_slot(int id, int src, int dst) {
    if (id < 256) {
        const int grb = id >> 3, qa = (id >> 1) & 3, o = grb >> 2, p = grb & 3;
        const int rb = o == dst ? p : o ^ dst;
        if (o != dst && (rb < 4 || p != (dst & 3))) return -1;
        return (rb * 32 + src * 4 + qa) * 2 + (id & 1);
    }
    const int ct = (id - 256) >> 2;
    if ((ct >> 5) != (dst >> 2)) return -1;
    return 512 + (ct & 31) * 32 + src * 4 + (id & 3);
}
__host__ __device__ inline unsigned flag_of_slot(int slot, int fs) {
    const int i = slot < 512 ? (slot >> 6) * 64 + (slot & 1) * 32 + ((slot >> 1) & 31) : slot;
    return (i >> 6) * fs + (i & 63) * 4;
}

#define RLX_SYS __ATOMIC_RELAXED, __HIP_MEMORY_SCOPE_SYSTEM
#define RLX_AGT __ATOMIC_RELAXED, __HIP_MEMORY_SCOPE_AGENT
constexpr int AUX_SYS = 17;

__device__ __forceinline__ int uni(int v) { return __builtin_amdgcn_readfirstlane(v); }

#define STAMP_V(slot, val) do { } while (0)
#define STAMP_AT(slot) do { } while (0)
#define STAMP_OUT() do { } while (0)

__device__ __forceinline__ bool spin_pause(u64 t0, unsigned* tmo, unsigned call) {
    __builtin_amdgcn_s_sleep(2);
    u64 dt = __builtin_amdgcn_s_memrealtime() - t0;
    if (dt > 50000ull) {
        if (dt > DEADLINE_TICKS || (unsigned)uni((int)__hip_atomic_load(tmo, RLX_AGT)) == call) {
            __hip_atomic_store(tmo, call, RLX_AGT);
            return true;
        }
        __builtin_amdgcn_s_sleep(32);
    }
    return false;
}

__device__ __forceinline__ unsigned wave_max(unsigned v) {
    v = max(v, (unsigned)__builtin_amdgcn_update_dpp(0, (int)v, 0x111, 0xf, 0xf, true));
    v = max(v, (unsigned)__builtin_amdgcn_update_dpp(0, (int)v, 0x112, 0xf, 0xf, true));
    v = max(v, (unsigned)__builtin_amdgcn_update_dpp(0, (int)v, 0x114, 0xf, 0xf, true));
    v = max(v, (unsigned)__builtin_amdgcn_update_dpp(0, (int)v, 0x118, 0xf, 0xf, true));
    v = max(v, (unsigned)__builtin_amdgcn_update_dpp(0, (int)v, 0x142, 0xa, 0xf, true));
    v = max(v, (unsigned)__builtin_amdgcn_update_dpp(0, (int)v, 0x143, 0xc, 0xf, true));
    return v;
}

__device__ __forceinline__ void wait_until(unsigned tstart, int ticks) {
    while ((int)((unsigned)__builtin_amdgcn_s_memrealtime() - tstart) < ticks) __builtin_amdgcn_s_sleep(1);
}

#define BAR() asm volatile("s_waitcnt lgkmcnt(0)\n\ts_barrier" ::: "memory")

constexpr unsigned RDY_VALID = 0x100, RDY_DEAD = 0x200;

typedef float f32v __attribute__((ext_vector_type(32)));

constexpr int C_MAX = 64, C_CALL = 72, C_CNT = 73;

__global__ void __launch_bounds__(384, 2) gemm_rs_kernel(const Params P) {
    __shared__ __attribute__((aligned(1024))) char lds[4 * ITEM + 512];
    const int lane = threadIdx.x & 63;
    const int wave = uni(threadIdx.x >> 6);
    const int xcd = blockIdx.x & 7;
    const int slot = blockIdx.x >> 3;
    const int g = slot - NSEND;
    int code0 = 0, code1 = 0;
    if (wave >= 4) {
        if (g >= 0) { if (wave == 5) __builtin_amdgcn_endpgm(); }
        else {
            const int idx = 2 * NSEND + slot + NSEND * lane;
            asm volatile("global_load_ushort %0, %2, %3\n\tglobal_load_ushort %1, %2, %3 offset:512"
                         : "=&v"(code0), "=&v"(code1) : "v"(2 * (idx < MAXITEMS ? idx : MAXITEMS - 1)), "s"(__builtin_amdgcn_kernarg_segment_ptr()) : "memory");
        }
    }
    const u64 tstart64 = __builtin_amdgcn_s_memrealtime();
    const unsigned tstart = (unsigned)tstart64;
    const float* const px = P.x;
    const float* const pw = P.w;
    char* const myws = P.ws[0];
    char* const prws = P.wsx[xcd];
    const unsigned cv = P.conv[blockIdx.x];
    const int rank = P.rank & 7, stag0 = P.stag0, stag = P.stag, fs = P.fs;
    asm volatile("" :: "s"(tstart64), "s"(px), "s"(pw), "s"(myws), "s"(prws), "s"(cv), "s"(rank), "s"(stag0), "s"(stag), "s"(fs));
    const int link = (cv >> 12) & 7;
    const int peer = rank ^ link;
    const int pos = rank & 3;
    const bool mover = g < 0 && link != 0;
    unsigned* const tmo = (unsigned*)(myws + TMO);
    typedef volatile __attribute__((address_space(3))) unsigned* ldsu;
    const ldsu ctl = (ldsu)(lds + 4 * ITEM);

    const int c16 = lane & 15, q4 = lane >> 4;
    unsigned call = 0;

    if (wave < 4) {
        auto fetch = [&](unsigned id, f4 (&d)[8]) {
            const float* base;
            int bytes, s0, s1, vo;
            if (id < 256) {
                const int qa = (id >> 1) & 3;
                base = px + (size_t)((id >> 3) * 128 + (id & 1) * 64 + wave * 16) * KSH + qa * 128;
                bytes = 16 * KSH * 4 - qa * 512;
                s0 = 64; s1 = 256;
                vo = c16 * (KSH * 4) + q4 * 16;
            } else {
                const int col0 = ((id >> 2) & 63) * 64;
                base = pw + (size_t)((id & 3) * 128 + wave * 32) * NCOL + col0;
                bytes = 32 * NCOL * 4 - col0 * 4;
                s0 = NCOL * 4; s1 = 16 * NCOL * 4;
                vo = q4 * 4 * (NCOL * 4) + c16 * 16;
            }
            auto rs = __builtin_amdgcn_make_buffer_rsrc((void*)base, 0, bytes, 0x00020000);
#pragma unroll
            for (int j = 0; j < 8; ++j)
                d[j] = __builtin_bit_cast(f4, __builtin_amdgcn_raw_buffer_load_b128(rs, vo, (j >> 2) * s1 + (j & 3) * s0, 0));
        };
        auto convert = [&](unsigned id, f4 (&v)[8], u4 (&o)[4], int c) -> unsigned {
            float m = 0.f;
#pragma unroll
            for (int j = 0; j < 8; ++j) m = fmaxf(fmaxf(fmaxf(fmaxf(m, fabsf(v[j][0])), fabsf(v[j][1])), fabsf(v[j][2])), fabsf(v[j][3]));
            STAMP_AT(64 + c * 4);
            const unsigned wm = wave_max(__builtin_bit_cast(unsigned, m));
            if (lane == 63) ctl[C_MAX + c * 4 + wave] = wm;
            BAR();
            call = (unsigned)uni((int)ctl[C_CALL]);
            const u4 mp = *(volatile __attribute__((address_space(3))) u4*)(ctl + C_MAX + c * 4);
            const int e8 = max(((uni((int)max(max(mp[0], mp[1]), max(mp[2], mp[3]))) >> 23) - 134) >> 3, -15);
            const float sc = __builtin_bit_cast(float, (127 - 8 * e8) << 23);
#pragma unroll
            for (int j = 0; j < 8; ++j) v[j] *= sc;
            auto h = [](float f) { return (_Float16)f; };
            if (id < 256) {
                asm volatile("" : "+v"(v[0]), "+v"(v[7]));
#pragma unroll
                for (int f = 0; f < 4; ++f) {
                    const f4 lo = v[2 * f], hi = v[2 * f + 1];
                    h8 t = {h(lo[0]), h(lo[1]), h(lo[2]), h(lo[3]), h(hi[0]), h(hi[1]), h(hi[2]), h(hi[3])};
                    o[f] = __builtin_bit_cast(u4, t);
                }
            } else {
#pragma unroll
                for (int f = 0; f < 4; ++f) {
                    h8 t = {h(v[0][f]), h(v[1][f]), h(v[2][f]), h(v[3][f]), h(v[4][f]), h(v[5][f]), h(v[6][f]), h(v[7][f])};
                    o[f] = __builtin_bit_cast(u4, t);
                }
            }
            asm volatile("" : "+v"(o[0]), "+v"(o[1]), "+v"(o[2]), "+v"(o[3]));
            STAMP_AT(64 + c * 4 + 1);
            return (unsigned)(e8 + 32);
        };
        auto last_in = [&](int c) {
            unsigned v = 0;
            if (lane == 0) v = __hip_atomic_fetch_add((__attribute__((address_space(3))) unsigned*)(ctl + c), 1u, __ATOMIC_RELAXED, __HIP_MEMORY_SCOPE_WORKGROUP);
            return uni((int)v) == 3;
        };
        auto flag_off = [&](unsigned id) -> unsigned {
            const int sl = recv_slot(id, rank, (rank ^ lane) & 7);
            const unsigned p = lane == 0 || (mover && lane == link) ? CTRL + flag_of_slot(sl, fs) : RDY + lane * 2048 + id * 4;
            return lane >= NR || sl < 0 ? 0u : p;
        };
        auto peers = [&]() {
            u64 p;
            asm volatile("global_load_dwordx2 %0, %1, %2 offset:%3"
                         : "=&v"(p) : "v"((lane & 7) * 8), "s"(__builtin_amdgcn_kernarg_segment_ptr()), "n"(offsetof(Params, ws)) : "memory");
            return p;
        };
        auto publish = [&](u64 wsp, unsigned off, unsigned e) {
            if (off) __hip_atomic_store((unsigned*)(wsp + ((call & 1u) * BUF_BYTES + off)), call << 8 | e, RLX_SYS);
        };
        auto put = [&](char* ws, unsigned item, const u4 (&o)[4], bool paced) {
            auto r = __builtin_amdgcn_make_buffer_rsrc(ws + (call & 1u) * BUF_BYTES + item * ITEM + wave * 4096, 0, 4096, 0x00020000);
#pragma unroll
            for (int f = 0; f < 4; ++f) {
                if (paced) asm volatile("s_waitcnt vmcnt(3)" ::: "memory");
                __builtin_amdgcn_raw_buffer_store_b128(o[f], r, lane * 16 + f * 1024, 0, AUX_SYS);
            }
        };
        if (mover) {
            const unsigned ida = cv & 511u, idb = (cv >> 16) & 511u;
            f4 va[8], vb[8];
            u4 oa[4], ob[4];
            fetch(ida, va);
            fetch(idb, vb);
            __builtin_amdgcn_sched_barrier(0);
            unsigned fa = flag_off(ida), fb = flag_off(idb);
            int sla = recv_slot(ida, rank, peer) & 2047, slb = recv_slot(idb, rank, peer) & 2047;
            asm volatile("" : "+v"(fa), "+v"(fb));
            asm volatile("" : "+s"(sla), "+s"(slb));
            const unsigned ea = convert(ida, va, oa, 0);
            put(prws, sla, oa, false);
            __builtin_amdgcn_sched_barrier(0);
            asm volatile("" : "+v"(vb[0]) :: "memory");
            const unsigned eb = convert(idb, vb, ob, 1);
            put(prws, slb, ob, true);
            STAMP_AT(64 + 2);
            if (ida < 256) put(myws, STG / ITEM + ida, oa, false);
            if (idb < 256) put(myws, STG / ITEM + idb, ob, false);
            u64 wsp = peers();
            asm volatile("s_waitcnt vmcnt(0)" : "+v"(wsp) : "v"(oa[0]), "v"(oa[1]), "v"(oa[2]), "v"(oa[3]), "v"(ob[0]), "v"(ob[1]), "v"(ob[2]), "v"(ob[3]) : "memory");
            STAMP_AT(64 + 4 + 2);
            if (last_in(C_CNT)) { publish(wsp, fa, ea); publish(wsp, fb, eb); }
            STAMP_OUT();
            __builtin_amdgcn_endpgm();
        }
#pragma unroll 1
        for (int c = 0; c < 2; ++c) {
            const unsigned id = (cv >> (c * 16)) & 511u;
            f4 v[8];
            u4 o[4];
            if (g > 0 || (g == 0 && c)) wait_until(tstart, c ? P.bdel + ((g * P.stagb) >> 4) : stag0 + ((g * stag) >> 4));
            fetch(id, v);
            __builtin_amdgcn_sched_barrier(0);
            unsigned fp = flag_off(id);
            asm volatile("" : "+v"(fp));
            const unsigned e = convert(id, v, o, c);
            put(myws, STG / ITEM + id, o, false);
            u64 wsp = peers();
            asm volatile("s_waitcnt vmcnt(0)" : "+v"(wsp) : "v"(o[0]), "v"(o[1]), "v"(o[2]), "v"(o[3]) : "memory");
            STAMP_AT(64 + c * 4 + 2);
            if (last_in(C_CNT + c)) publish(wsp, fp, e);
        }
        if (g < 0) { STAMP_OUT(); __builtin_amdgcn_endpgm(); }
    } else {
        if (wave == 4) {
            unsigned* const ep = (unsigned*)(myws + EPOCH) + blockIdx.x;
            call = __hip_atomic_load(ep, RLX_AGT) + 1u;
            ctl[C_CALL + (lane & 3)] = lane & 3 ? 0u : call;
            if (lane == 0) __hip_atomic_store(ep, call, RLX_AGT);
        } else ctl[lane] = 0;
        BAR();
        if (g < 0 && !mover) __builtin_amdgcn_endpgm();
        call = ctl[C_CALL];
        BAR();
    }
    call = (unsigned)uni((int)call);
    const unsigned tag = call & 0xFFFFFFu, pb = (call & 1u) * BUF_BYTES;

    if (g < 0) {
        asm volatile("s_waitcnt vmcnt(0)" : "+v"(code0), "+v"(code1) :: "memory");
        const int code = link >> 2 ? code1 : code0;
        const int n = __builtin_popcountll(__ballot(code != 0xFFFF));
        const int type = code >> 8;
        unsigned src, dst, sfl, dfl;
        if (type == 3) {
            const unsigned quad = (code >> 3) & 7, rg = code & 7;
            src = YST + ((link & 3) * 128 + rg * 16) * 8192 + quad * 1024;
            dst = ((peer & 3) * 128 + rg * 16) * (NCOL * 4) + (peer >> 2) * 8192 + quad * 1024;
            sfl = pb + YRDY + ((link & 3) * 32 + quad * 4) * 4;
            dfl = 0;
        } else if (type == 2) {
            const unsigned t = (code >> 2) & 31, sj = 4 + (code & 3);
            const unsigned sl = 512 + t * 32 + ((rank ^ sj) & 7) * 4 + (peer & 3);
            src = dst = pb + sl * ITEM;
            sfl = pb + RDY + sj * 2048 + (256 + ((rank >> 2) * 32 + t) * 4 + (peer & 3)) * 4;
            dfl = pb + CTRL + flag_of_slot(sl, fs);
        } else {
            const int id = staged_id(code & 0x1FF, rank, link, P.xrot);
            const int sl = recv_slot(id, peer, rank) & 2047;
            src = pb + STG + id * ITEM;
            dst = pb + sl * ITEM;
            sfl = pb + RDY + link * 2048 + id * 4;
            dfl = pb + CTRL + flag_of_slot(sl, fs);
        }
        auto rws = __builtin_amdgcn_make_buffer_rsrc(myws, 0, (int)STAMP, 0x00020000);
        if (wave == 5) {
            bool done = lane >= n;
            u64 t0 = __builtin_amdgcn_s_memrealtime();
            for (;;) {
                const u64 pend = __ballot(!done);
                if (!pend) break;
                const int first = __builtin_ctzll(pend);
                bool ok = false;
                if (!done && lane < first + 6) {
                    const u4 f = __builtin_amdgcn_raw_buffer_load_b128(rws, sfl & ~15u, 0, AUX_SYS);
                    const unsigned c = (sfl >> 2) & 3;
                    const unsigned v = c == 0 ? f[0] : c == 1 ? f[1] : c == 2 ? f[2] : f[3];
                    ok = (v >> 8) == tag;
                    if (type == 3) ok = ok && (f[1] >> 8) == tag && (f[2] >> 8) == tag && (f[3] >> 8) == tag;
                    if (ok) { ctl[lane] = (v & 255u) | RDY_VALID; done = true; }
                }
                if (__any(ok)) t0 = __builtin_amdgcn_s_memrealtime();
                else if (spin_pause(t0, tmo, call)) { if (!done) ctl[lane] = RDY_DEAD; break; }
            }
            STAMP_OUT();
            __builtin_amdgcn_endpgm();
        }
        auto rpw = __builtin_amdgcn_make_buffer_rsrc(prws, 0, (int)STAMP, 0x00020000);
        float* const po = P.out;
        const int vo = lane * 16;
        const unsigned sstr = type == 3 ? 8192u : 1024u, dstr = type == 3 ? NCOL * 4u : 1024u;
        int k = 0;
        while (k < n) {
            unsigned r;
            while ((r = (unsigned)uni((int)ctl[k])) == 0) __builtin_amdgcn_s_sleep(1);
            if (r & RDY_DEAD) {
                if (type == 3 && lane == n - 1) po[dst >> 2] = __builtin_nanf("");
                break;
            }
            u4 R[16];
            unsigned la = (unsigned)(uintptr_t)lds + 4 * ITEM + k * 4, vt, vd, pfv, so, ss, dso, ds, k1, rn, pfo;
            u64 pm;
#define LD(f) "buffer_load_dwordx4 %" #f ", %[vo], %[rs], %[so] offen sc0 sc1\n\ts_add_u32 %[so], %[so], %[ss]\n\t"
#define ST(f) "buffer_store_dwordx4 %" #f ", %[vo], s[88:91], %[dso] offen sc0 sc1\n\ts_add_u32 %[dso], %[dso], %[ds]\n\t"
#define WT(n) "s_waitcnt vmcnt(" #n ")\n\t"
#define WTW WT(23)
#define SL(f) WTW ST(f) LD(f)
#define L0(f) WTW "buffer_load_dword %[vd], off, %[rf], 0\n\t" LD(f)
#define FLAG "s_mov_b64 exec, %[pm]\n\ts_cbranch_execz 2\n\tbuffer_store_dword %[pfv], off, %[rf], %[pfo] sc0 sc1\n\ts_mov_b64 exec, -1\n\t"
#define NEXTFLAG "v_readlane_b32 %[pfo], %[dfl], %[k]\n\ts_and_b32 %[r], %[r], 0xff\n\ts_or_b32 %[r], %[r], %[tag8]\n\t" \
                 "v_mov_b32 %[pfv], %[r]\n\ts_cmp_lg_u32 %[pfo], 0\n\ts_cselect_b64 %[pm], 1, 0\n\ts_mov_b32 %[k], %[k1]\n\t"
#define ASTAMP
#define ASTAMP_OP
#define ASTAMP_CL
            asm volatile("; OPAQUE\n\t"
                         "v_readlane_b32 %[so], %[src], %[k]\n\t"
                         "v_readlane_b32 %[ss], %[sstr], %[k]\n\t"
                         "s_mov_b64 %[pm], 0\n\t"
                         "s_mov_b32 %[pfo], 0\n\t"
                         "s_mov_b32 s91, 0x20000\n\t"
                         "s_nop 4\n\t"
                         L0(0) L0(1) L0(2) L0(3) L0(4) L0(5) L0(6) L0(7) L0(8) L0(9) L0(10) L0(11) L0(12) L0(13) L0(14) L0(15)
                         ".Lsend_loop%=:\n\t"
                         "ds_read_b32 %[vt], %[la] offset:4\n\t"
                         "s_add_u32 %[k1], %[k], 1\n\t"
                         "v_readlane_b32 %[dso], %[dst], %[k]\n\t"
                         "v_readlane_b32 %[ds], %[dstr], %[k]\n\t"
                         "v_readlane_b32 %[so], %[src], %[k1]\n\t"
                         "v_readlane_b32 %[ss], %[sstr], %[k1]\n\t"
                         "s_cmp_eq_u32 %[ds], 0x400\n\t"
                         "s_cselect_b64 s[88:89], %[pw], %[po]\n\t"
                         "s_cselect_b32 s90, %[szw], %[szo]\n\t"
                         "s_waitcnt lgkmcnt(0)\n\t"
                         "v_readfirstlane_b32 %[rn], %[vt]\n\t"
                         "v_add_u32 %[la], 4, %[la]\n\t"
                         ASTAMP
                         "s_bitcmp1_b32 %[rn], 8\n\t"
                         "s_cbranch_scc0 .Lsend_last%=\n\t"
                         SL(0) SL(1) SL(2) SL(3) SL(4) SL(5) SL(6) SL(7) SL(8) SL(9) SL(10)
                         WT(23) FLAG ST(11) LD(11) SL(12) SL(13) SL(14) SL(15)
                         NEXTFLAG
                         "s_mov_b32 %[r], %[rn]\n\t"
                         "s_branch .Lsend_loop%=\n\t"
                         ".Lsend_last%=:\n\t"
                         WT(30) ST(0) WT(29) ST(1) WT(28) ST(2) WT(27) ST(3) WT(26) ST(4) WT(25) ST(5) WT(24) ST(6) WT(23) ST(7)
                         WT(22) ST(8) WT(21) ST(9) WT(20) ST(10) WT(19) ST(11) WT(18) ST(12) WT(17) ST(13) WT(16) ST(14) WT(15) FLAG ST(15)
                         NEXTFLAG
                         WT(0)
                         FLAG
                         "s_nop 1"
                         : "=&v"(R[0]), "=&v"(R[1]), "=&v"(R[2]), "=&v"(R[3]), "=&v"(R[4]), "=&v"(R[5]), "=&v"(R[6]), "=&v"(R[7]),
                           "=&v"(R[8]), "=&v"(R[9]), "=&v"(R[10]), "=&v"(R[11]), "=&v"(R[12]), "=&v"(R[13]), "=&v"(R[14]), "=&v"(R[15]),
                           [la] "+v"(la), [vt] "=&v"(vt), [vd] "=&v"(vd), [pfv] "=&v"(pfv), [so] "=&s"(so), [ss] "=&s"(ss), [dso] "=&s"(dso), [ds] "=&s"(ds),
                           [k1] "=&s"(k1), [rn] "=&s"(rn), [pfo] "=&s"(pfo), [pm] "=&s"(pm), [k] "+s"(k), [r] "+s"(r) ASTAMP_OP
                         : [vo] "v"(vo), [src] "v"(src), [dst] "v"(dst), [dfl] "v"(dfl), [sstr] "v"(sstr), [dstr] "v"(dstr),
                           [rs] "s"(rpw), [rf] "s"(rws), [pw] "s"(myws), [po] "s"(po), [szw] "s"(STAMP), [szo] "s"(MSH * NCOL * 4), [tag8] "s"(tag << 8)
                         : "memory", "scc", "s88", "s89", "s90", "s91" ASTAMP_CL);
            STAMP_AT(63 + k);
        }
        STAMP_OUT();
        __builtin_amdgcn_endpgm();
    }

    const bool own = g >= 12;
    const int o = g - 12, omap = P.omap;
    const int fl = (cv >> 28) & 3, fh = (cv >> 30) & 1;
    const int rb = own ? (omap ? (o & 3) : (xcd & 1) * 2 + (o & 1)) : 4 + fl;
    const int grb = own ? rank * 4 + rb : ((rank ^ rb) & 7) * 4 + pos;
    const int njob = g < 4 ? 2 : 1;
    auto tile = [&](int job) {
        return own ? (omap ? xcd * 4 + (o >> 2) : (xcd >> 1) * 8 + (o >> 1)) : (job ? 3 : g >> 2) * 8 + fh * 4 + (g & 3);
    };
    if (wave == 4) {
        char* const ows = P.ws[rb];
#pragma unroll 1
        for (int job = 0; job < njob; ++job) {
            const int t = tile(job);
            const int rot = (g * P.rot) & 31, fastu = P.fastu, slp0 = P.slp0, slp1 = P.slp1;
            const int ls = lane & 31, lh = lane >> 5;
            const char* const fxp = myws + pb + CTRL + rb * fs + lane * 4;
            const char* const fwp = myws + pb + CTRL + flag_of_slot(512 + t * 32 + ls, fs);
            const bool loc = (ls >> 2) == rank;
            const unsigned offa = pb + (loc ? STG + ((grb * 4 + (ls & 3)) * 2 + lh) * ITEM : ((rb * 32 + ls) * 2 + lh) * ITEM);
            const unsigned offb = pb + (loc ? STG + (256 + ((rank >> 2) * 32 + t) * 4 + (ls & 3)) * ITEM : XR_BYTES + (t * 32 + ls) * ITEM);
            unsigned issued = 0, xa, wa, xb, wb;
            int u = 0;
            bool dead = false;
            u64 t0 = __builtin_amdgcn_s_memrealtime();
#define POLL(fx, fw) asm volatile("; POLL\n\tglobal_load_dword %0, %2, off sc0 sc1\n\tglobal_load_dword %1, %3, off sc0 sc1" \
                                  : "=&v"(fx), "=&v"(fw) : "v"(fxp), "v"(fwp) : "memory")
            auto tick = [&](unsigned& px_, unsigned& pw_) {
                unsigned fx, fw;
                asm volatile("; GOT\n\ts_waitcnt vmcnt(2)\n\tv_mov_b32 %0, %2\n\tv_mov_b32 %1, %3\n\t"
                             "global_load_dword %2, %4, off sc0 sc1\n\tglobal_load_dword %3, %5, off sc0 sc1"
                             : "=&v"(fx), "=&v"(fw), "+v"(px_), "+v"(pw_) : "v"(fxp), "v"(fwp) : "memory");
                const u64 bx = __ballot((fx >> 8) == tag);
                unsigned rdy = (unsigned)bx & (unsigned)(bx >> 32) & (unsigned)__ballot((fw >> 8) == tag);
                if (dead) rdy = ~0u;
                unsigned avail = rdy & ~issued;
                if (!avail) {
                    const int n = u < fastu ? slp0 : slp1;
                    for (int i = 0; i < n; ++i) __builtin_amdgcn_s_sleep(2);
                    dead = spin_pause(t0, tmo, call);
                    return;
                }
                const u4 rec = {offa, offb, (fx & 255u) + (fw & 255u) - 64u, dead};
                do {
                    const int s = (__builtin_ctz(__builtin_amdgcn_alignbit(avail, avail, rot)) + rot) & 31;
                    issued |= 1u << s;
                    avail &= ~(1u << s);
                    if (ls == s) *(volatile __attribute__((address_space(3))) u4*)(ctl + (u & 1) * 8 + lh * 4) = rec;
                    STAMP_AT(job * 32 + u);
                    STAMP_V(64 + job * 32 + u, s);
                    BAR();
                    ++u;
                } while (__builtin_popcount(avail) > (rdy == ~0u ? 0 : 2));
                t0 = __builtin_amdgcn_s_memrealtime();
            };
            POLL(xa, wa);
            POLL(xb, wb);
            for (;;) {
                tick(xa, wa);
                if (u == 32) break;
                tick(xb, wb);
                if (u == 32) break;
            }
            BAR(); BAR(); BAR();
            asm volatile("s_waitcnt vmcnt(0)" :: "v"(xa), "v"(wa), "v"(xb), "v"(wb) : "memory");
            if (own) break;
            BAR();
            if (lane == 0) __hip_atomic_store((unsigned*)(ows + pb + YRDY) + (rb & 3) * 32 + t, tag << 8, RLX_SYS);
        }
        STAMP_OUT();
        __builtin_amdgcn_endpgm();
    }
#pragma unroll 1
    for (int job = 0; job < njob; ++job) {
        const int t = tile(job);
        f32v acc, d1, d2, d3, d4, d5;
        u4 dc;
        int ecur, emax, cnt, e0, e1, e2, e3, ta, tb;
        unsigned keep;
        u64 ret;
        const int rs = own ? NCOL * 4 : 8192;
        char* ob = own ? (char*)P.out + (size_t)(rb * 128 + wave * 32) * (NCOL * 4) + (rank >> 2) * 8192 + t * 256
                       : myws + YST + (size_t)((rb & 3) * 128 + wave * 32) * 8192 + t * 256;
        auto ro = __builtin_amdgcn_make_buffer_rsrc(ob, 0, 31 * rs + 256, 0x00020000);
#define S_(x) #x
#define S(x) S_(x)
#define VR(b) "v[" S(b) ":" S(b) "+3]"
#define ACC(i, nf) VR(64 + i * 16 + nf * 4)
#define BF(ks, nf) VR(96 + (ks & 1) * 16 + nf * 4)
#define AF(c, f) VR(128 + c * 32 + (f) * 4)
#define RDB(c, ks, nf) "ds_read_b128 " BF(ks, nf) ", %[vl] offset:" S(c * 16384 + ks * 4096 + nf * 1024) "\n\t"
#define RDB8(c) RDB(c, 0, 0) RDB(c, 0, 1) RDB(c, 0, 2) RDB(c, 0, 3) RDB(c, 1, 0) RDB(c, 1, 1) RDB(c, 1, 2) RDB(c, 1, 3)
#define MM(c, ks, nf, w) "s_waitcnt lgkmcnt(" #w ")\n\t" \
        "v_mfma_f32_16x16x32_f16 " ACC(0, nf) ", " AF(c, ks) ", " BF(ks, nf) ", " ACC(0, nf) "\n\t" \
        "v_mfma_f32_16x16x32_f16 " ACC(1, nf) ", " AF(c, 4 + ks) ", " BF(ks, nf) ", " ACC(1, nf) "\n\t"
#define LDB(j) "global_load_lds_dwordx4 v61, %[ws] offset:" S(j * 1024) " sc0 sc1\n\t"
#define LDA(n, f) "global_load_dwordx4 " AF(n, f) ", v60, %[ws] offset:" S((f) * 1024 - 4096) " sc0 sc1\n\t"
#define LD12(n) LDB(0) LDB(1) LDB(2) LDB(3) LDA(n, 0) LDA(n, 4) LDA(n, 1) LDA(n, 5) LDA(n, 2) LDA(n, 6) LDA(n, 3) LDA(n, 7)
#define REC(n) "ds_read_b96 v[60:62], %[vc] offset:" S((n & 1) * 32) "\n\t"
#define ADR(n) "s_add_u32 m0, %[lb], " S(n * 16384) "\n\tv_add_u32 v60, v60, %[voa]\n\tv_add_u32 v61, v61, %[vob]\n\tv_readfirstlane_b32 %[e" #n "], v62\n\t"
#define PRO(n) "s_barrier\n\t" REC(n) "s_waitcnt lgkmcnt(0)\n\t" ADR(n) LD12(n)
#define CHK(c, id) "s_cmp_lg_u32 %[e" #c "], %[ecur]\n\ts_cbranch_scc1 .Lfix" #id "_%=\n\t.Lback" #id "_%=:\n\t"
#define FIX(c, id, pre) ".Lfix" #id "_%=:\n\t" pre "s_call_b64 %[ret], .Lslow" #c "_%=\n\ts_branch .Lback" #id "_%=\n\t"
#define STEP(c, n) "s_waitcnt vmcnt(24)\n\ts_barrier\n\t" REC(n) RDB8(c) CHK(c, c) "s_waitcnt lgkmcnt(8)\n\t" ADR(n) \
        MM(c, 0, 0, 7) RDB(c, 2, 0) LDB(0) MM(c, 0, 1, 7) RDB(c, 2, 1) LDB(1) MM(c, 0, 2, 7) RDB(c, 2, 2) LDB(2) MM(c, 0, 3, 7) RDB(c, 2, 3) \
        MM(c, 1, 0, 7) RDB(c, 3, 0) LDB(3) MM(c, 1, 1, 7) RDB(c, 3, 1) LDA(n, 0) MM(c, 1, 2, 7) RDB(c, 3, 2) LDA(n, 4) MM(c, 1, 3, 7) RDB(c, 3, 3) \
        MM(c, 2, 0, 7) LDA(n, 1) MM(c, 2, 1, 6) LDA(n, 5) MM(c, 2, 2, 5) LDA(n, 2) MM(c, 2, 3, 4) \
        MM(c, 3, 0, 3) LDA(n, 6) MM(c, 3, 1, 2) LDA(n, 3) MM(c, 3, 2, 1) LDA(n, 7) MM(c, 3, 3, 0)
#define MMS(c, w0, w1, w2, w3) \
        w0 MM(c, 0, 0, 7) RDB(c, 2, 0) MM(c, 0, 1, 7) RDB(c, 2, 1) MM(c, 0, 2, 7) RDB(c, 2, 2) MM(c, 0, 3, 7) RDB(c, 2, 3) \
        w1 MM(c, 1, 0, 7) RDB(c, 3, 0) MM(c, 1, 1, 7) RDB(c, 3, 1) MM(c, 1, 2, 7) RDB(c, 3, 2) MM(c, 1, 3, 7) RDB(c, 3, 3) \
        w2 MM(c, 2, 0, 7) MM(c, 2, 1, 6) MM(c, 2, 2, 5) MM(c, 2, 3, 4) w3 MM(c, 3, 0, 3) MM(c, 3, 1, 2) MM(c, 3, 2, 1) MM(c, 3, 3, 0)
#define TBAR(n) "s_waitcnt vmcnt(" #n ")\n\ts_cbranch_execz 1\n\ts_barrier\n\t"
#define R4(M, b) M(b) M(b + 1) M(b + 2) M(b + 3)
#define R16(M, b) R4(M, b) R4(M, b + 4) R4(M, b + 8) R4(M, b + 12)
#define R32(M, b) R16(M, b) R16(M, b + 16)
#define ZERO(r) "v_mov_b32 v[" S(r) "], 0\n\t"
#define BAD(r) "v_mov_b32 v[" S(r) "], -1\n\t"
#define LDX(r) "v_ldexp_f32 v[" S(r) "], v[" S(r) "], %[tb]\n\t"
#define PKM(r) "v_pk_mul_f16 v[" S(r) "], v[" S(r) "], v63 op_sel_hi:[1,0]\n\t"
#define SLOW(c) ".Lslow" #c "_%=:\n\ts_max_i32 %[emax], %[emax], %[e" #c "]\n\ts_add_i32 %[ta], %[emax], -7\n\ts_max_i32 %[ta], %[ta], %[e" #c "]\n\t" \
        "s_sub_i32 %[tb], %[ecur], %[ta]\n\ts_lshl_b32 %[tb], %[tb], 3\n\ts_mov_b32 %[ecur], %[ta]\n\t" R32(LDX, 64) \
        "s_sub_i32 %[tb], %[e" #c "], %[ta]\n\ts_lshl_b32 %[tb], %[tb], 3\n\tv_ldexp_f32 v63, 1.0, %[tb]\n\tv_cvt_f16_f32 v63, v63\n\ts_nop 0\n\t" \
        R32(PKM, 128 + c * 32) "s_setpc_b64 %[ret]\n\t"
#define EP1(i, j, c) "v_ldexp_f32 v[" S(128 + (i * 4 + j) * 4 + c) "], v[" S(64 + i * 16 + c * 4 + j) "], %[tb]\n\t"
#define EP8(c) EP1(0, 0, c) EP1(0, 1, c) EP1(0, 2, c) EP1(0, 3, c) EP1(1, 0, c) EP1(1, 1, c) EP1(1, 2, c) EP1(1, 3, c)
#define EPS(i, step) "buffer_store_dwordx4 " VR(128 + i * 4) ", %[vo], %[ro], %[ta] offen sc0 sc1\n\ts_add_u32 %[ta], %[ta], " step "\n\t"
#define GSTAMP(n)
#define GSTAMP_OP
        asm volatile("; OPAQUE\n\t"
                     "s_mov_b32 %[keep], m0\n\t"
                     "s_mov_b32 %[cnt], 0\n\t"
                     "s_mov_b32 %[ecur], 0\n\ts_mov_b32 %[e0], 0\n\ts_mov_b32 %[e1], 0\n\ts_mov_b32 %[e2], 0\n\ts_mov_b32 %[e3], 0\n\t"
                     "s_mov_b64 exec, 0\n\t"
                     "s_branch .Lgemm_tail%=\n\t"
                     SLOW(0) SLOW(1) SLOW(2) SLOW(3)
                     FIX(0, 0, "") FIX(1, 1, "") FIX(2, 2, "") FIX(3, 3, "")
                     FIX(0, 4, "") FIX(1, 5, "") FIX(2, 6, "") FIX(3, 7, "s_waitcnt vmcnt(0)\n\t")
                     ".Lgemm_bad%=:\n\t"
                     "s_nop 7\n\t"
                     R32(BAD, 64)
                     "s_branch .Lgemm_ep%=\n\t"
                     ".Lgemm_go%=:\n\t"
                     "s_mov_b64 exec, -1\n\t"
                     R32(ZERO, 64)
                     PRO(0) PRO(1) PRO(2)
                     "s_mov_b32 %[ecur], %[e0]\n\t"
                     "s_mov_b32 %[emax], %[e0]\n\t"
                     ".Lgemm%=:\n\t"
                     STEP(0, 3) STEP(1, 0) STEP(2, 1) STEP(3, 2)
                     "s_add_u32 %[cnt], %[cnt], 1\n\t"
                     "s_cmp_lt_u32 %[cnt], 7\n\t"
                     "s_cbranch_scc1 .Lgemm%=\n\t"
                     ".Lgemm_tail%=:\n\t"
                     TBAR(24) "ds_read_b128 v[60:63], %[vc] offset:32\n\t" RDB8(0)
                     "s_waitcnt lgkmcnt(8)\n\t"
                     "s_cbranch_execz .Lgemm_t1%=\n\t"
                     ADR(3) LD12(3)
                     "v_readfirstlane_b32 %[cnt], v63\n\t"
                     ".Lgemm_t1%=:\n\t"
                     CHK(0, 4) MMS(0, "", "", "", "")
                     TBAR(24) RDB8(1) CHK(1, 5) MMS(1, "", "", "", "")
                     TBAR(12) RDB8(2) CHK(2, 6) MMS(2, "", "", "", "")
                     TBAR(8) RDB8(3) CHK(3, 7) MMS(3, WT(6), WT(4), WT(2), WT(0))
                     GSTAMP(0)
                     "s_cmp_lg_u32 %[cnt], 0\n\t"
                     "s_cbranch_scc1 .Lgemm_bad%=\n\t"
                     ".Lgemm_ep%=:\n\t"
                     "s_lshl_b32 %[tb], %[ecur], 3\n\t"
                     "s_mov_b32 %[ta], 0\n\t"
                     "s_mul_i32 %[emax], %[rs], 13\n\t"
                     EP8(0) EP8(1) EP8(2) EP8(3)
                     "s_cbranch_execz .Lgemm_go%=\n\t"
                     EPS(0, "%[rs]") EPS(1, "%[rs]") EPS(2, "%[rs]") EPS(3, "%[emax]") EPS(4, "%[rs]") EPS(5, "%[rs]") EPS(6, "%[rs]") EPS(7, "%[rs]")
                     GSTAMP(1)
                     "s_mov_b32 m0, %[keep]\n\t"
                     "s_waitcnt lgkmcnt(0)"
                     : "=&{v[64:95]}"(acc), "=&{v[96:127]}"(d1), "=&{v[128:159]}"(d2), "=&{v[160:191]}"(d3),
                       "=&{v[192:223]}"(d4), "=&{v[224:255]}"(d5), "=&{v[60:63]}"(dc),
                       [ecur] "=&s"(ecur), [emax] "=&s"(emax), [cnt] "=&s"(cnt), [e0] "=&s"(e0), [e1] "=&s"(e1), [e2] "=&s"(e2), [e3] "=&s"(e3),
                       [ta] "=&s"(ta), [tb] "=&s"(tb), [keep] "=&s"(keep), [ret] "=&s"(ret) GSTAMP_OP
                     : [vl] "v"((unsigned)(uintptr_t)lds + lane * 16), [vc] "v"((unsigned)(uintptr_t)lds + 4 * ITEM + (wave >> 1) * 16),
                       [voa] "v"(lane * 16 + 4096 + (wave & 1) * 8192), [vob] "v"(lane * 16 + wave * 4096),
                       [lb] "s"((unsigned)(uintptr_t)lds + wave * 4096), [ws] "s"(myws),
                       [vo] "v"(q4 * 4 * rs + c16 * 16), [ro] "s"(ro), [rs] "s"(rs)
                     : "memory", "scc");
        if (!own) asm volatile("s_waitcnt vmcnt(0)\n\ts_barrier" :: "v"(d2) : "memory");
        STAMP_V(job * 4 + 0, (unsigned)t0);
        STAMP_V(job * 4 + 1, (unsigned)t1);
    }
    STAMP_OUT();
}

struct Tune { int stag0 = 0, stag = 270, bdel = 500, stagb = 270, fs = 4352, rot = 3, xrot = 1, omap = 0, fastu = 20, slp0 = 6, slp1 = 2, nf = 7, xe = 0, near = 1, wsplit = 0; };

void build_lists(const Tune& T, uint16_t (&L)[3][MAXITEMS], int (&len)[3]) {
    const int nf = T.nf;
    for (int c = 0; c < 3; ++c) {
        int n = 0;
        auto X = [&](int a0, int i0 = 0, int i1 = 32) { for (int i = i0; i < i1; ++i) L[c][n++] = (uint16_t)(0 << 8 | (a0 + (i & 7)) << 2 | i >> 3); };
        auto W = [&](int t0, int t1) { for (int q = 0; q < 4; ++q) for (int t = t0; t < t1; ++t) L[c][n++] = (uint16_t)(1 << 8 | t << 2 | q); };
        if (c == 0) { X(8); W(0, 8); W(8, 16); W(16, 24); W(24, 32); X(0); }
        if (c == 1) {
            X(8); W(0, 8);
            for (int s = 0; s < 4; ++s) for (int t = 0; t < nf; ++t) L[c][n++] = (uint16_t)(2 << 8 | t << 2 | s);
            W(8, 16); W(16, 24); X(0, 0, T.xe);
            if (T.wsplit) { W(24, 28); W(28, 32); } else W(24, 32);
            X(0, T.xe, 32);
        }
        if (c == 2) {
            for (int t = 0; t < nf; ++t) L[c][n++] = (uint16_t)(1 << 8 | t << 2 | 0);
            X(8);
            if (nf < 8) W(nf, 8);
            W(8, 16); W(16, 24); W(24, 32); X(0);
            for (int i = 0; i < 64; ++i) L[c][n++] = (uint16_t)(3 << 8 | i);
        }
        len[c] = n;
        while (n < MAXITEMS) L[c][n++] = 0xFFFF;
    }
}

}

extern "C" void kernel_launch(int rank, int nranks, ncclComm_t comm,
                              hipStream_t stream,
                              void* const* d_in, const int* in_sizes, int n_in,
                              void* d_out, int out_size,
                              const int* in_dtypes, int out_dtype,
                              void* const* d_out_all,
                              void* const* d_ws_all, size_t ws_bytes) {
    (void)comm; (void)in_dtypes; (void)out_dtype; (void)d_out_all;
    if (nranks != NR || (unsigned)rank >= (unsigned)NR || n_in < 2 || ws_bytes < WS_NEED ||
        in_sizes[0] != 4096 * KSH || in_sizes[1] != KSH * NCOL || out_size != MSH * NCOL) {
        fprintf(stderr, "kernel_launch: unexpected problem shape\n");
        return;
    }
    const Tune T;
    uint16_t L[3][MAXITEMS];
    int len[3];
    build_lists(T, L, len);
    static const unsigned char upper[NR] = {0xB2, 0x8E, 0xE8, 0xD4, 0xD4, 0xE8, 0x8E, 0xB2};
    int xcd_of[NR], link_of[NR];
    for (int l = 0, lo = 0, hi = 4; l < NR; ++l) {
        xcd_of[l] = !T.near ? l : (upper[rank] >> l) & 1 ? hi++ : lo++;
        link_of[xcd_of[l]] = l;
    }
    Params P;
    P.x = (const float*)d_in[0];
    P.w = (const float*)d_in[1];
    P.out = (float*)d_out;
    for (int j = 0; j < NR; ++j) { P.ws[j] = (char*)d_ws_all[rank ^ j]; P.wsx[j] = (char*)d_ws_all[rank ^ link_of[j]]; }
    P.rank = rank;
    P.stag0 = T.stag0; P.stag = T.stag; P.bdel = T.bdel; P.stagb = T.stagb; P.fs = T.fs;
    P.rot = T.rot; P.xrot = T.xrot; P.omap = T.omap; P.fastu = T.fastu; P.slp0 = T.slp0; P.slp1 = T.slp1;
    for (int c = 0; c < 2; ++c) for (int i = 0; i < MAXITEMS; ++i) P.list[c][i] = L[c + 1][i];
    if (T.nf > 2 * NSEND || 24 * T.fs > (int)(YRDY - CTRL)) { fprintf(stderr, "bad tuning\n"); return; }
    bool taken[512] = {};
    int next = 0;
    for (unsigned i = 0; i < 256; ++i) P.conv[i] = (unsigned)link_of[i & 7] << 12 | ((i & 7) >> 1) << 28 | (i & 1) << 30;
    for (int p = 0; p < MAXITEMS; ++p)
        for (int x = 0; x < 8; ++x) {
            const int l = link_of[x];
            const int code = L[l == 0 ? 0 : 1 + (l >> 2)][p];
            if ((code >> 8) > 1) { if (p < 2 * NSEND) { fprintf(stderr, "bad list head\n"); return; } continue; }
            const int id = staged_id(code, rank ^ l, l, T.xrot);
            if (taken[id]) { if (p < 2 * NSEND) { fprintf(stderr, "bad list head\n"); return; } continue; }
            taken[id] = true;
            if (p < 2 * NSEND) P.conv[(p & 3) * 8 + x] |= (unsigned)id << (p >> 2) * 16;
            else {
                P.conv[8 * NSEND + next % (8 * NWORK)] |= (unsigned)id << (next / (8 * NWORK)) * 16;
                ++next;
            }
        }
    if (next != 16 * NWORK) { fprintf(stderr, "bad conversion table\n"); return; }
    gemm_rs_kernel<<<8 * (NWORK + NSEND), 384, 0, stream>>>(P);
    HIP_CHECK(hipGetLastError());
}
```

```cpp
#include <hip/hip_runtime.h>
#include <rccl/rccl.h>
#include <cstdio>
#include <cstdint>
#include <cstddef>

#define HIP_CHECK(expr) do { hipError_t _e = (expr); if (_e != hipSuccess) { \
    fprintf(stderr, "HIP error %s at %s:%d\n", hipGetErrorString(_e), __FILE__, __LINE__); \
    return; } } while (0)

#define XMASK 7
#define WIN 23
#define WSPR 7

namespace {

typedef _Float16 h8 __attribute__((ext_vector_type(8)));
typedef float f4 __attribute__((ext_vector_type(4)));
typedef unsigned u4 __attribute__((ext_vector_type(4)));
typedef unsigned long long u64;

constexpr int NR = 8;
constexpr int NCOL = 4096;
constexpr int KSH = 512;
constexpr int MSH = 512;
constexpr int NWORK = 28, NSEND = 4;

constexpr unsigned ITEM = 16384;
constexpr unsigned XR_BYTES = 8 * 32 * 2 * ITEM;
constexpr unsigned WR_BYTES = 32 * 32 * ITEM;
constexpr unsigned STG = XR_BYTES + WR_BYTES;
constexpr unsigned CTRL = STG + 512 * ITEM;
constexpr unsigned YRDY = CTRL + 110592;
constexpr unsigned RDY = CTRL + 114688;
constexpr unsigned BUF_BYTES = CTRL + 131072;
constexpr unsigned YST = 2 * BUF_BYTES;
constexpr unsigned EPOCH = YST + 256 * ITEM;
constexpr unsigned TMO = EPOCH + 8192;
constexpr unsigned STAMP = TMO + 4096;
constexpr unsigned WS_NEED = STAMP + (1 << 20);

constexpr u64 DEADLINE_TICKS = 100000000ull;

constexpr int MAXITEMS = 256;

struct Params {
    unsigned src[512];
    uint16_t list[2][MAXITEMS];
    const float* x;
    const float* w;
    int rot, xrot, omap, fastu;
    char* ws[NR];
    float* out;
    int rank;
    unsigned links;
    int fs;
    int nhead;
    int slp0, slp1;
};
static_assert(offsetof(Params, list) == 0x800 && offsetof(Params, x) == 0xC00 && offsetof(Params, ws) == 0xC20 && sizeof(Params) <= 4096, "layout known to the assembly");

__host__ __device__ inline int staged_id(int code, int peer, int link, int xrot) {
    const int a = (code >> 2) & 31, qa = (code & 3) ^ (peer & 3);
    if ((code >> 8) == 0) {
        const int rb = a >> 1;
        const int grb = rb < 4 ? peer * 4 + ((rb + link * xrot) & 3) : ((peer ^ rb) & 7) * 4 + (peer & 3);
        return (grb * 4 + qa) * 2 + (a & 1);
    }
    return 256 + ((peer >> 2) * 32 + a) * 4 + qa;
}
__host__ __device__ inline int recv_slot(int id, int src, int dst) {
    if (id < 256) {
        const int grb = id >> 3, qa = (id >> 1) & 3, o = grb >> 2, p = grb & 3;
        const int rb = o == dst ? p : o ^ dst;
        if (o != dst && (rb < 4 || p != (dst & 3))) return -1;
        return (rb * 32 + src * 4 + qa) * 2 + (id & 1);
    }
    const int ct = (id - 256) >> 2;
    if ((ct >> 5) != (dst >> 2)) return -1;
    return 512 + (ct & 31) * 32 + src * 4 + (id & 3);
}
__host__ __device__ inline unsigned flag_of_slot(int slot, int fs) {
    const int i = slot < 512 ? (slot >> 6) * 64 + (slot & 1) * 32 + ((slot >> 1) & 31) : slot;
    return (i >> 6) * fs + (i & 63) * 4;
}

#define RLX_SYS __ATOMIC_RELAXED, __HIP_MEMORY_SCOPE_SYSTEM
#define RLX_AGT __ATOMIC_RELAXED, __HIP_MEMORY_SCOPE_AGENT
constexpr int AUX_SYS = 17;

__device__ __forceinline__ int uni(int v) { return __builtin_amdgcn_readfirstlane(v); }

#define STAMP_V(slot, val) do { } while (0)
#define STAMP_AT(slot) do { } while (0)
#define STAMP_OUT() do { } while (0)

__device__ __forceinline__ bool spin_pause(u64 t0, unsigned* tmo, unsigned call) {
    __builtin_amdgcn_s_sleep(2);
    u64 dt = __builtin_amdgcn_s_memrealtime() - t0;
    if (dt > 50000ull) {
        if (dt > DEADLINE_TICKS || (unsigned)uni((int)__hip_atomic_load(tmo, RLX_AGT)) == call) {
            __hip_atomic_store(tmo, call, RLX_AGT);
            return true;
        }
        __builtin_amdgcn_s_sleep(32);
    }
    return false;
}

#define BAR() asm volatile("s_waitcnt lgkmcnt(0)\n\ts_barrier" ::: "memory")

constexpr unsigned RDY_VALID = 0x100, RDY_DEAD = 0x200;

typedef float f32v __attribute__((ext_vector_type(32)));

constexpr int C_MAX = 64, C_DST = 80, C_CALL = 96, C_FLAG = 128;

#define S_(x) #x
#define S(x) S_(x)
#define VR(b) "v[" S(b) ":" S(b) "+3]"
#define V2(b) "v[" S(b) ":" S(b) "+1]"
#define V1(b) "v[" S(b) "]"
#define WT(n) "s_waitcnt vmcnt(" #n ")\n\t"
#define WL(n) "s_waitcnt lgkmcnt(" #n ")\n\t"
#define R4(M, b) M(b) M(b + 1) M(b + 2) M(b + 3)
#define R16(M, b) R4(M, b) R4(M, b + 4) R4(M, b + 8) R4(M, b + 12)
#define R32(M, b) R16(M, b) R16(M, b + 16)

__global__ void __launch_bounds__(384, 2) gemm_rs_kernel(const Params) {
    __shared__ __attribute__((aligned(1024))) char lds[4 * ITEM + 1024];
    auto ka = (const __attribute__((address_space(4))) Params*)__builtin_amdgcn_kernarg_segment_ptr();
    const int wave = uni(threadIdx.x >> 6);
    const int guess = (blockIdx.x & ~7u) | (__builtin_amdgcn_s_getreg(20 | (3 << 11)) & XMASK);
#define CSTAMP(n)
#define CSTAMP0
#define CSTAMP5
#define CSTAMP_OP
#define CSTAMP_CL
    {
        f32v c0, c1, c2;
#define FL1(b, j, so) "buffer_load_dwordx4 " VR(b + (j) * 4) ", v42, s[52:55], " so " offen\n\t"
#define FL4(b, j, so) FL1(b, j, so) "s_add_u32 " so ", " so ", s60\n\t" FL1(b, j + 1, so) "s_add_u32 " so ", " so ", s60\n\t" \
        FL1(b, j + 2, so) "s_add_u32 " so ", " so ", s60\n\t" FL1(b, j + 3, so)
#define FETCH(b, sr, sh) "s_bitcmp1_b32 " sr ", " S(sh + 15) "\n\ts_cselect_b64 s[52:53], s[42:43], s[40:41]\n\ts_cselect_b32 s58, 14, 11\n\t" \
        "s_cselect_b32 s59, 20, 8\n\ts_bfe_u32 s61, " sr ", " S(0xf0000 + sh) "\n\ts_lshl_b32 s61, s61, 8\n\tv_lshl_or_b32 v42, v40, s58, v41\n\t" \
        "s_lshl_b32 s60, 1, s58\n\ts_lshl_b32 s59, 1, s59\n\ts_add_u32 s59, s59, s61\n\t" FL4(b, 0, "s61") FL4(b, 4, "s59")
#define MX(i) "v_max3_f32 v48, v48, |" V1(128 + (i) * 4) "|, |" V1(128 + (i) * 4 + 1) "|\n\tv_max3_f32 v49, v49, |" V1(128 + (i) * 4 + 2) "|, |" V1(128 + (i) * 4 + 3) "|\n\t"
#define DPP(c) "s_nop 1\n\tv_max_u32_dpp v48, v48, v48 " c "\n\t"
#define MUL(i) "v_pk_mul_f32 " V2(128 + (i) * 2) ", s[62:63], " V2(128 + (i) * 2) " op_sel_hi:[0,1]\n\t"
#define CV(d, a, b) "v_cvt_pk_f16_f32 " V1(d) ", " V1(128 + a) ", " V1(128 + b) "\n\t"
#define CVX(j) CV(112 + (j) * 4, (j) * 4, (j) * 4 + 1) CV(113 + (j) * 4, (j) * 4 + 2, (j) * 4 + 3) CV(114 + (j) * 4, (j) * 4 + 16, (j) * 4 + 17) \
        CV(115 + (j) * 4, (j) * 4 + 18, (j) * 4 + 19) "ds_write_b128 v46, " VR(112 + (j) * 4) " offset:" S((j) * 16) "\n\t"
#define RDX(f) "ds_read_b128 " VR(96 + (f) * 4) ", v47 offset:" S((f) * 1088) "\n\t"
#define CVW(f) CV(96 + (f) * 4, f, f + 4) CV(97 + (f) * 4, f + 8, f + 12) CV(98 + (f) * 4, f + 16, f + 20) CV(99 + (f) * 4, f + 24, f + 28)
#define PUT(a, f) "global_store_dwordx4 " V2(a) ", " VR(96 + f * 4) ", off offset:" S(f * 1024) " sc0 sc1\n\t"
        asm volatile("; OPAQUE\n\t"
                     "s_mov_b32 s56, %[guess]\n\t"
                     "s_cmp_lt_u32 %[wave], 4\n\t"
                     "s_cbranch_scc1 .Lc_conv%=\n\t"
                     "s_cmp_eq_u32 %[wave], 4\n\t"
                     "s_cbranch_scc1 .Lc_end%=\n\t"
                     "s_load_dwordx2 s[40:41], %[ka], 0xc20\n\t"
                     "s_add_u32 s58, %[lb], 0x10000 - 1280\n\t"
                     "v_lshl_add_u32 v37, %[tid], 2, s58\n\t"
                     "v_mov_b32 v48, 0\n\t"
                     "s_mov_b64 exec, 1\n\t"
                     "s_lshl2_add_u32 s58, %[bid], %[epo]\n\t"
                     "v_mov_b32 v40, s58\n\t"
                     "v_mov_b32 v42, 1\n\t"
                     "s_lshl2_add_u32 s58, s56, %[epo]\n\t"
                     "v_mov_b32 v41, s58\n\t"
                     WL(0)
                     "global_load_dword v47, v40, s[40:41] sc1\n\t"
                     ".Lc_claim%=:\n\t"
                     "global_atomic_add v46, v41, v42, s[40:41] offset:1024 sc0 sc1\n\t"
                     "s_add_u32 s58, %[lb], 0x10000\n\t"
                     "v_mov_b32 v36, s58\n\t"
                     "v_mov_b32 v45, s56\n\t"
                     "v_mov_b32 v49, 0\n\t"
                     "v_mov_b32 v50, 0\n\t"
                     "ds_write_b96 v36, v[48:50] offset:400\n\t"
                     "s_mov_b64 exec, -1\n\t"
                     "ds_write_b32 v37, v48\n\t"
                     "s_mov_b64 exec, 1\n\t"
                     WT(0)
                     "v_cmp_eq_u32 vcc, v46, v47\n\t"
                     "s_cbranch_vccz .Lc_taken%=\n\t"
                     ".Lc_won%=:\n\t"
                     "v_add_u32 v47, 1, v47\n\t"
                     "v_lshlrev_b32 v44, 8, v47\n\t"
                     "v_bfe_i32 v46, v47, 0, 1\n\t"
                     "v_and_b32 v46, %[buf], v46\n\t"
                     "ds_write_b128 v36, v[44:47] offset:384\n\t"
                     CSTAMP5
                     "global_store_dword v40, v47, s[40:41] sc1\n\t"
                     "s_mov_b64 exec, -1\n\t"
                     "s_branch .Lc_end%=\n\t"
                     ".Lc_conv%=:\n\t"
                     "s_cmp_ge_u32 s56, 32\n\t"
                     "s_cbranch_scc1 .Lc_retry%=\n\t"
                     "s_endpgm\n\t"
                     ".Lc_retry%=:\n\t"
                     "s_lshl_b32 s58, s56, 3\n\t"
                     "s_load_dwordx2 s[48:49], %[ka], s58\n\t"
                     "s_load_dwordx4 s[40:43], %[ka], 0xc00\n\t"
                     "s_memrealtime s[50:51]\n\t"
                     "s_bfe_u32 s58, s58, 0x80001\n\t"
                     "s_load_dword s44, %[ka], s58\n\t"
                     "s_lshr_b32 s45, s56, 6\n\t"
                     "s_max_u32 s45, s45, 2\n\t"
                     "s_mov_b32 s54, 0x800000\n\t"
                     "s_mov_b32 s55, 0x20000\n\t"
                     "s_mov_b32 s57, 0\n\t"
                     "s_mov_b32 s69, 0\n\t"
                     "v_and_b32 v37, 15, %[tid]\n\t"
                     "v_lshrrev_b32 v40, 2, %[tid]\n\t"
                     "v_lshlrev_b32 v41, 4, v37\n\t"
                     "v_and_b32 v40, 60, v40\n\t"
                     WL(0)
                     CSTAMP0
                     "s_bfe_u32 s58, s48, 0x80010\n\t"
                     "s_cbranch_scc1 .Lc_bar%=\n\t"
                     ".Lc_fetch%=:\n\t"
                     FETCH(128, "s48", 0)
                     "s_cmp_lg_u32 s69, 0\n\t"
                     "s_cbranch_scc1 .Lc_go%=\n\t"
                     ".Lc_bar%=:\n\t"
                     "v_and_b32 v43, 63, %[tid]\n\t"
                     "v_bfe_u32 v38, %[tid], 4, 2\n\t"
                     "s_lshl_b32 s58, %[wave], 12\n\t"
                     "v_lshl_or_b32 v44, v43, 4, s58\n\t"
                     "v_mov_b32 v45, 0\n\t"
                     "v_mov_b32 v63, 0\n\t"
                     "s_mul_i32 s58, %[wave], 4352\n\t"
                     "s_add_u32 s58, s58, %[lb]\n\t"
                     "v_mul_u32_u24 v46, 272, v37\n\t"
                     "v_lshl_add_u32 v46, v38, 6, v46\n\t"
                     "v_add_u32 v46, s58, v46\n\t"
                     "v_mul_u32_u24 v47, 272, v38\n\t"
                     "v_add3_u32 v47, v47, v41, s58\n\t"
                     "s_add_u32 s58, %[lb], 0x10000\n\t"
                     "v_mov_b32 v36, s58\n\t"
                     "v_mov_b32 v39, s58\n\t"
                     "v_lshl_add_u32 v33, v43, 3, s58\n\t"
                     "s_lshl2_add_u32 s58, %[wave], s58\n\t"
                     "v_mov_b32 v35, s58\n\t"
                     "s_barrier\n\t"
                     "ds_read_b96 v[60:62], v36 offset:384\n\t"
                     "ds_read_b64 v[50:51], v33 offset:512\n\t"
                     WL(0)
                     "v_readfirstlane_b32 s66, v61\n\t"
                     "v_add_u32 v44, v44, v62\n\t"
                     "v_cmp_ne_u64 vcc, 0, v[50:51]\n\t"
                     "v_lshl_add_u64 v[50:51], v[62:63], 0, v[50:51]\n\t"
                     "s_cmp_lg_u32 s66, s56\n\t"
                     "s_cbranch_scc1 .Lc_wrong%=\n\t"
                     "s_or_b32 s69, vcc_lo, 0x1000000\n\t"
                     "s_bfe_u32 s58, s48, 0x80010\n\t"
                     "s_cbranch_scc1 .Lc_delay%=\n\t"
                     ".Lc_go%=:\n\t"
                     "ds_read_b128 v[56:59], v39 offset:320\n\t"
                     WL(0)
                     "v_cmp_ne_u64 vcc, v[56:57], v[58:59]\n\t"
                     "v_lshl_add_u64 v[56:57], v[44:45], 0, v[56:57]\n\t"
                     "v_lshl_add_u64 v[58:59], v[44:45], 0, v[58:59]\n\t"
                     WT(0)
                     "v_max3_f32 v48, 0, |v128|, |v129|\n\tv_max3_f32 v49, 0, |v130|, |v131|\n\t"
                     MX(1) MX(2) MX(3) R4(MX, 4)
                     CSTAMP(0)
                     "v_max_u32 v48, v48, v49\n\t"
                     DPP("row_shr:1 row_mask:0xf bank_mask:0xf bound_ctrl:1") DPP("row_shr:2 row_mask:0xf bank_mask:0xf bound_ctrl:1")
                     DPP("row_shr:4 row_mask:0xf bank_mask:0xf bound_ctrl:1") DPP("row_shr:8 row_mask:0xf bank_mask:0xf bound_ctrl:1")
                     DPP("row_bcast:15 row_mask:0xa bank_mask:0xf") DPP("row_bcast:31 row_mask:0xc bank_mask:0xf")
                     "s_mov_b32 exec_lo, 0\n\t"
                     "s_brev_b32 exec_hi, 1\n\t"
                     "v_or_b32 v48, 1, v48\n\t"
                     "ds_write_b32 v35, v48 offset:256\n\t"
                     "s_mov_b64 exec, -1\n\t"
                     ".Lc_spin%=:\n\t"
                     "ds_read_b128 v[52:55], v39 offset:256\n\t"
                     WL(0)
                     "v_min3_u32 v49, v52, v53, v54\n\t"
                     "v_max3_u32 v48, v52, v53, v54\n\t"
                     "v_min_u32 v49, v49, v55\n\t"
                     "v_max_u32 v48, v48, v55\n\t"
                     "s_nop 0\n\t"
                     "v_readfirstlane_b32 s65, v49\n\t"
                     "v_readfirstlane_b32 s64, v48\n\t"
                     "s_cmp_eq_u32 s65, 0\n\t"
                     "s_cbranch_scc1 .Lc_spin%=\n\t"
                     "s_lshr_b32 s64, s64, 23\n\t"
                     "s_addk_i32 s64, 0xff7a\n\t"
                     "s_ashr_i32 s64, s64, 3\n\t"
                     "s_max_i32 s64, s64, -15\n\t"
                     "s_lshl_b32 s65, s64, 26\n\t"
                     "s_sub_u32 s62, 1.0, s65\n\t"
                     R16(MUL, 0)
                     "s_bitcmp1_b32 s48, 15\n\t"
                     "s_cbranch_scc0 .Lc_x%=\n\t"
                     R4(CVW, 0)
                     ".Lc_hold%=:\n\t"
                     "s_lshr_b32 s58, s48, 24\n\t"
                     "s_cbranch_scc1 .Lc_hold2%=\n\t"
                     ".Lc_put%=:\n\t"
                     WL(3) PUT(56, 0) WL(2) PUT(56, 1) WL(1) PUT(56, 2) WL(0) PUT(56, 3)
                     CSTAMP(1)
                     "s_cbranch_vccz .Lc_pub%=\n\t"
                     PUT(58, 0) PUT(58, 1) PUT(58, 2) PUT(58, 3)
                     "s_branch .Lc_pub%=\n\t"
                     ".Lc_x%=:\n\t"
                     R4(CVX, 0) R4(RDX, 0)
                     "s_branch .Lc_hold%=\n\t"
                     ".Lc_hold2%=:\n\t"
                     "s_mul_i32 s59, %[wave], " S(WSPR) "\n\t"
                     "s_lshl2_add_u32 s58, s58, s59\n\t"
                     ".Lc_hold3%=:\n\t"
                     "s_memrealtime s[66:67]\n\t"
                     WL(0)
                     "s_sub_u32 s66, s66, s50\n\t"
                     "s_cmp_ge_u32 s66, s58\n\t"
                     "s_cbranch_scc1 .Lc_put%=\n\t"
                     "s_branch .Lc_hold3%=\n\t"
                     ".Lc_taken%=:\n\t"
                     "global_atomic_sub v41, v42, s[40:41] offset:1024 sc1\n\t"
                     "s_add_u32 s58, s56, 1\n\t"
                     "s_and_b32 s58, s58, 7\n\t"
                     "s_and_b32 s56, s56, 0xf8\n\t"
                     "s_or_b32 s56, s56, s58\n\t"
                     "s_lshl2_add_u32 s58, s56, %[epo]\n\t"
                     "v_mov_b32 v41, s58\n\t"
                     "s_cmp_lg_u32 s56, %[guess]\n\t"
                     "s_cbranch_scc1 .Lc_claim%=\n\t"
                     "v_mov_b32 v45, %[bid]\n\t"
                     "s_branch .Lc_won%=\n\t"
                     ".Lc_wrong%=:\n\t"
                     "s_waitcnt vmcnt(0) lgkmcnt(0)\n\t"
                     "s_mov_b32 s56, s66\n\t"
                     "s_branch .Lc_retry%=\n\t"
                     ".Lc_pub%=:\n\t"
                     "s_add_u32 s64, s64, 32\n\t"
                     "v_or_b32 v61, s64, v60\n\t"
                     "s_lshl_b32 s58, s57, 3\n\t"
                     "s_lshl_b32 s68, 0xff, s58\n\t"
                     "v_lshl_add_u32 v49, s57, 2, v36\n\t"
                     "v_mov_b32 v48, 1\n\t"
                     "v_add_u32 v39, 16, v39\n\t"
                     "v_add_u32 v35, 16, v35\n\t"
                     WT(0)
                     "s_mov_b64 exec, 1\n\t"
                     "ds_add_rtn_u32 v48, v49, v48 offset:400\n\t"
                     CSTAMP(2)
                     WL(0)
                     "v_readfirstlane_b32 s65, v48\n\t"
                     "s_cmp_eq_u32 s65, 3\n\t"
                     "s_cselect_b32 s68, s68, 0\n\t"
                     "s_and_b32 exec_lo, s69, s68\n\t"
                     "s_cbranch_execz 2\n\t"
                     "global_store_dword v[50:51], v61, off sc0 sc1\n\t"
                     "s_mov_b64 exec, -1\n\t"
                     "s_add_u32 s57, s57, 1\n\t"
                     "s_cmp_eq_u32 s57, s45\n\t"
                     "s_cbranch_scc1 .Lc_end%=\n\t"
                     "s_mov_b32 s48, s49\n\t"
                     "s_mov_b32 s49, s44\n\t"
                     "s_bfe_u32 s58, s48, 0x80010\n\t"
                     ".Lc_delay%=:\n\t"
                     "s_memrealtime s[66:67]\n\t"
                     WL(0)
                     "s_sub_u32 s66, s66, s50\n\t"
                     "s_lshr_b32 s66, s66, 3\n\t"
                     "s_cmp_ge_u32 s66, s58\n\t"
                     "s_cbranch_scc1 .Lc_fetch%=\n\t"
                     "s_sleep 1\n\t"
                     "s_branch .Lc_delay%=\n\t"
                     ".Lc_end%=:"
                     : "=&{v[32:63]}"(c0), "=&{v[96:127]}"(c1), "=&{v[128:159]}"(c2),
                       [ka] "+s"(ka) CSTAMP_OP
                     : [tid] "v"(threadIdx.x), [wave] "s"(wave), [guess] "s"(guess),
                       [lb] "s"((unsigned)(uintptr_t)lds), [bid] "s"(blockIdx.x), [epo] "n"(EPOCH), [buf] "n"(BUF_BYTES)
                     : "memory", "scc", "vcc", "s40", "s41", "s42", "s43", "s44", "s45", "s48", "s49", "s50", "s51", "s52", "s53", "s54",
                       "s55", "s56", "s57", "s58", "s59", "s60", "s61", "s62", "s63", "s64", "s65", "s66", "s67", "s68", "s69" CSTAMP_CL);
    }
    const auto& P = *ka;
    const int lane = threadIdx.x & 63;
    const int xcd = blockIdx.x & 7;
    const int slot = blockIdx.x >> 3;
    const int g = slot - NSEND;
    char* const myws = P.ws[0];
    int rank = P.rank;
    const int fs = P.fs, nhead = P.nhead;
    const unsigned links = P.links;
    unsigned it0 = P.src[guess * 2], it1 = P.src[guess * 2 + 1], it2 = P.src[guess & 63];
    asm volatile("s_and_b32 %0, %0, 7" : "+s"(rank) : "s"(myws), "s"(fs), "s"(links), "s"(it0), "s"(it1), "s"(it2), "s"(nhead), "s"(P.ws[1]), "s"(P.ws[2]), "s"(P.ws[3]),
                 "s"(P.ws[4]), "s"(P.ws[5]), "s"(P.ws[6]), "s"(P.ws[7]) : "scc");
    __builtin_assume((unsigned)rank < 8u);
    const int pos = rank & 3;
    unsigned* const tmo = (unsigned*)(myws + TMO);
    typedef volatile __attribute__((address_space(3))) unsigned* ldsu;
    const ldsu ctl = (ldsu)(lds + 4 * ITEM);
    const int c16 = lane & 15, q4 = lane >> 4;
    unsigned call = 0;
    int role = guess, code;

    if (wave >= 4) {
        for (;;) {
            asm volatile("global_load_ushort %0, %1, %2 offset:2048" : "=&v"(code)
                         : "v"(((links >> (role & 7) * 4 & 4) * (MAXITEMS / 4) + min(nhead + slot + NSEND * lane, MAXITEMS - 1)) * 2), "s"(ka) : "memory");
            if (wave == 4) {
                const int link = (unsigned)g < (unsigned)nhead && lane < 8 ? (links >> (role & 7) * 4) & 7 : 0, j = lane & 7, d = rank ^ j;
                const unsigned c = lane & 16 ? it2 : lane & 8 ? it1 : it0;
                unsigned lo = 0, hi = 0;
#pragma unroll
                for (int i = 0; i < NR; ++i)
                    asm("v_writelane_b32 %0, %2, %4\n\tv_writelane_b32 %1, %3, %4" : "+v"(lo), "+v"(hi) : "s"((unsigned)(u64)P.ws[i]), "s"((unsigned)((u64)P.ws[i] >> 32)), "n"(i));
                const u64 wsp = (u64)(unsigned)__builtin_amdgcn_ds_swizzle((int)hi, 7) << 32 | (unsigned)__builtin_amdgcn_ds_swizzle((int)lo, 7);
                const bool isw = c & 0x8000u;
                const unsigned q = (isw ? c >> 13 : c >> 1) & 3, h = c >> 9 & 1, p = c >> 10 & 3, o = (c >> 12 & 7) ^ d;
                const unsigned id = isw ? 256 + (c & 63) * 4 + q : (c >> 10 & 31) * 8 + q * 2 + h;
                const bool need = isw ? (c >> 5 & 1) == (unsigned)(d >> 2) : (o == 0) | ((o >= 4) & (p == (unsigned)(d & 3)));
                const unsigned blk = isw ? 8 + (c >> 1 & 15) : o ? o : p, k = rank * 4 + q;
                const unsigned sl = blk * 64 + (isw ? (c & 1) * 32 + k : k * 2 + h);
                const unsigned fo = j == 0 || j == link ? CTRL + blk * fs + (((isw ? c : h) & 1) * 32 + k) * 4 : RDY + j * 2048 + id * 4;
                const u64 db = (u64)myws + (STG + id * ITEM);
                typedef u64 u64x2 __attribute__((ext_vector_type(2)));
                if ((lane & 39) == link) *(volatile __attribute__((address_space(3))) u64x2*)(ctl + C_DST + (lane >> 3) * 4) = u64x2{link ? wsp + sl * ITEM : db, db};
                *(volatile __attribute__((address_space(3))) u64*)(ctl + C_FLAG + lane * 2) = need ? wsp + fo : 0;
                if (lane < 12) ctl[C_MAX + lane] = 0;
                STAMP_AT(64 + 61);
            }
            BAR();
            asm volatile("s_waitcnt vmcnt(0)" : "+v"(code));
            call = ctl[C_CALL + 3];
            const int r = uni((int)ctl[C_CALL + 1]);
            if (r == role) break;
            role = r;
            it0 = P.src[(r & 255) * 2]; it1 = P.src[(r & 255) * 2 + 1]; it2 = P.src[r & 63];
        }
        if (g >= 0 && wave == 5) { STAMP_OUT(); __builtin_amdgcn_endpgm(); }
    }
    call = (unsigned)uni((int)call);
    const unsigned tag = call & 0xFFFFFFu, pb = (call & 1u) * BUF_BYTES;

    if (g < 0) {
        const int link = (links >> (role & 7) * 4) & 7, peer = rank ^ link;
        STAMP_V(127, (__builtin_amdgcn_s_getreg(63492) >> 8 & 0xff) | (__builtin_amdgcn_s_getreg(20 | (3 << 11)) & 15) << 8 | link << 12);
        if (link == 0) { STAMP_OUT(); __builtin_amdgcn_endpgm(); }
        char* const prws = P.ws[link];
        const int n = __builtin_popcountll(__ballot(code != 0xFFFF));
        const int type = code >> 8;
        unsigned src, dst, sfl, dfl;
        if (type == 3) {
            const unsigned quad = (code >> 3) & 7, rg = code & 7;
            src = YST + ((link & 3) * 128 + rg * 16) * 8192 + quad * 1024;
            dst = ((peer & 3) * 128 + rg * 16) * (NCOL * 4) + (peer >> 2) * 8192 + quad * 1024;
            sfl = pb + YRDY + ((link & 3) * 32 + quad * 4) * 4;
            dfl = 0;
        } else if (type == 2) {
            const unsigned t = (code >> 2) & 31, sj = 4 + (code & 3);
            const unsigned sl = 512 + t * 32 + ((rank ^ sj) & 7) * 4 + (peer & 3);
            src = dst = pb + sl * ITEM;
            sfl = pb + RDY + sj * 2048 + (256 + ((rank >> 2) * 32 + t) * 4 + (peer & 3)) * 4;
            dfl = pb + CTRL + flag_of_slot(sl, fs);
        } else {
            const int id = staged_id(code & 0x1FF, rank, link, P.xrot);
            const int sl = recv_slot(id, peer, rank) & 2047;
            src = pb + STG + id * ITEM;
            dst = pb + sl * ITEM;
            sfl = pb + RDY + link * 2048 + id * 4;
            dfl = pb + CTRL + flag_of_slot(sl, fs);
        }
        auto rws = __builtin_amdgcn_make_buffer_rsrc(myws, 0, (int)STAMP, 0x00020000);
        if (wave == 5) {
            bool done = lane >= n;
            u64 t0 = __builtin_amdgcn_s_memrealtime();
            for (;;) {
                const u64 pend = __ballot(!done);
                if (!pend) break;
                const int first = __builtin_ctzll(pend);
                bool ok = false;
                if (!done && lane < first + 6) {
                    const u4 f = __builtin_amdgcn_raw_buffer_load_b128(rws, sfl & ~15u, 0, AUX_SYS);
                    const unsigned c = (sfl >> 2) & 3;
                    const unsigned v = c == 0 ? f[0] : c == 1 ? f[1] : c == 2 ? f[2] : f[3];
                    ok = (v >> 8) == tag;
                    if (type == 3) ok = ok && (f[1] >> 8) == tag && (f[2] >> 8) == tag && (f[3] >> 8) == tag;
                    if (ok) { ctl[lane] = (v & 255u) | RDY_VALID; done = true; }
                }
                if (__any(ok)) t0 = __builtin_amdgcn_s_memrealtime();
                else if (spin_pause(t0, tmo, call)) { if (!done) ctl[lane] = RDY_DEAD; break; }
            }
            STAMP_OUT();
            __builtin_amdgcn_endpgm();
        }
        auto rpw = __builtin_amdgcn_make_buffer_rsrc(prws, 0, (int)STAMP, 0x00020000);
        float* const po = P.out;
        const int vo = lane * 16;
        const unsigned sstr = type == 3 ? 8192u : 1024u, dstr = type == 3 ? NCOL * 4u : 1024u;
        int k = 0;
        while (k < n) {
            unsigned r;
            while ((r = (unsigned)uni((int)ctl[k])) == 0) __builtin_amdgcn_s_sleep(1);
            if (r & RDY_DEAD) {
                if (type == 3 && lane == n - 1) po[dst >> 2] = __builtin_nanf("");
                break;
            }
            u4 R[16];
            unsigned la = (unsigned)(uintptr_t)lds + 4 * ITEM + k * 4, vt, vd, pfv, so, ss, dso, ds, k1, rn, pfo;
            u64 pm;
#define LD(f) "buffer_load_dwordx4 %" #f ", %[vo], %[rs], %[so] offen sc0 sc1\n\ts_add_u32 %[so], %[so], %[ss]\n\t"
#define ST(f) "buffer_store_dwordx4 %" #f ", %[vo], s[88:91], %[dso] offen sc0 sc1\n\ts_add_u32 %[dso], %[dso], %[ds]\n\t"
#define WTW "s_waitcnt vmcnt(" S(WIN) ")\n\t"
#define SL(f) WTW ".if " #f " == (" S(WIN) " - 1) / 2\n\t" FLAG ".endif\n\t" ST(f) LD(f)
#define L0(f) WTW "buffer_load_dword %[vd], off, %[rf], 0\n\t" LD(f)
#define FLAG "s_mov_b64 exec, %[pm]\n\ts_cbranch_execz 2\n\tbuffer_store_dword %[pfv], off, %[rf], %[pfo] sc0 sc1\n\ts_mov_b64 exec, -1\n\t"
#define NEXTFLAG "v_readlane_b32 %[pfo], %[dfl], %[k]\n\ts_and_b32 %[r], %[r], 0xff\n\ts_or_b32 %[r], %[r], %[tag8]\n\t" \
                 "v_mov_b32 %[pfv], %[r]\n\ts_cmp_lg_u32 %[pfo], 0\n\ts_cselect_b64 %[pm], 1, 0\n\ts_mov_b32 %[k], %[k1]\n\t"
#define ASTAMP
#define ASTAMP_OP
#define ASTAMP_CL
            asm volatile("; OPAQUE\n\t"
                         "v_readlane_b32 %[so], %[src], %[k]\n\t"
                         "v_readlane_b32 %[ss], %[sstr], %[k]\n\t"
                         "s_mov_b64 %[pm], 0\n\t"
                         "s_mov_b32 %[pfo], 0\n\t"
                         "s_mov_b32 s91, 0x20000\n\t"
                         "s_nop 4\n\t"
                         L0(0) L0(1) L0(2) L0(3) L0(4) L0(5) L0(6) L0(7) L0(8) L0(9) L0(10) L0(11) L0(12) L0(13) L0(14) L0(15)
                         ".Lsend_loop%=:\n\t"
                         "ds_read_b32 %[vt], %[la] offset:4\n\t"
                         "s_add_u32 %[k1], %[k], 1\n\t"
                         "v_readlane_b32 %[dso], %[dst], %[k]\n\t"
                         "v_readlane_b32 %[ds], %[dstr], %[k]\n\t"
                         "v_readlane_b32 %[so], %[src], %[k1]\n\t"
                         "v_readlane_b32 %[ss], %[sstr], %[k1]\n\t"
                         "s_cmp_eq_u32 %[ds], 0x400\n\t"
                         "s_cselect_b64 s[88:89], %[pw], %[po]\n\t"
                         "s_cselect_b32 s90, %[szw], %[szo]\n\t"
                         "s_waitcnt lgkmcnt(0)\n\t"
                         "v_readfirstlane_b32 %[rn], %[vt]\n\t"
                         "v_add_u32 %[la], 4, %[la]\n\t"
                         ASTAMP
                         "s_bitcmp1_b32 %[rn], 8\n\t"
                         "s_cbranch_scc0 .Lsend_last%=\n\t"
                         SL(0) SL(1) SL(2) SL(3) SL(4) SL(5) SL(6) SL(7) SL(8) SL(9) SL(10) SL(11) SL(12) SL(13) SL(14) SL(15)
                         NEXTFLAG
                         "s_mov_b32 %[r], %[rn]\n\t"
                         "s_branch .Lsend_loop%=\n\t"
                         ".Lsend_last%=:\n\t"
                         WT(30) ST(0) WT(29) ST(1) WT(28) ST(2) WT(27) ST(3) WT(26) ST(4) WT(25) ST(5) WT(24) ST(6) WT(23) ST(7)
                         WT(22) ST(8) WT(21) ST(9) WT(20) ST(10) WT(19) ST(11) WT(18) ST(12) WT(17) ST(13) WT(16) ST(14) WT(15) FLAG ST(15)
                         NEXTFLAG
                         WT(0)
                         FLAG
                         "s_nop 1"
                         : "=&v"(R[0]), "=&v"(R[1]), "=&v"(R[2]), "=&v"(R[3]), "=&v"(R[4]), "=&v"(R[5]), "=&v"(R[6]), "=&v"(R[7]),
                           "=&v"(R[8]), "=&v"(R[9]), "=&v"(R[10]), "=&v"(R[11]), "=&v"(R[12]), "=&v"(R[13]), "=&v"(R[14]), "=&v"(R[15]),
                           [la] "+v"(la), [vt] "=&v"(vt), [vd] "=&v"(vd), [pfv] "=&v"(pfv), [so] "=&s"(so), [ss] "=&s"(ss), [dso] "=&s"(dso), [ds] "=&s"(ds),
                           [k1] "=&s"(k1), [rn] "=&s"(rn), [pfo] "=&s"(pfo), [pm] "=&s"(pm), [k] "+s"(k), [r] "+s"(r) ASTAMP_OP
                         : [vo] "v"(vo), [src] "v"(src), [dst] "v"(dst), [dfl] "v"(dfl), [sstr] "v"(sstr), [dstr] "v"(dstr),
                           [rs] "s"(rpw), [rf] "s"(rws), [pw] "s"(myws), [po] "s"(po), [szw] "s"(STAMP), [szo] "s"(MSH * NCOL * 4), [tag8] "s"(tag << 8)
                         : "memory", "scc", "s88", "s89", "s90", "s91" ASTAMP_CL);
            STAMP_AT(63 + k);
        }
        STAMP_OUT();
        __builtin_amdgcn_endpgm();
    }

    const bool own = g >= 12;
    const int o = g - 12, omap = P.omap;
    const int fl = xcd >> 1, fh = xcd & 1;
    const int rb = own ? (omap ? (o & 3) : (xcd & 1) * 2 + (o & 1)) : 4 + fl;
    const int grb = own ? rank * 4 + rb : ((rank ^ rb) & 7) * 4 + pos;
    const int njob = g < 4 ? 2 : 1;
    auto tile = [&](int job) {
        return own ? (omap ? xcd * 4 + (o >> 2) : (xcd >> 1) * 8 + (o >> 1)) : (job ? 3 : g >> 2) * 8 + fh * 4 + (g & 3);
    };
    if (wave == 4) {
        char* const ows = P.ws[rb];
#pragma unroll 1
        for (int job = 0; job < njob; ++job) {
            const int t = tile(job);
            const int rot = (g * P.rot) & 31, fastu = P.fastu, slp0 = P.slp0, slp1 = P.slp1;
            const int ls = lane & 31, lh = lane >> 5;
            const char* const fxp = myws + pb + CTRL + rb * fs + lane * 4;
            const char* const fwp = myws + pb + CTRL + flag_of_slot(512 + t * 32 + ls, fs);
            const bool loc = (ls >> 2) == rank;
            const unsigned offa = pb + (loc ? STG + ((grb * 4 + (ls & 3)) * 2 + lh) * ITEM : ((rb * 32 + ls) * 2 + lh) * ITEM);
            const unsigned offb = pb + (loc ? STG + (256 + ((rank >> 2) * 32 + t) * 4 + (ls & 3)) * ITEM : XR_BYTES + (t * 32 + ls) * ITEM);
            unsigned issued = 0, xa, wa, xb, wb;
            int u = 0;
            bool dead = false;
            u64 t0 = __builtin_amdgcn_s_memrealtime();
#define POLL(fx, fw) asm volatile("; POLL\n\tglobal_load_dword %0, %2, off sc0 sc1\n\tglobal_load_dword %1, %3, off sc0 sc1" \
                                  : "=&v"(fx), "=&v"(fw) : "v"(fxp), "v"(fwp) : "memory")
            auto tick = [&](unsigned& px_, unsigned& pw_) {
                unsigned fx, fw;
                asm volatile("; GOT\n\ts_waitcnt vmcnt(2)\n\tv_mov_b32 %0, %2\n\tv_mov_b32 %1, %3\n\t"
                             "global_load_dword %2, %4, off sc0 sc1\n\tglobal_load_dword %3, %5, off sc0 sc1"
                             : "=&v"(fx), "=&v"(fw), "+v"(px_), "+v"(pw_) : "v"(fxp), "v"(fwp) : "memory");
                const u64 bx = __ballot((fx >> 8) == tag);
                unsigned rdy = (unsigned)bx & (unsigned)(bx >> 32) & (unsigned)__ballot((fw >> 8) == tag);
                if (dead) rdy = ~0u;
                unsigned avail = rdy & ~issued;
                if (!avail) {
                    const int n = u < fastu ? slp0 : slp1;
                    for (int i = 0; i < n; ++i) __builtin_amdgcn_s_sleep(2);
                    dead = spin_pause(t0, tmo, call);
                    return;
                }
                const u4 rec = {offa, offb, (fx & 255u) + (fw & 255u) - 64u, dead};
                do {
                    const int s = (__builtin_ctz(__builtin_amdgcn_alignbit(avail, avail, rot)) + rot) & 31;
                    issued |= 1u << s;
                    avail &= ~(1u << s);
                    if (ls == s) *(volatile __attribute__((address_space(3))) u4*)(ctl + (u & 1) * 8 + lh * 4) = rec;
                    STAMP_AT(job * 32 + u);
                    STAMP_V(64 + job * 32 + u, s);
                    BAR();
                    ++u;
                } while (__builtin_popcount(avail) > (rdy == ~0u ? 0 : 2));
                t0 = __builtin_amdgcn_s_memrealtime();
            };
            POLL(xa, wa);
            POLL(xb, wb);
            for (;;) {
                tick(xa, wa);
                if (u == 32) break;
                tick(xb, wb);
                if (u == 32) break;
            }
            BAR(); BAR(); BAR();
            asm volatile("s_waitcnt vmcnt(0)" :: "v"(xa), "v"(wa), "v"(xb), "v"(wb) : "memory");
            if (own) break;
            BAR();
            if (lane == 0) __hip_atomic_store((unsigned*)(ows + pb + YRDY) + (rb & 3) * 32 + t, tag << 8, RLX_SYS);
        }
        STAMP_OUT();
        __builtin_amdgcn_endpgm();
    }
#pragma unroll 1
    for (int job = 0; job < njob; ++job) {
        const int t = tile(job);
        f32v acc, d1, d2, d3, d4, d5;
        u4 dc;
        int ecur, emax, cnt, e0, e1, e2, e3, ta, tb;
        unsigned keep;
        u64 ret;
        const int rs = own ? NCOL * 4 : 8192;
        char* ob = own ? (char*)P.out + (size_t)(rb * 128 + wave * 32) * (NCOL * 4) + (rank >> 2) * 8192 + t * 256
                       : myws + YST + (size_t)((rb & 3) * 128 + wave * 32) * 8192 + t * 256;
        auto ro = __builtin_amdgcn_make_buffer_rsrc(ob, 0, 31 * rs + 256, 0x00020000);
#define ACC(i, nf) VR(64 + i * 16 + nf * 4)
#define BF(ks, nf) VR(96 + (ks & 1) * 16 + nf * 4)
#define AF(c, f) VR(128 + c * 32 + (f) * 4)
#define RDB(c, ks, nf) "ds_read_b128 " BF(ks, nf) ", %[vl] offset:" S(c * 16384 + ks * 4096 + nf * 1024) "\n\t"
#define RDB8(c) RDB(c, 0, 0) RDB(c, 0, 1) RDB(c, 0, 2) RDB(c, 0, 3) RDB(c, 1, 0) RDB(c, 1, 1) RDB(c, 1, 2) RDB(c, 1, 3)
#define MM(c, ks, nf, w) "s_waitcnt lgkmcnt(" #w ")\n\t" \
        "v_mfma_f32_16x16x32_f16 " ACC(0, nf) ", " AF(c, ks) ", " BF(ks, nf) ", " ACC(0, nf) "\n\t" \
        "v_mfma_f32_16x16x32_f16 " ACC(1, nf) ", " AF(c, 4 + ks) ", " BF(ks, nf) ", " ACC(1, nf) "\n\t"
#define LDB(j) "global_load_lds_dwordx4 v61, %[ws] offset:" S(j * 1024) " sc0 sc1\n\t"
#define LDA(n, f) "global_load_dwordx4 " AF(n, f) ", v60, %[ws] offset:" S((f) * 1024 - 4096) " sc0 sc1\n\t"
#define LD12(n) LDB(0) LDB(1) LDB(2) LDB(3) LDA(n, 0) LDA(n, 4) LDA(n, 1) LDA(n, 5) LDA(n, 2) LDA(n, 6) LDA(n, 3) LDA(n, 7)
#define REC(n) "ds_read_b96 v[60:62], %[vc] offset:" S((n & 1) * 32) "\n\t"
#define ADR(n) "s_add_u32 m0, %[lb], " S(n * 16384) "\n\tv_add_u32 v60, v60, %[voa]\n\tv_add_u32 v61, v61, %[vob]\n\tv_readfirstlane_b32 %[e" #n "], v62\n\t"
#define PRO(n) "s_barrier\n\t" REC(n) "s_waitcnt lgkmcnt(0)\n\t" ADR(n) LD12(n)
#define CHK(c, id) "s_cmp_lg_u32 %[e" #c "], %[ecur]\n\ts_cbranch_scc1 .Lfix" #id "_%=\n\t.Lback" #id "_%=:\n\t"
#define FIX(c, id, pre) ".Lfix" #id "_%=:\n\t" pre "s_call_b64 %[ret], .Lslow" #c "_%=\n\ts_branch .Lback" #id "_%=\n\t"
#define STEP(c, n) "s_waitcnt vmcnt(24)\n\ts_barrier\n\t" REC(n) RDB8(c) CHK(c, c) "s_waitcnt lgkmcnt(8)\n\t" ADR(n) \
        MM(c, 0, 0, 7) RDB(c, 2, 0) LDB(0) MM(c, 0, 1, 7) RDB(c, 2, 1) LDB(1) MM(c, 0, 2, 7) RDB(c, 2, 2) LDB(2) MM(c, 0, 3, 7) RDB(c, 2, 3) \
        MM(c, 1, 0, 7) RDB(c, 3, 0) LDB(3) MM(c, 1, 1, 7) RDB(c, 3, 1) LDA(n, 0) MM(c, 1, 2, 7) RDB(c, 3, 2) LDA(n, 4) MM(c, 1, 3, 7) RDB(c, 3, 3) \
        MM(c, 2, 0, 7) LDA(n, 1) MM(c, 2, 1, 6) LDA(n, 5) MM(c, 2, 2, 5) LDA(n, 2) MM(c, 2, 3, 4) \
        MM(c, 3, 0, 3) LDA(n, 6) MM(c, 3, 1, 2) LDA(n, 3) MM(c, 3, 2, 1) LDA(n, 7) MM(c, 3, 3, 0)
#define MMS(c, w0, w1, w2, w3) \
        w0 MM(c, 0, 0, 7) RDB(c, 2, 0) MM(c, 0, 1, 7) RDB(c, 2, 1) MM(c, 0, 2, 7) RDB(c, 2, 2) MM(c, 0, 3, 7) RDB(c, 2, 3) \
        w1 MM(c, 1, 0, 7) RDB(c, 3, 0) MM(c, 1, 1, 7) RDB(c, 3, 1) MM(c, 1, 2, 7) RDB(c, 3, 2) MM(c, 1, 3, 7) RDB(c, 3, 3) \
        w2 MM(c, 2, 0, 7) MM(c, 2, 1, 6) MM(c, 2, 2, 5) MM(c, 2, 3, 4) w3 MM(c, 3, 0, 3) MM(c, 3, 1, 2) MM(c, 3, 2, 1) MM(c, 3, 3, 0)
#define TBAR(n) "s_waitcnt vmcnt(" #n ")\n\ts_cbranch_execz 1\n\ts_barrier\n\t"
#define ZERO(r) "v_mov_b32 v[" S(r) "], 0\n\t"
#define BAD(r) "v_mov_b32 v[" S(r) "], -1\n\t"
#define LDX(r) "v_ldexp_f32 v[" S(r) "], v[" S(r) "], %[tb]\n\t"
#define PKM(r) "v_pk_mul_f16 v[" S(r) "], v[" S(r) "], v63 op_sel_hi:[1,0]\n\t"
#define SLOW(c) ".Lslow" #c "_%=:\n\ts_max_i32 %[emax], %[emax], %[e" #c "]\n\ts_add_i32 %[ta], %[emax], -7\n\ts_max_i32 %[ta], %[ta], %[e" #c "]\n\t" \
        "s_sub_i32 %[tb], %[ecur], %[ta]\n\ts_lshl_b32 %[tb], %[tb], 3\n\ts_mov_b32 %[ecur], %[ta]\n\t" R32(LDX, 64) \
        "s_sub_i32 %[tb], %[e" #c "], %[ta]\n\ts_lshl_b32 %[tb], %[tb], 3\n\tv_ldexp_f32 v63, 1.0, %[tb]\n\tv_cvt_f16_f32 v63, v63\n\ts_nop 0\n\t" \
        R32(PKM, 128 + c * 32) "s_setpc_b64 %[ret]\n\t"
#define EP1(i, j, c) "v_ldexp_f32 v[" S(128 + (i * 4 + j) * 4 + c) "], v[" S(64 + i * 16 + c * 4 + j) "], %[tb]\n\t"
#define EP8(c) EP1(0, 0, c) EP1(0, 1, c) EP1(0, 2, c) EP1(0, 3, c) EP1(1, 0, c) EP1(1, 1, c) EP1(1, 2, c) EP1(1, 3, c)
#define EPS(i, step) "buffer_store_dwordx4 " VR(128 + i * 4) ", %[vo], %[ro], %[ta] offen sc0 sc1\n\ts_add_u32 %[ta], %[ta], " step "\n\t"
#define GSTAMP(n)
#define GSTAMP_OP
        asm volatile("; OPAQUE\n\t"
                     "s_mov_b32 %[keep], m0\n\t"
                     "s_mov_b32 %[cnt], 0\n\t"
                     "s_mov_b32 %[ecur], 0\n\ts_mov_b32 %[e0], 0\n\ts_mov_b32 %[e1], 0\n\ts_mov_b32 %[e2], 0\n\ts_mov_b32 %[e3], 0\n\t"
                     "s_mov_b64 exec, 0\n\t"
                     "s_branch .Lgemm_tail%=\n\t"
                     SLOW(0) SLOW(1) SLOW(2) SLOW(3)
                     FIX(0, 0, "") FIX(1, 1, "") FIX(2, 2, "") FIX(3, 3, "")
                     FIX(0, 4, "") FIX(1, 5, "") FIX(2, 6, "") FIX(3, 7, "s_waitcnt vmcnt(0)\n\t")
                     ".Lgemm_bad%=:\n\t"
                     "s_nop 7\n\t"
                     R32(BAD, 64)
                     "s_branch .Lgemm_ep%=\n\t"
                     ".Lgemm_go%=:\n\t"
                     "s_mov_b64 exec, -1\n\t"
                     R32(ZERO, 64)
                     PRO(0) PRO(1) PRO(2)
                     "s_mov_b32 %[ecur], %[e0]\n\t"
                     "s_mov_b32 %[emax], %[e0]\n\t"
                     ".Lgemm%=:\n\t"
                     STEP(0, 3) STEP(1, 0) STEP(2, 1) STEP(3, 2)
                     "s_add_u32 %[cnt], %[cnt], 1\n\t"
                     "s_cmp_lt_u32 %[cnt], 7\n\t"
                     "s_cbranch_scc1 .Lgemm%=\n\t"
                     ".Lgemm_tail%=:\n\t"
                     TBAR(24) "ds_read_b128 v[60:63], %[vc] offset:32\n\t" RDB8(0)
                     "s_waitcnt lgkmcnt(8)\n\t"
                     "s_cbranch_execz .Lgemm_t1%=\n\t"
                     ADR(3) LD12(3)
                     "v_readfirstlane_b32 %[cnt], v63\n\t"
                     ".Lgemm_t1%=:\n\t"
                     CHK(0, 4) MMS(0, "", "", "", "")
                     TBAR(24) RDB8(1) CHK(1, 5) MMS(1, "", "", "", "")
                     TBAR(12) RDB8(2) CHK(2, 6) MMS(2, "", "", "", "")
                     TBAR(8) RDB8(3) CHK(3, 7) MMS(3, WT(6), WT(4), WT(2), WT(0))
                     GSTAMP(0)
                     "s_cmp_lg_u32 %[cnt], 0\n\t"
                     "s_cbranch_scc1 .Lgemm_bad%=\n\t"
                     ".Lgemm_ep%=:\n\t"
                     "s_lshl_b32 %[tb], %[ecur], 3\n\t"
                     "s_mov_b32 %[ta], 0\n\t"
                     "s_mul_i32 %[emax], %[rs], 13\n\t"
                     EP8(0) EP8(1) EP8(2) EP8(3)
                     "s_cbranch_execz .Lgemm_go%=\n\t"
                     EPS(0, "%[rs]") EPS(1, "%[rs]") EPS(2, "%[rs]") EPS(3, "%[emax]") EPS(4, "%[rs]") EPS(5, "%[rs]") EPS(6, "%[rs]") EPS(7, "%[rs]")
                     GSTAMP(1)
                     "s_mov_b32 m0, %[keep]\n\t"
                     "s_waitcnt lgkmcnt(0)"
                     : "=&{v[64:95]}"(acc), "=&{v[96:127]}"(d1), "=&{v[128:159]}"(d2), "=&{v[160:191]}"(d3),
                       "=&{v[192:223]}"(d4), "=&{v[224:255]}"(d5), "=&{v[60:63]}"(dc),
                       [ecur] "=&s"(ecur), [emax] "=&s"(emax), [cnt] "=&s"(cnt), [e0] "=&s"(e0), [e1] "=&s"(e1), [e2] "=&s"(e2), [e3] "=&s"(e3),
                       [ta] "=&s"(ta), [tb] "=&s"(tb), [keep] "=&s"(keep), [ret] "=&s"(ret) GSTAMP_OP
                     : [vl] "v"((unsigned)(uintptr_t)lds + lane * 16), [vc] "v"((unsigned)(uintptr_t)lds + 4 * ITEM + (wave >> 1) * 16),
                       [voa] "v"(lane * 16 + 4096 + (wave & 1) * 8192), [vob] "v"(lane * 16 + wave * 4096),
                       [lb] "s"((unsigned)(uintptr_t)lds + wave * 4096), [ws] "s"(myws),
                       [vo] "v"(q4 * 4 * rs + c16 * 16), [ro] "s"(ro), [rs] "s"(rs)
                     : "memory", "scc");
        if (!own) asm volatile("s_waitcnt vmcnt(0)\n\ts_barrier" :: "v"(d2) : "memory");
        STAMP_V(job * 4 + 0, (unsigned)t0);
        STAMP_V(job * 4 + 1, (unsigned)t1);
    }
    STAMP_OUT();
}

struct Tune { int nhead = 12, hfree = 2, hstag = 430, ht0 = 160, hlead = 200, tgo = 315, gstag = 0, stag0 = 40, stag = 430, fs = 4352, rot = 3, xrot = 1, omap = 0, fastu = 20, slp0 = 6, slp1 = 2, nf = 7, xe = 0, near = 1, wsplit = 0; };

void build_lists(const Tune& T, uint16_t (&L)[3][MAXITEMS], int (&len)[3]) {
    const int nf = T.nf;
    for (int c = 0; c < 3; ++c) {
        int n = 0;
        auto X = [&](int a0, int i0 = 0, int i1 = 32) { for (int i = i0; i < i1; ++i) L[c][n++] = (uint16_t)(0 << 8 | (a0 + (i & 7)) << 2 | i >> 3); };
        auto W = [&](int t0, int t1) { for (int q = 0; q < 4; ++q) for (int t = t0; t < t1; ++t) L[c][n++] = (uint16_t)(1 << 8 | t << 2 | q); };
        if (c == 0) { X(8); W(0, 8); W(8, 16); W(16, 24); W(24, 32); X(0); }
        if (c == 1) {
            X(8); W(0, 8);
            for (int s = 0; s < 4; ++s) for (int t = 0; t < nf; ++t) L[c][n++] = (uint16_t)(2 << 8 | t << 2 | s);
            W(8, 16); W(16, 24); X(0, 0, T.xe);
            if (T.wsplit) { W(24, 28); W(28, 32); } else W(24, 32);
            X(0, T.xe, 32);
        }
        if (c == 2) {
            for (int t = 0; t < nf; ++t) L[c][n++] = (uint16_t)(1 << 8 | t << 2 | 0);
            X(8);
            if (nf < 8) W(nf, 8);
            W(8, 16); W(16, 24); W(24, 32); X(0);
            for (int i = 0; i < 64; ++i) L[c][n++] = (uint16_t)(3 << 8 | i);
        }
        len[c] = n;
        while (n < MAXITEMS) L[c][n++] = 0xFFFF;
    }
}

}

extern "C" void kernel_launch(int rank, int nranks, ncclComm_t comm,
                              hipStream_t stream,
                              void* const* d_in, const int* in_sizes, int n_in,
                              void* d_out, int out_size,
                              const int* in_dtypes, int out_dtype,
                              void* const* d_out_all,
                              void* const* d_ws_all, size_t ws_bytes) {
    (void)comm; (void)in_dtypes; (void)out_dtype; (void)d_out_all;
    if (nranks != NR || (unsigned)rank >= (unsigned)NR || n_in < 2 || ws_bytes < WS_NEED ||
        in_sizes[0] != 4096 * KSH || in_sizes[1] != KSH * NCOL || out_size != MSH * NCOL) {
        fprintf(stderr, "kernel_launch: unexpected problem shape\n");
        return;
    }
    const Tune T;
    uint16_t L[3][MAXITEMS];
    int len[3];
    build_lists(T, L, len);
    static const unsigned char upper[NR] = {0xB2, 0x8E, 0xE8, 0xD4, 0xD4, 0xE8, 0x8E, 0xB2};
    int xcd_of[NR], link_of[NR];
    for (int l = 0, lo = 0, hi = 4; l < NR; ++l) {
        xcd_of[l] = !T.near ? l : (upper[rank] >> l) & 1 ? hi++ : lo++;
        link_of[xcd_of[l]] = l;
    }
    Params P;
    P.x = (const float*)d_in[0];
    P.w = (const float*)d_in[1];
    P.out = (float*)d_out;
    P.links = 0;
    for (int j = 0; j < NR; ++j) {
        P.ws[j] = (char*)d_ws_all[rank ^ j];
        P.links |= (unsigned)link_of[j] << j * 4;
    }
    P.rank = rank;
    P.fs = T.fs; P.nhead = T.nhead;
    P.rot = T.rot; P.xrot = T.xrot; P.omap = T.omap; P.fastu = T.fastu; P.slp0 = T.slp0; P.slp1 = T.slp1;
    for (int c = 0; c < 2; ++c) for (int i = 0; i < MAXITEMS; ++i) P.list[c][i] = L[c + 1][i];
    if (T.nf > T.nhead || T.nf > 8 || T.nhead > NWORK || 24 * T.fs > (int)(YRDY - CTRL)) { fprintf(stderr, "bad tuning\n"); return; }
    auto push_at = [&](int p) { return p < T.hfree ? 0 : T.ht0 + (p + 1 - T.hfree) * T.hstag / 16; };
    auto times = [&](int ld, int st) { return (unsigned)(ld > 0 ? ld / 8 : 0) << 16 | (unsigned)(st / 4) << 24; };
    const int nb = 64 - T.nhead;
    int word[64], freeat[32], cnt[32];
    unsigned when[64];
    for (int s = NSEND; s < 32; ++s) { cnt[s] = s < NSEND + T.nhead; freeat[s] = cnt[s] ? (s < NSEND + T.hfree ? T.ht0 : push_at(s - NSEND)) + 100 : 0; }
    for (int i = 0; i < nb; ++i) {
        int s = -1;
        for (int c = NSEND; c < 32; ++c)
            if (cnt[c] < (i >= nb - 8 ? 3 : 2) && (i < nb - 8 || c >= 24) && (s < 0 || freeat[c] < freeat[s])) s = c;
        const int st = i < NSEND ? T.tgo + i * T.gstag / 16 : 0, ld = i < NSEND ? st - T.hlead : T.stag0 + (i - NSEND) * T.stag / 16;
        word[i] = cnt[s] < 2 ? s * 16 + cnt[s] : (s - 24) * 8;
        when[i] = times(ld, st);
        freeat[s] = (ld > freeat[s] ? ld : freeat[s]) + 200;
        ++cnt[s];
    }
    bool taken[512] = {};
    int next = 0;
    for (int p = 0; p < MAXITEMS; ++p)
        for (int x = 0; x < 8; ++x) {
            const int l = link_of[x];
            const int code = L[l == 0 ? 0 : 1 + (l >> 2)][p];
            if ((code >> 8) > 1) { if (p < T.nhead) { fprintf(stderr, "bad list head\n"); return; } continue; }
            const int id = staged_id(code, rank ^ l, l, T.xrot);
            if (taken[id]) { if (p < T.nhead) { fprintf(stderr, "bad list head\n"); return; } continue; }
            taken[id] = true;
            const unsigned c = id < 256 ? (id >> 3) << 10 | (id & 1) << 9 | (id >> 1 & 3) << 1 : 0x8000 | (id & 3) << 13 | (id >> 2 & 63);
            if (p < T.nhead) P.src[((NSEND + p) * 8 + x) * 2] = c | times(push_at(p) - T.hlead, l ? push_at(p) : 0);
            else {
                const int w = word[next >> 3];
                P.src[w + (next & 7) * (w < 64 ? 1 : 2)] = c | when[next >> 3];
                ++next;
            }
        }
    if (next != 8 * nb) { fprintf(stderr, "bad conversion table\n"); return; }
    gemm_rs_kernel<<<8 * (NWORK + NSEND), 384, 0, stream>>>(P);
    HIP_CHECK(hipGetLastError());
}
```

```cpp
#include <hip/hip_runtime.h>
#include <rccl/rccl.h>
#include <cstdio>
#include <cstdint>
#include <cstddef>

#define HIP_CHECK(expr) do { hipError_t _e = (expr); if (_e != hipSuccess) { \
    fprintf(stderr, "HIP error %s at %s:%d\n", hipGetErrorString(_e), __FILE__, __LINE__); \
    return; } } while (0)

#define XMASK 7
#define WIN 35
#define WSPR 7

namespace {

typedef _Float16 h8 __attribute__((ext_vector_type(8)));
typedef float f4 __attribute__((ext_vector_type(4)));
typedef unsigned u4 __attribute__((ext_vector_type(4)));
typedef unsigned long long u64;

constexpr int NR = 8;
constexpr int NCOL = 4096;
constexpr int KSH = 512;
constexpr int MSH = 512;
constexpr int NWORK = 28, NSEND = 4;

constexpr unsigned ITEM = 16384;
constexpr unsigned XR_BYTES = 8 * 32 * 2 * ITEM;
constexpr unsigned WR_BYTES = 32 * 32 * ITEM;
constexpr unsigned STG = XR_BYTES + WR_BYTES;
constexpr unsigned CTRL = STG + 512 * ITEM;
constexpr unsigned YRDY = CTRL + 110592;
constexpr unsigned LRDY = YRDY + 2048;
constexpr unsigned RDY = CTRL + 114688;
constexpr unsigned YST = CTRL + 131072;
constexpr unsigned BUF_BYTES = YST + 256 * ITEM;
constexpr unsigned EPOCH = 2 * BUF_BYTES;
constexpr unsigned TMO = EPOCH + 8192;
constexpr unsigned STAMP = TMO + 4096;
constexpr unsigned WS_NEED = STAMP + (1 << 20);

constexpr u64 DEADLINE_TICKS = 100000000ull;

constexpr int MAXITEMS = 256;

struct Params {
    unsigned src[512];
    uint16_t list[2][MAXITEMS];
    const float* x;
    const float* w;
    int rot, xrot, omap, fastu;
    char* ws[NR];
    float* out;
    int rank;
    unsigned links;
    int fs;
    int nhead;
    int slp0, slp1, ahead, wide;
};
static_assert(offsetof(Params, list) == 0x800 && offsetof(Params, x) == 0xC00 && offsetof(Params, ws) == 0xC20 && sizeof(Params) <= 4096, "layout known to the assembly");

__host__ __device__ inline int staged_id(int code, int peer, int link, int xrot) {
    const int a = (code >> 2) & 31, qa = (code & 3) ^ (peer & 3);
    if ((code >> 8) == 0) {
        const int rb = a >> 1;
        const int grb = rb < 4 ? peer * 4 + ((rb + link * xrot) & 3) : ((peer ^ rb ^ (link & 3)) & 7) * 4 + (peer & 3);
        return (grb * 4 + qa) * 2 + (a & 1);
    }
    return 256 + ((peer >> 2) * 32 + a) * 4 + qa;
}
__host__ __device__ inline int recv_slot(int id, int src, int dst) {
    if (id < 256) {
        const int grb = id >> 3, qa = (id >> 1) & 3, o = grb >> 2, p = grb & 3;
        const int rb = o == dst ? p : o ^ dst;
        if (o != dst && (rb < 4 || p != (dst & 3))) return -1;
        return (rb * 32 + src * 4 + qa) * 2 + (id & 1);
    }
    const int ct = (id - 256) >> 2;
    if ((ct >> 5) != (dst >> 2)) return -1;
    return 512 + (ct & 31) * 32 + src * 4 + (id & 3);
}
__host__ __device__ inline unsigned flag_of_slot(int slot, int fs) {
    const int i = slot < 512 ? (slot >> 6) * 64 + (slot & 1) * 32 + ((slot >> 1) & 31) : slot;
    return (i >> 6) * fs + (i & 63) * 4;
}

__host__ __device__ inline unsigned rdy_of(unsigned id) {
    return RDY + (id < 256 ? (id & 135) | (id & 24) << 2 | (id & 96) >> 2 : id) * 4;
}

#define RLX_SYS __ATOMIC_RELAXED, __HIP_MEMORY_SCOPE_SYSTEM
#define RLX_AGT __ATOMIC_RELAXED, __HIP_MEMORY_SCOPE_AGENT
constexpr int AUX_SYS = 17;

__device__ __forceinline__ int uni(int v) { return __builtin_amdgcn_readfirstlane(v); }

#define STAMP_V(slot, val) do { } while (0)
#define STAMP_AT(slot) do { } while (0)
#define STAMP_OUT() do { } while (0)

__device__ __forceinline__ bool spin_pause(u64 t0, unsigned* tmo, unsigned call) {
    __builtin_amdgcn_s_sleep(2);
    u64 dt = __builtin_amdgcn_s_memrealtime() - t0;
    if (dt > 50000ull) {
        if (dt > DEADLINE_TICKS || (unsigned)uni((int)__hip_atomic_load(tmo, RLX_AGT)) == call) {
            __hip_atomic_store(tmo, call, RLX_AGT);
            return true;
        }
        __builtin_amdgcn_s_sleep(32);
    }
    return false;
}

#define BAR() asm volatile("s_waitcnt lgkmcnt(0)\n\ts_barrier" ::: "memory")

constexpr unsigned RDY_VALID = 0x100, RDY_DEAD = 0x200;

typedef float f32v __attribute__((ext_vector_type(32)));

constexpr int C_MAX = 64, C_DST = 80, C_CALL = 96, C_FLAG = 128;

#define S_(x) #x
#define S(x) S_(x)
#define VR(b) "v[" S(b) ":" S(b) "+3]"
#define V2(b) "v[" S(b) ":" S(b) "+1]"
#define V1(b) "v[" S(b) "]"
#define WT(n) "s_waitcnt vmcnt(" #n ")\n\t"
#define WL(n) "s_waitcnt lgkmcnt(" #n ")\n\t"
#define R4(M, b) M(b) M(b + 1) M(b + 2) M(b + 3)
#define R16(M, b) R4(M, b) R4(M, b + 4) R4(M, b + 8) R4(M, b + 12)
#define R32(M, b) R16(M, b) R16(M, b + 16)

__global__ void __launch_bounds__(384, 2) gemm_rs_kernel(const Params) {
    __shared__ __attribute__((aligned(1024))) char lds[4 * ITEM + 1024];
    auto ka = (const __attribute__((address_space(4))) Params*)__builtin_amdgcn_kernarg_segment_ptr();
    const int wave = uni(threadIdx.x >> 6);
    const int guess = (blockIdx.x & ~7u) | (__builtin_amdgcn_s_getreg(20 | (3 << 11)) & XMASK);
#define CSTAMP(n)
#define CSTAMP0
#define CSTAMP5
#define CSTAMP_OP
#define CSTAMP_CL
    {
        f32v c0, c1, c2;
#define FL1(b, j, so) "buffer_load_dwordx4 " VR(b + (j) * 4) ", v42, s[52:55], " so " offen\n\t"
#define FL4(b, j, so) FL1(b, j, so) "s_add_u32 " so ", " so ", s60\n\t" FL1(b, j + 1, so) "s_add_u32 " so ", " so ", s60\n\t" \
        FL1(b, j + 2, so) "s_add_u32 " so ", " so ", s60\n\t" FL1(b, j + 3, so)
#define FETCH(b, sr, sh) "s_bitcmp1_b32 " sr ", " S(sh + 15) "\n\ts_cselect_b64 s[52:53], s[42:43], s[40:41]\n\ts_cselect_b32 s58, 14, 11\n\t" \
        "s_cselect_b32 s59, 20, 8\n\ts_bfe_u32 s61, " sr ", " S(0xf0000 + sh) "\n\ts_lshl_b32 s61, s61, 8\n\tv_lshl_or_b32 v42, v40, s58, v41\n\t" \
        "s_lshl_b32 s60, 1, s58\n\ts_lshl_b32 s59, 1, s59\n\ts_add_u32 s59, s59, s61\n\t" FL4(b, 0, "s61") FL4(b, 4, "s59")
#define MX(i) "v_max3_f32 v48, v48, |" V1(128 + (i) * 4) "|, |" V1(128 + (i) * 4 + 1) "|\n\tv_max3_f32 v49, v49, |" V1(128 + (i) * 4 + 2) "|, |" V1(128 + (i) * 4 + 3) "|\n\t"
#define DPP(c) "s_nop 1\n\tv_max_u32_dpp v48, v48, v48 " c "\n\t"
#define MUL(i) "v_pk_mul_f32 " V2(128 + (i) * 2) ", s[62:63], " V2(128 + (i) * 2) " op_sel_hi:[0,1]\n\t"
#define CV(d, a, b) "v_cvt_pk_f16_f32 " V1(d) ", " V1(128 + a) ", " V1(128 + b) "\n\t"
#define CVX(j) CV(112 + (j) * 4, (j) * 4, (j) * 4 + 1) CV(113 + (j) * 4, (j) * 4 + 2, (j) * 4 + 3) CV(114 + (j) * 4, (j) * 4 + 16, (j) * 4 + 17) \
        CV(115 + (j) * 4, (j) * 4 + 18, (j) * 4 + 19) "ds_write_b128 v46, " VR(112 + (j) * 4) " offset:" S((j) * 16) "\n\t"
#define RDX(f) "ds_read_b128 " VR(96 + (f) * 4) ", v47 offset:" S((f) * 1088) "\n\t"
#define CVW(f) CV(96 + (f) * 4, f, f + 4) CV(97 + (f) * 4, f + 8, f + 12) CV(98 + (f) * 4, f + 16, f + 20) CV(99 + (f) * 4, f + 24, f + 28)
#define PUT(a, f) "global_store_dwordx4 " V2(a) ", " VR(96 + f * 4) ", off offset:" S(f * 1024) " sc0 sc1\n\t"
        asm volatile("; OPAQUE\n\t"
                     "s_mov_b32 s56, %[guess]\n\t"
                     "s_cmp_lt_u32 %[wave], 4\n\t"
                     "s_cbranch_scc1 .Lc_conv%=\n\t"
                     "s_cmp_eq_u32 %[wave], 4\n\t"
                     "s_cbranch_scc1 .Lc_end%=\n\t"
                     "s_load_dwordx2 s[40:41], %[ka], 0xc20\n\t"
                     "s_add_u32 s58, %[lb], 0x10000 - 1280\n\t"
                     "v_lshl_add_u32 v37, %[tid], 2, s58\n\t"
                     "v_mov_b32 v48, 0\n\t"
                     "s_mov_b64 exec, 1\n\t"
                     "s_lshl2_add_u32 s58, %[bid], %[epo]\n\t"
                     "v_mov_b32 v40, s58\n\t"
                     "v_mov_b32 v42, 1\n\t"
                     "s_lshl2_add_u32 s58, s56, %[epo]\n\t"
                     "v_mov_b32 v41, s58\n\t"
                     WL(0)
                     "global_load_dword v47, v40, s[40:41] sc1\n\t"
                     ".Lc_claim%=:\n\t"
                     "global_atomic_add v46, v41, v42, s[40:41] offset:1024 sc0 sc1\n\t"
                     "s_add_u32 s58, %[lb], 0x10000\n\t"
                     "v_mov_b32 v36, s58\n\t"
                     "v_mov_b32 v45, s56\n\t"
                     "v_mov_b32 v49, 0\n\t"
                     "v_mov_b32 v50, 0\n\t"
                     "ds_write_b96 v36, v[48:50] offset:400\n\t"
                     "s_mov_b64 exec, -1\n\t"
                     "ds_write_b32 v37, v48\n\t"
                     "s_mov_b64 exec, 1\n\t"
                     WT(0)
                     "v_cmp_eq_u32 vcc, v46, v47\n\t"
                     "s_cbranch_vccz .Lc_taken%=\n\t"
                     ".Lc_won%=:\n\t"
                     "v_add_u32 v47, 1, v47\n\t"
                     "v_lshlrev_b32 v44, 8, v47\n\t"
                     "v_bfe_i32 v46, v47, 0, 1\n\t"
                     "v_and_b32 v46, %[buf], v46\n\t"
                     "ds_write_b128 v36, v[44:47] offset:384\n\t"
                     CSTAMP5
                     "global_store_dword v40, v47, s[40:41] sc1\n\t"
                     "s_mov_b64 exec, -1\n\t"
                     "s_branch .Lc_end%=\n\t"
                     ".Lc_conv%=:\n\t"
                     "s_cmp_ge_u32 s56, 32\n\t"
                     "s_cbranch_scc1 .Lc_retry%=\n\t"
                     "s_endpgm\n\t"
                     ".Lc_retry%=:\n\t"
                     "s_lshl_b32 s58, s56, 3\n\t"
                     "s_load_dwordx2 s[48:49], %[ka], s58\n\t"
                     "s_load_dwordx4 s[40:43], %[ka], 0xc00\n\t"
                     "s_memrealtime s[50:51]\n\t"
                     "s_bfe_u32 s58, s58, 0x80001\n\t"
                     "s_load_dword s44, %[ka], s58\n\t"
                     "s_lshr_b32 s45, s56, 6\n\t"
                     "s_max_u32 s45, s45, 2\n\t"
                     "s_mov_b32 s54, 0x800000\n\t"
                     "s_mov_b32 s55, 0x20000\n\t"
                     "s_mov_b32 s57, 0\n\t"
                     "s_mov_b32 s69, 0\n\t"
                     "v_and_b32 v37, 15, %[tid]\n\t"
                     "v_lshrrev_b32 v40, 2, %[tid]\n\t"
                     "v_lshlrev_b32 v41, 4, v37\n\t"
                     "v_and_b32 v40, 60, v40\n\t"
                     WL(0)
                     CSTAMP0
                     "s_bfe_u32 s58, s48, 0x80010\n\t"
                     "s_cbranch_scc1 .Lc_bar%=\n\t"
                     ".Lc_fetch%=:\n\t"
                     FETCH(128, "s48", 0)
                     "s_cmp_lg_u32 s69, 0\n\t"
                     "s_cbranch_scc1 .Lc_go%=\n\t"
                     ".Lc_bar%=:\n\t"
                     "v_and_b32 v43, 63, %[tid]\n\t"
                     "v_bfe_u32 v38, %[tid], 4, 2\n\t"
                     "s_lshl_b32 s58, %[wave], 12\n\t"
                     "v_lshl_or_b32 v44, v43, 4, s58\n\t"
                     "v_mov_b32 v45, 0\n\t"
                     "v_mov_b32 v63, 0\n\t"
                     "s_mul_i32 s58, %[wave], 4352\n\t"
                     "s_add_u32 s58, s58, %[lb]\n\t"
                     "v_mul_u32_u24 v46, 272, v37\n\t"
                     "v_lshl_add_u32 v46, v38, 6, v46\n\t"
                     "v_add_u32 v46, s58, v46\n\t"
                     "v_mul_u32_u24 v47, 272, v38\n\t"
                     "v_add3_u32 v47, v47, v41, s58\n\t"
                     "s_add_u32 s58, %[lb], 0x10000\n\t"
                     "v_mov_b32 v36, s58\n\t"
                     "v_mov_b32 v39, s58\n\t"
                     "v_lshl_add_u32 v33, v43, 3, s58\n\t"
                     "s_lshl2_add_u32 s58, %[wave], s58\n\t"
                     "v_mov_b32 v35, s58\n\t"
                     "s_barrier\n\t"
                     "ds_read_b96 v[60:62], v36 offset:384\n\t"
                     "ds_read_b64 v[50:51], v33 offset:512\n\t"
                     WL(0)
                     "v_readfirstlane_b32 s66, v61\n\t"
                     "v_add_u32 v44, v44, v62\n\t"
                     "v_cmp_ne_u64 vcc, 0, v[50:51]\n\t"
                     "v_lshl_add_u64 v[50:51], v[62:63], 0, v[50:51]\n\t"
                     "s_cmp_lg_u32 s66, s56\n\t"
                     "s_cbranch_scc1 .Lc_wrong%=\n\t"
                     "s_or_b32 s69, vcc_lo, 0x1000000\n\t"
                     "s_bfe_u32 s58, s48, 0x80010\n\t"
                     "s_cbranch_scc1 .Lc_delay%=\n\t"
                     ".Lc_go%=:\n\t"
                     "ds_read_b128 v[56:59], v39 offset:320\n\t"
                     WL(0)
                     "v_cmp_ne_u64 vcc, v[56:57], v[58:59]\n\t"
                     "v_lshl_add_u64 v[56:57], v[44:45], 0, v[56:57]\n\t"
                     "v_lshl_add_u64 v[58:59], v[44:45], 0, v[58:59]\n\t"
                     WT(0)
                     "v_max3_f32 v48, 0, |v128|, |v129|\n\tv_max3_f32 v49, 0, |v130|, |v131|\n\t"
                     MX(1) MX(2) MX(3) R4(MX, 4)
                     CSTAMP(0)
                     "v_max_u32 v48, v48, v49\n\t"
                     DPP("row_shr:1 row_mask:0xf bank_mask:0xf bound_ctrl:1") DPP("row_shr:2 row_mask:0xf bank_mask:0xf bound_ctrl:1")
                     DPP("row_shr:4 row_mask:0xf bank_mask:0xf bound_ctrl:1") DPP("row_shr:8 row_mask:0xf bank_mask:0xf bound_ctrl:1")
                     DPP("row_bcast:15 row_mask:0xa bank_mask:0xf") DPP("row_bcast:31 row_mask:0xc bank_mask:0xf")
                     "s_mov_b32 exec_lo, 0\n\t"
                     "s_brev_b32 exec_hi, 1\n\t"
                     "v_or_b32 v48, 1, v48\n\t"
                     "ds_write_b32 v35, v48 offset:256\n\t"
                     "s_mov_b64 exec, -1\n\t"
                     ".Lc_spin%=:\n\t"
                     "ds_read_b128 v[52:55], v39 offset:256\n\t"
                     WL(0)
                     "v_min3_u32 v49, v52, v53, v54\n\t"
                     "v_max3_u32 v48, v52, v53, v54\n\t"
                     "v_min_u32 v49, v49, v55\n\t"
                     "v_max_u32 v48, v48, v55\n\t"
                     "s_nop 0\n\t"
                     "v_readfirstlane_b32 s65, v49\n\t"
                     "v_readfirstlane_b32 s64, v48\n\t"
                     "s_cmp_eq_u32 s65, 0\n\t"
                     "s_cbranch_scc1 .Lc_spin%=\n\t"
                     "s_lshr_b32 s64, s64, 23\n\t"
                     "s_addk_i32 s64, 0xff7a\n\t"
                     "s_ashr_i32 s64, s64, 3\n\t"
                     "s_max_i32 s64, s64, -15\n\t"
                     "s_lshl_b32 s65, s64, 26\n\t"
                     "s_sub_u32 s62, 1.0, s65\n\t"
                     R16(MUL, 0)
                     "s_bitcmp1_b32 s48, 15\n\t"
                     "s_cbranch_scc0 .Lc_x%=\n\t"
                     R4(CVW, 0)
                     ".Lc_hold%=:\n\t"
                     "s_lshr_b32 s58, s48, 24\n\t"
                     "s_cbranch_scc1 .Lc_hold2%=\n\t"
                     ".Lc_put%=:\n\t"
                     WL(3) PUT(56, 0) WL(2) PUT(56, 1) WL(1) PUT(56, 2) WL(0) PUT(56, 3)
                     CSTAMP(1)
                     "s_cbranch_vccz .Lc_pub%=\n\t"
                     PUT(58, 0) PUT(58, 1) PUT(58, 2) PUT(58, 3)
                     "s_branch .Lc_pub%=\n\t"
                     ".Lc_x%=:\n\t"
                     R4(CVX, 0) R4(RDX, 0)
                     "s_branch .Lc_hold%=\n\t"
                     ".Lc_hold2%=:\n\t"
                     "s_mul_i32 s59, %[wave], " S(WSPR) "\n\t"
                     "s_lshl2_add_u32 s58, s58, s59\n\t"
                     ".Lc_hold3%=:\n\t"
                     "s_memrealtime s[66:67]\n\t"
                     WL(0)
                     "s_sub_u32 s66, s66, s50\n\t"
                     "s_cmp_ge_u32 s66, s58\n\t"
                     "s_cbranch_scc1 .Lc_put%=\n\t"
                     "s_branch .Lc_hold3%=\n\t"
                     ".Lc_taken%=:\n\t"
                     "global_atomic_sub v41, v42, s[40:41] offset:1024 sc1\n\t"
                     "s_add_u32 s58, s56, 1\n\t"
                     "s_and_b32 s58, s58, 7\n\t"
                     "s_and_b32 s56, s56, 0xf8\n\t"
                     "s_or_b32 s56, s56, s58\n\t"
                     "s_lshl2_add_u32 s58, s56, %[epo]\n\t"
                     "v_mov_b32 v41, s58\n\t"
                     "s_cmp_lg_u32 s56, %[guess]\n\t"
                     "s_cbranch_scc1 .Lc_claim%=\n\t"
                     "v_mov_b32 v45, %[bid]\n\t"
                     "s_branch .Lc_won%=\n\t"
                     ".Lc_wrong%=:\n\t"
                     "s_waitcnt vmcnt(0) lgkmcnt(0)\n\t"
                     "s_mov_b32 s56, s66\n\t"
                     "s_branch .Lc_retry%=\n\t"
                     ".Lc_pub%=:\n\t"
                     "s_add_u32 s64, s64, 32\n\t"
                     "v_or_b32 v61, s64, v60\n\t"
                     "s_lshl_b32 s58, s57, 3\n\t"
                     "s_lshl_b32 s68, 0xff, s58\n\t"
                     "v_lshl_add_u32 v49, s57, 2, v36\n\t"
                     "v_mov_b32 v48, 1\n\t"
                     "v_add_u32 v39, 16, v39\n\t"
                     "v_add_u32 v35, 16, v35\n\t"
                     WT(0)
                     "s_mov_b64 exec, 1\n\t"
                     "ds_add_rtn_u32 v48, v49, v48 offset:400\n\t"
                     CSTAMP(2)
                     WL(0)
                     "v_readfirstlane_b32 s65, v48\n\t"
                     "s_cmp_eq_u32 s65, 3\n\t"
                     "s_cselect_b32 s68, s68, 0\n\t"
                     "s_and_b32 exec_lo, s69, s68\n\t"
                     "s_cbranch_execz 2\n\t"
                     "global_store_dword v[50:51], v61, off sc0 sc1\n\t"
                     "s_mov_b64 exec, -1\n\t"
                     "s_add_u32 s57, s57, 1\n\t"
                     "s_cmp_eq_u32 s57, s45\n\t"
                     "s_cbranch_scc1 .Lc_end%=\n\t"
                     "s_mov_b32 s48, s49\n\t"
                     "s_mov_b32 s49, s44\n\t"
                     "s_bfe_u32 s58, s48, 0x80010\n\t"
                     ".Lc_delay%=:\n\t"
                     "s_memrealtime s[66:67]\n\t"
                     WL(0)
                     "s_sub_u32 s66, s66, s50\n\t"
                     "s_lshr_b32 s66, s66, 3\n\t"
                     "s_cmp_ge_u32 s66, s58\n\t"
                     "s_cbranch_scc1 .Lc_fetch%=\n\t"
                     "s_sleep 1\n\t"
                     "s_branch .Lc_delay%=\n\t"
                     ".Lc_end%=:"
                     : "=&{v[32:63]}"(c0), "=&{v[96:127]}"(c1), "=&{v[128:159]}"(c2),
                       [ka] "+s"(ka) CSTAMP_OP
                     : [tid] "v"(threadIdx.x), [wave] "s"(wave), [guess] "s"(guess),
                       [lb] "s"((unsigned)(uintptr_t)lds), [bid] "s"(blockIdx.x), [epo] "n"(EPOCH), [buf] "n"(BUF_BYTES)
                     : "memory", "scc", "vcc", "s40", "s41", "s42", "s43", "s44", "s45", "s48", "s49", "s50", "s51", "s52", "s53", "s54",
                       "s55", "s56", "s57", "s58", "s59", "s60", "s61", "s62", "s63", "s64", "s65", "s66", "s67", "s68", "s69" CSTAMP_CL);
    }
    const auto& P = *ka;
    const int lane = threadIdx.x & 63;
    const int xcd = blockIdx.x & 7;
    const int slot = blockIdx.x >> 3;
    const int g = slot - NSEND;
    char* const myws = P.ws[0];
    int rank = P.rank;
    const int fs = P.fs, nhead = P.nhead;
    const unsigned links = P.links;
    unsigned it0 = P.src[guess * 2], it1 = P.src[guess * 2 + 1], it2 = P.src[guess & 63];
    asm volatile("s_and_b32 %0, %0, 7" : "+s"(rank) : "s"(myws), "s"(fs), "s"(links), "s"(it0), "s"(it1), "s"(it2), "s"(nhead), "s"(P.ws[1]), "s"(P.ws[2]), "s"(P.ws[3]),
                 "s"(P.ws[4]), "s"(P.ws[5]), "s"(P.ws[6]), "s"(P.ws[7]) : "scc");
    __builtin_assume((unsigned)rank < 8u);
    const int pos = rank & 3;
    unsigned* const tmo = (unsigned*)(myws + TMO);
    typedef volatile __attribute__((address_space(3))) unsigned* ldsu;
    const ldsu ctl = (ldsu)(lds + 4 * ITEM);
    const int c16 = lane & 15, q4 = lane >> 4;
    unsigned call = 0;
    int role = guess, code;

    if (wave >= 4) {
        for (;;) {
            asm volatile("global_load_ushort %0, %1, %2 offset:2048" : "=&v"(code)
                         : "v"(((links >> (role & 7) * 4 & 4) * (MAXITEMS / 4) + min(nhead + slot + NSEND * lane, MAXITEMS - 1)) * 2), "s"(ka) : "memory");
            if (wave == 4) {
                const int link = (unsigned)g < (unsigned)nhead ? (links >> (role & 7) * 4) & 7 : 0, j = lane & 7, d = lane == 2 ? rank ^ link : rank;
                const unsigned c = lane & 16 ? it2 : lane & 8 ? it1 : it0;
                unsigned lo = 0, hi = 0;
#pragma unroll
                for (int i = 0; i < NR; ++i)
                    asm("v_writelane_b32 %0, %2, %4\n\tv_writelane_b32 %1, %3, %4" : "+v"(lo), "+v"(hi) : "s"((unsigned)(u64)P.ws[i]), "s"((unsigned)((u64)P.ws[i] >> 32)), "n"(i));
                const u64 wsl = (u64)(unsigned)__builtin_amdgcn_readlane((int)hi, link) << 32 | (unsigned)__builtin_amdgcn_readlane((int)lo, link);
                const u64 wsp = lane == 2 ? wsl : (u64)myws;
                const bool isw = c & 0x8000u, push = lane == 2 && link;
                const unsigned q = (isw ? c >> 13 : c >> 1) & 3, h = c >> 9 & 1, p = c >> 10 & 3, o = (c >> 12 & 7) ^ d;
                const unsigned id = isw ? 256 + (c & 63) * 4 + q : (c >> 10 & 31) * 8 + q * 2 + h;
                const bool need = isw ? (c >> 5 & 1) == (unsigned)(d >> 2) : (o == 0) | ((o >= 4) & (p == (unsigned)(d & 3)));
                const unsigned blk = isw ? 8 + (c >> 1 & 15) : o ? o : p, k = rank * 4 + q;
                const unsigned sl = blk * 64 + (isw ? (c & 1) * 32 + k : k * 2 + h);
                const unsigned fo = j == 1 ? rdy_of(id) : CTRL + blk * fs + (((isw ? c : h) & 1) * 32 + k) * 4;
                const u64 db = (u64)myws + (STG + id * ITEM);
                typedef u64 u64x2 __attribute__((ext_vector_type(2)));
                if ((lane & 39) == 2) *(volatile __attribute__((address_space(3))) u64x2*)(ctl + C_DST + (lane >> 3) * 4) = u64x2{push ? wsp + sl * ITEM : db, db};
                *(volatile __attribute__((address_space(3))) u64*)(ctl + C_FLAG + lane * 2) = ((j == 0 && need) | (j == 1) | push) ? wsp + fo : 0;
                if (lane < 12) ctl[C_MAX + lane] = 0;
                STAMP_AT(64 + 61);
            }
            BAR();
            asm volatile("s_waitcnt vmcnt(0)" : "+v"(code));
            call = ctl[C_CALL + 3];
            const int r = uni((int)ctl[C_CALL + 1]);
            if (r == role) break;
            role = r;
            it0 = P.src[(r & 255) * 2]; it1 = P.src[(r & 255) * 2 + 1]; it2 = P.src[r & 63];
        }
        if (g >= 0 && wave == 5) { STAMP_OUT(); __builtin_amdgcn_endpgm(); }
    }
    call = (unsigned)uni((int)call);
    const unsigned tag = call & 0xFFFFFFu, pb = (call & 1u) * BUF_BYTES;

    if (g < 0) {
        const int link = (links >> (role & 7) * 4) & 7, peer = rank ^ link;
        STAMP_V(127, (__builtin_amdgcn_s_getreg(63492) >> 8 & 0xff) | (__builtin_amdgcn_s_getreg(20 | (3 << 11)) & 15) << 8 | link << 12);
        if (link == 0) { STAMP_OUT(); __builtin_amdgcn_endpgm(); }
        char* const prws = P.ws[link];
        const int n = __builtin_popcountll(__ballot(code != 0xFFFF));
        const int type = code >> 8;
        unsigned src, dst, sfl, dfl, lfl = STAMP;
        if (type == 3) {
            const unsigned quad = (code >> 3) & 7, rg = code & 7;
            src = pb + YST + ((link & 3) * 128 + rg * 16) * 8192 + quad * 1024;
            dst = ((peer & 3) * 128 + rg * 16) * (NCOL * 4) + (peer >> 2) * 8192 + quad * 1024;
            sfl = pb + YRDY + ((link & 3) * 32 + quad * 4) * 4;
            lfl = pb + LRDY + ((peer & 3) * 32 + quad * 4) * 4;
            dfl = 0;
        } else if (type == 2) {
            const unsigned t = (code >> 2) & 31, sj = 4 + (code & 3);
            const unsigned sl = 512 + t * 32 + ((rank ^ sj) & 7) * 4 + (peer & 3);
            src = dst = pb + sl * ITEM;
            sfl = dfl = pb + CTRL + flag_of_slot(sl, fs);
        } else {
            const int id = staged_id(code & 0x1FF, rank, link, P.xrot);
            const int sl = recv_slot(id, peer, rank) & 2047;
            src = pb + STG + id * ITEM;
            dst = pb + sl * ITEM;
            sfl = pb + rdy_of(id);
            dfl = pb + CTRL + flag_of_slot(sl, fs);
        }
        auto rws = __builtin_amdgcn_make_buffer_rsrc(myws, 0, (int)STAMP, 0x00020000);
        auto rpw = __builtin_amdgcn_make_buffer_rsrc(prws, 0, (int)STAMP, 0x00020000);
        if (wave == 5) {
            bool done = lane >= n;
            const int ahead = P.ahead, wide = P.wide;
            u64 t0 = __builtin_amdgcn_s_memrealtime();
            for (;;) {
                const u64 pend = __ballot(!done);
                if (!pend) break;
                const int first = __builtin_ctzll(pend);
                if (first > uni((int)ctl[63]) + ahead) {
                    __builtin_amdgcn_s_sleep(4);
                    t0 = __builtin_amdgcn_s_memrealtime();
                    continue;
                }
                bool ok = false;
                if (!done && lane < first + wide) {
                    const u4 f = __builtin_amdgcn_raw_buffer_load_b128(rpw, sfl & ~15u, 0, AUX_SYS);
                    const u4 e = __builtin_amdgcn_raw_buffer_load_b128(rws, lfl, 0, AUX_SYS);
                    const unsigned c = (sfl >> 2) & 3;
                    const unsigned v = c == 0 ? f[0] : c == 1 ? f[1] : c == 2 ? f[2] : f[3];
                    ok = (v >> 8) == tag;
                    if (type == 3) ok = ok && f[1] == v && f[2] == v && f[3] == v && e[0] == v && e[1] == v && e[2] == v && e[3] == v;
                    if (ok) { ctl[lane] = (v & 255u) | RDY_VALID; done = true; }
                }
                if (__any(ok)) t0 = __builtin_amdgcn_s_memrealtime();
                else if (spin_pause(t0, tmo, call)) { if (!done) ctl[lane] = RDY_DEAD; break; }
            }
            STAMP_OUT();
            __builtin_amdgcn_endpgm();
        }
        float* const po = P.out;
        const int vo = lane * 16;
        const unsigned sstr = type == 3 ? 8192u : 1024u, dstr = type == 3 ? NCOL * 4u : 1024u;
        int k = 0;
        while (k < n) {
            unsigned r;
            while ((r = (unsigned)uni((int)ctl[k])) == 0) __builtin_amdgcn_s_sleep(1);
            if (r & RDY_DEAD) {
                if (type == 3 && lane == n - 1) po[dst >> 2] = __builtin_nanf("");
                break;
            }
            f32v r0, r1, r2, r3;
            unsigned la = (unsigned)(uintptr_t)lds + 4 * ITEM + k * 4, vt, vd, vd2, pfv, so, ss, lo, dso, ds, k1, rn, pfo;
            u64 pm, ym;
#define LD(f) "buffer_load_dwordx4 " VR(128 + f * 4) ", %[vo], %[rs], %[so] offen sc0 sc1\n\ts_add_u32 %[so], %[so], %[ss]\n\t" \
              "buffer_load_dwordx4 " VR(192 + f * 4) ", %[vo], s[84:87], %[lo] offen sc0 sc1\n\ts_add_u32 %[lo], %[lo], 0x4000\n\t"
#define ST(f) "s_mov_b64 exec, %[ym]\n\tv_pk_add_f32 " V2(128 + f * 4) ", " V2(128 + f * 4) ", " V2(192 + f * 4) "\n\t" \
              "v_pk_add_f32 " V2(130 + f * 4) ", " V2(130 + f * 4) ", " V2(194 + f * 4) "\n\ts_mov_b64 exec, -1\n\t" \
              "buffer_store_dwordx4 " VR(128 + f * 4) ", %[vo], s[88:91], %[dso] offen sc0 sc1\n\ts_add_u32 %[dso], %[dso], %[ds]\n\t"
#define WTW "s_waitcnt vmcnt(" S(WIN) ")\n\t"
#define SL(f) WTW ".if " #f " == " S(WIN) " / 3\n\t" FLAG ".endif\n\t" ST(f) LD(f)
#define SE(f) "s_waitcnt vmcnt(45 - 2 * " #f ")\n\t.if " #f " == 15\n\t" FLAG ".endif\n\t" ST(f)
#define L0(f) WTW "buffer_load_dword %[vd], off, %[rf], 0\n\t" LD(f)
#define FLAG "s_mov_b64 exec, %[pm]\n\ts_cbranch_execz 2\n\tbuffer_store_dword %[pfv], off, %[rf], %[pfo] sc0 sc1\n\ts_mov_b64 exec, -1\n\t"
#define NEXTFLAG "v_readlane_b32 %[pfo], %[dfl], %[k]\n\ts_and_b32 %[r], %[r], 0xff\n\ts_or_b32 %[r], %[r], %[tag8]\n\t" \
                 "v_mov_b32 %[pfv], %[r]\n\ts_cmp_lg_u32 %[pfo], 0\n\ts_cselect_b64 %[pm], 1, 0\n\ts_mov_b32 %[k], %[k1]\n\t"
#define ASTAMP
#define ASTAMP_OP
#define ASTAMP_CL
            asm volatile("; OPAQUE\n\t"
                         "v_readlane_b32 %[so], %[src], %[k]\n\t"
                         "v_readlane_b32 %[ss], %[sstr], %[k]\n\t"
                         "v_readlane_b32 %[lo], %[dst], %[k]\n\t"
                         "s_mov_b64 %[pm], 0\n\t"
                         "s_mov_b32 %[pfo], 0\n\t"
                         "s_mov_b32 s91, 0x20000\n\t"
                         "s_mov_b32 s87, 0x20000\n\t"
                         "s_mov_b64 s[84:85], %[po]\n\t"
                         "s_cmp_eq_u32 %[ss], 0x400\n\t"
                         "s_cselect_b32 s86, 0, %[szo]\n\t"
                         "s_nop 4\n\t"
                         L0(0) L0(1) L0(2) L0(3) L0(4) L0(5) L0(6) L0(7) L0(8) L0(9) L0(10) L0(11) L0(12) L0(13) L0(14) L0(15)
                         ".Lsend_loop%=:\n\t"
                         "ds_read_b32 %[vt], %[la] offset:4\n\t"
                         "s_add_u32 %[k1], %[k], 1\n\t"
                         "v_mov_b32 %[vd2], %[k1]\n\t"
                         "ds_write_b32 %[lp], %[vd2]\n\t"
                         "v_readlane_b32 %[dso], %[dst], %[k]\n\t"
                         "v_readlane_b32 %[ds], %[dstr], %[k]\n\t"
                         "v_readlane_b32 %[so], %[src], %[k1]\n\t"
                         "v_readlane_b32 %[ss], %[sstr], %[k1]\n\t"
                         "v_readlane_b32 %[lo], %[dst], %[k1]\n\t"
                         "s_cmp_eq_u32 %[ds], 0x400\n\t"
                         "s_cselect_b64 s[88:89], %[pw], %[po]\n\t"
                         "s_cselect_b32 s90, %[szw], %[szo]\n\t"
                         "s_cselect_b64 %[ym], 0, -1\n\t"
                         "s_cmp_eq_u32 %[ss], 0x400\n\t"
                         "s_cselect_b32 s86, 0, %[szo]\n\t"
                         "s_waitcnt lgkmcnt(0)\n\t"
                         "v_readfirstlane_b32 %[rn], %[vt]\n\t"
                         "v_add_u32 %[la], 4, %[la]\n\t"
                         ASTAMP
                         "s_bitcmp1_b32 %[rn], 8\n\t"
                         "s_cbranch_scc0 .Lsend_last%=\n\t"
                         SL(0) SL(1) SL(2) SL(3) SL(4) SL(5) SL(6) SL(7) SL(8) SL(9) SL(10) SL(11) SL(12) SL(13) SL(14) SL(15)
                         NEXTFLAG
                         "s_mov_b32 %[r], %[rn]\n\t"
                         "s_branch .Lsend_loop%=\n\t"
                         ".Lsend_last%=:\n\t"
                         SE(0) SE(1) SE(2) SE(3) SE(4) SE(5) SE(6) SE(7) SE(8) SE(9) SE(10) SE(11) SE(12) SE(13) SE(14) SE(15)
                         NEXTFLAG
                         WT(0)
                         FLAG
                         "s_nop 1"
                         : "=&{v[128:159]}"(r0), "=&{v[160:191]}"(r1), "=&{v[192:223]}"(r2), "=&{v[224:255]}"(r3), [lo] "=&s"(lo), [ym] "=&s"(ym),
                           [la] "+v"(la), [vt] "=&v"(vt), [vd] "=&v"(vd), [vd2] "=&v"(vd2), [pfv] "=&v"(pfv), [so] "=&s"(so), [ss] "=&s"(ss), [dso] "=&s"(dso), [ds] "=&s"(ds),
                           [k1] "=&s"(k1), [rn] "=&s"(rn), [pfo] "=&s"(pfo), [pm] "=&s"(pm), [k] "+s"(k), [r] "+s"(r) ASTAMP_OP
                         : [vo] "v"(vo), [lp] "v"((unsigned)(uintptr_t)lds + 4 * ITEM + 252), [src] "v"(src), [dst] "v"(dst), [dfl] "v"(dfl), [sstr] "v"(sstr), [dstr] "v"(dstr),
                           [rs] "s"(rpw), [rf] "s"(rws), [pw] "s"(myws), [po] "s"(po), [szw] "s"(STAMP), [szo] "s"(MSH * NCOL * 4), [tag8] "s"(tag << 8)
                         : "memory", "scc", "s84", "s85", "s86", "s87", "s88", "s89", "s90", "s91" ASTAMP_CL);
            STAMP_AT(63 + k);
        }
        STAMP_OUT();
        __builtin_amdgcn_endpgm();
    }

    const bool own = g >= 12;
    const int o = g - 12, omap = P.omap;
    const int fl = xcd >> 1, fh = xcd & 1;
    const int rb = own ? (omap ? (o & 3) : (xcd & 1) * 2 + (o & 1)) : 4 + fl;
    const int grb = own ? rank * 4 + rb : ((rank ^ rb) & 7) * 4 + pos;
    const int njob = g < 4 || own ? 2 : 1;
    auto tile = [&](int job) {
        return own ? (omap ? xcd * 4 + (o >> 2) : (xcd >> 1) * 8 + (o >> 1)) : (job ? 3 : g >> 2) * 8 + fh * 4 + (g & 3);
    };
    if (wave == 4) {
#pragma unroll 1
        for (int job = 0; job < njob; ++job) {
            const int t = tile(job);
            const int rot = (g * P.rot) & 31, fastu = P.fastu, slp0 = P.slp0, slp1 = P.slp1;
            const int ls = lane & 31, lh = lane >> 5;
            const char* const fxp = myws + pb + CTRL + rb * fs + lane * 4;
            const bool mini = own && !job;
            const unsigned wid = 256 + (((rank >> 2) ^ mini) * 32 + t) * 4 + (ls & 3);
            const char* const fwp = myws + pb + (mini ? rdy_of(wid) : CTRL + flag_of_slot(512 + t * 32 + ls, fs));
            const unsigned skip = mini ? ~(15u << rank * 4) : own ? 0 : 15u << (rank ^ rb) * 4;
            const bool loc = (ls >> 2) == rank;
            const unsigned offa = pb + (loc ? STG + ((grb * 4 + (ls & 3)) * 2 + lh) * ITEM : ((rb * 32 + ls) * 2 + lh) * ITEM);
            const unsigned offb = pb + (loc ? STG + wid * ITEM : XR_BYTES + (t * 32 + ls) * ITEM);
            unsigned issued = skip, xa, wa, xb, wb;
            int u = __builtin_popcount(skip);
            bool dead = false;
            u64 t0 = __builtin_amdgcn_s_memrealtime();
#define POLL(fx, fw) asm volatile("; POLL\n\tglobal_load_dword %0, %2, off sc0 sc1\n\tglobal_load_dword %1, %3, off sc0 sc1" \
                                  : "=&v"(fx), "=&v"(fw) : "v"(fxp), "v"(fwp) : "memory")
            auto tick = [&](unsigned& px_, unsigned& pw_) {
                unsigned fx, fw;
                asm volatile("; GOT\n\ts_waitcnt vmcnt(2)\n\tv_mov_b32 %0, %2\n\tv_mov_b32 %1, %3\n\t"
                             "global_load_dword %2, %4, off sc0 sc1\n\tglobal_load_dword %3, %5, off sc0 sc1"
                             : "=&v"(fx), "=&v"(fw), "+v"(px_), "+v"(pw_) : "v"(fxp), "v"(fwp) : "memory");
                const u64 bx = __ballot((fx >> 8) == tag);
                unsigned rdy = (unsigned)bx & (unsigned)(bx >> 32) & (unsigned)__ballot((fw >> 8) == tag);
                rdy |= skip;
                if (dead) rdy = ~0u;
                unsigned avail = rdy & ~issued;
                if (!avail) {
                    const int n = u < fastu || mini ? slp0 : slp1;
                    for (int i = 0; i < n; ++i) __builtin_amdgcn_s_sleep(2);
                    dead = spin_pause(t0, tmo, call);
                    return;
                }
                const u4 rec = {offa, offb, (fx & 255u) + (fw & 255u) - 64u, dead};
                do {
                    const int s = (__builtin_ctz(__builtin_amdgcn_alignbit(avail, avail, rot)) + rot) & 31;
                    issued |= 1u << s;
                    avail &= ~(1u << s);
                    if (ls == s) *(volatile __attribute__((address_space(3))) u4*)(ctl + (u & 1) * 8 + lh * 4) = rec;
                    STAMP_AT(job * 32 + u);
                    STAMP_V(64 + job * 32 + u, s);
                    BAR();
                    ++u;
                } while (__builtin_popcount(avail) > (rdy == ~0u ? 0 : 2));
                t0 = __builtin_amdgcn_s_memrealtime();
            };
            POLL(xa, wa);
            POLL(xb, wb);
            for (;;) {
                tick(xa, wa);
                if (u == 32) break;
                tick(xb, wb);
                if (u == 32) break;
            }
            BAR(); BAR(); BAR();
            asm volatile("s_waitcnt vmcnt(0)" :: "v"(xa), "v"(wa), "v"(xb), "v"(wb) : "memory");
            if (own && job) break;
            BAR();
            if (lane == 0) __hip_atomic_store((unsigned*)(myws + pb + (own ? LRDY : YRDY)) + (rb & 3) * 32 + t, tag << 8, RLX_SYS);
        }
        STAMP_OUT();
        __builtin_amdgcn_endpgm();
    }
#pragma unroll 1
    for (int job = 0; job < njob; ++job) {
        const int t = tile(job);
        f32v acc, d1, d2, d3, d4, d5;
        u4 dc;
        int ecur, emax, cnt, e0, e1, e2, e3, ta, tb;
        unsigned keep;
        u64 ret;
        const int rs = own ? NCOL * 4 : 8192;
        char* ob = own ? (char*)P.out + (size_t)(rb * 128 + wave * 32) * (NCOL * 4) + ((rank >> 2) ^ job ^ 1) * 8192 + t * 256
                       : myws + (unsigned)uni((int)ctl[C_CALL + 2]) + YST + (size_t)((rb & 3) * 128 + wave * 32) * 8192 + t * 256;
        auto ro = __builtin_amdgcn_make_buffer_rsrc(ob, 0, 31 * rs + 256, 0x00020000);
#define ACC(i, nf) VR(64 + i * 16 + nf * 4)
#define BF(ks, nf) VR(96 + (ks & 1) * 16 + nf * 4)
#define AF(c, f) VR(128 + c * 32 + (f) * 4)
#define RDB(c, ks, nf) "ds_read_b128 " BF(ks, nf) ", %[vl] offset:" S(c * 16384 + ks * 4096 + nf * 1024) "\n\t"
#define RDB8(c) RDB(c, 0, 0) RDB(c, 0, 1) RDB(c, 0, 2) RDB(c, 0, 3) RDB(c, 1, 0) RDB(c, 1, 1) RDB(c, 1, 2) RDB(c, 1, 3)
#define MM(c, ks, nf, w) "s_waitcnt lgkmcnt(" #w ")\n\t" \
        "v_mfma_f32_16x16x32_f16 " ACC(0, nf) ", " AF(c, ks) ", " BF(ks, nf) ", " ACC(0, nf) "\n\t" \
        "v_mfma_f32_16x16x32_f16 " ACC(1, nf) ", " AF(c, 4 + ks) ", " BF(ks, nf) ", " ACC(1, nf) "\n\t"
#define LDB(j) "global_load_lds_dwordx4 v61, %[ws] offset:" S(j * 1024) " sc0 sc1\n\t"
#define LDA(n, f) "global_load_dwordx4 " AF(n, f) ", v60, %[ws] offset:" S((f) * 1024 - 4096) " sc0 sc1\n\t"
#define LD12(n) LDB(0) LDB(1) LDB(2) LDB(3) LDA(n, 0) LDA(n, 4) LDA(n, 1) LDA(n, 5) LDA(n, 2) LDA(n, 6) LDA(n, 3) LDA(n, 7)
#define REC(n) "ds_read_b96 v[60:62], %[vc] offset:" S((n & 1) * 32) "\n\t"
#define ADR(n) "s_add_u32 m0, %[lb], " S(n * 16384) "\n\tv_add_u32 v60, v60, %[voa]\n\tv_add_u32 v61, v61, %[vob]\n\tv_readfirstlane_b32 %[e" #n "], v62\n\t"
#define PRO(n) "s_barrier\n\t" REC(n) "s_waitcnt lgkmcnt(0)\n\t" ADR(n) LD12(n)
#define CHK(c, id) "s_cmp_lg_u32 %[e" #c "], %[ecur]\n\ts_cbranch_scc1 .Lfix" #id "_%=\n\t.Lback" #id "_%=:\n\t"
#define FIX(c, id, pre) ".Lfix" #id "_%=:\n\t" pre "s_call_b64 %[ret], .Lslow" #c "_%=\n\ts_branch .Lback" #id "_%=\n\t"
#define STEP(c, n) "s_waitcnt vmcnt(24)\n\ts_barrier\n\t" REC(n) RDB8(c) CHK(c, c) "s_waitcnt lgkmcnt(8)\n\t" ADR(n) \
        MM(c, 0, 0, 7) RDB(c, 2, 0) LDB(0) MM(c, 0, 1, 7) RDB(c, 2, 1) LDB(1) MM(c, 0, 2, 7) RDB(c, 2, 2) LDB(2) MM(c, 0, 3, 7) RDB(c, 2, 3) \
        MM(c, 1, 0, 7) RDB(c, 3, 0) LDB(3) MM(c, 1, 1, 7) RDB(c, 3, 1) LDA(n, 0) MM(c, 1, 2, 7) RDB(c, 3, 2) LDA(n, 4) MM(c, 1, 3, 7) RDB(c, 3, 3) \
        MM(c, 2, 0, 7) LDA(n, 1) MM(c, 2, 1, 6) LDA(n, 5) MM(c, 2, 2, 5) LDA(n, 2) MM(c, 2, 3, 4) \
        MM(c, 3, 0, 3) LDA(n, 6) MM(c, 3, 1, 2) LDA(n, 3) MM(c, 3, 2, 1) LDA(n, 7) MM(c, 3, 3, 0)
#define MMS(c, w0, w1, w2, w3) \
        w0 MM(c, 0, 0, 7) RDB(c, 2, 0) MM(c, 0, 1, 7) RDB(c, 2, 1) MM(c, 0, 2, 7) RDB(c, 2, 2) MM(c, 0, 3, 7) RDB(c, 2, 3) \
        w1 MM(c, 1, 0, 7) RDB(c, 3, 0) MM(c, 1, 1, 7) RDB(c, 3, 1) MM(c, 1, 2, 7) RDB(c, 3, 2) MM(c, 1, 3, 7) RDB(c, 3, 3) \
        w2 MM(c, 2, 0, 7) MM(c, 2, 1, 6) MM(c, 2, 2, 5) MM(c, 2, 3, 4) w3 MM(c, 3, 0, 3) MM(c, 3, 1, 2) MM(c, 3, 2, 1) MM(c, 3, 3, 0)
#define TBAR(n) "s_waitcnt vmcnt(" #n ")\n\ts_cbranch_execz 1\n\ts_barrier\n\t"
#define ZERO(r) "v_mov_b32 v[" S(r) "], 0\n\t"
#define BAD(r) "v_mov_b32 v[" S(r) "], -1\n\t"
#define LDX(r) "v_ldexp_f32 v[" S(r) "], v[" S(r) "], %[tb]\n\t"
#define PKM(r) "v_pk_mul_f16 v[" S(r) "], v[" S(r) "], v63 op_sel_hi:[1,0]\n\t"
#define SLOW(c) ".Lslow" #c "_%=:\n\ts_max_i32 %[emax], %[emax], %[e" #c "]\n\ts_add_i32 %[ta], %[emax], -7\n\ts_max_i32 %[ta], %[ta], %[e" #c "]\n\t" \
        "s_sub_i32 %[tb], %[ecur], %[ta]\n\ts_lshl_b32 %[tb], %[tb], 3\n\ts_mov_b32 %[ecur], %[ta]\n\t" R32(LDX, 64) \
        "s_sub_i32 %[tb], %[e" #c "], %[ta]\n\ts_lshl_b32 %[tb], %[tb], 3\n\tv_ldexp_f32 v63, 1.0, %[tb]\n\tv_cvt_f16_f32 v63, v63\n\ts_nop 0\n\t" \
        R32(PKM, 128 + c * 32) "s_setpc_b64 %[ret]\n\t"
#define EP1(i, j, c) "v_ldexp_f32 v[" S(128 + (i * 4 + j) * 4 + c) "], v[" S(64 + i * 16 + c * 4 + j) "], %[tb]\n\t"
#define EP8(c) EP1(0, 0, c) EP1(0, 1, c) EP1(0, 2, c) EP1(0, 3, c) EP1(1, 0, c) EP1(1, 1, c) EP1(1, 2, c) EP1(1, 3, c)
#define EPS(i, step) "buffer_store_dwordx4 " VR(128 + i * 4) ", %[vo], %[ro], %[ta] offen sc0 sc1\n\ts_add_u32 %[ta], %[ta], " step "\n\t"
#define GSTAMP(n)
#define GSTAMP_OP
        asm volatile("; OPAQUE\n\t"
                     "s_mov_b32 %[keep], m0\n\t"
                     "s_mov_b32 %[cnt], 0\n\t"
                     "s_mov_b32 %[ecur], 0\n\ts_mov_b32 %[e0], 0\n\ts_mov_b32 %[e1], 0\n\ts_mov_b32 %[e2], 0\n\ts_mov_b32 %[e3], 0\n\t"
                     "s_mov_b64 exec, 0\n\t"
                     "s_branch .Lgemm_tail%=\n\t"
                     SLOW(0) SLOW(1) SLOW(2) SLOW(3)
                     FIX(0, 0, "") FIX(1, 1, "") FIX(2, 2, "") FIX(3, 3, "")
                     FIX(0, 4, "") FIX(1, 5, "") FIX(2, 6, "") FIX(3, 7, "s_waitcnt vmcnt(0)\n\t")
                     ".Lgemm_bad%=:\n\t"
                     "s_nop 7\n\t"
                     R32(BAD, 64)
                     "s_branch .Lgemm_ep%=\n\t"
                     ".Lgemm_go%=:\n\t"
                     "s_mov_b64 exec, -1\n\t"
                     R32(ZERO, 64)
                     PRO(0) PRO(1) PRO(2)
                     "s_mov_b32 %[ecur], %[e0]\n\t"
                     "s_mov_b32 %[emax], %[e0]\n\t"
                     "s_mov_b32 %[cnt], %[cnt0]\n\t"
                     "s_cmp_eq_u32 %[cnt], 7\n\t"
                     "s_cbranch_scc1 .Lgemm_tail%=\n\t"
                     ".Lgemm%=:\n\t"
                     STEP(0, 3) STEP(1, 0) STEP(2, 1) STEP(3, 2)
                     "s_add_u32 %[cnt], %[cnt], 1\n\t"
                     "s_cmp_lt_u32 %[cnt], 7\n\t"
                     "s_cbranch_scc1 .Lgemm%=\n\t"
                     ".Lgemm_tail%=:\n\t"
                     TBAR(24) "ds_read_b128 v[60:63], %[vc] offset:32\n\t" RDB8(0)
                     "s_waitcnt lgkmcnt(8)\n\t"
                     "s_cbranch_execz .Lgemm_t1%=\n\t"
                     ADR(3) LD12(3)
                     "v_readfirstlane_b32 %[cnt], v63\n\t"
                     ".Lgemm_t1%=:\n\t"
                     CHK(0, 4) MMS(0, "", "", "", "")
                     TBAR(24) RDB8(1) CHK(1, 5) MMS(1, "", "", "", "")
                     TBAR(12) RDB8(2) CHK(2, 6) MMS(2, "", "", "", "")
                     TBAR(8) RDB8(3) CHK(3, 7) MMS(3, WT(6), WT(4), WT(2), WT(0))
                     GSTAMP(0)
                     "s_cmp_lg_u32 %[cnt], 0\n\t"
                     "s_cbranch_scc1 .Lgemm_bad%=\n\t"
                     ".Lgemm_ep%=:\n\t"
                     "s_lshl_b32 %[tb], %[ecur], 3\n\t"
                     "s_mov_b32 %[ta], 0\n\t"
                     "s_mul_i32 %[emax], %[rs], 13\n\t"
                     EP8(0) EP8(1) EP8(2) EP8(3)
                     "s_cbranch_execz .Lgemm_go%=\n\t"
                     EPS(0, "%[rs]") EPS(1, "%[rs]") EPS(2, "%[rs]") EPS(3, "%[emax]") EPS(4, "%[rs]") EPS(5, "%[rs]") EPS(6, "%[rs]") EPS(7, "%[rs]")
                     GSTAMP(1)
                     "s_mov_b32 m0, %[keep]\n\t"
                     "s_waitcnt lgkmcnt(0)"
                     : "=&{v[64:95]}"(acc), "=&{v[96:127]}"(d1), "=&{v[128:159]}"(d2), "=&{v[160:191]}"(d3),
                       "=&{v[192:223]}"(d4), "=&{v[224:255]}"(d5), "=&{v[60:63]}"(dc),
                       [ecur] "=&s"(ecur), [emax] "=&s"(emax), [cnt] "=&s"(cnt), [e0] "=&s"(e0), [e1] "=&s"(e1), [e2] "=&s"(e2), [e3] "=&s"(e3),
                       [ta] "=&s"(ta), [tb] "=&s"(tb), [keep] "=&s"(keep), [ret] "=&s"(ret) GSTAMP_OP
                     : [vl] "v"((unsigned)(uintptr_t)lds + lane * 16), [vc] "v"((unsigned)(uintptr_t)lds + 4 * ITEM + (wave >> 1) * 16),
                       [voa] "v"(lane * 16 + 4096 + (wave & 1) * 8192), [vob] "v"(lane * 16 + wave * 4096),
                       [lb] "s"((unsigned)(uintptr_t)lds + wave * 4096), [ws] "s"(myws),
                       [vo] "v"(q4 * 4 * rs + c16 * 16), [ro] "s"(ro), [rs] "s"(rs), [cnt0] "s"(!own ? 1 : job ? 0 : 7)
                     : "memory", "scc");
        if (!(own && job)) asm volatile("s_waitcnt vmcnt(0)\n\ts_barrier" :: "v"(d2) : "memory");
        STAMP_V(job * 4 + 0, (unsigned)t0);
        STAMP_V(job * 4 + 1, (unsigned)t1);
    }
    STAMP_OUT();
}

struct Tune { int xc = 2, nfast = 28, stag2 = 860, pref = 1, ahead = 2, wide = 4, nhead = 12, hfree = 2, hstag = 430, ht0 = 160, hlead = 200, tgo = 315, gstag = 0, stag0 = 40, stag = 430, fs = 4352, rot = 3, xrot = 1, omap = 0, fastu = 20, slp0 = 6, slp1 = 2, nf = 6, xe = 0, near = 1, wsplit = 0; };

void build_lists(const Tune& T, uint16_t (&L)[3][MAXITEMS], int (&len)[3]) {
    const int nf = T.nf;
    for (int c = 0; c < 3; ++c) {
        int n = 0;
        auto X = [&](int a0, int i0 = 0, int i1 = 32) { for (int i = i0; i < i1; ++i) L[c][n++] = (uint16_t)(0 << 8 | (a0 + (i & 7)) << 2 | i >> 3); };
        auto W = [&](int t0, int t1) { for (int q = 0; q < 4; ++q) for (int t = t0; t < t1; ++t) L[c][n++] = (uint16_t)(1 << 8 | t << 2 | q); };
        if (c == 0) { X(8); W(0, 8); W(8, 16); W(16, 24); W(24, 32); X(0); }
        if (c == 1) {
            X(8); W(0, 8);
            for (int s = 0; s < 4; ++s) for (int t = 0; t < nf; ++t) L[c][n++] = (uint16_t)(2 << 8 | t << 2 | s);
            W(8, 16); W(16, 24); X(0, 0, T.xe);
            if (T.wsplit) { W(24, 28); W(28, 32); } else W(24, 32);
            X(0, T.xe, 32);
        }
        if (c == 2) {
            for (int t = 0; t < nf; ++t) L[c][n++] = (uint16_t)(1 << 8 | t << 2 | 0);
            for (int i = 0; i < 24; ++i) L[c][n++] = (uint16_t)(0 << 8 | (10 + i % 6) << 2 | i / 6);
            if (nf < 8) W(nf, 8);
            for (int i = 3; i >= 0; --i) { if (T.xc == i) X(0); if (i) W(32 - 8 * i, 40 - 8 * i); }
            for (int i = 0; i < 64; ++i) L[c][n++] = (uint16_t)(3 << 8 | i);
        }
        len[c] = n;
        while (n < MAXITEMS) L[c][n++] = 0xFFFF;
    }
}

}

extern "C" void kernel_launch(int rank, int nranks, ncclComm_t comm,
                              hipStream_t stream,
                              void* const* d_in, const int* in_sizes, int n_in,
                              void* d_out, int out_size,
                              const int* in_dtypes, int out_dtype,
                              void* const* d_out_all,
                              void* const* d_ws_all, size_t ws_bytes) {
    (void)comm; (void)in_dtypes; (void)out_dtype; (void)d_out_all;
    if (nranks != NR || (unsigned)rank >= (unsigned)NR || n_in < 2 || ws_bytes < WS_NEED ||
        in_sizes[0] != 4096 * KSH || in_sizes[1] != KSH * NCOL || out_size != MSH * NCOL) {
        fprintf(stderr, "kernel_launch: unexpected problem shape\n");
        return;
    }
    const Tune T;
    uint16_t L[3][MAXITEMS];
    int len[3];
    build_lists(T, L, len);
    static const unsigned char upper[NR] = {0xB2, 0x8E, 0xE8, 0xD4, 0xD4, 0xE8, 0x8E, 0xB2};
    int xcd_of[NR], link_of[NR];
    for (int l = 0, lo = 0, hi = 4; l < NR; ++l) {
        xcd_of[l] = !T.near ? l : (upper[rank] >> l) & 1 ? hi++ : lo++;
        link_of[xcd_of[l]] = l;
    }
    Params P;
    P.x = (const float*)d_in[0];
    P.w = (const float*)d_in[1];
    P.out = (float*)d_out;
    P.links = 0;
    for (int j = 0; j < NR; ++j) {
        P.ws[j] = (char*)d_ws_all[rank ^ j];
        P.links |= (unsigned)link_of[j] << j * 4;
    }
    P.rank = rank;
    P.fs = T.fs; P.nhead = T.nhead;
    P.rot = T.rot; P.xrot = T.xrot; P.omap = T.omap; P.fastu = T.fastu; P.slp0 = T.slp0; P.slp1 = T.slp1; P.ahead = T.ahead; P.wide = T.wide;
    for (int c = 0; c < 2; ++c) for (int i = 0; i < MAXITEMS; ++i) P.list[c][i] = L[c + 1][i];
    if (T.nf > 8 || len[2] - T.nhead > 62 * NSEND || T.nhead > NWORK || 24 * T.fs > (int)(YRDY - CTRL)) { fprintf(stderr, "bad tuning\n"); return; }
    auto push_at = [&](int p) { return p < T.hfree ? 0 : T.ht0 + (p + 1 - T.hfree) * T.hstag / 16; };
    auto times = [&](int ld, int st) { return (unsigned)(ld > 0 ? ld / 8 : 0) << 16 | (unsigned)(st / 4) << 24; };
    const int nb = 64 - T.nhead;
    int word[64], freeat[32], cnt[32];
    unsigned when[64];
    for (int s = NSEND; s < 32; ++s) { cnt[s] = s < NSEND + T.nhead; freeat[s] = cnt[s] ? (s < NSEND + T.hfree ? T.ht0 : push_at(s - NSEND)) + 100 : 0; }
    for (int i = 0; i < nb; ++i) {
        const int st = i < NSEND ? T.tgo + i * T.gstag / 16 : 0;
        const int ld = i < NSEND ? st - T.hlead : T.stag0 + (i - NSEND) * T.stag / 16 + (i > T.nfast ? (i - T.nfast) * (T.stag2 - T.stag) / 16 : 0);
        if (ld > 2040) { fprintf(stderr, "bad tuning\n"); return; }
        int s = -1;
        for (int c = NSEND; c < 32; ++c)
            if (cnt[c] < (i >= nb - 8 ? 3 : 2) && (i < nb - 8 || c >= 24) && (s < 0 || freeat[c] < freeat[s]) && !(T.pref && s >= 0 && freeat[s] <= ld)) s = c;
        word[i] = cnt[s] < 2 ? s * 16 + cnt[s] : (s - 24) * 8;
        when[i] = times(ld, st);
        freeat[s] = (ld > freeat[s] ? ld : freeat[s]) + 200;
        ++cnt[s];
    }
    bool taken[512] = {};
    int next = 0;
    for (int p = 0; p < MAXITEMS; ++p)
        for (int x = 0; x < 8; ++x) {
            const int l = link_of[x];
            const int code = L[l == 0 ? 0 : 1 + (l >> 2)][p];
            if ((code >> 8) > 1) { if (p < T.nhead) { fprintf(stderr, "bad list head\n"); return; } continue; }
            const int id = staged_id(code, rank ^ l, l, T.xrot);
            if (taken[id]) { if (p < T.nhead) { fprintf(stderr, "bad list head\n"); return; } continue; }
            taken[id] = true;
            const unsigned c = id < 256 ? (id >> 3) << 10 | (id & 1) << 9 | (id >> 1 & 3) << 1 : 0x8000 | (id & 3) << 13 | (id >> 2 & 63);
            if (p < T.nhead) P.src[((NSEND + p) * 8 + x) * 2] = c | times(push_at(p) - T.hlead, l ? push_at(p) : 0);
            else {
                const int w = word[next >> 3];
                P.src[w + (next & 7) * (w < 64 ? 1 : 2)] = c | when[next >> 3];
                ++next;
            }
        }
    if (next != 8 * nb) { fprintf(stderr, "bad conversion table\n"); return; }
    gemm_rs_kernel<<<8 * (NWORK + NSEND), 384, 0, stream>>>(P);
    HIP_CHECK(hipGetLastError());
}
```

```cpp
#include <hip/hip_runtime.h>
#include <rccl/rccl.h>
#include <cstdio>
#include <cstdint>
#include <cstddef>

#define HIP_CHECK(expr) do { hipError_t _e = (expr); if (_e != hipSuccess) { \
    fprintf(stderr, "HIP error %s at %s:%d\n", hipGetErrorString(_e), __FILE__, __LINE__); \
    return; } } while (0)

#define XMASK 7
#define WIN 35
#define WSPR 7

namespace {

typedef _Float16 h8 __attribute__((ext_vector_type(8)));
typedef float f4 __attribute__((ext_vector_type(4)));
typedef unsigned u4 __attribute__((ext_vector_type(4)));
typedef unsigned long long u64;

constexpr int NR = 8;
constexpr int NCOL = 4096;
constexpr int KSH = 512;
constexpr int MSH = 512;
constexpr int NWORK = 28, NSEND = 4;

constexpr unsigned ITEM = 16384;
constexpr unsigned XR_BYTES = 8 * 32 * 2 * ITEM;
constexpr unsigned WR_BYTES = 32 * 32 * ITEM;
constexpr unsigned STG = XR_BYTES + WR_BYTES;
constexpr unsigned CTRL = STG + 512 * ITEM;
constexpr unsigned YRDY = CTRL + 110592;
constexpr unsigned LRDY = YRDY + 2048;
constexpr unsigned RDY = CTRL + 114688;
constexpr unsigned YST = CTRL + 131072;
constexpr unsigned OX = YST + 256 * ITEM;
constexpr unsigned BUF_BYTES = OX + 40 * ITEM;
constexpr unsigned EPOCH = 2 * BUF_BYTES;
constexpr unsigned TMO = EPOCH + 8192;
constexpr unsigned STAMP = TMO + 4096;
constexpr unsigned WS_NEED = STAMP + (1 << 20);

constexpr u64 DEADLINE_TICKS = 100000000ull;

constexpr int MAXITEMS = 256;

struct Params {
    unsigned src[512];
    uint16_t list[2][MAXITEMS];
    const float* x;
    const float* w;
    int rot, xrot, omap, fastu;
    char* ws[NR];
    float* out;
    int rank;
    unsigned links;
    int fs;
    int nhead;
    int slp0, slp1, ahead, wide;
    int gate[4];
};
static_assert(offsetof(Params, list) == 0x800 && offsetof(Params, x) == 0xC00 && offsetof(Params, ws) == 0xC20 && sizeof(Params) <= 4096, "layout known to the assembly");

__host__ __device__ inline int staged_id(int code, int peer, int link, int xrot) {
    const int a = (code >> 2) & 31, qa = (code & 3) ^ (peer & 3);
    if ((code >> 8) == 0) {
        const int rb = a >> 1;
        const int grb = rb < 4 ? peer * 4 + ((rb + link * xrot) & 3) : ((peer ^ rb ^ (link & 3)) & 7) * 4 + (peer & 3);
        return (grb * 4 + qa) * 2 + (a & 1);
    }
    return 256 + ((peer >> 2) * 32 + a) * 4 + qa;
}
__host__ __device__ inline int recv_slot(int id, int src, int dst) {
    if (id < 256) {
        const int grb = id >> 3, qa = (id >> 1) & 3, o = grb >> 2, p = grb & 3;
        const int rb = o == dst ? p : o ^ dst;
        if (o != dst && (rb < 4 || p != (dst & 3))) return -1;
        return (rb * 32 + src * 4 + qa) * 2 + (id & 1);
    }
    const int ct = (id - 256) >> 2;
    if ((ct >> 5) != (dst >> 2)) return -1;
    return 512 + (ct & 31) * 32 + src * 4 + (id & 3);
}
__host__ __device__ inline unsigned flag_of_slot(int slot, int fs) {
    const int i = slot < 512 ? (slot >> 6) * 64 + (slot & 1) * 32 + ((slot >> 1) & 31) : slot;
    return (i >> 6) * fs + (i & 63) * 4;
}

__host__ __device__ inline unsigned rdy_of(unsigned id) {
    return RDY + (id < 256 ? (id & 135) | (id & 24) << 2 | (id & 96) >> 2 : id) * 4;
}

#define RLX_SYS __ATOMIC_RELAXED, __HIP_MEMORY_SCOPE_SYSTEM
#define RLX_AGT __ATOMIC_RELAXED, __HIP_MEMORY_SCOPE_AGENT
constexpr int AUX_SYS = 17;

__device__ __forceinline__ int uni(int v) { return __builtin_amdgcn_readfirstlane(v); }

#define STAMP_V(slot, val) do { } while (0)
#define STAMP_AT(slot) do { } while (0)
#define STAMP_OUT() do { } while (0)

__device__ __forceinline__ bool spin_pause(u64 t0, unsigned* tmo, unsigned call) {
    __builtin_amdgcn_s_sleep(2);
    u64 dt = __builtin_amdgcn_s_memrealtime() - t0;
    if (dt > 50000ull) {
        if (dt > DEADLINE_TICKS || (unsigned)uni((int)__hip_atomic_load(tmo, RLX_AGT)) == call) {
            __hip_atomic_store(tmo, call, RLX_AGT);
            return true;
        }
        __builtin_amdgcn_s_sleep(32);
    }
    return false;
}

#define BAR() asm volatile("s_waitcnt lgkmcnt(0)\n\ts_barrier" ::: "memory")

constexpr unsigned RDY_VALID = 0x100, RDY_DEAD = 0x200;

typedef float f32v __attribute__((ext_vector_type(32)));

constexpr int C_MAX = 64, C_DST = 80, C_CALL = 96, C_FLAG = 128;

#define S_(x) #x
#define S(x) S_(x)
#define VR(b) "v[" S(b) ":" S(b) "+3]"
#define V2(b) "v[" S(b) ":" S(b) "+1]"
#define V1(b) "v[" S(b) "]"
#define WT(n) "s_waitcnt vmcnt(" #n ")\n\t"
#define WL(n) "s_waitcnt lgkmcnt(" #n ")\n\t"
#define R4(M, b) M(b) M(b + 1) M(b + 2) M(b + 3)
#define R16(M, b) R4(M, b) R4(M, b + 4) R4(M, b + 8) R4(M, b + 12)
#define R32(M, b) R16(M, b) R16(M, b + 16)

__global__ void __launch_bounds__(384, 2) gemm_rs_kernel(const Params) {
    __shared__ __attribute__((aligned(1024))) char lds[4 * ITEM + 1024];
    auto ka = (const __attribute__((address_space(4))) Params*)__builtin_amdgcn_kernarg_segment_ptr();
    const int wave = uni(threadIdx.x >> 6);
    const int guess = (blockIdx.x & ~7u) | (__builtin_amdgcn_s_getreg(20 | (3 << 11)) & XMASK);
#define CSTAMP(n)
#define CSTAMP0
#define CSTAMP5
#define CSTAMP_OP
#define CSTAMP_CL
    {
        f32v c0, c1, c2;
#define FL1(b, j, so) "buffer_load_dwordx4 " VR(b + (j) * 4) ", v42, s[52:55], " so " offen\n\t"
#define FL4(b, j, so) FL1(b, j, so) "s_add_u32 " so ", " so ", s60\n\t" FL1(b, j + 1, so) "s_add_u32 " so ", " so ", s60\n\t" \
        FL1(b, j + 2, so) "s_add_u32 " so ", " so ", s60\n\t" FL1(b, j + 3, so)
#define FETCH(b, sr, sh) "s_bitcmp1_b32 " sr ", " S(sh + 15) "\n\ts_cselect_b64 s[52:53], s[42:43], s[40:41]\n\ts_cselect_b32 s58, 14, 11\n\t" \
        "s_cselect_b32 s59, 20, 8\n\ts_bfe_u32 s61, " sr ", " S(0xf0000 + sh) "\n\ts_lshl_b32 s61, s61, 8\n\tv_lshl_or_b32 v42, v40, s58, v41\n\t" \
        "s_lshl_b32 s60, 1, s58\n\ts_lshl_b32 s59, 1, s59\n\ts_add_u32 s59, s59, s61\n\t" FL4(b, 0, "s61") FL4(b, 4, "s59")
#define MX(i) "v_max3_f32 v48, v48, |" V1(128 + (i) * 4) "|, |" V1(128 + (i) * 4 + 1) "|\n\tv_max3_f32 v49, v49, |" V1(128 + (i) * 4 + 2) "|, |" V1(128 + (i) * 4 + 3) "|\n\t"
#define DPP(c) "s_nop 1\n\tv_max_u32_dpp v48, v48, v48 " c "\n\t"
#define MUL(i) "v_pk_mul_f32 " V2(128 + (i) * 2) ", s[62:63], " V2(128 + (i) * 2) " op_sel_hi:[0,1]\n\t"
#define CV(d, a, b) "v_cvt_pk_f16_f32 " V1(d) ", " V1(128 + a) ", " V1(128 + b) "\n\t"
#define CVX(j) CV(112 + (j) * 4, (j) * 4, (j) * 4 + 1) CV(113 + (j) * 4, (j) * 4 + 2, (j) * 4 + 3) CV(114 + (j) * 4, (j) * 4 + 16, (j) * 4 + 17) \
        CV(115 + (j) * 4, (j) * 4 + 18, (j) * 4 + 19) "ds_write_b128 v46, " VR(112 + (j) * 4) " offset:" S((j) * 16) "\n\t"
#define RDX(f) "ds_read_b128 " VR(96 + (f) * 4) ", v47 offset:" S((f) * 1088) "\n\t"
#define CVW(f) CV(96 + (f) * 4, f, f + 4) CV(97 + (f) * 4, f + 8, f + 12) CV(98 + (f) * 4, f + 16, f + 20) CV(99 + (f) * 4, f + 24, f + 28)
#define PUT(a, f) "global_store_dwordx4 " V2(a) ", " VR(96 + f * 4) ", off offset:" S(f * 1024) " sc0 sc1\n\t"
        asm volatile("; OPAQUE\n\t"
                     "s_mov_b32 s56, %[guess]\n\t"
                     "s_cmp_lt_u32 %[wave], 4\n\t"
                     "s_cbranch_scc1 .Lc_conv%=\n\t"
                     "s_cmp_eq_u32 %[wave], 4\n\t"
                     "s_cbranch_scc1 .Lc_end%=\n\t"
                     "s_load_dwordx2 s[40:41], %[ka], 0xc20\n\t"
                     "s_add_u32 s58, %[lb], 0x10000 - 1280\n\t"
                     "v_lshl_add_u32 v37, %[tid], 2, s58\n\t"
                     "v_mov_b32 v48, 0\n\t"
                     "s_mov_b64 exec, 1\n\t"
                     "s_lshl2_add_u32 s58, %[bid], %[epo]\n\t"
                     "v_mov_b32 v40, s58\n\t"
                     "v_mov_b32 v42, 1\n\t"
                     "s_lshl2_add_u32 s58, s56, %[epo]\n\t"
                     "v_mov_b32 v41, s58\n\t"
                     WL(0)
                     "global_load_dword v47, v40, s[40:41] sc1\n\t"
                     ".Lc_claim%=:\n\t"
                     "global_atomic_add v46, v41, v42, s[40:41] offset:1024 sc0 sc1\n\t"
                     "s_add_u32 s58, %[lb], 0x10000\n\t"
                     "v_mov_b32 v36, s58\n\t"
                     "v_mov_b32 v45, s56\n\t"
                     "v_mov_b32 v49, 0\n\t"
                     "v_mov_b32 v50, 0\n\t"
                     "ds_write_b96 v36, v[48:50] offset:400\n\t"
                     "s_mov_b64 exec, -1\n\t"
                     "ds_write_b32 v37, v48\n\t"
                     "s_mov_b64 exec, 1\n\t"
                     WT(0)
                     "v_cmp_eq_u32 vcc, v46, v47\n\t"
                     "s_cbranch_vccz .Lc_taken%=\n\t"
                     ".Lc_won%=:\n\t"
                     "v_add_u32 v47, 1, v47\n\t"
                     "v_lshlrev_b32 v44, 8, v47\n\t"
                     "v_bfe_i32 v46, v47, 0, 1\n\t"
                     "v_and_b32 v46, %[buf], v46\n\t"
                     "ds_write_b128 v36, v[44:47] offset:384\n\t"
                     CSTAMP5
                     "global_store_dword v40, v47, s[40:41] sc1\n\t"
                     "s_mov_b64 exec, -1\n\t"
                     "s_branch .Lc_end%=\n\t"
                     ".Lc_conv%=:\n\t"
                     "s_cmp_ge_u32 s56, 32\n\t"
                     "s_cbranch_scc1 .Lc_retry%=\n\t"
                     "s_endpgm\n\t"
                     ".Lc_retry%=:\n\t"
                     "s_lshl_b32 s58, s56, 3\n\t"
                     "s_load_dwordx2 s[48:49], %[ka], s58\n\t"
                     "s_load_dwordx4 s[40:43], %[ka], 0xc00\n\t"
                     "s_memrealtime s[50:51]\n\t"
                     "s_bfe_u32 s58, s58, 0x80001\n\t"
                     "s_load_dword s44, %[ka], s58\n\t"
                     "s_lshr_b32 s45, s56, 6\n\t"
                     "s_max_u32 s45, s45, 2\n\t"
                     "s_mov_b32 s54, 0x800000\n\t"
                     "s_mov_b32 s55, 0x20000\n\t"
                     "s_mov_b32 s57, 0\n\t"
                     "s_mov_b32 s69, 0\n\t"
                     "v_and_b32 v37, 15, %[tid]\n\t"
                     "v_lshrrev_b32 v40, 2, %[tid]\n\t"
                     "v_lshlrev_b32 v41, 4, v37\n\t"
                     "v_and_b32 v40, 60, v40\n\t"
                     WL(0)
                     CSTAMP0
                     "s_bfe_u32 s58, s48, 0x80010\n\t"
                     "s_cbranch_scc1 .Lc_bar%=\n\t"
                     ".Lc_fetch%=:\n\t"
                     FETCH(128, "s48", 0)
                     "s_cmp_lg_u32 s69, 0\n\t"
                     "s_cbranch_scc1 .Lc_go%=\n\t"
                     ".Lc_bar%=:\n\t"
                     "v_and_b32 v43, 63, %[tid]\n\t"
                     "v_bfe_u32 v38, %[tid], 4, 2\n\t"
                     "s_lshl_b32 s58, %[wave], 12\n\t"
                     "v_lshl_or_b32 v44, v43, 4, s58\n\t"
                     "v_mov_b32 v45, 0\n\t"
                     "v_mov_b32 v63, 0\n\t"
                     "s_mul_i32 s58, %[wave], 4352\n\t"
                     "s_add_u32 s58, s58, %[lb]\n\t"
                     "v_mul_u32_u24 v46, 272, v37\n\t"
                     "v_lshl_add_u32 v46, v38, 6, v46\n\t"
                     "v_add_u32 v46, s58, v46\n\t"
                     "v_mul_u32_u24 v47, 272, v38\n\t"
                     "v_add3_u32 v47, v47, v41, s58\n\t"
                     "s_add_u32 s58, %[lb], 0x10000\n\t"
                     "v_mov_b32 v36, s58\n\t"
                     "v_mov_b32 v39, s58\n\t"
                     "v_lshl_add_u32 v33, v43, 3, s58\n\t"
                     "s_lshl2_add_u32 s58, %[wave], s58\n\t"
                     "v_mov_b32 v35, s58\n\t"
                     "s_barrier\n\t"
                     "ds_read_b96 v[60:62], v36 offset:384\n\t"
                     "ds_read_b64 v[50:51], v33 offset:512\n\t"
                     WL(0)
                     "v_readfirstlane_b32 s66, v61\n\t"
                     "v_add_u32 v44, v44, v62\n\t"
                     "v_cmp_ne_u64 vcc, 0, v[50:51]\n\t"
                     "v_lshl_add_u64 v[50:51], v[62:63], 0, v[50:51]\n\t"
                     "s_cmp_lg_u32 s66, s56\n\t"
                     "s_cbranch_scc1 .Lc_wrong%=\n\t"
                     "s_or_b32 s69, vcc_lo, 0x1000000\n\t"
                     "s_bfe_u32 s58, s48, 0x80010\n\t"
                     "s_cbranch_scc1 .Lc_delay%=\n\t"
                     ".Lc_go%=:\n\t"
                     "ds_read_b128 v[56:59], v39 offset:320\n\t"
                     WL(0)
                     "v_cmp_ne_u64 vcc, v[56:57], v[58:59]\n\t"
                     "v_lshl_add_u64 v[56:57], v[44:45], 0, v[56:57]\n\t"
                     "v_lshl_add_u64 v[58:59], v[44:45], 0, v[58:59]\n\t"
                     WT(0)
                     "v_max3_f32 v48, 0, |v128|, |v129|\n\tv_max3_f32 v49, 0, |v130|, |v131|\n\t"
                     MX(1) MX(2) MX(3) R4(MX, 4)
                     CSTAMP(0)
                     "v_max_u32 v48, v48, v49\n\t"
                     DPP("row_shr:1 row_mask:0xf bank_mask:0xf bound_ctrl:1") DPP("row_shr:2 row_mask:0xf bank_mask:0xf bound_ctrl:1")
                     DPP("row_shr:4 row_mask:0xf bank_mask:0xf bound_ctrl:1") DPP("row_shr:8 row_mask:0xf bank_mask:0xf bound_ctrl:1")
                     DPP("row_bcast:15 row_mask:0xa bank_mask:0xf") DPP("row_bcast:31 row_mask:0xc bank_mask:0xf")
                     "s_mov_b32 exec_lo, 0\n\t"
                     "s_brev_b32 exec_hi, 1\n\t"
                     "v_or_b32 v48, 1, v48\n\t"
                     "ds_write_b32 v35, v48 offset:256\n\t"
                     "s_mov_b64 exec, -1\n\t"
                     ".Lc_spin%=:\n\t"
                     "ds_read_b128 v[52:55], v39 offset:256\n\t"
                     WL(0)
                     "v_min3_u32 v49, v52, v53, v54\n\t"
                     "v_max3_u32 v48, v52, v53, v54\n\t"
                     "v_min_u32 v49, v49, v55\n\t"
                     "v_max_u32 v48, v48, v55\n\t"
                     "s_nop 0\n\t"
                     "v_readfirstlane_b32 s65, v49\n\t"
                     "v_readfirstlane_b32 s64, v48\n\t"
                     "s_cmp_eq_u32 s65, 0\n\t"
                     "s_cbranch_scc1 .Lc_spin%=\n\t"
                     "s_lshr_b32 s64, s64, 23\n\t"
                     "s_addk_i32 s64, 0xff7a\n\t"
                     "s_ashr_i32 s64, s64, 3\n\t"
                     "s_max_i32 s64, s64, -15\n\t"
                     "s_lshl_b32 s65, s64, 26\n\t"
                     "s_sub_u32 s62, 1.0, s65\n\t"
                     R16(MUL, 0)
                     "s_bitcmp1_b32 s48, 15\n\t"
                     "s_cbranch_scc0 .Lc_x%=\n\t"
                     R4(CVW, 0)
                     ".Lc_hold%=:\n\t"
                     "s_lshr_b32 s58, s48, 24\n\t"
                     "s_cbranch_scc1 .Lc_hold2%=\n\t"
                     ".Lc_put%=:\n\t"
                     WL(3) PUT(56, 0) WL(2) PUT(56, 1) WL(1) PUT(56, 2) WL(0) PUT(56, 3)
                     CSTAMP(1)
                     "s_cbranch_vccz .Lc_pub%=\n\t"
                     PUT(58, 0) PUT(58, 1) PUT(58, 2) PUT(58, 3)
                     "s_branch .Lc_pub%=\n\t"
                     ".Lc_x%=:\n\t"
                     R4(CVX, 0) R4(RDX, 0)
                     "s_branch .Lc_hold%=\n\t"
                     ".Lc_hold2%=:\n\t"
                     "s_mul_i32 s59, %[wave], " S(WSPR) "\n\t"
                     "s_lshl2_add_u32 s58, s58, s59\n\t"
                     ".Lc_hold3%=:\n\t"
                     "s_memrealtime s[66:67]\n\t"
                     WL(0)
                     "s_sub_u32 s66, s66, s50\n\t"
                     "s_cmp_ge_u32 s66, s58\n\t"
                     "s_cbranch_scc1 .Lc_put%=\n\t"
                     "s_branch .Lc_hold3%=\n\t"
                     ".Lc_taken%=:\n\t"
                     "global_atomic_sub v41, v42, s[40:41] offset:1024 sc1\n\t"
                     "s_add_u32 s58, s56, 1\n\t"
                     "s_and_b32 s58, s58, 7\n\t"
                     "s_and_b32 s56, s56, 0xf8\n\t"
                     "s_or_b32 s56, s56, s58\n\t"
                     "s_lshl2_add_u32 s58, s56, %[epo]\n\t"
                     "v_mov_b32 v41, s58\n\t"
                     "s_cmp_lg_u32 s56, %[guess]\n\t"
                     "s_cbranch_scc1 .Lc_claim%=\n\t"
                     "v_mov_b32 v45, %[bid]\n\t"
                     "s_branch .Lc_won%=\n\t"
                     ".Lc_wrong%=:\n\t"
                     "s_waitcnt vmcnt(0) lgkmcnt(0)\n\t"
                     "s_mov_b32 s56, s66\n\t"
                     "s_branch .Lc_retry%=\n\t"
                     ".Lc_pub%=:\n\t"
                     "s_add_u32 s64, s64, 32\n\t"
                     "v_or_b32 v61, s64, v60\n\t"
                     "s_lshl_b32 s58, s57, 3\n\t"
                     "s_lshl_b32 s68, 0xff, s58\n\t"
                     "v_lshl_add_u32 v49, s57, 2, v36\n\t"
                     "v_mov_b32 v48, 1\n\t"
                     "v_add_u32 v39, 16, v39\n\t"
                     "v_add_u32 v35, 16, v35\n\t"
                     WT(0)
                     "s_mov_b64 exec, 1\n\t"
                     "ds_add_rtn_u32 v48, v49, v48 offset:400\n\t"
                     CSTAMP(2)
                     WL(0)
                     "v_readfirstlane_b32 s65, v48\n\t"
                     "s_cmp_eq_u32 s65, 3\n\t"
                     "s_cselect_b32 s68, s68, 0\n\t"
                     "s_and_b32 exec_lo, s69, s68\n\t"
                     "s_cbranch_execz 2\n\t"
                     "global_store_dword v[50:51], v61, off sc0 sc1\n\t"
                     "s_mov_b64 exec, -1\n\t"
                     "s_add_u32 s57, s57, 1\n\t"
                     "s_cmp_eq_u32 s57, s45\n\t"
                     "s_cbranch_scc1 .Lc_end%=\n\t"
                     "s_mov_b32 s48, s49\n\t"
                     "s_mov_b32 s49, s44\n\t"
                     "s_bfe_u32 s58, s48, 0x80010\n\t"
                     ".Lc_delay%=:\n\t"
                     "s_memrealtime s[66:67]\n\t"
                     WL(0)
                     "s_sub_u32 s66, s66, s50\n\t"
                     "s_lshr_b32 s66, s66, 3\n\t"
                     "s_cmp_ge_u32 s66, s58\n\t"
                     "s_cbranch_scc1 .Lc_fetch%=\n\t"
                     "s_sleep 1\n\t"
                     "s_branch .Lc_delay%=\n\t"
                     ".Lc_end%=:"
                     : "=&{v[32:63]}"(c0), "=&{v[96:127]}"(c1), "=&{v[128:159]}"(c2),
                       [ka] "+s"(ka) CSTAMP_OP
                     : [tid] "v"(threadIdx.x), [wave] "s"(wave), [guess] "s"(guess),
                       [lb] "s"((unsigned)(uintptr_t)lds), [bid] "s"(blockIdx.x), [epo] "n"(EPOCH), [buf] "n"(BUF_BYTES)
                     : "memory", "scc", "vcc", "s40", "s41", "s42", "s43", "s44", "s45", "s48", "s49", "s50", "s51", "s52", "s53", "s54",
                       "s55", "s56", "s57", "s58", "s59", "s60", "s61", "s62", "s63", "s64", "s65", "s66", "s67", "s68", "s69" CSTAMP_CL);
    }
    const auto& P = *ka;
    const int lane = threadIdx.x & 63;
    const int xcd = blockIdx.x & 7;
    const int slot = blockIdx.x >> 3;
    const int g = slot - NSEND;
    char* const myws = P.ws[0];
    int rank = P.rank;
    const int fs = P.fs, nhead = P.nhead;
    const unsigned links = P.links;
    unsigned it0 = P.src[guess * 2], it1 = P.src[guess * 2 + 1], it2 = P.src[guess & 63];
    asm volatile("s_and_b32 %0, %0, 7" : "+s"(rank) : "s"(myws), "s"(fs), "s"(links), "s"(it0), "s"(it1), "s"(it2), "s"(nhead), "s"(P.ws[1]), "s"(P.ws[2]), "s"(P.ws[3]),
                 "s"(P.ws[4]), "s"(P.ws[5]), "s"(P.ws[6]), "s"(P.ws[7]) : "scc");
    __builtin_assume((unsigned)rank < 8u);
    const int pos = rank & 3;
    unsigned* const tmo = (unsigned*)(myws + TMO);
    typedef volatile __attribute__((address_space(3))) unsigned* ldsu;
    const ldsu ctl = (ldsu)(lds + 4 * ITEM);
    const int c16 = lane & 15, q4 = lane >> 4;
    unsigned call = 0;
    int role = guess, code;
    const unsigned tstart = (unsigned)__builtin_amdgcn_s_memrealtime();

    if (wave >= 4) {
        for (;;) {
            asm volatile("global_load_ushort %0, %1, %2 offset:2048" : "=&v"(code)
                         : "v"(((links >> (role & 7) * 4 & 4) * (MAXITEMS / 4) + min(nhead + slot + NSEND * lane, MAXITEMS - 1)) * 2), "s"(ka) : "memory");
            if (wave == 4) {
                const int link = (unsigned)g < (unsigned)nhead ? (links >> (role & 7) * 4) & 7 : 0, j = lane & 7, d = lane == 2 ? rank ^ link : rank;
                const unsigned c = lane & 16 ? it2 : lane & 8 ? it1 : it0;
                unsigned lo = 0, hi = 0;
#pragma unroll
                for (int i = 0; i < NR; ++i)
                    asm("v_writelane_b32 %0, %2, %4\n\tv_writelane_b32 %1, %3, %4" : "+v"(lo), "+v"(hi) : "s"((unsigned)(u64)P.ws[i]), "s"((unsigned)((u64)P.ws[i] >> 32)), "n"(i));
                const u64 wsl = (u64)(unsigned)__builtin_amdgcn_readlane((int)hi, link) << 32 | (unsigned)__builtin_amdgcn_readlane((int)lo, link);
                const u64 wsp = lane == 2 ? wsl : (u64)myws;
                const bool isw = c & 0x8000u, push = lane == 2 && link;
                const unsigned q = (isw ? c >> 13 : c >> 1) & 3, h = c >> 9 & 1, p = c >> 10 & 3, o = (c >> 12 & 7) ^ d;
                const unsigned id = isw ? 256 + (c & 63) * 4 + q : (c >> 10 & 31) * 8 + q * 2 + h;
                const bool need = isw ? (c >> 5 & 1) == (unsigned)(d >> 2) : (o == 0) | ((o >= 4) & (p == (unsigned)(d & 3)));
                const unsigned blk = isw ? 8 + (c >> 1 & 15) : o ? o : p, k = rank * 4 + q;
                const unsigned sl = blk * 64 + (isw ? (c & 1) * 32 + k : k * 2 + h);
                const unsigned fo = j == 1 ? rdy_of(id) : CTRL + blk * fs + (((isw ? c : h) & 1) * 32 + k) * 4;
                const u64 db = (u64)myws + (STG + id * ITEM);
                typedef u64 u64x2 __attribute__((ext_vector_type(2)));
                if ((lane & 39) == 2) *(volatile __attribute__((address_space(3))) u64x2*)(ctl + C_DST + (lane >> 3) * 4) = u64x2{push ? wsp + sl * ITEM : db, db};
                *(volatile __attribute__((address_space(3))) u64*)(ctl + C_FLAG + lane * 2) = ((j == 0 && need) | (j == 1) | push) ? wsp + fo : 0;
                if (lane < 12) ctl[C_MAX + lane] = 0;
                STAMP_AT(64 + 61);
            }
            BAR();
            asm volatile("s_waitcnt vmcnt(0)" : "+v"(code));
            call = ctl[C_CALL + 3];
            const int r = uni((int)ctl[C_CALL + 1]);
            if (r == role) break;
            role = r;
            it0 = P.src[(r & 255) * 2]; it1 = P.src[(r & 255) * 2 + 1]; it2 = P.src[r & 63];
        }
        if (g >= 0 && wave == 5) { STAMP_OUT(); __builtin_amdgcn_endpgm(); }
    }
    call = (unsigned)uni((int)call);
    const unsigned tag = call & 0xFFFFFFu, pb = (call & 1u) * BUF_BYTES;

    if (g < 0) {
        const int link = (links >> (role & 7) * 4) & 7, peer = rank ^ link;
        STAMP_V(127, (__builtin_amdgcn_s_getreg(63492) >> 8 & 0xff) | (__builtin_amdgcn_s_getreg(20 | (3 << 11)) & 15) << 8 | link << 12);
        if (link == 0) { STAMP_OUT(); __builtin_amdgcn_endpgm(); }
        char* const prws = P.ws[link];
        const int n = __builtin_popcountll(__ballot(code != 0xFFFF));
        const int type = code >> 8;
        unsigned src, dst, sfl, dfl, lfl = STAMP;
        if (type == 3) {
            const unsigned quad = (code >> 3) & 7, rg = code & 7;
            src = pb + YST + ((link & 3) * 128 + rg * 16) * 8192 + quad * 1024;
            dst = ((peer & 3) * 128 + rg * 16) * (NCOL * 4) + (peer >> 2) * 8192 + quad * 1024;
            sfl = pb + YRDY + ((link & 3) * 32 + quad * 4) * 4;
            lfl = pb + LRDY + ((peer & 3) * 32 + quad * 4) * 4;
            dfl = 0;
        } else if (type == 2) {
            const unsigned t = (code >> 2) & 31, sj = 4 + (code & 3);
            const unsigned sl = 512 + t * 32 + ((rank ^ sj) & 7) * 4 + (peer & 3);
            src = dst = pb + sl * ITEM;
            sfl = dfl = pb + CTRL + flag_of_slot(sl, fs);
        } else {
            const int id = staged_id(code & 0x1FF, rank, link, P.xrot);
            const int sl = recv_slot(id, peer, rank) & 2047;
            src = pb + STG + id * ITEM;
            dst = pb + sl * ITEM;
            sfl = pb + rdy_of(id);
            dfl = sl < 256 ? pb + OX + flag_of_slot(sl, 32 * fs) + 1 : pb + CTRL + flag_of_slot(sl, fs);
        }
        auto rws = __builtin_amdgcn_make_buffer_rsrc(myws, 0, (int)STAMP, 0x00020000);
        auto rpw = __builtin_amdgcn_make_buffer_rsrc(prws, 0, (int)STAMP, 0x00020000);
        if (wave == 5) {
            bool done = lane >= n;
            const int ahead = P.ahead, wide = P.wide;
            u64 t0 = __builtin_amdgcn_s_memrealtime();
            for (;;) {
                const u64 pend = __ballot(!done);
                if (!pend) break;
                const int first = __builtin_ctzll(pend);
                if (first > uni((int)ctl[63]) + ahead) {
                    __builtin_amdgcn_s_sleep(4);
                    t0 = __builtin_amdgcn_s_memrealtime();
                    continue;
                }
                bool ok = false;
                if (!done && lane < first + wide) {
                    const u4 f = __builtin_amdgcn_raw_buffer_load_b128(rpw, sfl & ~15u, 0, AUX_SYS);
                    const u4 e = __builtin_amdgcn_raw_buffer_load_b128(rws, lfl, 0, AUX_SYS);
                    const unsigned c = (sfl >> 2) & 3;
                    const unsigned v = c == 0 ? f[0] : c == 1 ? f[1] : c == 2 ? f[2] : f[3];
                    ok = (v >> 8) == tag;
                    if (type == 3) ok = ok && f[1] == v && f[2] == v && f[3] == v && e[0] == v && e[1] == v && e[2] == v && e[3] == v;
                    if (ok) { ctl[lane] = (v & 255u) | RDY_VALID; done = true; }
                }
                if (__any(ok)) t0 = __builtin_amdgcn_s_memrealtime();
                else if (spin_pause(t0, tmo, call)) { if (!done) ctl[lane] = RDY_DEAD; break; }
            }
            STAMP_OUT();
            __builtin_amdgcn_endpgm();
        }
        float* const po = P.out;
        const int vo = lane * 16;
        const unsigned sstr = type == 3 ? 8192u : 1024u, dstr = type == 3 ? NCOL * 4u : 1024u;
        int k = 0;
        while (k < n) {
            unsigned r;
            while ((r = (unsigned)uni((int)ctl[k])) == 0) __builtin_amdgcn_s_sleep(1);
            if (r & RDY_DEAD) {
                if (type == 3 && lane == n - 1) po[dst >> 2] = __builtin_nanf("");
                break;
            }
            f32v r0, r1, r2, r3;
            unsigned la = (unsigned)(uintptr_t)lds + 4 * ITEM + k * 4, vt, vd, vd2, pfv, so, ss, lo, dso, ds, k1, rn, pfo;
            u64 pm, ym;
#define LD(f) "buffer_load_dwordx4 " VR(128 + f * 4) ", %[vo], %[rs], %[so] offen sc0 sc1\n\ts_add_u32 %[so], %[so], %[ss]\n\t" \
              "buffer_load_dwordx4 " VR(192 + f * 4) ", %[vo], s[84:87], %[lo] offen sc0 sc1\n\ts_add_u32 %[lo], %[lo], 0x4000\n\t"
#define ST(f) "s_mov_b64 exec, %[ym]\n\tv_pk_add_f32 " V2(128 + f * 4) ", " V2(128 + f * 4) ", " V2(192 + f * 4) "\n\t" \
              "v_pk_add_f32 " V2(130 + f * 4) ", " V2(130 + f * 4) ", " V2(194 + f * 4) "\n\ts_mov_b64 exec, -1\n\t" \
              "buffer_store_dwordx4 " VR(128 + f * 4) ", %[vo], s[88:91], %[dso] offen sc0 sc1\n\ts_add_u32 %[dso], %[dso], %[ds]\n\t"
#define WTW "s_waitcnt vmcnt(" S(WIN) ")\n\t"
#define SL(f) WTW ".if " #f " == " S(WIN) " / 3\n\t" FLAG ".endif\n\t" ST(f) LD(f)
#define SE(f) "s_waitcnt vmcnt(45 - 2 * " #f ")\n\t.if " #f " == 15\n\t" FLAG ".endif\n\t" ST(f)
#define L0(f) WTW "buffer_load_dword %[vd], off, %[rf], 0\n\t" LD(f)
#define FLAG "s_mov_b64 exec, %[pm]\n\ts_cbranch_execz 2\n\tbuffer_store_dword %[pfv], %[vf], %[rf], %[pfo] offen sc0 sc1\n\ts_mov_b64 exec, -1\n\t"
#define NEXTFLAG "v_readlane_b32 %[pfo], %[dfl], %[k]\n\ts_and_b32 %[r], %[r], 0xff\n\ts_or_b32 %[r], %[r], %[tag8]\n\t" \
                 "v_mov_b32 %[pfv], %[r]\n\ts_cmp_lg_u32 %[pfo], 0\n\ts_cselect_b64 %[pm], 1, 0\n\ts_bitcmp1_b32 %[pfo], 0\n\t" \
                 "s_cselect_b64 %[pm], %[m32], %[pm]\n\ts_bitset0_b32 %[pfo], 0\n\ts_mov_b32 %[k], %[k1]\n\t"
#define ASTAMP
#define ASTAMP_OP
#define ASTAMP_CL
            asm volatile("; OPAQUE\n\t"
                         "v_readlane_b32 %[so], %[src], %[k]\n\t"
                         "v_readlane_b32 %[ss], %[sstr], %[k]\n\t"
                         "v_readlane_b32 %[lo], %[dst], %[k]\n\t"
                         "s_mov_b64 %[pm], 0\n\t"
                         "s_mov_b32 %[pfo], 0\n\t"
                         "s_mov_b32 s91, 0x20000\n\t"
                         "s_mov_b32 s87, 0x20000\n\t"
                         "s_mov_b64 s[84:85], %[po]\n\t"
                         "s_cmp_eq_u32 %[ss], 0x400\n\t"
                         "s_cselect_b32 s86, 0, %[szo]\n\t"
                         "s_nop 4\n\t"
                         L0(0) L0(1) L0(2) L0(3) L0(4) L0(5) L0(6) L0(7) L0(8) L0(9) L0(10) L0(11) L0(12) L0(13) L0(14) L0(15)
                         ".Lsend_loop%=:\n\t"
                         "ds_read_b32 %[vt], %[la] offset:4\n\t"
                         "s_add_u32 %[k1], %[k], 1\n\t"
                         "v_mov_b32 %[vd2], %[k1]\n\t"
                         "ds_write_b32 %[lp], %[vd2]\n\t"
                         "v_readlane_b32 %[dso], %[dst], %[k]\n\t"
                         "v_readlane_b32 %[ds], %[dstr], %[k]\n\t"
                         "v_readlane_b32 %[so], %[src], %[k1]\n\t"
                         "v_readlane_b32 %[ss], %[sstr], %[k1]\n\t"
                         "v_readlane_b32 %[lo], %[dst], %[k1]\n\t"
                         "s_cmp_eq_u32 %[ds], 0x400\n\t"
                         "s_cselect_b64 s[88:89], %[pw], %[po]\n\t"
                         "s_cselect_b32 s90, %[szw], %[szo]\n\t"
                         "s_cselect_b64 %[ym], 0, -1\n\t"
                         "s_cmp_eq_u32 %[ss], 0x400\n\t"
                         "s_cselect_b32 s86, 0, %[szo]\n\t"
                         "s_waitcnt lgkmcnt(0)\n\t"
                         "v_readfirstlane_b32 %[rn], %[vt]\n\t"
                         "v_add_u32 %[la], 4, %[la]\n\t"
                         ASTAMP
                         "s_bitcmp1_b32 %[rn], 8\n\t"
                         "s_cbranch_scc0 .Lsend_last%=\n\t"
                         SL(0) SL(1) SL(2) SL(3) SL(4) SL(5) SL(6) SL(7) SL(8) SL(9) SL(10) SL(11) SL(12) SL(13) SL(14) SL(15)
                         NEXTFLAG
                         "s_mov_b32 %[r], %[rn]\n\t"
                         "s_branch .Lsend_loop%=\n\t"
                         ".Lsend_last%=:\n\t"
                         SE(0) SE(1) SE(2) SE(3) SE(4) SE(5) SE(6) SE(7) SE(8) SE(9) SE(10) SE(11) SE(12) SE(13) SE(14) SE(15)
                         NEXTFLAG
                         WT(0)
                         FLAG
                         "s_nop 1"
                         : "=&{v[128:159]}"(r0), "=&{v[160:191]}"(r1), "=&{v[192:223]}"(r2), "=&{v[224:255]}"(r3), [lo] "=&s"(lo), [ym] "=&s"(ym),
                           [la] "+v"(la), [vt] "=&v"(vt), [vd] "=&v"(vd), [vd2] "=&v"(vd2), [pfv] "=&v"(pfv), [so] "=&s"(so), [ss] "=&s"(ss), [dso] "=&s"(dso), [ds] "=&s"(ds),
                           [k1] "=&s"(k1), [rn] "=&s"(rn), [pfo] "=&s"(pfo), [pm] "=&s"(pm), [k] "+s"(k), [r] "+s"(r) ASTAMP_OP
                         : [vo] "v"(vo), [vf] "v"(lane * fs), [m32] "s"(0xFFFFFFFFull), [lp] "v"((unsigned)(uintptr_t)lds + 4 * ITEM + 252), [src] "v"(src), [dst] "v"(dst), [dfl] "v"(dfl), [sstr] "v"(sstr), [dstr] "v"(dstr),
                           [rs] "s"(rpw), [rf] "s"(rws), [pw] "s"(myws), [po] "s"(po), [szw] "s"(STAMP), [szo] "s"(MSH * NCOL * 4), [tag8] "s"(tag << 8)
                         : "memory", "scc", "s84", "s85", "s86", "s87", "s88", "s89", "s90", "s91" ASTAMP_CL);
            STAMP_AT(63 + k);
        }
        STAMP_OUT();
        __builtin_amdgcn_endpgm();
    }

    const bool own = g >= 12;
    const int o = g - 12, omap = P.omap;
    const int fl = xcd >> 1, fh = xcd & 1;
    const int rb = own ? (omap ? (o & 3) : (xcd & 1) * 2 + (o & 1)) : 4 + fl;
    const int grb = own ? rank * 4 + rb : ((rank ^ rb) & 7) * 4 + pos;
    const int njob = g < 4 || own ? 2 : 1;
    auto tile = [&](int job) {
        return own ? (omap ? xcd * 4 + (o >> 2) : (xcd >> 1) * 8 + (o >> 1)) : (job ? 3 : g >> 2) * 8 + fh * 4 + (g & 3);
    };
    if (wave == 4) {
        auto rws = __builtin_amdgcn_make_buffer_rsrc(myws, 0, (int)STAMP, 0x00020000);
        const unsigned gate = P.gate[min(g >> 2, 3)];
        while ((unsigned)__builtin_amdgcn_s_memrealtime() - tstart < gate) __builtin_amdgcn_s_sleep(8);
        unsigned kx = 0;
#pragma unroll 1
        for (int job = 0; job < njob; ++job) {
            const int t = tile(job);
            const int rot = (g * P.rot) & 31, fastu = P.fastu, slp0 = P.slp0, slp1 = P.slp1;
            const int ls = lane & 31, lh = lane >> 5;
            const bool mini = own && !job;
            const unsigned wid = 256 + (((rank >> 2) ^ mini) * 32 + t) * 4 + (ls & 3);
            const unsigned skip = mini ? ~(15u << rank * 4) : own ? 0 : 15u << (rank ^ rb) * 4;
            const bool off = skip >> ls & 1;
            unsigned fxo = off || kx ? STAMP : pb + (own && job ? OX + (rb * 32 + t) * fs : CTRL + rb * fs) + lane * 4;
            unsigned fwo = off || lh ? STAMP : pb + (mini ? rdy_of(wid) : CTRL + flag_of_slot(512 + t * 32 + ls, fs));
            const bool loc = (ls >> 2) == rank;
            const unsigned offa = pb + (loc ? STG + ((grb * 4 + (ls & 3)) * 2 + lh) * ITEM : ((rb * 32 + ls) * 2 + lh) * ITEM);
            const unsigned offb = pb + (loc ? STG + wid * ITEM : XR_BYTES + (t * 32 + ls) * ITEM);
            unsigned issued = skip, xa, wa, xb, wb, kw = 0;
            int u = __builtin_popcount(skip);
            bool dead = false;
            u64 t0 = __builtin_amdgcn_s_memrealtime();
#define POLL(c, fx, fw, so) asm volatile("; POLL\n\tbuffer_load_dword %0, %2, %4, %5 offen sc0 sc1\n\tbuffer_load_dword %1, %3, %4, %5 offen sc0 sc1" \
                                  : c(fx), c(fw) : "v"(fxo), "v"(fwo), "s"(rws), "s"(so) : "memory")
            auto tick = [&](unsigned& px_, unsigned& pw_, bool second) {
                asm volatile("; GOT\n\ts_waitcnt vmcnt(2)" : "+v"(px_), "+v"(pw_) :: "memory");
                if (px_ + ~(tag << 8) < 255u) { kx = px_; fxo = STAMP; }
                if (pw_ + ~(tag << 8) < 255u) { kw = pw_; fwo = STAMP; }
                const u64 bx = __ballot(kx != 0);
                unsigned rdy = (unsigned)bx & (unsigned)(bx >> 32) & (unsigned)__ballot(kw != 0);
                rdy |= skip;
                if (dead) rdy = ~0u;
                unsigned avail = rdy & ~issued;
                const bool slow = u < fastu || mini;
                if (!avail && !(slow && second)) {
                    const int n = slow ? slp0 : slp1;
                    for (int i = 0; i < n; ++i) __builtin_amdgcn_s_sleep(2);
                    dead = spin_pause(t0, tmo, call);
                }
                POLL("+v", px_, pw_, slow && second ? STAMP : 0u);
                if (!avail) return;
                do {
                    const int s = (__builtin_ctz(__builtin_amdgcn_alignbit(avail, avail, rot)) + rot) & 31;
                    issued |= 1u << s;
                    avail &= ~(1u << s);
                    const u4 rec = {offa, offb, (kx & 255u) + ((unsigned)__builtin_amdgcn_readlane((int)kw, s) & 255u) - 64u, dead};
                    if (ls == s) *(volatile __attribute__((address_space(3))) u4*)(ctl + (u & 1) * 8 + lh * 4) = rec;
                    STAMP_AT(job * 32 + u);
                    STAMP_V(64 + job * 32 + u, s);
                    BAR();
                    ++u;
                } while (__builtin_popcount(avail) > (rdy == ~0u ? 0 : 2));
                t0 = __builtin_amdgcn_s_memrealtime();
            };
            POLL("=&v", xa, wa, 0u);
            POLL("=&v", xb, wb, STAMP);
            for (;;) {
                tick(xa, wa, false);
                if (u == 32) break;
                tick(xb, wb, true);
                if (u == 32) break;
            }
            BAR(); BAR(); BAR();
            asm volatile("s_waitcnt vmcnt(0)" :: "v"(xa), "v"(wa), "v"(xb), "v"(wb) : "memory");
            if (own && job) break;
            BAR();
            if (lane == 0) __hip_atomic_store((unsigned*)(myws + pb + (own ? LRDY : YRDY)) + (rb & 3) * 32 + t, tag << 8, RLX_SYS);
        }
        STAMP_OUT();
        __builtin_amdgcn_endpgm();
    }
#pragma unroll 1
    for (int job = 0; job < njob; ++job) {
        const int t = tile(job);
        f32v acc, d1, d2, d3, d4, d5;
        u4 dc;
        int ecur, emax, cnt, e0, e1, e2, e3, ta, tb;
        unsigned keep;
        u64 ret;
        const int rs = own ? NCOL * 4 : 8192;
        char* ob = own ? (char*)P.out + (size_t)(rb * 128 + wave * 32) * (NCOL * 4) + ((rank >> 2) ^ job ^ 1) * 8192 + t * 256
                       : myws + (unsigned)uni((int)ctl[C_CALL + 2]) + YST + (size_t)((rb & 3) * 128 + wave * 32) * 8192 + t * 256;
        auto ro = __builtin_amdgcn_make_buffer_rsrc(ob, 0, 31 * rs + 256, 0x00020000);
#define ACC(i, nf) VR(64 + i * 16 + nf * 4)
#define BF(ks, nf) VR(96 + (ks & 1) * 16 + nf * 4)
#define AF(c, f) VR(128 + c * 32 + (f) * 4)
#define RDB(c, ks, nf) "ds_read_b128 " BF(ks, nf) ", %[vl] offset:" S(c * 16384 + ks * 4096 + nf * 1024) "\n\t"
#define RDB8(c) RDB(c, 0, 0) RDB(c, 0, 1) RDB(c, 0, 2) RDB(c, 0, 3) RDB(c, 1, 0) RDB(c, 1, 1) RDB(c, 1, 2) RDB(c, 1, 3)
#define MM(c, ks, nf, w) "s_waitcnt lgkmcnt(" #w ")\n\t" \
        "v_mfma_f32_16x16x32_f16 " ACC(0, nf) ", " AF(c, ks) ", " BF(ks, nf) ", " ACC(0, nf) "\n\t" \
        "v_mfma_f32_16x16x32_f16 " ACC(1, nf) ", " AF(c, 4 + ks) ", " BF(ks, nf) ", " ACC(1, nf) "\n\t"
#define LDB(j) "global_load_lds_dwordx4 v61, %[ws] offset:" S(j * 1024) " sc0 sc1\n\t"
#define LDA(n, f) "global_load_dwordx4 " AF(n, f) ", v60, %[ws] offset:" S((f) * 1024 - 4096) " sc0 sc1\n\t"
#define LD12(n) LDB(0) LDB(1) LDB(2) LDB(3) LDA(n, 0) LDA(n, 4) LDA(n, 1) LDA(n, 5) LDA(n, 2) LDA(n, 6) LDA(n, 3) LDA(n, 7)
#define REC(n) "ds_read_b96 v[60:62], %[vc] offset:" S((n & 1) * 32) "\n\t"
#define ADR(n) "s_add_u32 m0, %[lb], " S(n * 16384) "\n\tv_add_u32 v60, v60, %[voa]\n\tv_add_u32 v61, v61, %[vob]\n\tv_readfirstlane_b32 %[e" #n "], v62\n\t"
#define PRO(n) "s_barrier\n\t" REC(n) "s_waitcnt lgkmcnt(0)\n\t" ADR(n) LD12(n)
#define CHK(c, id) "s_cmp_lg_u32 %[e" #c "], %[ecur]\n\ts_cbranch_scc1 .Lfix" #id "_%=\n\t.Lback" #id "_%=:\n\t"
#define FIX(c, id, pre) ".Lfix" #id "_%=:\n\t" pre "s_call_b64 %[ret], .Lslow" #c "_%=\n\ts_branch .Lback" #id "_%=\n\t"
#define STEP(c, n) "s_waitcnt vmcnt(24)\n\ts_barrier\n\t" REC(n) RDB8(c) CHK(c, c) "s_waitcnt lgkmcnt(8)\n\t" ADR(n) \
        MM(c, 0, 0, 7) RDB(c, 2, 0) LDB(0) MM(c, 0, 1, 7) RDB(c, 2, 1) LDB(1) MM(c, 0, 2, 7) RDB(c, 2, 2) LDB(2) MM(c, 0, 3, 7) RDB(c, 2, 3) \
        MM(c, 1, 0, 7) RDB(c, 3, 0) LDB(3) MM(c, 1, 1, 7) RDB(c, 3, 1) LDA(n, 0) MM(c, 1, 2, 7) RDB(c, 3, 2) LDA(n, 4) MM(c, 1, 3, 7) RDB(c, 3, 3) \
        MM(c, 2, 0, 7) LDA(n, 1) MM(c, 2, 1, 6) LDA(n, 5) MM(c, 2, 2, 5) LDA(n, 2) MM(c, 2, 3, 4) \
        MM(c, 3, 0, 3) LDA(n, 6) MM(c, 3, 1, 2) LDA(n, 3) MM(c, 3, 2, 1) LDA(n, 7) MM(c, 3, 3, 0)
#define MMS(c, w0, w1, w2, w3) \
        w0 MM(c, 0, 0, 7) RDB(c, 2, 0) MM(c, 0, 1, 7) RDB(c, 2, 1) MM(c, 0, 2, 7) RDB(c, 2, 2) MM(c, 0, 3, 7) RDB(c, 2, 3) \
        w1 MM(c, 1, 0, 7) RDB(c, 3, 0) MM(c, 1, 1, 7) RDB(c, 3, 1) MM(c, 1, 2, 7) RDB(c, 3, 2) MM(c, 1, 3, 7) RDB(c, 3, 3) \
        w2 MM(c, 2, 0, 7) MM(c, 2, 1, 6) MM(c, 2, 2, 5) MM(c, 2, 3, 4) w3 MM(c, 3, 0, 3) MM(c, 3, 1, 2) MM(c, 3, 2, 1) MM(c, 3, 3, 0)
#define TBAR(n) "s_waitcnt vmcnt(" #n ")\n\ts_cbranch_execz 1\n\ts_barrier\n\t"
#define ZERO(r) "v_mov_b32 v[" S(r) "], 0\n\t"
#define BAD(r) "v_mov_b32 v[" S(r) "], -1\n\t"
#define LDX(r) "v_ldexp_f32 v[" S(r) "], v[" S(r) "], %[tb]\n\t"
#define PKM(r) "v_pk_mul_f16 v[" S(r) "], v[" S(r) "], v63 op_sel_hi:[1,0]\n\t"
#define SLOW(c) ".Lslow" #c "_%=:\n\ts_max_i32 %[emax], %[emax], %[e" #c "]\n\ts_add_i32 %[ta], %[emax], -7\n\ts_max_i32 %[ta], %[ta], %[e" #c "]\n\t" \
        "s_sub_i32 %[tb], %[ecur], %[ta]\n\ts_lshl_b32 %[tb], %[tb], 3\n\ts_mov_b32 %[ecur], %[ta]\n\t" R32(LDX, 64) \
        "s_sub_i32 %[tb], %[e" #c "], %[ta]\n\ts_lshl_b32 %[tb], %[tb], 3\n\tv_ldexp_f32 v63, 1.0, %[tb]\n\tv_cvt_f16_f32 v63, v63\n\ts_nop 0\n\t" \
        R32(PKM, 128 + c * 32) "s_setpc_b64 %[ret]\n\t"
#define EP1(i, j, c) "v_ldexp_f32 v[" S(128 + (i * 4 + j) * 4 + c) "], v[" S(64 + i * 16 + c * 4 + j) "], %[tb]\n\t"
#define EP8(c) EP1(0, 0, c) EP1(0, 1, c) EP1(0, 2, c) EP1(0, 3, c) EP1(1, 0, c) EP1(1, 1, c) EP1(1, 2, c) EP1(1, 3, c)
#define EPS(i, step) "buffer_store_dwordx4 " VR(128 + i * 4) ", %[vo], %[ro], %[ta] offen sc0 sc1\n\ts_add_u32 %[ta], %[ta], " step "\n\t"
#define GSTAMP(n)
#define GSTAMP_OP
        asm volatile("; OPAQUE\n\t"
                     "s_mov_b32 %[keep], m0\n\t"
                     "s_mov_b32 %[cnt], 0\n\t"
                     "s_mov_b32 %[ecur], 0\n\ts_mov_b32 %[e0], 0\n\ts_mov_b32 %[e1], 0\n\ts_mov_b32 %[e2], 0\n\ts_mov_b32 %[e3], 0\n\t"
                     "s_mov_b64 exec, 0\n\t"
                     "s_branch .Lgemm_tail%=\n\t"
                     SLOW(0) SLOW(1) SLOW(2) SLOW(3)
                     FIX(0, 0, "") FIX(1, 1, "") FIX(2, 2, "") FIX(3, 3, "")
                     FIX(0, 4, "") FIX(1, 5, "") FIX(2, 6, "") FIX(3, 7, "s_waitcnt vmcnt(0)\n\t")
                     ".Lgemm_bad%=:\n\t"
                     "s_nop 7\n\t"
                     R32(BAD, 64)
                     "s_branch .Lgemm_ep%=\n\t"
                     ".Lgemm_go%=:\n\t"
                     "s_mov_b64 exec, -1\n\t"
                     R32(ZERO, 64)
                     PRO(0) PRO(1) PRO(2)
                     "s_mov_b32 %[ecur], %[e0]\n\t"
                     "s_mov_b32 %[emax], %[e0]\n\t"
                     "s_mov_b32 %[cnt], %[cnt0]\n\t"
                     "s_cmp_eq_u32 %[cnt], 7\n\t"
                     "s_cbranch_scc1 .Lgemm_tail%=\n\t"
                     ".Lgemm%=:\n\t"
                     STEP(0, 3) STEP(1, 0) STEP(2, 1) STEP(3, 2)
                     "s_add_u32 %[cnt], %[cnt], 1\n\t"
                     "s_cmp_lt_u32 %[cnt], 7\n\t"
                     "s_cbranch_scc1 .Lgemm%=\n\t"
                     ".Lgemm_tail%=:\n\t"
                     TBAR(24) "ds_read_b128 v[60:63], %[vc] offset:32\n\t" RDB8(0)
                     "s_waitcnt lgkmcnt(8)\n\t"
                     "s_cbranch_execz .Lgemm_t1%=\n\t"
                     ADR(3) LD12(3)
                     "v_readfirstlane_b32 %[cnt], v63\n\t"
                     ".Lgemm_t1%=:\n\t"
                     CHK(0, 4) MMS(0, "", "", "", "")
                     TBAR(24) RDB8(1) CHK(1, 5) MMS(1, "", "", "", "")
                     TBAR(12) RDB8(2) CHK(2, 6) MMS(2, "", "", "", "")
                     TBAR(8) RDB8(3) CHK(3, 7) MMS(3, WT(6), WT(4), WT(2), WT(0))
                     GSTAMP(0)
                     "s_cmp_lg_u32 %[cnt], 0\n\t"
                     "s_cbranch_scc1 .Lgemm_bad%=\n\t"
                     ".Lgemm_ep%=:\n\t"
                     "s_lshl_b32 %[tb], %[ecur], 3\n\t"
                     "s_mov_b32 %[ta], 0\n\t"
                     "s_mul_i32 %[emax], %[rs], 13\n\t"
                     EP8(0) EP8(1) EP8(2) EP8(3)
                     "s_cbranch_execz .Lgemm_go%=\n\t"
                     EPS(0, "%[rs]") EPS(1, "%[rs]") EPS(2, "%[rs]") EPS(3, "%[emax]") EPS(4, "%[rs]") EPS(5, "%[rs]") EPS(6, "%[rs]") EPS(7, "%[rs]")
                     GSTAMP(1)
                     "s_mov_b32 m0, %[keep]\n\t"
                     "s_waitcnt lgkmcnt(0)"
                     : "=&{v[64:95]}"(acc), "=&{v[96:127]}"(d1), "=&{v[128:159]}"(d2), "=&{v[160:191]}"(d3),
                       "=&{v[192:223]}"(d4), "=&{v[224:255]}"(d5), "=&{v[60:63]}"(dc),
                       [ecur] "=&s"(ecur), [emax] "=&s"(emax), [cnt] "=&s"(cnt), [e0] "=&s"(e0), [e1] "=&s"(e1), [e2] "=&s"(e2), [e3] "=&s"(e3),
                       [ta] "=&s"(ta), [tb] "=&s"(tb), [keep] "=&s"(keep), [ret] "=&s"(ret) GSTAMP_OP
                     : [vl] "v"((unsigned)(uintptr_t)lds + lane * 16), [vc] "v"((unsigned)(uintptr_t)lds + 4 * ITEM + (wave >> 1) * 16),
                       [voa] "v"(lane * 16 + 4096 + (wave & 1) * 8192), [vob] "v"(lane * 16 + wave * 4096),
                       [lb] "s"((unsigned)(uintptr_t)lds + wave * 4096), [ws] "s"(myws),
                       [vo] "v"(q4 * 4 * rs + c16 * 16), [ro] "s"(ro), [rs] "s"(rs), [cnt0] "s"(!own ? 1 : job ? 0 : 7)
                     : "memory", "scc");
        if (!(own && job)) asm volatile("s_waitcnt vmcnt(0)\n\ts_barrier" :: "v"(d2) : "memory");
        STAMP_V(job * 4 + 0, (unsigned)t0);
        STAMP_V(job * 4 + 1, (unsigned)t1);
    }
    STAMP_OUT();
}

struct Tune { int g0 = 0, g1 = 1800, g2 = 2600, g3 = 2050, cyc = 0, flip = 0, xc = 2, nfast = 28, stag2 = 860, pref = 1, ahead = 2, wide = 4, nhead = 12, hfree = 2, hstag = 430, ht0 = 160, hlead = 200, tgo = 315, gstag = 0, stag0 = 40, stag = 430, fs = 4352, rot = 3, xrot = 1, omap = 0, fastu = 20, slp0 = 20, slp1 = 2, nf = 6, xe = 0, near = 1, wsplit = 0; };

void build_lists(const Tune& T, uint16_t (&L)[3][MAXITEMS], int (&len)[3]) {
    const int nf = T.nf;
    for (int c = 0; c < 3; ++c) {
        int n = 0;
        auto X = [&](int a0, int i0 = 0, int i1 = 32) { for (int i = i0; i < i1; ++i) L[c][n++] = (uint16_t)(0 << 8 | (a0 + (i & 7)) << 2 | i >> 3); };
        auto W = [&](int t0, int t1) { for (int q = 0; q < 4; ++q) for (int t = t0; t < t1; ++t) L[c][n++] = (uint16_t)(1 << 8 | t << 2 | q); };
        if (c == 0) { X(8); W(0, 8); W(8, 16); W(16, 24); W(24, 32); X(0); }
        if (c == 1) {
            X(8); W(0, 8);
            for (int s = 0; s < 4; ++s) for (int t = 0; t < nf; ++t) L[c][n++] = (uint16_t)(2 << 8 | t << 2 | s);
            W(8, 16); W(16, 24); X(0, 0, T.xe);
            if (T.wsplit) { W(24, 28); W(28, 32); } else W(24, 32);
            X(0, T.xe, 32);
        }
        if (c == 2) {
            for (int t = 0; t < nf; ++t) L[c][n++] = (uint16_t)(1 << 8 | t << 2 | 0);
            for (int i = 0; i < 24; ++i) L[c][n++] = (uint16_t)(0 << 8 | (10 + i % 6) << 2 | i / 6);
            if (nf < 8) W(nf, 8);
            for (int i = 3; i >= 0; --i) { if (T.xc == i) X(0); if (i) W(32 - 8 * i, 40 - 8 * i); }
            for (int i = 0; i < 64; ++i) L[c][n++] = (uint16_t)(3 << 8 | i);
        }
        len[c] = n;
        while (n < MAXITEMS) L[c][n++] = 0xFFFF;
    }
}

}

extern "C" void kernel_launch(int rank, int nranks, ncclComm_t comm,
                              hipStream_t stream,
                              void* const* d_in, const int* in_sizes, int n_in,
                              void* d_out, int out_size,
                              const int* in_dtypes, int out_dtype,
                              void* const* d_out_all,
                              void* const* d_ws_all, size_t ws_bytes) {
    (void)comm; (void)in_dtypes; (void)out_dtype; (void)d_out_all;
    if (nranks != NR || (unsigned)rank >= (unsigned)NR || n_in < 2 || ws_bytes < WS_NEED ||
        in_sizes[0] != 4096 * KSH || in_sizes[1] != KSH * NCOL || out_size != MSH * NCOL) {
        fprintf(stderr, "kernel_launch: unexpected problem shape\n");
        return;
    }
    const Tune T;
    uint16_t L[3][MAXITEMS];
    int len[3];
    build_lists(T, L, len);
    static const unsigned char upper[NR] = {0xB2, 0x8E, 0xE8, 0xD4, 0xD4, 0xE8, 0x8E, 0xB2};
    int xcd_of[NR], link_of[NR];
    for (int l = 0, lo = 0, hi = 4; l < NR; ++l) {
        xcd_of[l] = !T.near ? l : (upper[rank] >> l) & 1 ? hi++ : lo++;
        xcd_of[l] = (xcd_of[l] & 4) | (((xcd_of[l] ^ T.flip) + T.cyc) & 3);
        link_of[xcd_of[l]] = l;
    }
    Params P;
    P.x = (const float*)d_in[0];
    P.w = (const float*)d_in[1];
    P.out = (float*)d_out;
    P.links = 0;
    for (int j = 0; j < NR; ++j) {
        P.ws[j] = (char*)d_ws_all[rank ^ j];
        P.links |= (unsigned)link_of[j] << j * 4;
    }
    P.rank = rank;
    P.fs = T.fs; P.nhead = T.nhead;
    P.rot = T.rot; P.xrot = T.xrot; P.omap = T.omap; P.fastu = T.fastu; P.slp0 = T.slp0; P.slp1 = T.slp1; P.ahead = T.ahead; P.wide = T.wide;
    P.gate[0] = T.g0; P.gate[1] = T.g1; P.gate[2] = T.g2; P.gate[3] = T.g3;
    for (int c = 0; c < 2; ++c) for (int i = 0; i < MAXITEMS; ++i) P.list[c][i] = L[c + 1][i];
    if (T.nf > 8 || len[2] - T.nhead > 62 * NSEND || T.nhead > NWORK || 24 * T.fs > (int)(YRDY - CTRL) || 128 * T.fs > (int)(BUF_BYTES - OX)) { fprintf(stderr, "bad tuning\n"); return; }
    auto push_at = [&](int p) { return p < T.hfree ? 0 : T.ht0 + (p + 1 - T.hfree) * T.hstag / 16; };
    auto times = [&](int ld, int st) { return (unsigned)(ld > 0 ? ld / 8 : 0) << 16 | (unsigned)(st / 4) << 24; };
    const int nb = 64 - T.nhead;
    int word[64], freeat[32], cnt[32];
    unsigned when[64];
    for (int s = NSEND; s < 32; ++s) { cnt[s] = s < NSEND + T.nhead; freeat[s] = cnt[s] ? (s < NSEND + T.hfree ? T.ht0 : push_at(s - NSEND)) + 100 : 0; }
    for (int i = 0; i < nb; ++i) {
        const int st = i < NSEND ? T.tgo + i * T.gstag / 16 : 0;
        const int ld = i < NSEND ? st - T.hlead : T.stag0 + (i - NSEND) * T.stag / 16 + (i > T.nfast ? (i - T.nfast) * (T.stag2 - T.stag) / 16 : 0);
        if (ld > 2040) { fprintf(stderr, "bad tuning\n"); return; }
        int s = -1;
        for (int c = NSEND; c < 32; ++c)
            if (cnt[c] < (i >= nb - 8 ? 3 : 2) && (i < nb - 8 || c >= 24) && (s < 0 || freeat[c] < freeat[s]) && !(T.pref && s >= 0 && freeat[s] <= ld)) s = c;
        word[i] = cnt[s] < 2 ? s * 16 + cnt[s] : (s - 24) * 8;
        when[i] = times(ld, st);
        freeat[s] = (ld > freeat[s] ? ld : freeat[s]) + 200;
        ++cnt[s];
    }
    bool taken[512] = {};
    int next = 0;
    for (int p = 0; p < MAXITEMS; ++p)
        for (int x = 0; x < 8; ++x) {
            const int l = link_of[x];
            const int code = L[l == 0 ? 0 : 1 + (l >> 2)][p];
            if ((code >> 8) > 1) { if (p < T.nhead) { fprintf(stderr, "bad list head\n"); return; } continue; }
            const int id = staged_id(code, rank ^ l, l, T.xrot);
            if (taken[id]) { if (p < T.nhead) { fprintf(stderr, "bad list head\n"); return; } continue; }
            taken[id] = true;
            const unsigned c = id < 256 ? (id >> 3) << 10 | (id & 1) << 9 | (id >> 1 & 3) << 1 : 0x8000 | (id & 3) << 13 | (id >> 2 & 63);
            if (p < T.nhead) P.src[((NSEND + p) * 8 + x) * 2] = c | times(push_at(p) - T.hlead, l ? push_at(p) : 0);
            else {
                const int w = word[next >> 3];
                P.src[w + (next & 7) * (w < 64 ? 1 : 2)] = c | when[next >> 3];
                ++next;
            }
        }
    if (next != 8 * nb) { fprintf(stderr, "bad conversion table\n"); return; }
    gemm_rs_kernel<<<8 * (NWORK + NSEND), 384, 0, stream>>>(P);
    HIP_CHECK(hipGetLastError());
}
```

```cpp
#include <hip/hip_runtime.h>
#include <rccl/rccl.h>
#include <cstdio>
#include <cstdint>
#include <cstddef>

#define HIP_CHECK(expr) do { hipError_t _e = (expr); if (_e != hipSuccess) { \
    fprintf(stderr, "HIP error %s at %s:%d\n", hipGetErrorString(_e), __FILE__, __LINE__); \
    return; } } while (0)

#define XMASK 7
#define WIN 35
#define WSPR 7

namespace {

typedef _Float16 h8 __attribute__((ext_vector_type(8)));
typedef float f4 __attribute__((ext_vector_type(4)));
typedef unsigned u4 __attribute__((ext_vector_type(4)));
typedef unsigned long long u64;

constexpr int NR = 8;
constexpr int NCOL = 4096;
constexpr int KSH = 512;
constexpr int MSH = 512;
constexpr int NWORK = 28, NSEND = 4;

constexpr unsigned ITEM = 16384;
constexpr unsigned XR_BYTES = 8 * 32 * 2 * ITEM;
constexpr unsigned WR_BYTES = 32 * 32 * ITEM;
constexpr unsigned STG = XR_BYTES + WR_BYTES;
constexpr unsigned CTRL = STG + 512 * ITEM;
constexpr unsigned YRDY = CTRL + 110592;
constexpr unsigned LRDY = YRDY + 2048;
constexpr unsigned PACE = LRDY + 1024;
constexpr unsigned RDY = CTRL + 114688;
constexpr unsigned YST = CTRL + 131072;
constexpr unsigned OX = YST + 256 * ITEM;
constexpr unsigned BUF_BYTES = OX + 40 * ITEM;
constexpr unsigned EPOCH = 2 * BUF_BYTES;
constexpr unsigned TMO = EPOCH + 8192;
constexpr unsigned STAMP = TMO + 4096;
constexpr unsigned WS_NEED = STAMP + (1 << 20);

constexpr u64 DEADLINE_TICKS = 100000000ull;

constexpr int MAXITEMS = 256;

struct Params {
    unsigned src[512];
    uint16_t list[2][MAXITEMS];
    const float* x;
    const float* w;
    int rot, xrot, omap, fastu;
    char* ws[NR];
    float* out;
    int rank;
    unsigned links;
    int fs;
    int nhead;
    int slp0, slp1, ahead, wide;
    int gate[4];
    int dyn[2];
    int kmeas, kplan, base, gain;
};
static_assert(offsetof(Params, list) == 0x800 && offsetof(Params, x) == 0xC00 && offsetof(Params, ws) == 0xC20 && sizeof(Params) <= 4096, "layout known to the assembly");

__host__ __device__ inline int staged_id(int code, int peer, int link, int xrot) {
    const int a = (code >> 2) & 31, qa = (code & 3) ^ (peer & 3);
    if ((code >> 8) == 0) {
        const int rb = a >> 1;
        const int grb = rb < 4 ? peer * 4 + ((rb + link * xrot) & 3) : ((peer ^ rb ^ (link & 3)) & 7) * 4 + (peer & 3);
        return (grb * 4 + qa) * 2 + (a & 1);
    }
    return 256 + ((peer >> 2) * 32 + a) * 4 + qa;
}
__host__ __device__ inline int recv_slot(int id, int src, int dst) {
    if (id < 256) {
        const int grb = id >> 3, qa = (id >> 1) & 3, o = grb >> 2, p = grb & 3;
        const int rb = o == dst ? p : o ^ dst;
        if (o != dst && (rb < 4 || p != (dst & 3))) return -1;
        return (rb * 32 + src * 4 + qa) * 2 + (id & 1);
    }
    const int ct = (id - 256) >> 2;
    if ((ct >> 5) != (dst >> 2)) return -1;
    return 512 + (ct & 31) * 32 + src * 4 + (id & 3);
}
__host__ __device__ inline unsigned flag_of_slot(int slot, int fs) {
    const int i = slot < 512 ? (slot >> 6) * 64 + (slot & 1) * 32 + ((slot >> 1) & 31) : slot;
    return (i >> 6) * fs + (i & 63) * 4;
}

__host__ __device__ inline unsigned rdy_of(unsigned id) {
    return RDY + (id < 256 ? (id & 135) | (id & 24) << 2 | (id & 96) >> 2 : id) * 4;
}

#define RLX_SYS __ATOMIC_RELAXED, __HIP_MEMORY_SCOPE_SYSTEM
#define RLX_AGT __ATOMIC_RELAXED, __HIP_MEMORY_SCOPE_AGENT
constexpr int AUX_SYS = 17;

__device__ __forceinline__ int uni(int v) { return __builtin_amdgcn_readfirstlane(v); }

#define STAMP_V(slot, val) do { } while (0)
#define STAMP_AT(slot) do { } while (0)
#define STAMP_OUT() do { } while (0)

__device__ __forceinline__ bool spin_pause(u64 t0, unsigned* tmo, unsigned call) {
    __builtin_amdgcn_s_sleep(2);
    u64 dt = __builtin_amdgcn_s_memrealtime() - t0;
    if (dt > 50000ull) {
        if (dt > DEADLINE_TICKS || (unsigned)uni((int)__hip_atomic_load(tmo, RLX_AGT)) == call) {
            __hip_atomic_store(tmo, call, RLX_AGT);
            return true;
        }
        __builtin_amdgcn_s_sleep(32);
    }
    return false;
}

#define BAR() asm volatile("s_waitcnt lgkmcnt(0)\n\ts_barrier" ::: "memory")

constexpr unsigned RDY_VALID = 0x100, RDY_END = 0x200, RDY_Y = 0x400;

typedef float f32v __attribute__((ext_vector_type(32)));

constexpr int C_MAX = 64, C_DST = 80, C_CALL = 96, C_FLAG = 128;

#define S_(x) #x
#define S(x) S_(x)
#define VR(b) "v[" S(b) ":" S(b) "+3]"
#define V2(b) "v[" S(b) ":" S(b) "+1]"
#define V1(b) "v[" S(b) "]"
#define WT(n) "s_waitcnt vmcnt(" #n ")\n\t"
#define WL(n) "s_waitcnt lgkmcnt(" #n ")\n\t"
#define R4(M, b) M(b) M(b + 1) M(b + 2) M(b + 3)
#define R16(M, b) R4(M, b) R4(M, b + 4) R4(M, b + 8) R4(M, b + 12)
#define R32(M, b) R16(M, b) R16(M, b + 16)

__global__ void __launch_bounds__(384, 2) gemm_rs_kernel(const Params) {
    __shared__ __attribute__((aligned(1024))) char lds[4 * ITEM + 1024];
    auto ka = (const __attribute__((address_space(4))) Params*)__builtin_amdgcn_kernarg_segment_ptr();
    const int wave = uni(threadIdx.x >> 6);
    const int guess = (blockIdx.x & ~7u) | (__builtin_amdgcn_s_getreg(20 | (3 << 11)) & XMASK);
#define CSTAMP(n)
#define CSTAMP0
#define CSTAMP5
#define CSTAMP_OP
#define CSTAMP_CL
    {
        f32v c0, c1, c2;
#define FL1(b, j, so) "buffer_load_dwordx4 " VR(b + (j) * 4) ", v42, s[52:55], " so " offen\n\t"
#define FL4(b, j, so) FL1(b, j, so) "s_add_u32 " so ", " so ", s60\n\t" FL1(b, j + 1, so) "s_add_u32 " so ", " so ", s60\n\t" \
        FL1(b, j + 2, so) "s_add_u32 " so ", " so ", s60\n\t" FL1(b, j + 3, so)
#define FETCH(b, sr, sh) "s_bitcmp1_b32 " sr ", " S(sh + 15) "\n\ts_cselect_b64 s[52:53], s[42:43], s[40:41]\n\ts_cselect_b32 s58, 14, 11\n\t" \
        "s_cselect_b32 s59, 20, 8\n\ts_bfe_u32 s61, " sr ", " S(0xf0000 + sh) "\n\ts_lshl_b32 s61, s61, 8\n\tv_lshl_or_b32 v42, v40, s58, v41\n\t" \
        "s_lshl_b32 s60, 1, s58\n\ts_lshl_b32 s59, 1, s59\n\ts_add_u32 s59, s59, s61\n\t" FL4(b, 0, "s61") FL4(b, 4, "s59")
#define MX(i) "v_max3_f32 v48, v48, |" V1(128 + (i) * 4) "|, |" V1(128 + (i) * 4 + 1) "|\n\tv_max3_f32 v49, v49, |" V1(128 + (i) * 4 + 2) "|, |" V1(128 + (i) * 4 + 3) "|\n\t"
#define DPP(c) "s_nop 1\n\tv_max_u32_dpp v48, v48, v48 " c "\n\t"
#define MUL(i) "v_pk_mul_f32 " V2(128 + (i) * 2) ", s[62:63], " V2(128 + (i) * 2) " op_sel_hi:[0,1]\n\t"
#define CV(d, a, b) "v_cvt_pk_f16_f32 " V1(d) ", " V1(128 + a) ", " V1(128 + b) "\n\t"
#define CVX(j) CV(112 + (j) * 4, (j) * 4, (j) * 4 + 1) CV(113 + (j) * 4, (j) * 4 + 2, (j) * 4 + 3) CV(114 + (j) * 4, (j) * 4 + 16, (j) * 4 + 17) \
        CV(115 + (j) * 4, (j) * 4 + 18, (j) * 4 + 19) "ds_write_b128 v46, " VR(112 + (j) * 4) " offset:" S((j) * 16) "\n\t"
#define RDX(f) "ds_read_b128 " VR(96 + (f) * 4) ", v47 offset:" S((f) * 1088) "\n\t"
#define CVW(f) CV(96 + (f) * 4, f, f + 4) CV(97 + (f) * 4, f + 8, f + 12) CV(98 + (f) * 4, f + 16, f + 20) CV(99 + (f) * 4, f + 24, f + 28)
#define PUT(a, f) "global_store_dwordx4 " V2(a) ", " VR(96 + f * 4) ", off offset:" S(f * 1024) " sc0 sc1\n\t"
        asm volatile("; OPAQUE\n\t"
                     "s_mov_b32 s56, %[guess]\n\t"
                     "s_cmp_lt_u32 %[wave], 4\n\t"
                     "s_cbranch_scc1 .Lc_conv%=\n\t"
                     "s_cmp_eq_u32 %[wave], 4\n\t"
                     "s_cbranch_scc1 .Lc_end%=\n\t"
                     "s_load_dwordx2 s[40:41], %[ka], 0xc20\n\t"
                     "s_add_u32 s58, %[lb], 0x10000 - 1280\n\t"
                     "v_lshl_add_u32 v37, %[tid], 2, s58\n\t"
                     "v_mov_b32 v48, 0\n\t"
                     "s_mov_b64 exec, 1\n\t"
                     "s_lshl2_add_u32 s58, %[bid], %[epo]\n\t"
                     "v_mov_b32 v40, s58\n\t"
                     "v_mov_b32 v42, 1\n\t"
                     "s_lshl2_add_u32 s58, s56, %[epo]\n\t"
                     "v_mov_b32 v41, s58\n\t"
                     WL(0)
                     "global_load_dword v47, v40, s[40:41] sc1\n\t"
                     ".Lc_claim%=:\n\t"
                     "global_atomic_add v46, v41, v42, s[40:41] offset:1024 sc0 sc1\n\t"
                     "s_add_u32 s58, %[lb], 0x10000\n\t"
                     "v_mov_b32 v36, s58\n\t"
                     "v_mov_b32 v45, s56\n\t"
                     "v_mov_b32 v49, 0\n\t"
                     "v_mov_b32 v50, 0\n\t"
                     "ds_write_b96 v36, v[48:50] offset:400\n\t"
                     "s_mov_b64 exec, -1\n\t"
                     "ds_write_b32 v37, v48\n\t"
                     "s_mov_b64 exec, 1\n\t"
                     WT(0)
                     "v_cmp_eq_u32 vcc, v46, v47\n\t"
                     "s_cbranch_vccz .Lc_taken%=\n\t"
                     ".Lc_won%=:\n\t"
                     "v_add_u32 v47, 1, v47\n\t"
                     "v_lshlrev_b32 v44, 8, v47\n\t"
                     "v_bfe_i32 v46, v47, 0, 1\n\t"
                     "v_and_b32 v46, %[buf], v46\n\t"
                     "ds_write_b128 v36, v[44:47] offset:384\n\t"
                     CSTAMP5
                     "global_store_dword v40, v47, s[40:41] sc1\n\t"
                     "s_mov_b64 exec, -1\n\t"
                     "s_branch .Lc_end%=\n\t"
                     ".Lc_conv%=:\n\t"
                     "s_cmp_ge_u32 s56, 32\n\t"
                     "s_cbranch_scc1 .Lc_retry%=\n\t"
                     "s_endpgm\n\t"
                     ".Lc_retry%=:\n\t"
                     "s_lshl_b32 s58, s56, 3\n\t"
                     "s_load_dwordx2 s[48:49], %[ka], s58\n\t"
                     "s_load_dwordx4 s[40:43], %[ka], 0xc00\n\t"
                     "s_memrealtime s[50:51]\n\t"
                     "s_bfe_u32 s58, s58, 0x80001\n\t"
                     "s_load_dword s44, %[ka], s58\n\t"
                     "s_lshr_b32 s45, s56, 6\n\t"
                     "s_max_u32 s45, s45, 2\n\t"
                     "s_mov_b32 s54, 0x800000\n\t"
                     "s_mov_b32 s55, 0x20000\n\t"
                     "s_mov_b32 s57, 0\n\t"
                     "s_mov_b32 s69, 0\n\t"
                     "v_and_b32 v37, 15, %[tid]\n\t"
                     "v_lshrrev_b32 v40, 2, %[tid]\n\t"
                     "v_lshlrev_b32 v41, 4, v37\n\t"
                     "v_and_b32 v40, 60, v40\n\t"
                     WL(0)
                     CSTAMP0
                     "s_bfe_u32 s58, s48, 0x80010\n\t"
                     "s_cbranch_scc1 .Lc_bar%=\n\t"
                     ".Lc_fetch%=:\n\t"
                     FETCH(128, "s48", 0)
                     "s_cmp_lg_u32 s69, 0\n\t"
                     "s_cbranch_scc1 .Lc_go%=\n\t"
                     ".Lc_bar%=:\n\t"
                     "v_and_b32 v43, 63, %[tid]\n\t"
                     "v_bfe_u32 v38, %[tid], 4, 2\n\t"
                     "s_lshl_b32 s58, %[wave], 12\n\t"
                     "v_lshl_or_b32 v44, v43, 4, s58\n\t"
                     "v_mov_b32 v45, 0\n\t"
                     "v_mov_b32 v63, 0\n\t"
                     "s_mul_i32 s58, %[wave], 4352\n\t"
                     "s_add_u32 s58, s58, %[lb]\n\t"
                     "v_mul_u32_u24 v46, 272, v37\n\t"
                     "v_lshl_add_u32 v46, v38, 6, v46\n\t"
                     "v_add_u32 v46, s58, v46\n\t"
                     "v_mul_u32_u24 v47, 272, v38\n\t"
                     "v_add3_u32 v47, v47, v41, s58\n\t"
                     "s_add_u32 s58, %[lb], 0x10000\n\t"
                     "v_mov_b32 v36, s58\n\t"
                     "v_mov_b32 v39, s58\n\t"
                     "v_lshl_add_u32 v33, v43, 3, s58\n\t"
                     "s_lshl2_add_u32 s58, %[wave], s58\n\t"
                     "v_mov_b32 v35, s58\n\t"
                     "s_barrier\n\t"
                     "ds_read_b96 v[60:62], v36 offset:384\n\t"
                     "ds_read_b64 v[50:51], v33 offset:512\n\t"
                     WL(0)
                     "v_readfirstlane_b32 s66, v61\n\t"
                     "v_add_u32 v44, v44, v62\n\t"
                     "v_cmp_ne_u64 vcc, 0, v[50:51]\n\t"
                     "v_lshl_add_u64 v[50:51], v[62:63], 0, v[50:51]\n\t"
                     "s_cmp_lg_u32 s66, s56\n\t"
                     "s_cbranch_scc1 .Lc_wrong%=\n\t"
                     "s_or_b32 s69, vcc_lo, 0x1000000\n\t"
                     "s_bfe_u32 s58, s48, 0x80010\n\t"
                     "s_cbranch_scc1 .Lc_delay%=\n\t"
                     ".Lc_go%=:\n\t"
                     "ds_read_b128 v[56:59], v39 offset:320\n\t"
                     WL(0)
                     "v_cmp_ne_u64 vcc, v[56:57], v[58:59]\n\t"
                     "v_lshl_add_u64 v[56:57], v[44:45], 0, v[56:57]\n\t"
                     "v_lshl_add_u64 v[58:59], v[44:45], 0, v[58:59]\n\t"
                     WT(0)
                     "v_max3_f32 v48, 0, |v128|, |v129|\n\tv_max3_f32 v49, 0, |v130|, |v131|\n\t"
                     MX(1) MX(2) MX(3) R4(MX, 4)
                     CSTAMP(0)
                     "v_max_u32 v48, v48, v49\n\t"
                     DPP("row_shr:1 row_mask:0xf bank_mask:0xf bound_ctrl:1") DPP("row_shr:2 row_mask:0xf bank_mask:0xf bound_ctrl:1")
                     DPP("row_shr:4 row_mask:0xf bank_mask:0xf bound_ctrl:1") DPP("row_shr:8 row_mask:0xf bank_mask:0xf bound_ctrl:1")
                     DPP("row_bcast:15 row_mask:0xa bank_mask:0xf") DPP("row_bcast:31 row_mask:0xc bank_mask:0xf")
                     "s_mov_b32 exec_lo, 0\n\t"
                     "s_brev_b32 exec_hi, 1\n\t"
                     "v_or_b32 v48, 1, v48\n\t"
                     "ds_write_b32 v35, v48 offset:256\n\t"
                     "s_mov_b64 exec, -1\n\t"
                     ".Lc_spin%=:\n\t"
                     "ds_read_b128 v[52:55], v39 offset:256\n\t"
                     WL(0)
                     "v_min3_u32 v49, v52, v53, v54\n\t"
                     "v_max3_u32 v48, v52, v53, v54\n\t"
                     "v_min_u32 v49, v49, v55\n\t"
                     "v_max_u32 v48, v48, v55\n\t"
                     "s_nop 0\n\t"
                     "v_readfirstlane_b32 s65, v49\n\t"
                     "v_readfirstlane_b32 s64, v48\n\t"
                     "s_cmp_eq_u32 s65, 0\n\t"
                     "s_cbranch_scc1 .Lc_spin%=\n\t"
                     "s_lshr_b32 s64, s64, 23\n\t"
                     "s_addk_i32 s64, 0xff7a\n\t"
                     "s_ashr_i32 s64, s64, 3\n\t"
                     "s_max_i32 s64, s64, -15\n\t"
                     "s_lshl_b32 s65, s64, 26\n\t"
                     "s_sub_u32 s62, 1.0, s65\n\t"
                     R16(MUL, 0)
                     "s_bitcmp1_b32 s48, 15\n\t"
                     "s_cbranch_scc0 .Lc_x%=\n\t"
                     R4(CVW, 0)
                     ".Lc_hold%=:\n\t"
                     "s_lshr_b32 s58, s48, 24\n\t"
                     "s_cbranch_scc1 .Lc_hold2%=\n\t"
                     ".Lc_put%=:\n\t"
                     WL(3) PUT(56, 0) WL(2) PUT(56, 1) WL(1) PUT(56, 2) WL(0) PUT(56, 3)
                     CSTAMP(1)
                     "s_cbranch_vccz .Lc_pub%=\n\t"
                     PUT(58, 0) PUT(58, 1) PUT(58, 2) PUT(58, 3)
                     "s_branch .Lc_pub%=\n\t"
                     ".Lc_x%=:\n\t"
                     R4(CVX, 0) R4(RDX, 0)
                     "s_branch .Lc_hold%=\n\t"
                     ".Lc_hold2%=:\n\t"
                     "s_mul_i32 s59, %[wave], " S(WSPR) "\n\t"
                     "s_lshl2_add_u32 s58, s58, s59\n\t"
                     ".Lc_hold3%=:\n\t"
                     "s_memrealtime s[66:67]\n\t"
                     WL(0)
                     "s_sub_u32 s66, s66, s50\n\t"
                     "s_cmp_ge_u32 s66, s58\n\t"
                     "s_cbranch_scc1 .Lc_put%=\n\t"
                     "s_branch .Lc_hold3%=\n\t"
                     ".Lc_taken%=:\n\t"
                     "global_atomic_sub v41, v42, s[40:41] offset:1024 sc1\n\t"
                     "s_add_u32 s58, s56, 1\n\t"
                     "s_and_b32 s58, s58, 7\n\t"
                     "s_and_b32 s56, s56, 0xf8\n\t"
                     "s_or_b32 s56, s56, s58\n\t"
                     "s_lshl2_add_u32 s58, s56, %[epo]\n\t"
                     "v_mov_b32 v41, s58\n\t"
                     "s_cmp_lg_u32 s56, %[guess]\n\t"
                     "s_cbranch_scc1 .Lc_claim%=\n\t"
                     "v_mov_b32 v45, %[bid]\n\t"
                     "s_branch .Lc_won%=\n\t"
                     ".Lc_wrong%=:\n\t"
                     "s_waitcnt vmcnt(0) lgkmcnt(0)\n\t"
                     "s_mov_b32 s56, s66\n\t"
                     "s_branch .Lc_retry%=\n\t"
                     ".Lc_pub%=:\n\t"
                     "s_add_u32 s64, s64, 32\n\t"
                     "v_or_b32 v61, s64, v60\n\t"
                     "s_lshl_b32 s58, s57, 3\n\t"
                     "s_lshl_b32 s68, 0xff, s58\n\t"
                     "v_lshl_add_u32 v49, s57, 2, v36\n\t"
                     "v_mov_b32 v48, 1\n\t"
                     "v_add_u32 v39, 16, v39\n\t"
                     "v_add_u32 v35, 16, v35\n\t"
                     WT(0)
                     "s_mov_b64 exec, 1\n\t"
                     "ds_add_rtn_u32 v48, v49, v48 offset:400\n\t"
                     CSTAMP(2)
                     WL(0)
                     "v_readfirstlane_b32 s65, v48\n\t"
                     "s_cmp_eq_u32 s65, 3\n\t"
                     "s_cselect_b32 s68, s68, 0\n\t"
                     "s_and_b32 exec_lo, s69, s68\n\t"
                     "s_cbranch_execz 2\n\t"
                     "global_store_dword v[50:51], v61, off sc0 sc1\n\t"
                     "s_mov_b64 exec, -1\n\t"
                     "s_add_u32 s57, s57, 1\n\t"
                     "s_cmp_eq_u32 s57, s45\n\t"
                     "s_cbranch_scc1 .Lc_end%=\n\t"
                     "s_mov_b32 s48, s49\n\t"
                     "s_mov_b32 s49, s44\n\t"
                     "s_bfe_u32 s58, s48, 0x80010\n\t"
                     ".Lc_delay%=:\n\t"
                     "s_memrealtime s[66:67]\n\t"
                     WL(0)
                     "s_sub_u32 s66, s66, s50\n\t"
                     "s_lshr_b32 s66, s66, 3\n\t"
                     "s_cmp_ge_u32 s66, s58\n\t"
                     "s_cbranch_scc1 .Lc_fetch%=\n\t"
                     "s_sleep 1\n\t"
                     "s_branch .Lc_delay%=\n\t"
                     ".Lc_end%=:"
                     : "=&{v[32:63]}"(c0), "=&{v[96:127]}"(c1), "=&{v[128:159]}"(c2),
                       [ka] "+s"(ka) CSTAMP_OP
                     : [tid] "v"(threadIdx.x), [wave] "s"(wave), [guess] "s"(guess),
                       [lb] "s"((unsigned)(uintptr_t)lds), [bid] "s"(blockIdx.x), [epo] "n"(EPOCH), [buf] "n"(BUF_BYTES)
                     : "memory", "scc", "vcc", "s40", "s41", "s42", "s43", "s44", "s45", "s48", "s49", "s50", "s51", "s52", "s53", "s54",
                       "s55", "s56", "s57", "s58", "s59", "s60", "s61", "s62", "s63", "s64", "s65", "s66", "s67", "s68", "s69" CSTAMP_CL);
    }
    const auto& P = *ka;
    const int lane = threadIdx.x & 63;
    const int xcd = blockIdx.x & 7;
    const int slot = blockIdx.x >> 3;
    const int g = slot - NSEND;
    char* const myws = P.ws[0];
    int rank = P.rank;
    const int fs = P.fs, nhead = P.nhead;
    const unsigned links = P.links;
    unsigned it0 = P.src[guess * 2], it1 = P.src[guess * 2 + 1], it2 = P.src[guess & 63];
    asm volatile("s_and_b32 %0, %0, 7" : "+s"(rank) : "s"(myws), "s"(fs), "s"(links), "s"(it0), "s"(it1), "s"(it2), "s"(nhead), "s"(P.ws[1]), "s"(P.ws[2]), "s"(P.ws[3]),
                 "s"(P.ws[4]), "s"(P.ws[5]), "s"(P.ws[6]), "s"(P.ws[7]) : "scc");
    __builtin_assume((unsigned)rank < 8u);
    const int pos = rank & 3;
    unsigned* const tmo = (unsigned*)(myws + TMO);
    typedef volatile __attribute__((address_space(3))) unsigned* ldsu;
    const ldsu ctl = (ldsu)(lds + 4 * ITEM);
    const int c16 = lane & 15, q4 = lane >> 4;
    unsigned call = 0;
    int role = guess, code;
    const unsigned tstart = (unsigned)__builtin_amdgcn_s_memrealtime();

    if (wave >= 4) {
        for (;;) {
            asm volatile("global_load_ushort %0, %1, %2 offset:2048" : "=&v"(code)
                         : "v"(((links >> (role & 7) * 4 & 4) * (MAXITEMS / 4) + min(nhead + slot + NSEND * lane, MAXITEMS - 1)) * 2), "s"(ka) : "memory");
            if (wave == 4) {
                const int link = (unsigned)g < (unsigned)nhead ? (links >> (role & 7) * 4) & 7 : 0, j = lane & 7, d = lane == 2 ? rank ^ link : rank;
                const unsigned c = lane & 16 ? it2 : lane & 8 ? it1 : it0;
                unsigned lo = 0, hi = 0;
#pragma unroll
                for (int i = 0; i < NR; ++i)
                    asm("v_writelane_b32 %0, %2, %4\n\tv_writelane_b32 %1, %3, %4" : "+v"(lo), "+v"(hi) : "s"((unsigned)(u64)P.ws[i]), "s"((unsigned)((u64)P.ws[i] >> 32)), "n"(i));
                const u64 wsl = (u64)(unsigned)__builtin_amdgcn_readlane((int)hi, link) << 32 | (unsigned)__builtin_amdgcn_readlane((int)lo, link);
                const u64 wsp = lane == 2 ? wsl : (u64)myws;
                const bool isw = c & 0x8000u, push = lane == 2 && link;
                const unsigned q = (isw ? c >> 13 : c >> 1) & 3, h = c >> 9 & 1, p = c >> 10 & 3, o = (c >> 12 & 7) ^ d;
                const unsigned id = isw ? 256 + (c & 63) * 4 + q : (c >> 10 & 31) * 8 + q * 2 + h;
                const bool need = isw ? (c >> 5 & 1) == (unsigned)(d >> 2) : (o == 0) | ((o >= 4) & (p == (unsigned)(d & 3)));
                const unsigned blk = isw ? 8 + (c >> 1 & 15) : o ? o : p, k = rank * 4 + q;
                const unsigned sl = blk * 64 + (isw ? (c & 1) * 32 + k : k * 2 + h);
                const unsigned fo = j == 1 ? rdy_of(id) : CTRL + blk * fs + (((isw ? c : h) & 1) * 32 + k) * 4;
                const u64 db = (u64)myws + (STG + id * ITEM);
                typedef u64 u64x2 __attribute__((ext_vector_type(2)));
                if ((lane & 39) == 2) *(volatile __attribute__((address_space(3))) u64x2*)(ctl + C_DST + (lane >> 3) * 4) = u64x2{push ? wsp + sl * ITEM : db, db};
                *(volatile __attribute__((address_space(3))) u64*)(ctl + C_FLAG + lane * 2) = ((j == 0 && need) | (j == 1) | push) ? wsp + fo : 0;
                if (lane < 12) ctl[C_MAX + lane] = 0;
                STAMP_AT(64 + 61);
            }
            BAR();
            asm volatile("s_waitcnt vmcnt(0)" : "+v"(code));
            call = ctl[C_CALL + 3];
            const int r = uni((int)ctl[C_CALL + 1]);
            if (r == role) break;
            role = r;
            it0 = P.src[(r & 255) * 2]; it1 = P.src[(r & 255) * 2 + 1]; it2 = P.src[r & 63];
        }
        if (g >= 0 && wave == 5) { STAMP_OUT(); __builtin_amdgcn_endpgm(); }
    }
    call = (unsigned)uni((int)call);
    const unsigned tag = call & 0xFFFFFFu, pb = (call & 1u) * BUF_BYTES;

    if (g < 0) {
        const int link = (links >> (role & 7) * 4) & 7, peer = rank ^ link;
        STAMP_V(127, (__builtin_amdgcn_s_getreg(63492) >> 8 & 0xff) | (__builtin_amdgcn_s_getreg(20 | (3 << 11)) & 15) << 8 | link << 12);
        if (link == 0) { STAMP_OUT(); __builtin_amdgcn_endpgm(); }
        char* const prws = P.ws[link];
        float* const po = P.out;
        auto rws = __builtin_amdgcn_make_buffer_rsrc(myws, 0, (int)STAMP, 0x00020000);
        auto rpw = __builtin_amdgcn_make_buffer_rsrc(prws, 0, (int)STAMP, 0x00020000);
        typedef volatile __attribute__((address_space(3))) u4* ldsq;
        const ldsq desc = (ldsq)lds;
        if (wave == 5) {
            int type;
            unsigned dst, sfl, lfl;
            auto describe = [&]() {
                unsigned src, dfl;
                type = code >> 8;
                lfl = STAMP;
                if (type == 3) {
                    const unsigned quad = (code >> 3) & 7, rg = code & 7;
                    src = pb + YST + ((link & 3) * 128 + rg * 16) * 8192 + quad * 1024;
                    dst = ((peer & 3) * 128 + rg * 16) * (NCOL * 4) + (peer >> 2) * 8192 + quad * 1024;
                    sfl = pb + YRDY + ((link & 3) * 32 + quad * 4) * 4;
                    lfl = pb + LRDY + ((peer & 3) * 32 + quad * 4) * 4;
                    dfl = 0;
                } else if (type == 2) {
                    const unsigned t = (code >> 2) & 31, sj = 4 + (code & 3);
                    const unsigned sl = 512 + t * 32 + ((rank ^ sj) & 7) * 4 + (peer & 3);
                    src = dst = pb + sl * ITEM;
                    sfl = dfl = pb + CTRL + flag_of_slot(sl, fs);
                } else {
                    const int id = staged_id(code & 0x1FF, rank, link, P.xrot);
                    const int sl = recv_slot(id, peer, rank) & 2047;
                    src = pb + STG + id * ITEM;
                    dst = pb + sl * ITEM;
                    sfl = pb + rdy_of(id);
                    dfl = sl < 256 ? pb + OX + flag_of_slot(sl, 32 * fs) + 1 : pb + CTRL + flag_of_slot(sl, fs);
                }
                desc[lane] = u4{src, dst, dfl, 0};
            };
            describe();
            const int ahead = P.ahead, wide = P.wide, kmeas = P.kmeas, kplan = P.kplan;
            const int rel = nhead + slot + NSEND * lane - P.dyn[link >> 2];
            int n = __builtin_popcountll(__ballot(rel < 0)), stage = 0;
            bool done = false;
            u64 t0 = __builtin_amdgcn_s_memrealtime();
            for (;;) {
                const u64 pend = __ballot(!done);
                if (!pend) break;
                const int first = __builtin_ctzll(pend), at = uni((int)ctl[63]);
                bool ok = false;
                if (stage == 0 && at > kmeas) {
                    typedef unsigned u2 __attribute__((ext_vector_type(2)));
                    const unsigned t = ((ldsu)lds)[kmeas * 4 + 3];
                    if (lane == 0) __builtin_amdgcn_raw_buffer_store_b64(u2{t, call}, rws, pb + PACE + (link * 4 + (slot & 3)) * 8, 0, AUX_SYS);
                    stage = 1;
                }
                if (stage == 1 && ((at >= kplan && first > at + 4) || first >= n)) {
                    const auto pc = __builtin_amdgcn_raw_buffer_load_b64(rws, lane < 32 ? pb + PACE + lane * 8 : STAMP, 0, AUX_SYS);
                    if ((__ballot(pc[1] == call) & 0xFFFFFFF0ull) == 0xFFFFFFF0ull) {
                        int d = min(max((int)(pc[0] - (unsigned)__builtin_amdgcn_readlane((int)pc[0], 4)), -2000), 2000);
                        d += __builtin_amdgcn_mov_dpp(d, 0xB1, 15, 15, true);
                        d += __builtin_amdgcn_mov_dpp(d, 0x4E, 15, 15, true);
                        int e[8], in = 0, all = 0;
#pragma unroll
                        for (int l = 1; l < 8; ++l) { e[l] = __builtin_amdgcn_readlane(d, l * 4); all += e[l]; if (l < 4) in += e[l]; }
                        const int mine = __builtin_amdgcn_readlane(d, link * 4), base = P.base, gain = P.gain;
                        int c = link < 4 ? 2 << 8 : 1 << 8, left = rel;
                        bool found = false;
#pragma unroll
                        for (int i = 0; i < 4; ++i) {
                            const int sx = link < 4 ? i : link & 3, p = link < 4 ? link : i;
                            const int late = 28 * (link < 4 ? e[4 + i] : mine) - 21 * (link < 4 ? mine : e[i ? i : 1]) + 7 * in - 4 * all;
                            const int share = min(max((base + late * gain + (((sx + 3 * p) & 3) << 18)) >> 20, 0), 16);
                            const int cnt = link < 4 ? share : i ? 16 - share : 0;
                            if (!found && left < cnt) { found = true; c |= (8 + left + (link < 4 ? 0 : share)) << 2 | i; }
                            left -= cnt;
                            STAMP_V(64 + i, share);
                        }
                        if (rel >= 0) {
                            code = P.list[link >> 2][min(max(P.dyn[link >> 2] + left, 0), MAXITEMS - 1)];
                            if (found) code = c;
                            describe();
                        }
                        n = __builtin_popcountll(__ballot(code != 0xFFFF));
                        if (lane >= n) done = true;
                        if (lane == n) ctl[lane] = RDY_END;
                        stage = 2;
                        STAMP_AT(64 + 4);
                        continue;
                    }
                }
                if (first < n) {
                    if (first > at + (stage < 2 && at + 2 >= kplan ? 5 : ahead)) {
                        __builtin_amdgcn_s_sleep(4);
                        t0 = __builtin_amdgcn_s_memrealtime();
                        continue;
                    }
                    if (!done && lane < min(first + wide, n)) {
                        const u4 f = __builtin_amdgcn_raw_buffer_load_b128(rpw, sfl & ~15u, 0, AUX_SYS);
                        const u4 e = __builtin_amdgcn_raw_buffer_load_b128(rws, lfl, 0, AUX_SYS);
                        const unsigned c = (sfl >> 2) & 3;
                        const unsigned v = c == 0 ? f[0] : c == 1 ? f[1] : c == 2 ? f[2] : f[3];
                        ok = (v >> 8) == tag;
                        if (type == 3) ok = ok && f[1] == v && f[2] == v && f[3] == v && e[0] == v && e[1] == v && e[2] == v && e[3] == v;
                        if (ok) { ctl[lane] = (v & 255u) | (type == 3 ? RDY_VALID | RDY_Y : RDY_VALID); done = true; }
                    }
                }
                if (__any(ok)) t0 = __builtin_amdgcn_s_memrealtime();
                else if (spin_pause(t0, tmo, call)) {
                    if (!done && lane < 63) ctl[lane] = RDY_END;
                    if (!done && type == 3) po[dst >> 2] = __builtin_nanf("");
                    break;
                }
            }
            STAMP_OUT();
            __builtin_amdgcn_endpgm();
        }
        const int vo = lane * 16;
        for (int k = 0;;) {
            unsigned r;
            while ((r = (unsigned)uni((int)ctl[k])) == 0) __builtin_amdgcn_s_sleep(1);
            if (r & RDY_END) break;
            const u4 d0 = desc[k];
            unsigned so = uni(d0[0]), cdst = uni(d0[1]), cdfl = uni(d0[2]);
            f32v r0, r1, r2, r3;
            unsigned la = (unsigned)(uintptr_t)lds + 4 * ITEM + k * 4, lq = (unsigned)(uintptr_t)lds + k * 16, vt, vd, vd2, pfv, ss, lo, dso, ds, ndfl, rn, pfo;
            u64 pm, ym;
            u4 vq;
#define LD(f) "buffer_load_dwordx4 " VR(128 + f * 4) ", %[vo], %[rs], %[so] offen sc0 sc1\n\ts_add_u32 %[so], %[so], %[ss]\n\t" \
              "buffer_load_dwordx4 " VR(192 + f * 4) ", %[vo], s[84:87], %[lo] offen sc0 sc1\n\ts_add_u32 %[lo], %[lo], 0x4000\n\t"
#define ST(f) "s_mov_b64 exec, %[ym]\n\tv_pk_add_f32 " V2(128 + f * 4) ", " V2(128 + f * 4) ", " V2(192 + f * 4) "\n\t" \
              "v_pk_add_f32 " V2(130 + f * 4) ", " V2(130 + f * 4) ", " V2(194 + f * 4) "\n\ts_mov_b64 exec, -1\n\t" \
              "buffer_store_dwordx4 " VR(128 + f * 4) ", %[vo], s[88:91], %[dso] offen sc0 sc1\n\ts_add_u32 %[dso], %[dso], %[ds]\n\t"
#define WTW "s_waitcnt vmcnt(" S(WIN) ")\n\t"
#define SL(f) WTW ".if " #f " == " S(WIN) " / 3\n\t" FLAG ".endif\n\t" ST(f) LD(f)
#define SE(f) "s_waitcnt vmcnt(45 - 2 * " #f ")\n\t.if " #f " == 15\n\t" FLAG ".endif\n\t" ST(f)
#define L0(f) WTW "buffer_load_dword %[vd], off, %[rf], 0\n\t" LD(f)
#define FLAG "s_mov_b64 exec, %[pm]\n\ts_cbranch_execz 2\n\tbuffer_store_dword %[pfv], %[vf], %[rf], %[pfo] offen sc0 sc1\n\ts_mov_b64 exec, -1\n\t"
#define NEXTFLAG "s_mov_b32 %[pfo], %[cdfl]\n\ts_mov_b32 %[cdfl], %[ndfl]\n\ts_and_b32 %[r], %[r], 0xff\n\ts_or_b32 %[r], %[r], %[tag8]\n\t" \
                 "v_mov_b32 %[pfv], %[r]\n\ts_cmp_lg_u32 %[pfo], 0\n\ts_cselect_b64 %[pm], 1, 0\n\ts_bitcmp1_b32 %[pfo], 0\n\t" \
                 "s_cselect_b64 %[pm], %[m32], %[pm]\n\ts_bitset0_b32 %[pfo], 0\n\t"
#define ASTAMP
#define ASTAMP_OP
#define ASTAMP_CL
            asm volatile("; OPAQUE\n\t"
                         "s_mov_b32 %[lo], %[cdst]\n\t"
                         "s_mov_b64 %[pm], 0\n\t"
                         "s_mov_b32 %[pfo], 0\n\t"
                         "s_mov_b32 s91, 0x20000\n\t"
                         "s_mov_b32 s87, 0x20000\n\t"
                         "s_mov_b64 s[84:85], %[po]\n\t"
                         "s_movk_i32 %[ss], 0x400\n\t"
                         "s_bitcmp1_b32 %[r], 10\n\t"
                         "s_cselect_b32 s86, %[szo], 0\n\t"
                         "s_cmovk_i32 %[ss], 0x2000\n\t"
                         "s_nop 4\n\t"
                         L0(0) L0(1) L0(2) L0(3) L0(4) L0(5) L0(6) L0(7) L0(8) L0(9) L0(10) L0(11) L0(12) L0(13) L0(14) L0(15)
                         ".Lsend_loop%=:\n\t"
                         "s_memrealtime s[92:93]\n\t"
                         "ds_read_b32 %[vt], %[la] offset:4\n\t"
                         "ds_read_b96 v[120:122], %[lq] offset:16\n\t"
                         "s_add_u32 %[k], %[k], 1\n\t"
                         "v_mov_b32 %[vd2], %[k]\n\t"
                         "s_mov_b32 %[dso], %[cdst]\n\t"
                         "s_movk_i32 %[ds], 0x400\n\t"
                         "s_movk_i32 %[ss], 0x400\n\t"
                         "s_bitcmp1_b32 %[r], 10\n\t"
                         "s_cselect_b64 s[88:89], %[po], %[pw]\n\t"
                         "s_cselect_b32 s90, %[szo], %[szw]\n\t"
                         "s_cselect_b64 %[ym], -1, 0\n\t"
                         "s_cmovk_i32 %[ds], 0x4000\n\t"
                         "s_waitcnt lgkmcnt(0)\n\t"
                         "v_mov_b32 v123, s92\n\t"
                         "v_readfirstlane_b32 %[rn], %[vt]\n\t"
                         "v_readfirstlane_b32 %[so], v120\n\t"
                         "v_readfirstlane_b32 %[cdst], v121\n\t"
                         "v_readfirstlane_b32 %[ndfl], v122\n\t"
                         "ds_write_b32 %[lq], v123 offset:12\n\t"
                         "ds_write_b32 %[lp], %[vd2]\n\t"
                         "s_mov_b32 %[lo], %[cdst]\n\t"
                         "s_bitcmp1_b32 %[rn], 10\n\t"
                         "s_cselect_b32 s86, %[szo], 0\n\t"
                         "s_cmovk_i32 %[ss], 0x2000\n\t"
                         "v_add_u32 %[la], 4, %[la]\n\t"
                         "v_add_u32 %[lq], 16, %[lq]\n\t"
                         ASTAMP
                         "s_bitcmp1_b32 %[rn], 8\n\t"
                         "s_cbranch_scc0 .Lsend_last%=\n\t"
                         SL(0) SL(1) SL(2) SL(3) SL(4) SL(5) SL(6) SL(7) SL(8) SL(9) SL(10) SL(11) SL(12) SL(13) SL(14) SL(15)
                         NEXTFLAG
                         "s_mov_b32 %[r], %[rn]\n\t"
                         "s_branch .Lsend_loop%=\n\t"
                         ".Lsend_last%=:\n\t"
                         SE(0) SE(1) SE(2) SE(3) SE(4) SE(5) SE(6) SE(7) SE(8) SE(9) SE(10) SE(11) SE(12) SE(13) SE(14) SE(15)
                         NEXTFLAG
                         WT(0)
                         FLAG
                         "s_nop 1"
                         : "=&{v[128:159]}"(r0), "=&{v[160:191]}"(r1), "=&{v[192:223]}"(r2), "=&{v[224:255]}"(r3), [lo] "=&s"(lo), [ym] "=&s"(ym),
                           [la] "+v"(la), [lq] "+v"(lq), "=&{v[120:123]}"(vq), [vt] "=&v"(vt), [vd] "=&v"(vd), [vd2] "=&v"(vd2), [pfv] "=&v"(pfv), [so] "+s"(so), [cdst] "+s"(cdst), [cdfl] "+s"(cdfl), [ss] "=&s"(ss), [dso] "=&s"(dso), [ds] "=&s"(ds),
                           [ndfl] "=&s"(ndfl), [rn] "=&s"(rn), [pfo] "=&s"(pfo), [pm] "=&s"(pm), [k] "+s"(k), [r] "+s"(r) ASTAMP_OP
                         : [vo] "v"(vo), [vf] "v"(lane * fs), [m32] "s"(0xFFFFFFFFull), [lp] "v"((unsigned)(uintptr_t)lds + 4 * ITEM + 252),
                           [rs] "s"(rpw), [rf] "s"(rws), [pw] "s"(myws), [po] "s"(po), [szw] "s"(STAMP), [szo] "s"(MSH * NCOL * 4), [tag8] "s"(tag << 8)
                         : "memory", "scc", "s84", "s85", "s86", "s87", "s88", "s89", "s90", "s91", "s92", "s93" ASTAMP_CL);
            STAMP_AT(63 + k);
        }
        STAMP_OUT();
        __builtin_amdgcn_endpgm();
    }

    const bool own = g >= 12;
    const int o = g - 12, omap = P.omap;
    const int fl = xcd >> 1, fh = xcd & 1;
    const int rb = own ? (omap ? (o & 3) : (xcd & 1) * 2 + (o & 1)) : 4 + fl;
    const int grb = own ? rank * 4 + rb : ((rank ^ rb) & 7) * 4 + pos;
    const int njob = g < 4 || own ? 2 : 1;
    auto tile = [&](int job) {
        return own ? (omap ? xcd * 4 + (o >> 2) : (xcd >> 1) * 8 + (o >> 1)) : (job ? 3 : g >> 2) * 8 + fh * 4 + (g & 3);
    };
    if (wave == 4) {
        auto rws = __builtin_amdgcn_make_buffer_rsrc(myws, 0, (int)STAMP, 0x00020000);
        const unsigned gate = P.gate[min(g >> 2, 3)];
        while ((unsigned)__builtin_amdgcn_s_memrealtime() - tstart < gate) __builtin_amdgcn_s_sleep(8);
        __builtin_amdgcn_s_setprio(3);
        unsigned kx = 0;
#pragma unroll 1
        for (int job = 0; job < njob; ++job) {
            const int t = tile(job);
            const int rot = (g * P.rot) & 31, fastu = P.fastu, slp0 = P.slp0, slp1 = P.slp1;
            const int ls = lane & 31, lh = lane >> 5;
            const bool mini = own && !job;
            const unsigned wid = 256 + (((rank >> 2) ^ mini) * 32 + t) * 4 + (ls & 3);
            const unsigned skip = mini ? ~(15u << rank * 4) : own ? 0 : 15u << (rank ^ rb) * 4;
            const bool off = skip >> ls & 1;
            unsigned fxo = off || kx ? STAMP : pb + (own && job ? OX + (rb * 32 + t) * fs : CTRL + rb * fs) + lane * 4;
            unsigned fwo = off || lh ? STAMP : pb + (mini ? rdy_of(wid) : CTRL + flag_of_slot(512 + t * 32 + ls, fs));
            const bool loc = (ls >> 2) == rank;
            const unsigned offa = pb + (loc ? STG + ((grb * 4 + (ls & 3)) * 2 + lh) * ITEM : ((rb * 32 + ls) * 2 + lh) * ITEM);
            const unsigned offb = pb + (loc ? STG + wid * ITEM : XR_BYTES + (t * 32 + ls) * ITEM);
            unsigned issued = skip, xa, wa, xb, wb, kw = 0;
            int u = __builtin_popcount(skip);
            bool dead = false;
            u64 t0 = __builtin_amdgcn_s_memrealtime();
#define POLL(c, fx, fw, so) asm volatile("; POLL\n\tbuffer_load_dword %0, %2, %4, %5 offen sc0 sc1\n\tbuffer_load_dword %1, %3, %4, %5 offen sc0 sc1" \
                                  : c(fx), c(fw) : "v"(fxo), "v"(fwo), "s"(rws), "s"(so) : "memory")
            auto tick = [&](unsigned& px_, unsigned& pw_, bool second) {
                asm volatile("; GOT\n\ts_waitcnt vmcnt(2)" : "+v"(px_), "+v"(pw_) :: "memory");
                if (px_ + ~(tag << 8) < 255u) { kx = px_; fxo = STAMP; }
                if (pw_ + ~(tag << 8) < 255u) { kw = pw_; fwo = STAMP; }
                const u64 bx = __ballot(kx != 0);
                unsigned rdy = (unsigned)bx & (unsigned)(bx >> 32) & (unsigned)__ballot(kw != 0);
                rdy |= skip;
                if (dead) rdy = ~0u;
                unsigned avail = rdy & ~issued;
                const bool slow = u < fastu || mini;
                if (!avail && !(slow && second)) {
                    const int n = slow ? slp0 : slp1;
                    for (int i = 0; i < n; ++i) __builtin_amdgcn_s_sleep(2);
                    dead = spin_pause(t0, tmo, call);
                }
                POLL("+v", px_, pw_, slow && second ? STAMP : 0u);
                if (!avail) return;
                do {
                    const int s = (__builtin_ctz(__builtin_amdgcn_alignbit(avail, avail, rot)) + rot) & 31;
                    issued |= 1u << s;
                    avail &= ~(1u << s);
                    const u4 rec = {offa, offb, (kx & 255u) + ((unsigned)__builtin_amdgcn_readlane((int)kw, s) & 255u) - 64u, dead};
                    if (ls == s) *(volatile __attribute__((address_space(3))) u4*)(ctl + (u & 1) * 8 + lh * 4) = rec;
                    STAMP_AT(job * 32 + u);
                    STAMP_V(64 + job * 32 + u, s);
                    BAR();
                    ++u;
                } while (__builtin_popcount(avail) > (rdy == ~0u ? 0 : 2));
                t0 = __builtin_amdgcn_s_memrealtime();
            };
            POLL("=&v", xa, wa, 0u);
            POLL("=&v", xb, wb, STAMP);
            for (;;) {
                tick(xa, wa, false);
                if (u == 32) break;
                tick(xb, wb, true);
                if (u == 32) break;
            }
            BAR(); BAR(); BAR();
            asm volatile("s_waitcnt vmcnt(0)" :: "v"(xa), "v"(wa), "v"(xb), "v"(wb) : "memory");
            if (own && job) break;
            BAR();
            if (lane == 0) __hip_atomic_store((unsigned*)(myws + pb + (own ? LRDY : YRDY)) + (rb & 3) * 32 + t, tag << 8, RLX_SYS);
        }
        STAMP_OUT();
        __builtin_amdgcn_endpgm();
    }
#pragma unroll 1
    for (int job = 0; job < njob; ++job) {
        const int t = tile(job);
        f32v acc, d1, d2, d3, d4, d5;
        u4 dc;
        int ecur, emax, cnt, e0, e1, e2, e3, ta, tb;
        unsigned keep;
        u64 ret;
        const int rs = own ? NCOL * 4 : 8192;
        char* ob = own ? (char*)P.out + (size_t)(rb * 128 + wave * 32) * (NCOL * 4) + ((rank >> 2) ^ job ^ 1) * 8192 + t * 256
                       : myws + (unsigned)uni((int)ctl[C_CALL + 2]) + YST + (size_t)((rb & 3) * 128 + wave * 32) * 8192 + t * 256;
        auto ro = __builtin_amdgcn_make_buffer_rsrc(ob, 0, 31 * rs + 256, 0x00020000);
#define ACC(i, nf) VR(64 + i * 16 + nf * 4)
#define BF(ks, nf) VR(96 + (ks & 1) * 16 + nf * 4)
#define AF(c, f) VR(128 + c * 32 + (f) * 4)
#define RDB(c, ks, nf) "ds_read_b128 " BF(ks, nf) ", %[vl] offset:" S(c * 16384 + ks * 4096 + nf * 1024) "\n\t"
#define RDB8(c) RDB(c, 0, 0) RDB(c, 0, 1) RDB(c, 0, 2) RDB(c, 0, 3) RDB(c, 1, 0) RDB(c, 1, 1) RDB(c, 1, 2) RDB(c, 1, 3)
#define MM(c, ks, nf, w) "s_waitcnt lgkmcnt(" #w ")\n\t" \
        "v_mfma_f32_16x16x32_f16 " ACC(0, nf) ", " AF(c, ks) ", " BF(ks, nf) ", " ACC(0, nf) "\n\t" \
        "v_mfma_f32_16x16x32_f16 " ACC(1, nf) ", " AF(c, 4 + ks) ", " BF(ks, nf) ", " ACC(1, nf) "\n\t"
#define LDB(j) "global_load_lds_dwordx4 v61, %[ws] offset:" S(j * 1024) " sc0 sc1\n\t"
#define LDA(n, f) "global_load_dwordx4 " AF(n, f) ", v60, %[ws] offset:" S((f) * 1024 - 4096) " sc0 sc1\n\t"
#define LD12(n) LDB(0) LDB(1) LDB(2) LDB(3) LDA(n, 0) LDA(n, 4) LDA(n, 1) LDA(n, 5) LDA(n, 2) LDA(n, 6) LDA(n, 3) LDA(n, 7)
#define REC(n) "ds_read_b96 v[60:62], %[vc] offset:" S((n & 1) * 32) "\n\t"
#define ADR(n) "s_add_u32 m0, %[lb], " S(n * 16384) "\n\tv_add_u32 v60, v60, %[voa]\n\tv_add_u32 v61, v61, %[vob]\n\tv_readfirstlane_b32 %[e" #n "], v62\n\t"
#define PRO(n) "s_barrier\n\t" REC(n) "s_waitcnt lgkmcnt(0)\n\t" ADR(n) LD12(n)
#define CHK(c, id) "s_cmp_lg_u32 %[e" #c "], %[ecur]\n\ts_cbranch_scc1 .Lfix" #id "_%=\n\t.Lback" #id "_%=:\n\t"
#define FIX(c, id, pre) ".Lfix" #id "_%=:\n\t" pre "s_call_b64 %[ret], .Lslow" #c "_%=\n\ts_branch .Lback" #id "_%=\n\t"
#define STEP(c, n) "s_waitcnt vmcnt(24)\n\ts_barrier\n\t" REC(n) RDB8(c) CHK(c, c) "s_waitcnt lgkmcnt(8)\n\t" ADR(n) \
        MM(c, 0, 0, 7) RDB(c, 2, 0) LDB(0) MM(c, 0, 1, 7) RDB(c, 2, 1) LDB(1) MM(c, 0, 2, 7) RDB(c, 2, 2) LDB(2) MM(c, 0, 3, 7) RDB(c, 2, 3) \
        MM(c, 1, 0, 7) RDB(c, 3, 0) LDB(3) MM(c, 1, 1, 7) RDB(c, 3, 1) LDA(n, 0) MM(c, 1, 2, 7) RDB(c, 3, 2) LDA(n, 4) MM(c, 1, 3, 7) RDB(c, 3, 3) \
        MM(c, 2, 0, 7) LDA(n, 1) MM(c, 2, 1, 6) LDA(n, 5) MM(c, 2, 2, 5) LDA(n, 2) MM(c, 2, 3, 4) \
        MM(c, 3, 0, 3) LDA(n, 6) MM(c, 3, 1, 2) LDA(n, 3) MM(c, 3, 2, 1) LDA(n, 7) MM(c, 3, 3, 0)
#define MMS(c, w0, w1, w2, w3) \
        w0 MM(c, 0, 0, 7) RDB(c, 2, 0) MM(c, 0, 1, 7) RDB(c, 2, 1) MM(c, 0, 2, 7) RDB(c, 2, 2) MM(c, 0, 3, 7) RDB(c, 2, 3) \
        w1 MM(c, 1, 0, 7) RDB(c, 3, 0) MM(c, 1, 1, 7) RDB(c, 3, 1) MM(c, 1, 2, 7) RDB(c, 3, 2) MM(c, 1, 3, 7) RDB(c, 3, 3) \
        w2 MM(c, 2, 0, 7) MM(c, 2, 1, 6) MM(c, 2, 2, 5) MM(c, 2, 3, 4) w3 MM(c, 3, 0, 3) MM(c, 3, 1, 2) MM(c, 3, 2, 1) MM(c, 3, 3, 0)
#define TBAR(n) "s_waitcnt vmcnt(" #n ")\n\ts_cbranch_execz 1\n\ts_barrier\n\t"
#define ZERO(r) "v_mov_b32 v[" S(r) "], 0\n\t"
#define BAD(r) "v_mov_b32 v[" S(r) "], -1\n\t"
#define LDX(r) "v_ldexp_f32 v[" S(r) "], v[" S(r) "], %[tb]\n\t"
#define PKM(r) "v_pk_mul_f16 v[" S(r) "], v[" S(r) "], v63 op_sel_hi:[1,0]\n\t"
#define SLOW(c) ".Lslow" #c "_%=:\n\ts_max_i32 %[emax], %[emax], %[e" #c "]\n\ts_add_i32 %[ta], %[emax], -7\n\ts_max_i32 %[ta], %[ta], %[e" #c "]\n\t" \
        "s_sub_i32 %[tb], %[ecur], %[ta]\n\ts_lshl_b32 %[tb], %[tb], 3\n\ts_mov_b32 %[ecur], %[ta]\n\t" R32(LDX, 64) \
        "s_sub_i32 %[tb], %[e" #c "], %[ta]\n\ts_lshl_b32 %[tb], %[tb], 3\n\tv_ldexp_f32 v63, 1.0, %[tb]\n\tv_cvt_f16_f32 v63, v63\n\ts_nop 0\n\t" \
        R32(PKM, 128 + c * 32) "s_setpc_b64 %[ret]\n\t"
#define EP1(i, j, c) "v_ldexp_f32 v[" S(128 + (i * 4 + j) * 4 + c) "], v[" S(64 + i * 16 + c * 4 + j) "], %[tb]\n\t"
#define EP8(c) EP1(0, 0, c) EP1(0, 1, c) EP1(0, 2, c) EP1(0, 3, c) EP1(1, 0, c) EP1(1, 1, c) EP1(1, 2, c) EP1(1, 3, c)
#define EPS(i, step) "buffer_store_dwordx4 " VR(128 + i * 4) ", %[vo], %[ro], %[ta] offen sc0 sc1\n\ts_add_u32 %[ta], %[ta], " step "\n\t"
#define GSTAMP(n)
#define GSTAMP_OP
        asm volatile("; OPAQUE\n\t"
                     "s_mov_b32 %[keep], m0\n\t"
                     "s_mov_b32 %[cnt], 0\n\t"
                     "s_mov_b32 %[ecur], 0\n\ts_mov_b32 %[e0], 0\n\ts_mov_b32 %[e1], 0\n\ts_mov_b32 %[e2], 0\n\ts_mov_b32 %[e3], 0\n\t"
                     "s_mov_b64 exec, 0\n\t"
                     "s_branch .Lgemm_tail%=\n\t"
                     SLOW(0) SLOW(1) SLOW(2) SLOW(3)
                     FIX(0, 0, "") FIX(1, 1, "") FIX(2, 2, "") FIX(3, 3, "")
                     FIX(0, 4, "") FIX(1, 5, "") FIX(2, 6, "") FIX(3, 7, "s_waitcnt vmcnt(0)\n\t")
                     ".Lgemm_bad%=:\n\t"
                     "s_nop 7\n\t"
                     R32(BAD, 64)
                     "s_branch .Lgemm_ep%=\n\t"
                     ".Lgemm_go%=:\n\t"
                     "s_mov_b64 exec, -1\n\t"
                     R32(ZERO, 64)
                     PRO(0) PRO(1) PRO(2)
                     "s_mov_b32 %[ecur], %[e0]\n\t"
                     "s_mov_b32 %[emax], %[e0]\n\t"
                     "s_mov_b32 %[cnt], %[cnt0]\n\t"
                     "s_cmp_eq_u32 %[cnt], 7\n\t"
                     "s_cbranch_scc1 .Lgemm_tail%=\n\t"
                     ".Lgemm%=:\n\t"
                     STEP(0, 3) STEP(1, 0) STEP(2, 1) STEP(3, 2)
                     "s_add_u32 %[cnt], %[cnt], 1\n\t"
                     "s_cmp_lt_u32 %[cnt], 7\n\t"
                     "s_cbranch_scc1 .Lgemm%=\n\t"
                     ".Lgemm_tail%=:\n\t"
                     TBAR(24) "ds_read_b128 v[60:63], %[vc] offset:32\n\t" RDB8(0)
                     "s_waitcnt lgkmcnt(8)\n\t"
                     "s_cbranch_execz .Lgemm_t1%=\n\t"
                     ADR(3) LD12(3)
                     "v_readfirstlane_b32 %[cnt], v63\n\t"
                     ".Lgemm_t1%=:\n\t"
                     CHK(0, 4) MMS(0, "", "", "", "")
                     TBAR(24) RDB8(1) CHK(1, 5) MMS(1, "", "", "", "")
                     TBAR(12) RDB8(2) CHK(2, 6) MMS(2, "", "", "", "")
                     TBAR(8) RDB8(3) CHK(3, 7) MMS(3, WT(6), WT(4), WT(2), WT(0))
                     GSTAMP(0)
                     "s_cmp_lg_u32 %[cnt], 0\n\t"
                     "s_cbranch_scc1 .Lgemm_bad%=\n\t"
                     ".Lgemm_ep%=:\n\t"
                     "s_lshl_b32 %[tb], %[ecur], 3\n\t"
                     "s_mov_b32 %[ta], 0\n\t"
                     "s_mul_i32 %[emax], %[rs], 13\n\t"
                     EP8(0) EP8(1) EP8(2) EP8(3)
                     "s_cbranch_execz .Lgemm_go%=\n\t"
                     EPS(0, "%[rs]") EPS(1, "%[rs]") EPS(2, "%[rs]") EPS(3, "%[emax]") EPS(4, "%[rs]") EPS(5, "%[rs]") EPS(6, "%[rs]") EPS(7, "%[rs]")
                     GSTAMP(1)
                     "s_mov_b32 m0, %[keep]\n\t"
                     "s_waitcnt lgkmcnt(0)"
                     : "=&{v[64:95]}"(acc), "=&{v[96:127]}"(d1), "=&{v[128:159]}"(d2), "=&{v[160:191]}"(d3),
                       "=&{v[192:223]}"(d4), "=&{v[224:255]}"(d5), "=&{v[60:63]}"(dc),
                       [ecur] "=&s"(ecur), [emax] "=&s"(emax), [cnt] "=&s"(cnt), [e0] "=&s"(e0), [e1] "=&s"(e1), [e2] "=&s"(e2), [e3] "=&s"(e3),
                       [ta] "=&s"(ta), [tb] "=&s"(tb), [keep] "=&s"(keep), [ret] "=&s"(ret) GSTAMP_OP
                     : [vl] "v"((unsigned)(uintptr_t)lds + lane * 16), [vc] "v"((unsigned)(uintptr_t)lds + 4 * ITEM + (wave >> 1) * 16),
                       [voa] "v"(lane * 16 + 4096 + (wave & 1) * 8192), [vob] "v"(lane * 16 + wave * 4096),
                       [lb] "s"((unsigned)(uintptr_t)lds + wave * 4096), [ws] "s"(myws),
                       [vo] "v"(q4 * 4 * rs + c16 * 16), [ro] "s"(ro), [rs] "s"(rs), [cnt0] "s"(!own ? 1 : job ? 0 : 7)
                     : "memory", "scc");
        if (!(own && job)) asm volatile("s_waitcnt vmcnt(0)\n\ts_barrier" :: "v"(d2) : "memory");
        STAMP_V(job * 4 + 0, (unsigned)t0);
        STAMP_V(job * 4 + 1, (unsigned)t1);
    }
    STAMP_OUT();
}

struct Tune { int g0 = 0, g1 = 1800, g2 = 2600, g3 = 2050, cyc = 0, flip = 0, xc = 2, nfast = 28, stag2 = 860, pref = 1, ahead = 2, wide = 4, nhead = 12, hfree = 2, hstag = 430, ht0 = 160, hlead = 200, tgo = 315, gstag = 0, stag0 = 40, stag = 430, fs = 4352, rot = 3, xrot = 1, omap = 0, fastu = 20, slp0 = 20, slp1 = 0, base = 620, gain = 100, kmeas = 13, kplan = 17, near = 1, wsplit = 1; };

void build_lists(const Tune& T, uint16_t (&L)[3][MAXITEMS], int (&len)[3], int (&dyn)[3]) {
    for (int c = 0; c < 3; ++c) {
        int n = 0;
        auto X = [&](int a0) { for (int i = 0; i < 32; ++i) L[c][n++] = (uint16_t)(0 << 8 | (a0 + (i & 7)) << 2 | i >> 3); };
        auto W = [&](int t0, int t1, int q0 = 0, int q1 = 4) { for (int q = q0; q < q1; ++q) for (int t = t0; t < t1; ++t) L[c][n++] = (uint16_t)(1 << 8 | t << 2 | q); };
        if (c < 2) {
            X(8); W(0, 8); W(8, 16);
            if (T.wsplit == 0) { W(16, 24); dyn[c] = n; W(24, 32); }
            if (T.wsplit == 1) { W(16, 32, 0, 1); dyn[c] = n; W(16, 32, 1, 2); for (int t = 16; t < 32; t += 4) W(t, t + 4, 2, 4); }
            if (T.wsplit == 2) { W(16, 32, 0, 1); dyn[c] = n; W(16, 32, 1, 2); W(16, 24, 2, 4); W(24, 32, 2, 4); }
            if (T.wsplit == 3) { W(16, 32, 0, 1); dyn[c] = n; W(16, 32, 1, 2); W(16, 24, 2, 3); W(16, 20, 3, 4); W(24, 28, 2, 3); W(20, 24, 3, 4); W(28, 32, 2, 3); W(24, 32, 3, 4); }
            if (T.wsplit == 4) { W(16, 24, 0, 2); dyn[c] = n; W(16, 24, 2, 4); W(24, 32); }
            X(0);
        } else {
            for (int i = 0; i < 24; ++i) L[c][n++] = (uint16_t)(0 << 8 | (10 + i % 6) << 2 | i / 6);
            W(0, 8); W(8, 24, 0, 1);
            if (T.xc == 2) X(0);
            if (T.xc == 3) W(24, 32);
            dyn[c] = n;
            if (T.xc == 1 || T.xc == 3) X(0);
            if (T.xc != 3) W(24, 32);
            for (int i = 0; i < 64; ++i) L[c][n++] = (uint16_t)(3 << 8 | i);
        }
        len[c] = n;
        while (n < MAXITEMS) L[c][n++] = 0xFFFF;
    }
}

}

extern "C" void kernel_launch(int rank, int nranks, ncclComm_t comm,
                              hipStream_t stream,
                              void* const* d_in, const int* in_sizes, int n_in,
                              void* d_out, int out_size,
                              const int* in_dtypes, int out_dtype,
                              void* const* d_out_all,
                              void* const* d_ws_all, size_t ws_bytes) {
    (void)comm; (void)in_dtypes; (void)out_dtype; (void)d_out_all;
    if (nranks != NR || (unsigned)rank >= (unsigned)NR || n_in < 2 || ws_bytes < WS_NEED ||
        in_sizes[0] != 4096 * KSH || in_sizes[1] != KSH * NCOL || out_size != MSH * NCOL) {
        fprintf(stderr, "kernel_launch: unexpected problem shape\n");
        return;
    }
    const Tune T;
    uint16_t L[3][MAXITEMS];
    int len[3], dyn[3];
    build_lists(T, L, len, dyn);
    static const unsigned char upper[NR] = {0xB2, 0x8E, 0xE8, 0xD4, 0xD4, 0xE8, 0x8E, 0xB2};
    int xcd_of[NR], link_of[NR];
    for (int l = 0, lo = 0, hi = 4; l < NR; ++l) {
        xcd_of[l] = !T.near ? l : (upper[rank] >> l) & 1 ? hi++ : lo++;
        xcd_of[l] = (xcd_of[l] & 4) | (((xcd_of[l] ^ T.flip) + T.cyc) & 3);
        link_of[xcd_of[l]] = l;
    }
    Params P;
    P.x = (const float*)d_in[0];
    P.w = (const float*)d_in[1];
    P.out = (float*)d_out;
    P.links = 0;
    for (int j = 0; j < NR; ++j) {
        P.ws[j] = (char*)d_ws_all[rank ^ j];
        P.links |= (unsigned)link_of[j] << j * 4;
    }
    P.rank = rank;
    P.fs = T.fs; P.nhead = T.nhead;
    P.rot = T.rot; P.xrot = T.xrot; P.omap = T.omap; P.fastu = T.fastu; P.slp0 = T.slp0; P.slp1 = T.slp1; P.ahead = T.ahead; P.wide = T.wide;
    P.gate[0] = T.g0; P.gate[1] = T.g1; P.gate[2] = T.g2; P.gate[3] = T.g3;
    P.dyn[0] = dyn[1]; P.dyn[1] = dyn[2]; P.kmeas = T.kmeas; P.kplan = T.kplan;
    P.base = (T.base << 20) / 100 + (1 << 17); P.gain = T.gain * 1166 / 1000;
    for (int c = 0; c < 2; ++c) for (int i = 0; i < MAXITEMS; ++i) P.list[c][i] = L[c + 1][i];
    if (len[1] + 64 - T.nhead > 62 * NSEND || len[2] + 48 - T.nhead > 62 * NSEND || T.nhead + (T.kmeas + 2) * NSEND > dyn[2] || T.gain > 200 || T.nhead > NWORK || 24 * T.fs > (int)(YRDY - CTRL) || 128 * T.fs > (int)(BUF_BYTES - OX)) { fprintf(stderr, "bad tuning\n"); return; }
    auto push_at = [&](int p) { return p < T.hfree ? 0 : T.ht0 + (p + 1 - T.hfree) * T.hstag / 16; };
    auto times = [&](int ld, int st) { return (unsigned)(ld > 0 ? ld / 8 : 0) << 16 | (unsigned)(st / 4) << 24; };
    const int nb = 64 - T.nhead;
    int word[64], freeat[32], cnt[32];
    unsigned when[64];
    for (int s = NSEND; s < 32; ++s) { cnt[s] = s < NSEND + T.nhead; freeat[s] = cnt[s] ? (s < NSEND + T.hfree ? T.ht0 : push_at(s - NSEND)) + 100 : 0; }
    for (int i = 0; i < nb; ++i) {
        const int st = i < NSEND ? T.tgo + i * T.gstag / 16 : 0;
        const int ld = i < NSEND ? st - T.hlead : T.stag0 + (i - NSEND) * T.stag / 16 + (i > T.nfast ? (i - T.nfast) * (T.stag2 - T.stag) / 16 : 0);
        if (ld > 2040) { fprintf(stderr, "bad tuning\n"); return; }
        int s = -1;
        for (int c = NSEND; c < 32; ++c)
            if (cnt[c] < (i >= nb - 8 ? 3 : 2) && (i < nb - 8 || c >= 24) && (s < 0 || freeat[c] < freeat[s]) && !(T.pref && s >= 0 && freeat[s] <= ld)) s = c;
        word[i] = cnt[s] < 2 ? s * 16 + cnt[s] : (s - 24) * 8;
        when[i] = times(ld, st);
        freeat[s] = (ld > freeat[s] ? ld : freeat[s]) + 200;
        ++cnt[s];
    }
    bool taken[512] = {};
    int next = 0;
    for (int p = 0; p < MAXITEMS; ++p)
        for (int x = 0; x < 8; ++x) {
            const int l = link_of[x];
            const int code = L[l == 0 ? 0 : 1 + (l >> 2)][p];
            if ((code >> 8) > 1) { if (p < T.nhead) { fprintf(stderr, "bad list head\n"); return; } continue; }
            const int id = staged_id(code, rank ^ l, l, T.xrot);
            if (taken[id]) { if (p < T.nhead) { fprintf(stderr, "bad list head\n"); return; } continue; }
            taken[id] = true;
            const unsigned c = id < 256 ? (id >> 3) << 10 | (id & 1) << 9 | (id >> 1 & 3) << 1 : 0x8000 | (id & 3) << 13 | (id >> 2 & 63);
            if (p < T.nhead) P.src[((NSEND + p) * 8 + x) * 2] = c | times(push_at(p) - T.hlead, l ? push_at(p) : 0);
            else {
                const int w = word[next >> 3];
                P.src[w + (next & 7) * (w < 64 ? 1 : 2)] = c | when[next >> 3];
                ++next;
            }
        }
    if (next != 8 * nb) { fprintf(stderr, "bad conversion table\n"); return; }
    gemm_rs_kernel<<<8 * (NWORK + NSEND), 384, 0, stream>>>(P);
    HIP_CHECK(hipGetLastError());
}
```
